```python
import numpy as np

kernel_name = "gpu_a2a_gemm_m4096_k4096_n4096_w8a8_dqrelu"


ABSMAX_RTOL = 0.001

COMPUTE_DTYPE = "int8"
EPILOGUE = "dequant_relu"
IN_QUANT = "int8"
OUT_QUANT = None
IN_DTYPES = {"x": "int8", "w_mat": "int8",
             "scale_x": "float32", "scale_w": "float32"}
WORLD_SIZE = 8

def setup_inputs():

    x = np.random.randint(-127, 128, size=(4096, 4096)).astype(np.int8)
    w_mat = np.random.randint(-127, 128, size=(4096, 4096)).astype(np.int8)
    scale_x = (0.02 * (1.0 + np.random.rand(1))).astype(np.float32)
    scale_w = (0.02 * (1.0 + np.random.rand(1))).astype(np.float32)
    return {"x": x, "w_mat": w_mat, "scale_x": scale_x, "scale_w": scale_w}

def reference(x, w_mat, scale_x, scale_w):
    acc = x.astype(np.float64) @ w_mat.astype(np.float64)
    y = (acc.astype(np.float32) * (scale_x * scale_w)).astype(np.float32)
    return np.maximum(y, 0.0).astype(np.float32)
```

```cpp
#include <hip/hip_runtime.h>
#include <rccl/rccl.h>
#include <cstdio>

typedef unsigned u32x4 __attribute__((ext_vector_type(4)));
typedef int i32x4 __attribute__((ext_vector_type(4)));
typedef int i32x2 __attribute__((ext_vector_type(2)));
typedef float f32x4 __attribute__((ext_vector_type(4)));
#define GAS __attribute__((address_space(1)))
#define LAS __attribute__((address_space(3)))

constexpr int NR = 8;
constexpr int MLOC = 512, KS = 512, KTOT = 4096, NTOT = 4096;
constexpr size_t CNT_OFF = 0;
constexpr size_t TMO_OFF = 4096;
constexpr size_t FLAG_OFF = 8192;
constexpr size_t A_OFF = 1 << 20;
constexpr size_t A_SLOT = 2 << 20;
constexpr size_t WS_NEED = A_OFF + 2 * A_SLOT;
constexpr unsigned long long DEADLINE = 50000000ull;

struct Args {
  const signed char* x; const signed char* w; const float* sx; const float* sw;
  float* out; char* ws[NR]; int rank; int pad;
};

__global__ __launch_bounds__(256, 1) void a2a_gemm(Args a) {
  __shared__ __attribute__((aligned(16))) char lds[2 * 24576 + 64];
  const int tid = threadIdx.x, lane = tid & 63;
  const int wave = __builtin_amdgcn_readfirstlane(tid >> 6);
  const int b = blockIdx.x;
  const int rank = a.rank;
  char* myws = a.ws[rank];
  GAS unsigned* cnt = (GAS unsigned*)(myws + CNT_OFF) + b;
  const unsigned call = __builtin_amdgcn_readfirstlane(__hip_atomic_load(cnt, __ATOMIC_RELAXED, __HIP_MEMORY_SCOPE_AGENT)) + 1u;
  const unsigned par = call & 1u;
  {
    const int d = (rank + 1 + (b & 7)) & 7;
    const int u = b >> 3, smt = u >> 2, q = u & 3;
    const int ks = 2 * q + (wave >> 1), mf0 = 2 * (wave & 1);
    char* dws = a.ws[0];
    #pragma unroll
    for (int i = 1; i < NR; ++i) if (d == i) dws = a.ws[i];
    const signed char* src = a.x + (size_t)(d * MLOC + smt * 64 + mf0 * 16 + (lane & 15)) * KS + ks * 64 + (lane >> 4) * 16;
    u32x4 v0 = *(const GAS u32x4*)src;
    u32x4 v1 = *(const GAS u32x4*)(src + 16 * KS);
    char* dst = dws + A_OFF + par * A_SLOT + smt * (256 << 10) + rank * (32 << 10) + (ks * 4 + mf0) * 1024 + lane * 16;
    asm volatile("global_store_dwordx4 %0, %1, off sc0 sc1\n\ts_nop 1" :: "v"(dst), "v"(v0) : "memory");
    asm volatile("global_store_dwordx4 %0, %1, off offset:1024 sc0 sc1\n\ts_nop 1" :: "v"(dst), "v"(v1) : "memory");
    asm volatile("s_waitcnt vmcnt(0)" ::: "memory");
    __syncthreads();
    if (tid == 0) {
      GAS unsigned* fl = (GAS unsigned*)(dws + FLAG_OFF) + smt * 32 + rank * 4 + q;
      __hip_atomic_store(fl, call, __ATOMIC_RELAXED, __HIP_MEMORY_SCOPE_SYSTEM);
      __hip_atomic_store(cnt, call, __ATOMIC_RELAXED, __HIP_MEMORY_SCOPE_AGENT);
    }
  }
  const int xcd = b & 7, idx = b >> 3;
  const int nt = xcd * 4 + (idx & 3), mt = idx >> 2;
  GAS unsigned* tmo = (GAS unsigned*)(myws + TMO_OFF);
  if (wave == 0) {
    GAS unsigned* fl = (GAS unsigned*)(myws + FLAG_OFF) + mt * 32 + (lane & 31);
    unsigned long long t0 = __builtin_amdgcn_s_memrealtime();
    while (true) {
      unsigned v = __hip_atomic_load(fl, __ATOMIC_RELAXED, __HIP_MEMORY_SCOPE_SYSTEM);
      if (__all(v >= call)) break;
      if (__builtin_amdgcn_readfirstlane(__hip_atomic_load(tmo, __ATOMIC_RELAXED, __HIP_MEMORY_SCOPE_AGENT)) == call ||
          __builtin_amdgcn_s_memrealtime() - t0 > DEADLINE) {
        __hip_atomic_store(tmo, call, __ATOMIC_RELAXED, __HIP_MEMORY_SCOPE_AGENT);
        break;
      }
      __builtin_amdgcn_s_sleep(2);
    }
    asm volatile("s_waitcnt vmcnt(0)" ::: "memory");
  }
  __syncthreads();
  const bool bad = __builtin_amdgcn_readfirstlane(__hip_atomic_load(tmo, __ATOMIC_RELAXED, __HIP_MEMORY_SCOPE_AGENT)) == call;

  i32x4 acc[4][2] = {};
  if (!bad) {
    const char* Ab = myws + A_OFF + par * A_SLOT + mt * (256 << 10) + lane * 16;
    const signed char* Bb0 = a.w + (size_t)(lane >> 3) * NTOT + nt * 128 + (((lane & 7) ^ ((lane >> 4) & 3)) * 16);
    const signed char* Bb1 = a.w + (size_t)(lane >> 3) * NTOT + nt * 128 + (((lane & 7) ^ (((lane >> 4) & 3) | 4)) * 16);
    auto stage = [&](int s, int buf) {
      char* L = lds + buf * 24576;
      #pragma unroll
      for (int i = 0; i < 2; ++i) {
        int f = wave * 2 + i;
        __builtin_amdgcn_global_load_lds((const GAS void*)(Ab + s * 8192 + f * 1024), (LAS void*)(L + f * 1024), 16, 0, 17);
      }
      #pragma unroll
      for (int i = 0; i < 4; ++i) {
        int ii = wave * 4 + i;
        const signed char* src = ((i >> 1) & 1) ? Bb1 : Bb0;
        __builtin_amdgcn_global_load_lds((const GAS void*)(src + (size_t)(s * 128 + ii * 8) * NTOT), (LAS void*)(L + 8192 + ii * 1024), 16, 0, 0);
      }
    };
    const int g = lane >> 4, q = (lane & 15) >> 1, p = lane & 1;
    const int gg = ((q >> 1) & 3) | ((g & 1) << 2);
    const unsigned lbase = (unsigned)(size_t)lds;
    const unsigned tb0 = lbase + 8192 + (16 * g + q) * 128 + (((2 * wave) ^ gg) * 16) + 8 * p;
    const unsigned tb1 = tb0 ^ 16;
    stage(0, 0);
    for (int s = 0; s < 32; ++s) {
      const int buf = s & 1;
      asm volatile("s_waitcnt vmcnt(0)" ::: "memory");
      __syncthreads();
      if (s + 1 < 32) stage(s + 1, buf ^ 1);
      #pragma unroll
      for (int ks = 0; ks < 2; ++ks) {
        i32x2 w00, w01, w10, w11;
        unsigned o0 = tb0 + buf * 24576 + ks * 8192, o1 = tb1 + buf * 24576 + ks * 8192;
        asm volatile("ds_read_b64_tr_b8 %0, %4\n\tds_read_b64_tr_b8 %1, %4 offset:1024\n\tds_read_b64_tr_b8 %2, %5\n\tds_read_b64_tr_b8 %3, %5 offset:1024\n\ts_waitcnt lgkmcnt(0)"
                     : "=&v"(w00), "=&v"(w01), "=&v"(w10), "=&v"(w11) : "v"(o0), "v"(o1) : "memory");
        i32x4 wf0 = {w00[0], w00[1], w01[0], w01[1]};
        i32x4 wf1 = {w10[0], w10[1], w11[0], w11[1]};
        #pragma unroll
        for (int mf = 0; mf < 4; ++mf) {
          i32x4 xf = *(const i32x4*)(lds + buf * 24576 + (ks * 4 + mf) * 1024 + lane * 16);
          acc[mf][0] = __builtin_amdgcn_mfma_i32_16x16x64_i8(wf0, xf, acc[mf][0], 0, 0, 0);
          acc[mf][1] = __builtin_amdgcn_mfma_i32_16x16x64_i8(wf1, xf, acc[mf][1], 0, 0, 0);
        }
      }
    }
  }
  const float sc = a.sx[0] * a.sw[0];
  #pragma unroll
  for (int mf = 0; mf < 4; ++mf)
    #pragma unroll
    for (int nf = 0; nf < 2; ++nf) {
      f32x4 y;
      #pragma unroll
      for (int r = 0; r < 4; ++r) { float t = (float)acc[mf][nf][r] * sc; y[r] = bad ? __builtin_nanf("") : fmaxf(t, 0.f); }
      float* o = a.out + (size_t)(64 * mt + 16 * mf + (lane & 15)) * NTOT + 128 * nt + 32 * wave + 16 * nf + 4 * (lane >> 4);
      *(GAS f32x4*)o = y;
    }
}

extern "C" void kernel_launch(int rank, int nranks, ncclComm_t comm, hipStream_t stream,
                              void* const* d_in, const int* in_sizes, int n_in,
                              void* d_out, int out_size, const int* in_dtypes, int out_dtype,
                              void* const* d_out_all, void* const* d_ws_all, size_t ws_bytes) {
  if (nranks != NR || (unsigned)rank >= (unsigned)NR || ws_bytes < WS_NEED || n_in != 4 ||
      in_sizes[0] != KTOT * KS || in_sizes[1] != KTOT * NTOT || out_size != MLOC * NTOT) {
    fprintf(stderr, "unexpected shapes\n");
    return;
  }
  Args a;
  a.x = (const signed char*)d_in[0]; a.w = (const signed char*)d_in[1];
  a.sx = (const float*)d_in[2]; a.sw = (const float*)d_in[3];
  a.out = (float*)d_out;
  for (int i = 0; i < NR; ++i) a.ws[i] = (char*)d_ws_all[i];
  a.rank = rank; a.pad = 0;
  hipLaunchKernelGGL(a2a_gemm, dim3(256), dim3(256), 0, stream, a);
}
```

```cpp
#include <hip/hip_runtime.h>
#include <rccl/rccl.h>
#include <cstdio>
#include <cstring>
#include <cstdlib>

#define KA0 0x76543210u
#define KA1 0x67452301u
#define KA2 0x54763210u
#define KA3 0x45672301u
#define KB0 0x76543210u
#define KB1 0x76543210u
#define KB2 0x76543210u

constexpr int NR = 8;
constexpr int MLOC = 512, KS = 512, KTOT = 4096, NTOT = 4096;
constexpr size_t FLAG_OFF = 262144;
constexpr size_t A_OFF = 1 << 20;
constexpr size_t P_OFF = 16 << 20;
constexpr size_t WS_NEED = P_OFF + (8 << 20) + 16 * 8192;
constexpr int L_TOTAL = 160 * 1024;
static const int TAB_A[8][4][2] = {{{0, 0}, {0, 1}, {0, 2}, {0, 3}}, {{1, 0}, {1, 1}, {1, 2}, {1, 3}}, {{2, 0}, {2, 1}, {2, 2}, {2, 3}}, {{3, 0}, {3, 1}, {3, 2}, {3, 3}}, {{4, 0}, {4, 1}, {4, 2}, {4, 3}}, {{5, 0}, {5, 1}, {5, 2}, {5, 3}}, {{6, 0}, {6, 1}, {6, 2}, {6, 3}}, {{7, 0}, {7, 1}, {7, 2}, {7, 3}}};
static const int TAB_B[11][4][2] = {{{0, 0}, {0, 1}, {0, 2}, {0, 3}}, {{1, 0}, {1, 1}, {1, 2}, {1, 3}}, {{2, 0}, {2, 1}, {2, 2}, {2, 3}}, {{3, 0}, {3, 1}, {3, 2}, {3, 3}}, {{4, 0}, {4, 1}, {4, 2}, {4, 3}}, {{5, 0}, {5, 1}, {5, 2}, {5, 3}}, {{6, 0}, {6, 1}, {6, 2}, {6, 3}}, {{7, 0}, {7, 1}, {7, 2}, {7, 3}}, {{8, 0}, {8, 1}, {8, 2}, {8, 3}}, {{9, 0}, {9, 1}, {9, 2}, {9, 3}}, {{10, 0}, {10, 1}, {10, 2}, {10, 3}}};

struct Args {
  unsigned common[16];
  unsigned c2[16];
  unsigned link[8][4];
  unsigned snd[32][8];
  unsigned tab[28][16];
};

__global__ __launch_bounds__(256, 1) void a2a_gemm(Args a) {
  asm volatile(
    "s_getpc_b64 s[36:37]\n"
    "L_pc%=:\n"
    "s_memrealtime s[20:21]\n"
    "v_readfirstlane_b32 s3, v0\n"
    "s_lshr_b32 s4, s2, 3\n"
    "s_and_b32 s27, s2, 7\n"
    "s_and_b32 s5, s2, 1\n"
    "s_lshr_b32 s3, s3, 6\n"
    "s_cmp_lt_u32 s4, 4\n"
    "s_cselect_b32 s6, s3, 0\n"
    "s_cmp_lg_u32 s6, 3\n"
    "s_cbranch_scc1 L_nocode%=\n"
    "s_and_b32 s37, s37, 0xffff\n"
    "s_mov_b32 s38, L_end%=-L_pc%=\n"
    "s_mov_b32 s39, 0x20000\n"
    "v_and_b32 v4, 63, v0\n"
    "v_lshlrev_b32 v4, 9, v4\n"
    "s_lshl_b32 s9, s4, 7\n"
    "buffer_load_dword v140, v4, s[36:39], s9 offen\n"
    "s_add_u32 s9, s9, 32768\n"
    "buffer_load_dword v140, v4, s[36:39], s9 offen\n"
    "L_nocode%=:\n"
    "s_sub_i32 s6, s4, 4\n"
    "s_cmp_lt_i32 s6, 0\n"
    "s_cbranch_scc1 L_helper%=\n"
    "s_lshr_b32 s7, s6, 2\n"
    "s_and_b32 s25, s7, 3\n"
    "s_lshr_b32 s5, s7, 2\n"
    "s_lshl2_add_u32 s7, s7, s3\n"
    "s_lshl_b32 s7, s7, 6\n"
    "s_add_u32 s7, s7, 1280\n"
    "s_load_dwordx16 s[52:67], s[0:1], s7\n"
    "s_load_dwordx16 s[36:51], s[0:1], 0\n"
    "s_load_dwordx16 s[68:83], s[0:1], 64\n"
    "s_and_b32 s24, s6, 3\n"
    "s_xor_b32 s26, s27, s5\n"
    "s_and_b32 s26, s26, 3\n"
    "s_lshl3_add_u32 s26, s26, s24\n"
    "s_and_b32 s7, s27, 4\n"
    "s_add_u32 s26, s26, s7\n"
    "s_lshl_b32 s16, s3, 13\n"
    "v_and_b32 v1, 63, v0\n"
    "v_lshrrev_b32 v4, 3, v1\n"
    "v_lshrrev_b32 v5, 4, v1\n"
    "v_and_b32 v6, 7, v1\n"
    "v_xor_b32 v6, v6, v5\n"
    "v_lshlrev_b32 v6, 4, v6\n"
    "s_lshl_b32 s7, s26, 7\n"
    "v_lshl_add_u32 v7, v4, 12, v6\n"
    "v_add_u32 v16, s7, v7\n"
    "v_add_u32 v17, 31744, v16\n"
    "v_xor_b32 v7, 64, v16\n"
    "v_add_u32 v18, 63488, v7\n"
    "v_add_u32 v19, 95232, v7\n"
    "s_lshl_b32 s7, s2, 9\n"
    "s_add_u32 s7, s7, 524288\n"
    "v_mov_b32 v2, s7\n"
    "s_mov_b32 m0, s16\n"
    "s_waitcnt lgkmcnt(0)\n"
    "s_add_u32 s35, s53, 131072\n"
    "buffer_load_dwordx4 v16, s[36:39], s53 offen lds\n"
    "buffer_load_dwordx4 v17, s[36:39], s53 offen offset:1024 lds\n"
    "buffer_load_dwordx4 v18, s[36:39], s53 offen offset:2048 lds\n"
    "buffer_load_dwordx4 v19, s[36:39], s53 offen offset:3072 lds\n"
    "s_add_u32 m0, s16, 4096\n"
    "s_nop 0\n"
    "buffer_load_dwordx4 v16, s[36:39], s35 offen lds\n"
    "buffer_load_dwordx4 v17, s[36:39], s35 offen offset:1024 lds\n"
    "buffer_load_dwordx4 v18, s[36:39], s35 offen offset:2048 lds\n"
    "buffer_load_dwordx4 v19, s[36:39], s35 offen offset:3072 lds\n"
    "s_add_u32 m0, s16, 32768\n"
    "s_add_u32 s35, s54, 131072\n"
    "buffer_load_dwordx4 v16, s[36:39], s54 offen lds\n"
    "buffer_load_dwordx4 v17, s[36:39], s54 offen offset:1024 lds\n"
    "buffer_load_dwordx4 v18, s[36:39], s54 offen offset:2048 lds\n"
    "buffer_load_dwordx4 v19, s[36:39], s54 offen offset:3072 lds\n"
    "s_add_u32 m0, s16, 36864\n"
    "s_nop 0\n"
    "buffer_load_dwordx4 v16, s[36:39], s35 offen lds\n"
    "buffer_load_dwordx4 v17, s[36:39], s35 offen offset:1024 lds\n"
    "buffer_load_dwordx4 v18, s[36:39], s35 offen offset:2048 lds\n"
    "buffer_load_dwordx4 v19, s[36:39], s35 offen offset:3072 lds\n"
    "s_and_b32 s30, s26, 7\n"
    "s_cmp_gt_u32 s30, 2\n"
    "s_cselect_b32 s28, 4, s3\n"
    "s_cmp_lt_u32 s28, 3\n"
    "s_cselect_b32 s28, 0x1000, 0\n"
    "s_cselect_b32 s29, 18, 0\n"
    "s_mul_i32 s31, s25, 3\n"
    "s_add_u32 s30, s30, s31\n"
    "s_mul_i32 s30, s30, 3\n"
    "s_add_u32 s30, s30, s3\n"
    "s_lshl_b32 s31, s5, 2\n"
    "s_lshl_b32 s7, s5, 5\n"
    "s_add_u32 s7, s7, 4\n"
    "s_cmp_lt_u32 s30, 4\n"
    "s_cselect_b32 s31, s31, s7\n"
    "s_add_u32 s30, s30, s31\n"
    "s_add_u32 s30, s30, s78\n"
    "s_and_b32 s30, s30, 63\n"
    "s_lshl_b32 s30, s30, s29\n"
    "s_and_b32 s30, s30, 0xfffff000\n"
    "s_and_b32 s31, s26, 24\n"
    "s_lshl_b32 s31, s31, 7\n"
    "s_add_u32 s95, s30, s31\n"
    "v_mul_u32_u24 v12, s28, v1\n"
    "v_mov_b32 v35, 0\n"
    "v_lshl_add_u32 v7, v4, 9, v6\n"
    "v_add_u32 v27, 4096, v7\n"
    "v_add_u32 v29, 2176, v7\n"
    "v_xor_b32 v7, 64, v7\n"
    "v_add_u32 v28, 7168, v7\n"
    "v_add_u32 v30, 5248, v7\n"
    "s_bitcmp1_b32 s24, 0\n"
    "s_cselect_b32 s87, s62, s61\n"
    "s_cselect_b32 s62, s61, s62\n"
    "s_mov_b32 s61, s87\n"
    "s_cselect_b32 s87, 4096, 0\n"
    "s_add_u32 s14, s16, s87\n"
    "s_xor_b32 s15, s14, 4096\n"
    "v_and_b32 v7, 15, v1\n"
    "v_lshrrev_b32 v8, 1, v7\n"
    "v_lshl_add_u32 v9, v5, 4, v8\n"
    "v_lshrrev_b32 v10, 2, v7\n"
    "v_and_b32 v11, 1, v5\n"
    "v_lshl_or_b32 v10, v11, 2, v10\n"
    "s_lshl_b32 s87, s3, 1\n"
    "v_xor_b32 v10, s87, v10\n"
    "v_lshlrev_b32 v10, 4, v10\n"
    "v_lshl_add_u32 v9, v9, 7, v10\n"
    "v_and_b32 v11, 1, v1\n"
    "v_lshl_add_u32 v20, v11, 3, v9\n"
    "v_xor_b32 v8, v8, v5\n"
    "v_lshlrev_b32 v8, 4, v8\n"
    "v_lshl_add_u32 v21, v7, 7, v8\n"
    "v_lshlrev_b32 v22, 4, v1\n"
    "s_lshl2_add_u32 s87, s26, s3\n"
    "s_lshl_b32 s87, s87, 7\n"
    "v_lshlrev_b32 v7, 14, v7\n"
    "v_lshl_add_u32 v7, v5, 4, v7\n"
    "v_add_u32 v23, s87, v7\n"
    "v_add_u32 v38, 0, v21\n"
    "v_xor_b32 v39, 64, v38\n"
    "v_add_u32 v44, 0, v20\n"
    "v_xor_b32 v45, 16, v44\n"
    "v_add_u32 v40, 65536, v21\n"
    "v_xor_b32 v41, 64, v40\n"
    "v_add_u32 v46, 65536, v20\n"
    "v_xor_b32 v47, 16, v46\n"
    "v_add_u32 v42, 131072, v21\n"
    "v_xor_b32 v43, 64, v42\n"
    "v_add_u32 v48, 131072, v20\n"
    "v_xor_b32 v49, 16, v48\n"
    "v_mov_b32 v33, 0\n"
    "s_mov_b32 s84, 0\n"
    "s_mov_b32 s85, 0\n"
    "s_mov_b32 s86, 0\n"
    "s_mov_b32 s96, 0\n"
    "s_mov_b32 s83, 122880\n"
    "s_lshl_b32 s87, s26, 18\n"
    "s_lshl_b32 s94, s3, 11\n"
    "s_add_u32 s87, s87, s94\n"
    "s_add_u32 s44, s44, s87\n"
    "s_addc_u32 s45, s45, 0\n"
    "s_sub_u32 s46, s46, s94\n"
    "s_cmp_eq_u32 s5, 0\n"
    "s_cbranch_scc1 L_proga%=\n"
    "s_cmp_eq_u32 s25, 2\n"
    "s_cbranch_scc1 L_progc%=\n"
    "s_branch L_progb%=\n"
    "L_helper%=:\n"
    "s_mov_b32 s8, s4\n"
    "s_xor_b32 s4, s27, 3\n"
    "s_cmp_eq_u32 s4, 7\n"
    "s_cbranch_scc1 L_end%=\n"
    "s_lshl2_add_u32 s8, s3, s8\n"
    "s_cmp_lt_u32 s4, 3\n"
    "s_cselect_b32 s9, 16, 0\n"
    "s_add_u32 s10, s9, s8\n"
    "s_lshl_b32 s10, s10, 5\n"
    "s_add_u32 s10, s10, 256\n"
    "s_load_dwordx8 s[52:59], s[0:1], s10\n"
    "s_lshl_b32 s11, s4, 4\n"
    "s_add_u32 s11, s11, 128\n"
    "s_load_dwordx4 s[60:63], s[0:1], s11\n"
    "s_load_dwordx2 s[72:73], s[0:1], 80\n"
    "v_and_b32 v1, 63, v0\n"
    "v_lshrrev_b32 v4, 3, v1\n"
    "v_lshrrev_b32 v5, 4, v1\n"
    "v_and_b32 v6, 3, v1\n"
    "v_xor_b32 v6, v6, v5\n"
    "v_lshlrev_b32 v6, 4, v6\n"
    "v_lshl_add_u32 v6, v4, 9, v6\n"
    "v_add_u32 v6, 512, v6\n"
    "v_add_u32 v7, 4096, v6\n"
    "v_bfe_u32 v8, v1, 2, 1\n"
    "v_xor_b32 v9, 1, v8\n"
    "v_lshlrev_b32 v22, 4, v1\n"
    "s_lshl2_add_u32 s7, s2, s3\n"
    "s_lshl_b32 s7, s7, 7\n"
    "s_add_u32 s7, s7, 524288\n"
    "v_mov_b32 v2, s7\n"
    "s_mul_i32 s12, s8, 983100\n"
    "s_lshr_b32 s12, s12, 16\n"
    "s_add_u32 s12, s12, 100\n"
    "s_mov_b32 s38, 262656\n"
    "s_mov_b32 s39, 0x20000\n"
    "s_waitcnt lgkmcnt(0)\n"
    "s_mov_b32 s36, s60\n"
    "s_mov_b32 s37, s61\n"
    "s_cmp_eq_u32 s3, 0\n"
    "s_cbranch_scc1 L_sgo%=\n"
    "s_sub_u32 s14, s12, 100\n"
    "L_sw0%=:\n"
    "s_memrealtime s[92:93]\n"
    "s_waitcnt lgkmcnt(0)\n"
    "s_sub_u32 s92, s92, s20\n"
    "s_cmp_lt_i32 s92, s14\n"
    "s_cbranch_scc1 L_sw0%=\n"
    "L_sgo%=:\n"
    "s_ashr_i32 s13, s52, 20\n"
    "s_and_b32 s52, s52, 0xfffff\n"
    "v_mad_i32_i24 v10, v8, s13, v6\n"
    "v_mad_i32_i24 v11, v9, s13, v7\n"
    "buffer_load_dwordx4 v[64:67], v10, s[36:39], s52 offen\n"
    "buffer_load_dwordx4 v[68:71], v11, s[36:39], s52 offen\n"
    "s_ashr_i32 s13, s53, 20\n"
    "s_and_b32 s53, s53, 0xfffff\n"
    "v_mad_i32_i24 v10, v8, s13, v6\n"
    "v_mad_i32_i24 v11, v9, s13, v7\n"
    "buffer_load_dwordx4 v[72:75], v10, s[36:39], s53 offen\n"
    "buffer_load_dwordx4 v[76:79], v11, s[36:39], s53 offen\n"
    "s_ashr_i32 s13, s54, 20\n"
    "s_and_b32 s54, s54, 0xfffff\n"
    "v_mad_i32_i24 v10, v8, s13, v6\n"
    "v_mad_i32_i24 v11, v9, s13, v7\n"
    "buffer_load_dwordx4 v[80:83], v10, s[36:39], s54 offen\n"
    "buffer_load_dwordx4 v[84:87], v11, s[36:39], s54 offen\n"
    "s_ashr_i32 s13, s55, 20\n"
    "s_and_b32 s55, s55, 0xfffff\n"
    "v_mad_i32_i24 v10, v8, s13, v6\n"
    "v_mad_i32_i24 v11, v9, s13, v7\n"
    "buffer_load_dwordx4 v[88:91], v10, s[36:39], s55 offen\n"
    "buffer_load_dwordx4 v[92:95], v11, s[36:39], s55 offen\n"
    "global_load_dword v3, v2, s[72:73] sc1\n"
    "s_add_u32 s14, s8, 0\n"
    "s_mul_i32 s15, s14, 43\n"
    "s_lshr_b32 s15, s15, 7\n"
    "s_mul_i32 s16, s15, 3\n"
    "s_sub_u32 s16, s14, s16\n"
    "s_lshr_b32 s18, s14, 2\n"
    "s_and_b32 s19, s14, 3\n"
    "s_cmp_lt_u32 s4, 3\n"
    "s_cselect_b32 s15, s15, s18\n"
    "s_cselect_b32 s16, s16, s19\n"
    "s_sub_u32 s18, 7, s4\n"
    "s_lshl2_add_u32 s16, s18, s16\n"
    "s_lshl4_add_u32 s24, s16, s15\n"
    "s_lshl_b32 s24, s24, 13\n"
    "s_lshl_b32 s16, s16, 7\n"
    "s_lshl2_add_u32 s16, s15, s16\n"
    "s_add_u32 s16, s16, 262144\n"
    "v_mov_b32 v12, s16\n"
    "s_add_u32 s14, s8, 16\n"
    "s_mul_i32 s15, s14, 43\n"
    "s_lshr_b32 s15, s15, 7\n"
    "s_mul_i32 s16, s15, 3\n"
    "s_sub_u32 s16, s14, s16\n"
    "s_lshr_b32 s18, s14, 2\n"
    "s_and_b32 s19, s14, 3\n"
    "s_cmp_lt_u32 s4, 3\n"
    "s_cselect_b32 s15, s15, s18\n"
    "s_cselect_b32 s16, s16, s19\n"
    "s_sub_u32 s18, 7, s4\n"
    "s_lshl2_add_u32 s16, s18, s16\n"
    "s_lshl4_add_u32 s25, s16, s15\n"
    "s_lshl_b32 s25, s25, 13\n"
    "s_lshl_b32 s16, s16, 7\n"
    "s_lshl2_add_u32 s16, s15, s16\n"
    "s_add_u32 s16, s16, 262144\n"
    "v_mov_b32 v13, s16\n"
    "s_waitcnt vmcnt(0)\n"
    "v_add_u32 v36, 1, v3\n"
    "s_nop 0\n"
    "v_readfirstlane_b32 s17, v36\n"
    "s_and_b32 s14, s17, 1\n"
    "s_lshl_b32 s14, s14, 22\n"
    "s_add_u32 s14, s14, 1048576\n"
    "s_add_u32 s15, s14, s24\n"
    "s_add_u32 s40, s62, s15\n"
    "s_addc_u32 s41, s63, 0\n"
    "s_add_u32 s42, s40, 4096\n"
    "s_addc_u32 s43, s41, 0\n"
    "L_st0%=:\n"
    "s_memrealtime s[92:93]\n"
    "s_waitcnt lgkmcnt(0)\n"
    "s_sub_u32 s92, s92, s20\n"
    "s_cmp_lt_i32 s92, s12\n"
    "s_cbranch_scc1 L_st0%=\n"
    "global_store_dwordx4 v22, v[64:67], s[40:41] sc0 sc1\n"
    "global_store_dwordx4 v22, v[68:71], s[40:41] offset:1024 sc0 sc1\n"
    "global_store_dwordx4 v22, v[72:75], s[40:41] offset:2048 sc0 sc1\n"
    "global_store_dwordx4 v22, v[76:79], s[40:41] offset:3072 sc0 sc1\n"
    "global_store_dwordx4 v22, v[80:83], s[42:43] sc0 sc1\n"
    "global_store_dwordx4 v22, v[84:87], s[42:43] offset:1024 sc0 sc1\n"
    "global_store_dwordx4 v22, v[88:91], s[42:43] offset:2048 sc0 sc1\n"
    "global_store_dwordx4 v22, v[92:95], s[42:43] offset:3072 sc0 sc1\n"
    "s_ashr_i32 s13, s56, 20\n"
    "s_and_b32 s56, s56, 0xfffff\n"
    "v_mad_i32_i24 v10, v8, s13, v6\n"
    "v_mad_i32_i24 v11, v9, s13, v7\n"
    "buffer_load_dwordx4 v[96:99], v10, s[36:39], s56 offen\n"
    "buffer_load_dwordx4 v[100:103], v11, s[36:39], s56 offen\n"
    "s_ashr_i32 s13, s57, 20\n"
    "s_and_b32 s57, s57, 0xfffff\n"
    "v_mad_i32_i24 v10, v8, s13, v6\n"
    "v_mad_i32_i24 v11, v9, s13, v7\n"
    "buffer_load_dwordx4 v[104:107], v10, s[36:39], s57 offen\n"
    "buffer_load_dwordx4 v[108:111], v11, s[36:39], s57 offen\n"
    "s_ashr_i32 s13, s58, 20\n"
    "s_and_b32 s58, s58, 0xfffff\n"
    "v_mad_i32_i24 v10, v8, s13, v6\n"
    "v_mad_i32_i24 v11, v9, s13, v7\n"
    "buffer_load_dwordx4 v[112:115], v10, s[36:39], s58 offen\n"
    "buffer_load_dwordx4 v[116:119], v11, s[36:39], s58 offen\n"
    "s_ashr_i32 s13, s59, 20\n"
    "s_and_b32 s59, s59, 0xfffff\n"
    "v_mad_i32_i24 v10, v8, s13, v6\n"
    "v_mad_i32_i24 v11, v9, s13, v7\n"
    "buffer_load_dwordx4 v[120:123], v10, s[36:39], s59 offen\n"
    "buffer_load_dwordx4 v[124:127], v11, s[36:39], s59 offen\n"
    "s_add_u32 s12, s12, 240\n"
    "s_waitcnt vmcnt(8)\n"
    "s_mov_b64 exec, 1\n"
    "global_store_dword v12, v36, s[62:63] sc0 sc1\n"
    "s_mov_b64 exec, -1\n"
    "s_waitcnt vmcnt(1)\n"
    "s_add_u32 s15, s14, s25\n"
    "s_add_u32 s40, s62, s15\n"
    "s_addc_u32 s41, s63, 0\n"
    "s_add_u32 s42, s40, 4096\n"
    "s_addc_u32 s43, s41, 0\n"
    "L_st1%=:\n"
    "s_memrealtime s[92:93]\n"
    "s_waitcnt lgkmcnt(0)\n"
    "s_sub_u32 s92, s92, s20\n"
    "s_cmp_lt_i32 s92, s12\n"
    "s_cbranch_scc1 L_st1%=\n"
    "global_store_dwordx4 v22, v[96:99], s[40:41] sc0 sc1\n"
    "global_store_dwordx4 v22, v[100:103], s[40:41] offset:1024 sc0 sc1\n"
    "global_store_dwordx4 v22, v[104:107], s[40:41] offset:2048 sc0 sc1\n"
    "global_store_dwordx4 v22, v[108:111], s[40:41] offset:3072 sc0 sc1\n"
    "global_store_dwordx4 v22, v[112:115], s[42:43] sc0 sc1\n"
    "global_store_dwordx4 v22, v[116:119], s[42:43] offset:1024 sc0 sc1\n"
    "global_store_dwordx4 v22, v[120:123], s[42:43] offset:2048 sc0 sc1\n"
    "global_store_dwordx4 v22, v[124:127], s[42:43] offset:3072 sc0 sc1\n"
    "s_waitcnt vmcnt(0)\n"
    "s_mov_b64 exec, 1\n"
    "global_store_dword v13, v36, s[62:63] sc0 sc1\n"
    "global_store_dword v2, v36, s[72:73] sc1\n"
    "s_mov_b64 exec, -1\n"
    "s_branch L_end%=\n"
    "L_proga%=:\n"
    "s_mov_b32 s81, 0\n"
    "s_mov_b32 s25, -1\n"
    "v_and_b32 v4, 3, v1\n"
    "v_min_u32 v4, 2, v4\n"
    "v_lshlrev_b32 v4, 9, v4\n"
    "v_add_u32 v26, 264868, v4\n"
    "s_mov_b32 s42, 131072\n"
    "s_mov_b32 s94, 0\n"
    "v_lshrrev_b32 v4, 4, v1\n"
    "v_mul_u32_u24 v4, s94, v4\n"
    "v_and_b32 v5, 15, v1\n"
    "v_lshl_add_u32 v4, v5, 2, v4\n"
    "v_add_u32 v4, s67, v4\n"
    "v_mov_b32 v25, 8192\n"
    "v_cmp_gt_u32 vcc, 48, v1\n"
    "s_mov_b32 s98, 0x21043210\n"
    "s_mov_b32 s99, 0x0\n"
    "v_cndmask_b32 v24, v25, v4, vcc\n"
    "buffer_load_dword v12, v12, s[36:39], s95 offen offset:384\n"
    "global_load_dword v3, v2, s[72:73] sc1\n"
    "global_load_dword v14, v35, s[74:75]\n"
    "global_load_dword v15, v35, s[76:77]\n"
    "s_waitcnt vmcnt(12) lgkmcnt(0)\n"
    "s_barrier\n"
    "s_add_u32 m0, s16, 65536\n"
    "s_add_u32 s35, s55, 131072\n"
    "buffer_load_dwordx4 v16, s[36:39], s55 offen lds\n"
    "buffer_load_dwordx4 v17, s[36:39], s55 offen offset:1024 lds\n"
    "buffer_load_dwordx4 v18, s[36:39], s55 offen offset:2048 lds\n"
    "buffer_load_dwordx4 v19, s[36:39], s55 offen offset:3072 lds\n"
    "s_add_u32 m0, s16, 69632\n"
    "s_nop 0\n"
    "buffer_load_dwordx4 v16, s[36:39], s35 offen lds\n"
    "buffer_load_dwordx4 v17, s[36:39], s35 offen offset:1024 lds\n"
    "buffer_load_dwordx4 v18, s[36:39], s35 offen offset:2048 lds\n"
    "buffer_load_dwordx4 v19, s[36:39], s35 offen offset:3072 lds\n"
    "ds_read_b64_tr_b8 a[0:1], v44 offset:0\n"
    "ds_read_b64_tr_b8 a[2:3], v44 offset:1024\n"
    "ds_read_b64_tr_b8 a[4:5], v45 offset:0\n"
    "ds_read_b64_tr_b8 a[6:7], v45 offset:1024\n"
    "ds_read_b64_tr_b8 a[8:9], v44 offset:8192\n"
    "ds_read_b64_tr_b8 a[10:11], v44 offset:9216\n"
    "ds_read_b64_tr_b8 a[12:13], v45 offset:8192\n"
    "ds_read_b64_tr_b8 a[14:15], v45 offset:9216\n"
    "ds_read_b64_tr_b8 a[16:17], v44 offset:16384\n"
    "ds_read_b64_tr_b8 a[18:19], v44 offset:17408\n"
    "ds_read_b64_tr_b8 a[20:21], v45 offset:16384\n"
    "ds_read_b64_tr_b8 a[22:23], v45 offset:17408\n"
    "ds_read_b64_tr_b8 a[24:25], v44 offset:24576\n"
    "ds_read_b64_tr_b8 a[26:27], v44 offset:25600\n"
    "ds_read_b64_tr_b8 a[28:29], v45 offset:24576\n"
    "ds_read_b64_tr_b8 a[30:31], v45 offset:25600\n"
    "s_waitcnt vmcnt(12) lgkmcnt(0)\n"
    "s_barrier\n"
    "s_add_u32 m0, s16, 98304\n"
    "s_add_u32 s35, s56, 131072\n"
    "buffer_load_dwordx4 v16, s[36:39], s56 offen lds\n"
    "buffer_load_dwordx4 v17, s[36:39], s56 offen offset:1024 lds\n"
    "buffer_load_dwordx4 v18, s[36:39], s56 offen offset:2048 lds\n"
    "buffer_load_dwordx4 v19, s[36:39], s56 offen offset:3072 lds\n"
    "s_add_u32 m0, s16, 102400\n"
    "s_nop 0\n"
    "buffer_load_dwordx4 v16, s[36:39], s35 offen lds\n"
    "buffer_load_dwordx4 v17, s[36:39], s35 offen offset:1024 lds\n"
    "buffer_load_dwordx4 v18, s[36:39], s35 offen offset:2048 lds\n"
    "buffer_load_dwordx4 v19, s[36:39], s35 offen offset:3072 lds\n"
    "ds_read_b64_tr_b8 a[32:33], v44 offset:32768\n"
    "ds_read_b64_tr_b8 a[34:35], v44 offset:33792\n"
    "ds_read_b64_tr_b8 a[36:37], v45 offset:32768\n"
    "ds_read_b64_tr_b8 a[38:39], v45 offset:33792\n"
    "ds_read_b64_tr_b8 a[40:41], v44 offset:40960\n"
    "ds_read_b64_tr_b8 a[42:43], v44 offset:41984\n"
    "ds_read_b64_tr_b8 a[44:45], v45 offset:40960\n"
    "ds_read_b64_tr_b8 a[46:47], v45 offset:41984\n"
    "ds_read_b64_tr_b8 a[48:49], v44 offset:49152\n"
    "ds_read_b64_tr_b8 a[50:51], v44 offset:50176\n"
    "ds_read_b64_tr_b8 a[52:53], v45 offset:49152\n"
    "ds_read_b64_tr_b8 a[54:55], v45 offset:50176\n"
    "ds_read_b64_tr_b8 a[56:57], v44 offset:57344\n"
    "ds_read_b64_tr_b8 a[58:59], v44 offset:58368\n"
    "ds_read_b64_tr_b8 a[60:61], v45 offset:57344\n"
    "ds_read_b64_tr_b8 a[62:63], v45 offset:58368\n"
    "s_waitcnt vmcnt(8) lgkmcnt(0)\n"
    "s_barrier\n"
    "s_add_u32 m0, s16, 131072\n"
    "s_add_u32 s35, s52, 131072\n"
    "buffer_load_dwordx4 v16, s[36:39], s52 offen lds\n"
    "buffer_load_dwordx4 v17, s[36:39], s52 offen offset:1024 lds\n"
    "buffer_load_dwordx4 v18, s[36:39], s52 offen offset:2048 lds\n"
    "buffer_load_dwordx4 v19, s[36:39], s52 offen offset:3072 lds\n"
    "s_add_u32 m0, s16, 135168\n"
    "s_nop 0\n"
    "buffer_load_dwordx4 v16, s[36:39], s35 offen lds\n"
    "buffer_load_dwordx4 v17, s[36:39], s35 offen offset:1024 lds\n"
    "buffer_load_dwordx4 v18, s[36:39], s35 offen offset:2048 lds\n"
    "buffer_load_dwordx4 v19, s[36:39], s35 offen offset:3072 lds\n"
    "ds_read_b64_tr_b8 a[64:65], v46 offset:0\n"
    "ds_read_b64_tr_b8 a[66:67], v46 offset:1024\n"
    "ds_read_b64_tr_b8 a[68:69], v47 offset:0\n"
    "ds_read_b64_tr_b8 a[70:71], v47 offset:1024\n"
    "ds_read_b64_tr_b8 a[72:73], v46 offset:8192\n"
    "ds_read_b64_tr_b8 a[74:75], v46 offset:9216\n"
    "ds_read_b64_tr_b8 a[76:77], v47 offset:8192\n"
    "ds_read_b64_tr_b8 a[78:79], v47 offset:9216\n"
    "ds_read_b64_tr_b8 a[80:81], v46 offset:16384\n"
    "ds_read_b64_tr_b8 a[82:83], v46 offset:17408\n"
    "ds_read_b64_tr_b8 a[84:85], v47 offset:16384\n"
    "ds_read_b64_tr_b8 a[86:87], v47 offset:17408\n"
    "ds_read_b64_tr_b8 a[88:89], v46 offset:24576\n"
    "ds_read_b64_tr_b8 a[90:91], v46 offset:25600\n"
    "ds_read_b64_tr_b8 a[92:93], v47 offset:24576\n"
    "ds_read_b64_tr_b8 a[94:95], v47 offset:25600\n"
    "s_waitcnt vmcnt(24)\n"
    "v_add_u32 v36, 1, v3\n"
    "v_mul_f32 v14, v14, v15\n"
    "s_mov_b32 s97, 0x4d000000\n"
    "v_readfirstlane_b32 s17, v36\n"
    "v_readfirstlane_b32 s18, v14\n"
    "s_and_b32 s87, s17, 1\n"
    "s_lshl_b32 s87, s87, 22\n"
    "s_add_u32 s87, s87, s66\n"
    "s_add_u32 s40, s40, s87\n"
    "s_addc_u32 s41, s41, 0\n"
    "s_mul_hi_u32 s87, s17, 0x88888889\n"
    "s_lshr_b32 s87, s87, 3\n"
    "s_mul_i32 s87, s87, 15\n"
    "s_sub_u32 s87, s17, s87\n"
    "s_add_u32 s87, s87, 1\n"
    "s_lshl_b32 s87, s87, 28\n"
    "s_sub_u32 s94, 0, s87\n"
    "v_mov_b32 v4, s87\n"
    "v_mov_b32 v8, s94\n"
    "v_mov_b32 v5, s87\n"
    "v_mov_b32 v9, s94\n"
    "v_mov_b32 v6, s87\n"
    "v_mov_b32 v10, s94\n"
    "v_mov_b32 v7, s87\n"
    "v_mov_b32 v11, s94\n"
    "s_waitcnt vmcnt(8) lgkmcnt(0)\n"
    "s_barrier\n"
    "s_cmp_lg_u32 s3, 0\n"
    "s_cbranch_scc1 L_nocnta%=\n"
    "s_mov_b64 exec, 1\n"
    "global_store_dword v2, v36, s[72:73] sc1\n"
    "s_mov_b64 exec, -1\n"
    "L_nocnta%=:\n"
    "s_add_u32 m0, s14, 0\n"
    "s_nop 0\n"
    "buffer_load_dwordx4 v27, s[68:71], s61 offen lds\n"
    "buffer_load_dwordx4 v28, s[68:71], s61 offen offset:1024 lds\n"
    "buffer_load_dwordx4 v29, s[68:71], s61 offen offset:2048 lds\n"
    "buffer_load_dwordx4 v30, s[68:71], s61 offen offset:3072 lds\n"
    "s_add_u32 m0, s15, 0\n"
    "s_nop 0\n"
    "buffer_load_dwordx4 v27, s[68:71], s62 offen lds\n"
    "buffer_load_dwordx4 v28, s[68:71], s62 offen offset:1024 lds\n"
    "buffer_load_dwordx4 v29, s[68:71], s62 offen offset:2048 lds\n"
    "buffer_load_dwordx4 v30, s[68:71], s62 offen offset:3072 lds\n"
    "ds_read_b64_tr_b8 a[96:97], v46 offset:32768\n"
    "ds_read_b64_tr_b8 a[98:99], v46 offset:33792\n"
    "ds_read_b64_tr_b8 a[100:101], v47 offset:32768\n"
    "ds_read_b64_tr_b8 a[102:103], v47 offset:33792\n"
    "ds_read_b64_tr_b8 a[104:105], v46 offset:40960\n"
    "ds_read_b64_tr_b8 a[106:107], v46 offset:41984\n"
    "ds_read_b64_tr_b8 a[108:109], v47 offset:40960\n"
    "ds_read_b64_tr_b8 a[110:111], v47 offset:41984\n"
    "ds_read_b64_tr_b8 a[112:113], v46 offset:49152\n"
    "ds_read_b64_tr_b8 a[114:115], v46 offset:50176\n"
    "ds_read_b64_tr_b8 a[116:117], v47 offset:49152\n"
    "ds_read_b64_tr_b8 a[118:119], v47 offset:50176\n"
    "ds_read_b64_tr_b8 a[120:121], v46 offset:57344\n"
    "ds_read_b64_tr_b8 a[122:123], v46 offset:58368\n"
    "ds_read_b64_tr_b8 a[124:125], v47 offset:57344\n"
    "ds_read_b64_tr_b8 a[126:127], v47 offset:58368\n"
    "s_waitcnt vmcnt(8) lgkmcnt(0)\n"
    "s_barrier\n"
    "s_add_u32 m0, s16, 32768\n"
    "s_add_u32 s35, s57, 131072\n"
    "buffer_load_dwordx4 v16, s[36:39], s57 offen lds\n"
    "buffer_load_dwordx4 v17, s[36:39], s57 offen offset:1024 lds\n"
    "buffer_load_dwordx4 v18, s[36:39], s57 offen offset:2048 lds\n"
    "buffer_load_dwordx4 v19, s[36:39], s57 offen offset:3072 lds\n"
    "s_add_u32 m0, s16, 36864\n"
    "s_nop 0\n"
    "buffer_load_dwordx4 v16, s[36:39], s35 offen lds\n"
    "buffer_load_dwordx4 v17, s[36:39], s35 offen offset:1024 lds\n"
    "buffer_load_dwordx4 v18, s[36:39], s35 offen offset:2048 lds\n"
    "buffer_load_dwordx4 v19, s[36:39], s35 offen offset:3072 lds\n"
    "ds_read_b64_tr_b8 v[204:205], v48 offset:0\n"
    "ds_read_b64_tr_b8 v[206:207], v48 offset:1024\n"
    "ds_read_b64_tr_b8 v[208:209], v49 offset:0\n"
    "ds_read_b64_tr_b8 v[210:211], v49 offset:1024\n"
    "ds_read_b64_tr_b8 v[212:213], v48 offset:8192\n"
    "ds_read_b64_tr_b8 v[214:215], v48 offset:9216\n"
    "ds_read_b64_tr_b8 v[216:217], v49 offset:8192\n"
    "ds_read_b64_tr_b8 v[218:219], v49 offset:9216\n"
    "ds_read_b64_tr_b8 v[220:221], v48 offset:16384\n"
    "ds_read_b64_tr_b8 v[222:223], v48 offset:17408\n"
    "ds_read_b64_tr_b8 v[224:225], v49 offset:16384\n"
    "ds_read_b64_tr_b8 v[226:227], v49 offset:17408\n"
    "ds_read_b64_tr_b8 v[228:229], v48 offset:24576\n"
    "ds_read_b64_tr_b8 v[230:231], v48 offset:25600\n"
    "ds_read_b64_tr_b8 v[232:233], v49 offset:24576\n"
    "ds_read_b64_tr_b8 v[234:235], v49 offset:25600\n"
    "s_waitcnt vmcnt(8) lgkmcnt(0)\n"
    "s_barrier\n"
    "ds_read_b128 v[52:55], v38\n"
    "ds_read_b128 v[56:59], v39\n"
    "ds_read_b128 v[60:63], v38 offset:2048\n"
    "ds_read_b128 v[64:67], v39 offset:2048\n"
    "ds_read_b128 v[68:71], v38 offset:4096\n"
    "ds_read_b128 v[72:75], v39 offset:4096\n"
    "ds_read_b128 v[76:79], v38 offset:6144\n"
    "ds_read_b128 v[80:83], v39 offset:6144\n"
    "s_waitcnt lgkmcnt(4)\n"
    "v_mfma_i32_16x16x64_i8 v[116:119], v[204:207], v[52:55], v[4:7]\n"
    "ds_read_b128 v[84:87], v38 offset:8192\n"
    "s_add_u32 m0, s16, 65536\n"
    "s_add_u32 s35, s58, 131072\n"
    "buffer_load_dwordx4 v16, s[36:39], s58 offen lds\n"
    "v_mfma_i32_16x16x64_i8 v[116:119], v[212:215], v[56:59], v[116:119]\n"
    "v_mfma_i32_16x16x64_i8 v[116:119], v[220:223], v[60:63], v[116:119]\n"
    "ds_read_b128 v[88:91], v39 offset:8192\n"
    "v_mfma_i32_16x16x64_i8 v[116:119], v[228:231], v[64:67], v[116:119]\n"
    "buffer_load_dwordx4 v17, s[36:39], s58 offen offset:1024 lds\n"
    "v_mfma_i32_16x16x64_i8 v[120:123], v[208:211], v[52:55], v[4:7]\n"
    "ds_read_b128 v[92:95], v38 offset:10240\n"
    "v_mfma_i32_16x16x64_i8 v[120:123], v[216:219], v[56:59], v[120:123]\n"
    "v_mfma_i32_16x16x64_i8 v[120:123], v[224:227], v[60:63], v[120:123]\n"
    "ds_read_b128 v[96:99], v39 offset:10240\n"
    "v_mfma_i32_16x16x64_i8 v[120:123], v[232:235], v[64:67], v[120:123]\n"
    "s_waitcnt lgkmcnt(4)\n"
    "buffer_load_dwordx4 v18, s[36:39], s58 offen offset:2048 lds\n"
    "v_mfma_i32_16x16x64_i8 v[128:131], v[208:211], v[68:71], v[4:7]\n"
    "ds_read_b128 v[100:103], v38 offset:12288\n"
    "v_mfma_i32_16x16x64_i8 v[128:131], v[216:219], v[72:75], v[128:131]\n"
    "buffer_load_dwordx4 v19, s[36:39], s58 offen offset:3072 lds\n"
    "v_mfma_i32_16x16x64_i8 v[128:131], v[224:227], v[76:79], v[128:131]\n"
    "ds_read_b128 v[104:107], v39 offset:12288\n"
    "v_mfma_i32_16x16x64_i8 v[128:131], v[232:235], v[80:83], v[128:131]\n"
    "s_add_u32 m0, s16, 69632\n"
    "v_mfma_i32_16x16x64_i8 v[124:127], v[204:207], v[68:71], v[4:7]\n"
    "ds_read_b128 v[108:111], v38 offset:14336\n"
    "v_mfma_i32_16x16x64_i8 v[124:127], v[212:215], v[72:75], v[124:127]\n"
    "buffer_load_dwordx4 v16, s[36:39], s35 offen lds\n"
    "v_mfma_i32_16x16x64_i8 v[124:127], v[220:223], v[76:79], v[124:127]\n"
    "ds_read_b128 v[112:115], v39 offset:14336\n"
    "v_mfma_i32_16x16x64_i8 v[124:127], v[228:231], v[80:83], v[124:127]\n"
    "s_waitcnt lgkmcnt(4)\n"
    "v_mfma_i32_16x16x64_i8 v[132:135], v[204:207], v[84:87], v[4:7]\n"
    "ds_read_b128 v[52:55], v38 offset:16384\n"
    "v_mfma_i32_16x16x64_i8 v[132:135], v[212:215], v[88:91], v[132:135]\n"
    "buffer_load_dwordx4 v17, s[36:39], s35 offen offset:1024 lds\n"
    "v_mfma_i32_16x16x64_i8 v[132:135], v[220:223], v[92:95], v[132:135]\n"
    "ds_read_b128 v[56:59], v39 offset:16384\n"
    "v_mfma_i32_16x16x64_i8 v[132:135], v[228:231], v[96:99], v[132:135]\n"
    "buffer_load_dwordx4 v18, s[36:39], s35 offen offset:2048 lds\n"
    "v_mfma_i32_16x16x64_i8 v[136:139], v[208:211], v[84:87], v[4:7]\n"
    "ds_read_b128 v[60:63], v38 offset:18432\n"
    "v_mfma_i32_16x16x64_i8 v[136:139], v[216:219], v[88:91], v[136:139]\n"
    "v_mfma_i32_16x16x64_i8 v[136:139], v[224:227], v[92:95], v[136:139]\n"
    "ds_read_b128 v[64:67], v39 offset:18432\n"
    "v_mfma_i32_16x16x64_i8 v[136:139], v[232:235], v[96:99], v[136:139]\n"
    "s_waitcnt lgkmcnt(4)\n"
    "buffer_load_dwordx4 v19, s[36:39], s35 offen offset:3072 lds\n"
    "v_mfma_i32_16x16x64_i8 v[144:147], v[208:211], v[100:103], v[4:7]\n"
    "ds_read_b128 v[68:71], v38 offset:20480\n"
    "v_mfma_i32_16x16x64_i8 v[144:147], v[216:219], v[104:107], v[144:147]\n"
    "v_mfma_i32_16x16x64_i8 v[144:147], v[224:227], v[108:111], v[144:147]\n"
    "ds_read_b128 v[72:75], v39 offset:20480\n"
    "v_mfma_i32_16x16x64_i8 v[144:147], v[232:235], v[112:115], v[144:147]\n"
    "global_load_dword v32, v24, s[72:73] sc0 sc1\n"
    "s_add_u32 m0, s16, 98304\n"
    "v_mfma_i32_16x16x64_i8 v[140:143], v[204:207], v[100:103], v[4:7]\n"
    "ds_read_b128 v[76:79], v38 offset:22528\n"
    "s_add_u32 s35, s59, 131072\n"
    "v_mfma_i32_16x16x64_i8 v[140:143], v[212:215], v[104:107], v[140:143]\n"
    "buffer_load_dwordx4 v16, s[36:39], s59 offen lds\n"
    "v_mfma_i32_16x16x64_i8 v[140:143], v[220:223], v[108:111], v[140:143]\n"
    "ds_read_b128 v[80:83], v39 offset:22528\n"
    "v_mfma_i32_16x16x64_i8 v[140:143], v[228:231], v[112:115], v[140:143]\n"
    "s_waitcnt lgkmcnt(4)\n"
    "v_mfma_i32_16x16x64_i8 v[148:151], v[204:207], v[52:55], v[8:11]\n"
    "ds_read_b128 v[84:87], v38 offset:24576\n"
    "v_mfma_i32_16x16x64_i8 v[148:151], v[212:215], v[56:59], v[148:151]\n"
    "buffer_load_dwordx4 v17, s[36:39], s59 offen offset:1024 lds\n"
    "v_mfma_i32_16x16x64_i8 v[148:151], v[220:223], v[60:63], v[148:151]\n"
    "ds_read_b128 v[88:91], v39 offset:24576\n"
    "v_mfma_i32_16x16x64_i8 v[148:151], v[228:231], v[64:67], v[148:151]\n"
    "v_mfma_i32_16x16x64_i8 v[152:155], v[208:211], v[52:55], v[8:11]\n"
    "ds_read_b128 v[92:95], v38 offset:26624\n"
    "v_mfma_i32_16x16x64_i8 v[152:155], v[216:219], v[56:59], v[152:155]\n"
    "buffer_load_dwordx4 v18, s[36:39], s59 offen offset:2048 lds\n"
    "v_mfma_i32_16x16x64_i8 v[152:155], v[224:227], v[60:63], v[152:155]\n"
    "ds_read_b128 v[96:99], v39 offset:26624\n"
    "v_mfma_i32_16x16x64_i8 v[152:155], v[232:235], v[64:67], v[152:155]\n"
    "s_waitcnt lgkmcnt(4)\n"
    "buffer_load_dwordx4 v19, s[36:39], s59 offen offset:3072 lds\n"
    "v_mfma_i32_16x16x64_i8 v[160:163], v[208:211], v[68:71], v[8:11]\n"
    "ds_read_b128 v[100:103], v38 offset:28672\n"
    "v_mfma_i32_16x16x64_i8 v[160:163], v[216:219], v[72:75], v[160:163]\n"
    "s_add_u32 m0, s16, 102400\n"
    "v_mfma_i32_16x16x64_i8 v[160:163], v[224:227], v[76:79], v[160:163]\n"
    "ds_read_b128 v[104:107], v39 offset:28672\n"
    "v_mfma_i32_16x16x64_i8 v[160:163], v[232:235], v[80:83], v[160:163]\n"
    "buffer_load_dwordx4 v16, s[36:39], s35 offen lds\n"
    "v_mfma_i32_16x16x64_i8 v[156:159], v[204:207], v[68:71], v[8:11]\n"
    "ds_read_b128 v[108:111], v38 offset:30720\n"
    "v_mfma_i32_16x16x64_i8 v[156:159], v[212:215], v[72:75], v[156:159]\n"
    "v_mfma_i32_16x16x64_i8 v[156:159], v[220:223], v[76:79], v[156:159]\n"
    "ds_read_b128 v[112:115], v39 offset:30720\n"
    "v_mfma_i32_16x16x64_i8 v[156:159], v[228:231], v[80:83], v[156:159]\n"
    "s_waitcnt lgkmcnt(4)\n"
    "buffer_load_dwordx4 v17, s[36:39], s35 offen offset:1024 lds\n"
    "v_mfma_i32_16x16x64_i8 v[164:167], v[204:207], v[84:87], v[8:11]\n"
    "v_mfma_i32_16x16x64_i8 v[164:167], v[212:215], v[88:91], v[164:167]\n"
    "buffer_load_dwordx4 v18, s[36:39], s35 offen offset:2048 lds\n"
    "v_mfma_i32_16x16x64_i8 v[164:167], v[220:223], v[92:95], v[164:167]\n"
    "v_mfma_i32_16x16x64_i8 v[164:167], v[228:231], v[96:99], v[164:167]\n"
    "v_mfma_i32_16x16x64_i8 v[168:171], v[208:211], v[84:87], v[8:11]\n"
    "v_mfma_i32_16x16x64_i8 v[168:171], v[216:219], v[88:91], v[168:171]\n"
    "buffer_load_dwordx4 v19, s[36:39], s35 offen offset:3072 lds\n"
    "v_mfma_i32_16x16x64_i8 v[168:171], v[224:227], v[92:95], v[168:171]\n"
    "v_mfma_i32_16x16x64_i8 v[168:171], v[232:235], v[96:99], v[168:171]\n"
    "s_waitcnt lgkmcnt(0)\n"
    "v_mfma_i32_16x16x64_i8 v[176:179], v[208:211], v[100:103], v[8:11]\n"
    "v_mfma_i32_16x16x64_i8 v[176:179], v[216:219], v[104:107], v[176:179]\n"
    "v_mfma_i32_16x16x64_i8 v[176:179], v[224:227], v[108:111], v[176:179]\n"
    "v_mfma_i32_16x16x64_i8 v[176:179], v[232:235], v[112:115], v[176:179]\n"
    "v_mfma_i32_16x16x64_i8 v[172:175], v[204:207], v[100:103], v[8:11]\n"
    "v_mfma_i32_16x16x64_i8 v[172:175], v[212:215], v[104:107], v[172:175]\n"
    "v_mfma_i32_16x16x64_i8 v[172:175], v[220:223], v[108:111], v[172:175]\n"
    "v_mfma_i32_16x16x64_i8 v[172:175], v[228:231], v[112:115], v[172:175]\n"
    "s_waitcnt vmcnt(17) lgkmcnt(0)\n"
    "s_barrier\n"
    "s_add_u32 m0, s16, 131072\n"
    "s_add_u32 s35, s60, 131072\n"
    "buffer_load_dwordx4 v16, s[36:39], s60 offen lds\n"
    "buffer_load_dwordx4 v17, s[36:39], s60 offen offset:1024 lds\n"
    "buffer_load_dwordx4 v18, s[36:39], s60 offen offset:2048 lds\n"
    "buffer_load_dwordx4 v19, s[36:39], s60 offen offset:3072 lds\n"
    "s_add_u32 m0, s16, 135168\n"
    "s_nop 0\n"
    "buffer_load_dwordx4 v16, s[36:39], s35 offen lds\n"
    "buffer_load_dwordx4 v17, s[36:39], s35 offen offset:1024 lds\n"
    "buffer_load_dwordx4 v18, s[36:39], s35 offen offset:2048 lds\n"
    "buffer_load_dwordx4 v19, s[36:39], s35 offen offset:3072 lds\n"
    "ds_read_b64_tr_b8 a[128:129], v44 offset:32768\n"
    "ds_read_b64_tr_b8 a[130:131], v44 offset:33792\n"
    "ds_read_b64_tr_b8 a[132:133], v45 offset:32768\n"
    "ds_read_b64_tr_b8 a[134:135], v45 offset:33792\n"
    "ds_read_b64_tr_b8 a[136:137], v44 offset:40960\n"
    "ds_read_b64_tr_b8 a[138:139], v44 offset:41984\n"
    "ds_read_b64_tr_b8 a[140:141], v45 offset:40960\n"
    "ds_read_b64_tr_b8 a[142:143], v45 offset:41984\n"
    "ds_read_b64_tr_b8 a[144:145], v44 offset:49152\n"
    "ds_read_b64_tr_b8 a[146:147], v44 offset:50176\n"
    "ds_read_b64_tr_b8 a[148:149], v45 offset:49152\n"
    "ds_read_b64_tr_b8 a[150:151], v45 offset:50176\n"
    "ds_read_b64_tr_b8 a[152:153], v44 offset:57344\n"
    "ds_read_b64_tr_b8 a[154:155], v44 offset:58368\n"
    "ds_read_b64_tr_b8 a[156:157], v45 offset:57344\n"
    "ds_read_b64_tr_b8 a[158:159], v45 offset:58368\n"
    "s_waitcnt vmcnt(17) lgkmcnt(0)\n"
    "s_barrier\n"
    "s_waitcnt vmcnt(16)\n"
    "v_cmp_le_u32 vcc, s17, v32\n"
    "s_ff0_i32_b32 s85, vcc_lo\n"
    "s_max_u32 s85, s85, s86\n"
    "s_cmp_eq_u32 s84, 0\n"
    "s_cselect_b32 s91, s85, 0\n"
    "s_cmp_gt_u32 s91, 0\n"
    "s_cselect_b32 s82, 0, s83\n"
    "s_addc_u32 s84, s84, 0\n"
    "s_add_u32 m0, s16, 0\n"
    "s_mov_b32 s34, s82\n"
    "buffer_load_dwordx4 v22, s[40:43], s34 offen sc0 sc1 lds\n"
    "buffer_load_dwordx4 v22, s[40:43], s34 offen offset:1024 sc0 sc1 lds\n"
    "buffer_load_dwordx4 v22, s[40:43], s34 offen offset:2048 sc0 sc1 lds\n"
    "buffer_load_dwordx4 v22, s[40:43], s34 offen offset:3072 sc0 sc1 lds\n"
    "s_add_u32 m0, m0, 4096\n"
    "s_add_u32 s34, s34, 4096\n"
    "buffer_load_dwordx4 v22, s[40:43], s34 offen sc0 sc1 lds\n"
    "buffer_load_dwordx4 v22, s[40:43], s34 offen offset:1024 sc0 sc1 lds\n"
    "buffer_load_dwordx4 v22, s[40:43], s34 offen offset:2048 sc0 sc1 lds\n"
    "buffer_load_dwordx4 v22, s[40:43], s34 offen offset:3072 sc0 sc1 lds\n"
    "s_mov_b32 s96, 1\n"
    "ds_read_b64_tr_b8 a[160:161], v46 offset:0\n"
    "ds_read_b64_tr_b8 a[162:163], v46 offset:1024\n"
    "ds_read_b64_tr_b8 a[164:165], v47 offset:0\n"
    "ds_read_b64_tr_b8 a[166:167], v47 offset:1024\n"
    "ds_read_b64_tr_b8 a[168:169], v46 offset:8192\n"
    "ds_read_b64_tr_b8 a[170:171], v46 offset:9216\n"
    "ds_read_b64_tr_b8 a[172:173], v47 offset:8192\n"
    "ds_read_b64_tr_b8 a[174:175], v47 offset:9216\n"
    "ds_read_b64_tr_b8 a[176:177], v46 offset:16384\n"
    "ds_read_b64_tr_b8 a[178:179], v46 offset:17408\n"
    "ds_read_b64_tr_b8 a[180:181], v47 offset:16384\n"
    "ds_read_b64_tr_b8 a[182:183], v47 offset:17408\n"
    "ds_read_b64_tr_b8 a[184:185], v46 offset:24576\n"
    "ds_read_b64_tr_b8 a[186:187], v46 offset:25600\n"
    "ds_read_b64_tr_b8 a[188:189], v47 offset:24576\n"
    "ds_read_b64_tr_b8 a[190:191], v47 offset:25600\n"
    "s_waitcnt vmcnt(16) lgkmcnt(0)\n"
    "s_barrier\n"
    "global_load_dword v32, v24, s[72:73] sc0 sc1\n"
    "s_cmp_eq_u32 s84, 1\n"
    "s_cselect_b32 s91, s85, 0\n"
    "s_cmp_gt_u32 s91, 1\n"
    "s_cselect_b32 s82, 8192, s83\n"
    "s_addc_u32 s84, s84, 0\n"
    "s_add_u32 m0, s16, 32768\n"
    "s_mov_b32 s34, s82\n"
    "buffer_load_dwordx4 v22, s[40:43], s34 offen sc0 sc1 lds\n"
    "buffer_load_dwordx4 v22, s[40:43], s34 offen offset:1024 sc0 sc1 lds\n"
    "buffer_load_dwordx4 v22, s[40:43], s34 offen offset:2048 sc0 sc1 lds\n"
    "buffer_load_dwordx4 v22, s[40:43], s34 offen offset:3072 sc0 sc1 lds\n"
    "s_add_u32 m0, m0, 4096\n"
    "s_add_u32 s34, s34, 4096\n"
    "buffer_load_dwordx4 v22, s[40:43], s34 offen sc0 sc1 lds\n"
    "buffer_load_dwordx4 v22, s[40:43], s34 offen offset:1024 sc0 sc1 lds\n"
    "buffer_load_dwordx4 v22, s[40:43], s34 offen offset:2048 sc0 sc1 lds\n"
    "buffer_load_dwordx4 v22, s[40:43], s34 offen offset:3072 sc0 sc1 lds\n"
    "s_mov_b32 s96, 2\n"
    "ds_read_b64_tr_b8 a[192:193], v46 offset:32768\n"
    "ds_read_b64_tr_b8 a[194:195], v46 offset:33792\n"
    "ds_read_b64_tr_b8 a[196:197], v47 offset:32768\n"
    "ds_read_b64_tr_b8 a[198:199], v47 offset:33792\n"
    "ds_read_b64_tr_b8 a[200:201], v46 offset:40960\n"
    "ds_read_b64_tr_b8 a[202:203], v46 offset:41984\n"
    "ds_read_b64_tr_b8 a[204:205], v47 offset:40960\n"
    "ds_read_b64_tr_b8 a[206:207], v47 offset:41984\n"
    "ds_read_b64_tr_b8 a[208:209], v46 offset:49152\n"
    "ds_read_b64_tr_b8 a[210:211], v46 offset:50176\n"
    "ds_read_b64_tr_b8 a[212:213], v47 offset:49152\n"
    "ds_read_b64_tr_b8 a[214:215], v47 offset:50176\n"
    "ds_read_b64_tr_b8 a[216:217], v46 offset:57344\n"
    "ds_read_b64_tr_b8 a[218:219], v46 offset:58368\n"
    "ds_read_b64_tr_b8 a[220:221], v47 offset:57344\n"
    "ds_read_b64_tr_b8 a[222:223], v47 offset:58368\n"
    "s_waitcnt vmcnt(17) lgkmcnt(0)\n"
    "s_barrier\n"
    "s_cmp_lt_u32 s84, s96\n"
    "s_getpc_b64 s[88:89]\n"
    "s_cbranch_scc1 L_mida%=\n"
    "ds_read_b64_tr_b8 a[224:225], v48 offset:0\n"
    "ds_read_b64_tr_b8 a[226:227], v48 offset:1024\n"
    "ds_read_b64_tr_b8 a[228:229], v49 offset:0\n"
    "ds_read_b64_tr_b8 a[230:231], v49 offset:1024\n"
    "ds_read_b64_tr_b8 a[232:233], v48 offset:8192\n"
    "ds_read_b64_tr_b8 a[234:235], v48 offset:9216\n"
    "ds_read_b64_tr_b8 a[236:237], v49 offset:8192\n"
    "ds_read_b64_tr_b8 a[238:239], v49 offset:9216\n"
    "ds_read_b64_tr_b8 a[240:241], v48 offset:16384\n"
    "ds_read_b64_tr_b8 a[242:243], v48 offset:17408\n"
    "ds_read_b64_tr_b8 a[244:245], v49 offset:16384\n"
    "ds_read_b64_tr_b8 a[246:247], v49 offset:17408\n"
    "ds_read_b64_tr_b8 a[248:249], v48 offset:24576\n"
    "ds_read_b64_tr_b8 a[250:251], v48 offset:25600\n"
    "ds_read_b64_tr_b8 a[252:253], v49 offset:24576\n"
    "ds_read_b64_tr_b8 a[254:255], v49 offset:25600\n"
    "s_mov_b32 s95, 0\n"
    "s_and_b32 s91, s84, s25\n"
    "s_cmp_gt_u32 s91, 0\n"
    "s_getpc_b64 s[88:89]\n"
    "s_cbranch_scc0 L_slowa%=\n"
    "s_waitcnt vmcnt(9) lgkmcnt(0)\n"
    "s_barrier\n"
    "ds_read_b128 v[52:55], v38\n"
    "ds_read_b128 v[56:59], v39\n"
    "ds_read_b128 v[60:63], v38 offset:8192\n"
    "ds_read_b128 v[64:67], v39 offset:8192\n"
    "ds_read_b128 v[68:71], v38 offset:16384\n"
    "ds_read_b128 v[72:75], v39 offset:16384\n"
    "ds_read_b128 v[76:79], v38 offset:24576\n"
    "ds_read_b128 v[80:83], v39 offset:24576\n"
    "s_waitcnt lgkmcnt(4)\n"
    "v_mfma_i32_16x16x64_i8 v[116:119], a[0:3], v[52:55], v[116:119]\n"
    "s_waitcnt vmcnt(8)\n"
    "ds_read_b128 v[84:87], v38 offset:2048\n"
    "v_mfma_i32_16x16x64_i8 v[116:119], a[32:35], v[56:59], v[116:119]\n"
    "v_cmp_le_u32 vcc, s17, v32\n"
    "s_ff0_i32_b32 s85, vcc_lo\n"
    "v_mfma_i32_16x16x64_i8 v[116:119], a[8:11], v[60:63], v[116:119]\n"
    "ds_read_b128 v[88:91], v39 offset:2048\n"
    "s_max_u32 s85, s85, s86\n"
    "v_mfma_i32_16x16x64_i8 v[116:119], a[40:43], v[64:67], v[116:119]\n"
    "s_min_u32 s91, s85, s96\n"
    "s_cmp_lt_u32 s84, s91\n"
    "s_getpc_b64 s[88:89]\n"
    "s_cbranch_scc1 L_catcha%=\n"
    "v_mfma_i32_16x16x64_i8 v[120:123], a[4:7], v[52:55], v[120:123]\n"
    "ds_read_b128 v[92:95], v38 offset:10240\n"
    "global_load_dword v32, v24, s[72:73] sc0 sc1\n"
    "v_mfma_i32_16x16x64_i8 v[120:123], a[36:39], v[56:59], v[120:123]\n"
    "s_cmp_eq_u32 s84, 2\n"
    "s_cselect_b32 s91, s85, 0\n"
    "s_cmp_gt_u32 s91, 2\n"
    "s_cselect_b32 s82, 16384, s83\n"
    "s_addc_u32 s84, s84, 0\n"
    "v_mfma_i32_16x16x64_i8 v[120:123], a[12:15], v[60:63], v[120:123]\n"
    "ds_read_b128 v[96:99], v39 offset:10240\n"
    "s_add_u32 m0, s16, 65536\n"
    "v_mfma_i32_16x16x64_i8 v[120:123], a[44:47], v[64:67], v[120:123]\n"
    "s_waitcnt lgkmcnt(4)\n"
    "s_mov_b32 s34, s82\n"
    "v_mfma_i32_16x16x64_i8 v[120:123], a[20:23], v[68:71], v[120:123]\n"
    "ds_read_b128 v[100:103], v38 offset:18432\n"
    "buffer_load_dwordx4 v22, s[40:43], s34 offen sc0 sc1 lds\n"
    "v_mfma_i32_16x16x64_i8 v[120:123], a[52:55], v[72:75], v[120:123]\n"
    "v_mfma_i32_16x16x64_i8 v[120:123], a[28:31], v[76:79], v[120:123]\n"
    "ds_read_b128 v[104:107], v39 offset:18432\n"
    "v_mfma_i32_16x16x64_i8 v[120:123], a[60:63], v[80:83], v[120:123]\n"
    "v_mfma_i32_16x16x64_i8 v[116:119], a[16:19], v[68:71], v[116:119]\n"
    "ds_read_b128 v[108:111], v38 offset:26624\n"
    "v_mfma_i32_16x16x64_i8 v[116:119], a[48:51], v[72:75], v[116:119]\n"
    "buffer_load_dwordx4 v22, s[40:43], s34 offen offset:1024 sc0 sc1 lds\n"
    "v_mfma_i32_16x16x64_i8 v[116:119], a[24:27], v[76:79], v[116:119]\n"
    "ds_read_b128 v[112:115], v39 offset:26624\n"
    "v_mfma_i32_16x16x64_i8 v[116:119], a[56:59], v[80:83], v[116:119]\n"
    "s_waitcnt lgkmcnt(4)\n"
    "s_cmp_lt_u32 s84, s96\n"
    "s_getpc_b64 s[88:89]\n"
    "s_cbranch_scc1 L_mida%=\n"
    "v_mfma_i32_16x16x64_i8 v[116:119], a[64:67], v[84:87], v[116:119]\n"
    "ds_read_b128 v[52:55], v38 offset:4096\n"
    "v_mfma_i32_16x16x64_i8 v[116:119], a[96:99], v[88:91], v[116:119]\n"
    "buffer_load_dwordx4 v22, s[40:43], s34 offen offset:2048 sc0 sc1 lds\n"
    "v_mfma_i32_16x16x64_i8 v[116:119], a[72:75], v[92:95], v[116:119]\n"
    "ds_read_b128 v[56:59], v39 offset:4096\n"
    "v_mfma_i32_16x16x64_i8 v[116:119], a[104:107], v[96:99], v[116:119]\n"
    "v_mfma_i32_16x16x64_i8 v[120:123], a[68:71], v[84:87], v[120:123]\n"
    "ds_read_b128 v[60:63], v38 offset:12288\n"
    "v_mfma_i32_16x16x64_i8 v[120:123], a[100:103], v[88:91], v[120:123]\n"
    "v_mfma_i32_16x16x64_i8 v[120:123], a[76:79], v[92:95], v[120:123]\n"
    "ds_read_b128 v[64:67], v39 offset:12288\n"
    "v_mfma_i32_16x16x64_i8 v[120:123], a[108:111], v[96:99], v[120:123]\n"
    "s_waitcnt lgkmcnt(4)\n"
    "buffer_load_dwordx4 v22, s[40:43], s34 offen offset:3072 sc0 sc1 lds\n"
    "v_mfma_i32_16x16x64_i8 v[120:123], a[84:87], v[100:103], v[120:123]\n"
    "ds_read_b128 v[68:71], v38 offset:20480\n"
    "v_mfma_i32_16x16x64_i8 v[120:123], a[116:119], v[104:107], v[120:123]\n"
    "v_mfma_i32_16x16x64_i8 v[120:123], a[92:95], v[108:111], v[120:123]\n"
    "ds_read_b128 v[72:75], v39 offset:20480\n"
    "v_mfma_i32_16x16x64_i8 v[120:123], a[124:127], v[112:115], v[120:123]\n"
    "s_add_u32 m0, m0, 4096\n"
    "s_add_u32 s34, s34, 4096\n"
    "v_mfma_i32_16x16x64_i8 v[116:119], a[80:83], v[100:103], v[116:119]\n"
    "ds_read_b128 v[76:79], v38 offset:28672\n"
    "v_mfma_i32_16x16x64_i8 v[116:119], a[112:115], v[104:107], v[116:119]\n"
    "buffer_load_dwordx4 v22, s[40:43], s34 offen sc0 sc1 lds\n"
    "v_mfma_i32_16x16x64_i8 v[116:119], a[88:91], v[108:111], v[116:119]\n"
    "ds_read_b128 v[80:83], v39 offset:28672\n"
    "v_mfma_i32_16x16x64_i8 v[116:119], a[120:123], v[112:115], v[116:119]\n"
    "s_waitcnt lgkmcnt(4)\n"
    "v_mfma_i32_16x16x64_i8 v[116:119], a[128:131], v[52:55], v[116:119]\n"
    "ds_read_b128 v[84:87], v38 offset:6144\n"
    "v_mfma_i32_16x16x64_i8 v[116:119], a[160:163], v[56:59], v[116:119]\n"
    "s_cmp_lt_u32 s84, s96\n"
    "s_getpc_b64 s[88:89]\n"
    "s_cbranch_scc1 L_mida%=\n"
    "v_mfma_i32_16x16x64_i8 v[116:119], a[136:139], v[60:63], v[116:119]\n"
    "ds_read_b128 v[88:91], v39 offset:6144\n"
    "v_mfma_i32_16x16x64_i8 v[116:119], a[168:171], v[64:67], v[116:119]\n"
    "buffer_load_dwordx4 v22, s[40:43], s34 offen offset:1024 sc0 sc1 lds\n"
    "v_mfma_i32_16x16x64_i8 v[120:123], a[132:135], v[52:55], v[120:123]\n"
    "ds_read_b128 v[92:95], v38 offset:14336\n"
    "v_mfma_i32_16x16x64_i8 v[120:123], a[164:167], v[56:59], v[120:123]\n"
    "v_mfma_i32_16x16x64_i8 v[120:123], a[140:143], v[60:63], v[120:123]\n"
    "ds_read_b128 v[96:99], v39 offset:14336\n"
    "v_mfma_i32_16x16x64_i8 v[120:123], a[172:175], v[64:67], v[120:123]\n"
    "s_waitcnt lgkmcnt(4)\n"
    "buffer_load_dwordx4 v22, s[40:43], s34 offen offset:2048 sc0 sc1 lds\n"
    "v_mfma_i32_16x16x64_i8 v[120:123], a[148:151], v[68:71], v[120:123]\n"
    "ds_read_b128 v[100:103], v38 offset:22528\n"
    "v_mfma_i32_16x16x64_i8 v[120:123], a[180:183], v[72:75], v[120:123]\n"
    "v_mfma_i32_16x16x64_i8 v[120:123], a[156:159], v[76:79], v[120:123]\n"
    "ds_read_b128 v[104:107], v39 offset:22528\n"
    "v_mfma_i32_16x16x64_i8 v[120:123], a[188:191], v[80:83], v[120:123]\n"
    "v_mfma_i32_16x16x64_i8 v[116:119], a[144:147], v[68:71], v[116:119]\n"
    "ds_read_b128 v[108:111], v38 offset:30720\n"
    "v_mfma_i32_16x16x64_i8 v[116:119], a[176:179], v[72:75], v[116:119]\n"
    "buffer_load_dwordx4 v22, s[40:43], s34 offen offset:3072 sc0 sc1 lds\n"
    "s_mov_b32 s96, 3\n"
    "v_mfma_i32_16x16x64_i8 v[116:119], a[152:155], v[76:79], v[116:119]\n"
    "ds_read_b128 v[112:115], v39 offset:30720\n"
    "v_mfma_i32_16x16x64_i8 v[116:119], a[184:187], v[80:83], v[116:119]\n"
    "s_waitcnt lgkmcnt(4)\n"
    "v_mfma_i32_16x16x64_i8 v[116:119], a[192:195], v[84:87], v[116:119]\n"
    "v_mfma_i32_16x16x64_i8 v[116:119], a[224:227], v[88:91], v[116:119]\n"
    "v_mfma_i32_16x16x64_i8 v[116:119], a[200:203], v[92:95], v[116:119]\n"
    "s_mov_b32 s95, 1\n"
    "s_and_b32 s91, s84, s25\n"
    "v_mfma_i32_16x16x64_i8 v[116:119], a[232:235], v[96:99], v[116:119]\n"
    "s_cmp_gt_u32 s91, 1\n"
    "s_getpc_b64 s[88:89]\n"
    "s_cbranch_scc0 L_slowa%=\n"
    "s_waitcnt vmcnt(9) lgkmcnt(0)\n"
    "s_barrier\n"
    "v_mfma_i32_16x16x64_i8 v[120:123], a[196:199], v[84:87], v[120:123]\n"
    "ds_read_b128 v[52:55], v38 offset:32768\n"
    "s_waitcnt vmcnt(8)\n"
    "v_mfma_i32_16x16x64_i8 v[120:123], a[228:231], v[88:91], v[120:123]\n"
    "ds_read_b128 v[56:59], v39 offset:32768\n"
    "v_mfma_i32_16x16x64_i8 v[120:123], a[204:207], v[92:95], v[120:123]\n"
    "ds_read_b128 v[60:63], v38 offset:40960\n"
    "v_cmp_le_u32 vcc, s17, v32\n"
    "v_mfma_i32_16x16x64_i8 v[120:123], a[236:239], v[96:99], v[120:123]\n"
    "ds_read_b128 v[64:67], v39 offset:40960\n"
    "v_mfma_i32_16x16x64_i8 v[120:123], a[212:215], v[100:103], v[120:123]\n"
    "ds_read_b128 v[68:71], v38 offset:49152\n"
    "v_mfma_i32_16x16x64_i8 v[120:123], a[244:247], v[104:107], v[120:123]\n"
    "ds_read_b128 v[72:75], v39 offset:49152\n"
    "s_ff0_i32_b32 s85, vcc_lo\n"
    "v_mfma_i32_16x16x64_i8 v[120:123], a[220:223], v[108:111], v[120:123]\n"
    "ds_read_b128 v[76:79], v38 offset:57344\n"
    "v_mfma_i32_16x16x64_i8 v[120:123], a[252:255], v[112:115], v[120:123]\n"
    "ds_read_b128 v[80:83], v39 offset:57344\n"
    "v_mfma_i32_16x16x64_i8 v[116:119], a[208:211], v[100:103], v[116:119]\n"
    "s_max_u32 s85, s85, s86\n"
    "v_mfma_i32_16x16x64_i8 v[116:119], a[240:243], v[104:107], v[116:119]\n"
    "s_min_u32 s91, s85, s96\n"
    "v_mfma_i32_16x16x64_i8 v[116:119], a[216:219], v[108:111], v[116:119]\n"
    "s_cmp_lt_u32 s84, s91\n"
    "s_getpc_b64 s[88:89]\n"
    "s_cbranch_scc1 L_catcha%=\n"
    "v_mfma_i32_16x16x64_i8 v[116:119], a[248:251], v[112:115], v[116:119]\n"
    "s_waitcnt lgkmcnt(4)\n"
    "v_mfma_i32_16x16x64_i8 v[124:127], a[0:3], v[52:55], v[124:127]\n"
    "global_load_dword v32, v24, s[72:73] sc0 sc1\n"
    "ds_read_b128 v[84:87], v38 offset:34816\n"
    "v_mfma_i32_16x16x64_i8 v[124:127], a[32:35], v[56:59], v[124:127]\n"
    "v_mfma_i32_16x16x64_i8 v[124:127], a[8:11], v[60:63], v[124:127]\n"
    "ds_read_b128 v[88:91], v39 offset:34816\n"
    "v_mfma_i32_16x16x64_i8 v[124:127], a[40:43], v[64:67], v[124:127]\n"
    "v_mfma_i32_16x16x64_i8 v[128:131], a[4:7], v[52:55], v[128:131]\n"
    "ds_read_b128 v[92:95], v38 offset:43008\n"
    "v_mfma_i32_16x16x64_i8 v[128:131], a[36:39], v[56:59], v[128:131]\n"
    "s_add_u32 s90, s64, 0\n"
    "buffer_store_dwordx4 v[116:119], v22, s[44:47], s90 offen sc0 sc1\n"
    "v_mfma_i32_16x16x64_i8 v[128:131], a[12:15], v[60:63], v[128:131]\n"
    "ds_read_b128 v[96:99], v39 offset:43008\n"
    "v_mfma_i32_16x16x64_i8 v[128:131], a[44:47], v[64:67], v[128:131]\n"
    "s_waitcnt lgkmcnt(4)\n"
    "v_mfma_i32_16x16x64_i8 v[128:131], a[20:23], v[68:71], v[128:131]\n"
    "ds_read_b128 v[100:103], v38 offset:51200\n"
    "v_mfma_i32_16x16x64_i8 v[128:131], a[52:55], v[72:75], v[128:131]\n"
    "buffer_store_dwordx4 v[120:123], v22, s[44:47], s90 offen offset:1024 sc0 sc1\n"
    "v_mfma_i32_16x16x64_i8 v[128:131], a[28:31], v[76:79], v[128:131]\n"
    "ds_read_b128 v[104:107], v39 offset:51200\n"
    "s_cmp_eq_u32 s84, 3\n"
    "s_cselect_b32 s91, s85, 0\n"
    "v_mfma_i32_16x16x64_i8 v[128:131], a[60:63], v[80:83], v[128:131]\n"
    "s_cmp_gt_u32 s91, 3\n"
    "s_cselect_b32 s82, 24576, s83\n"
    "s_addc_u32 s84, s84, 0\n"
    "v_mfma_i32_16x16x64_i8 v[124:127], a[16:19], v[68:71], v[124:127]\n"
    "ds_read_b128 v[108:111], v38 offset:59392\n"
    "s_add_u32 m0, s16, 98304\n"
    "v_mfma_i32_16x16x64_i8 v[124:127], a[48:51], v[72:75], v[124:127]\n"
    "s_mov_b32 s34, s82\n"
    "v_mfma_i32_16x16x64_i8 v[124:127], a[24:27], v[76:79], v[124:127]\n"
    "ds_read_b128 v[112:115], v39 offset:59392\n"
    "v_mfma_i32_16x16x64_i8 v[124:127], a[56:59], v[80:83], v[124:127]\n"
    "s_waitcnt lgkmcnt(4)\n"
    "buffer_load_dwordx4 v22, s[40:43], s34 offen sc0 sc1 lds\n"
    "s_cmp_lt_u32 s84, s96\n"
    "s_getpc_b64 s[88:89]\n"
    "s_cbranch_scc1 L_mida%=\n"
    "v_mfma_i32_16x16x64_i8 v[124:127], a[64:67], v[84:87], v[124:127]\n"
    "ds_read_b128 v[52:55], v38 offset:36864\n"
    "v_mfma_i32_16x16x64_i8 v[124:127], a[96:99], v[88:91], v[124:127]\n"
    "v_mfma_i32_16x16x64_i8 v[124:127], a[72:75], v[92:95], v[124:127]\n"
    "ds_read_b128 v[56:59], v39 offset:36864\n"
    "v_mfma_i32_16x16x64_i8 v[124:127], a[104:107], v[96:99], v[124:127]\n"
    "buffer_load_dwordx4 v22, s[40:43], s34 offen offset:1024 sc0 sc1 lds\n"
    "v_mfma_i32_16x16x64_i8 v[128:131], a[68:71], v[84:87], v[128:131]\n"
    "ds_read_b128 v[60:63], v38 offset:45056\n"
    "v_mfma_i32_16x16x64_i8 v[128:131], a[100:103], v[88:91], v[128:131]\n"
    "v_mfma_i32_16x16x64_i8 v[128:131], a[76:79], v[92:95], v[128:131]\n"
    "ds_read_b128 v[64:67], v39 offset:45056\n"
    "v_mfma_i32_16x16x64_i8 v[128:131], a[108:111], v[96:99], v[128:131]\n"
    "s_waitcnt lgkmcnt(4)\n"
    "buffer_load_dwordx4 v22, s[40:43], s34 offen offset:2048 sc0 sc1 lds\n"
    "v_mfma_i32_16x16x64_i8 v[128:131], a[84:87], v[100:103], v[128:131]\n"
    "ds_read_b128 v[68:71], v38 offset:53248\n"
    "v_mfma_i32_16x16x64_i8 v[128:131], a[116:119], v[104:107], v[128:131]\n"
    "v_mfma_i32_16x16x64_i8 v[128:131], a[92:95], v[108:111], v[128:131]\n"
    "ds_read_b128 v[72:75], v39 offset:53248\n"
    "v_mfma_i32_16x16x64_i8 v[128:131], a[124:127], v[112:115], v[128:131]\n"
    "v_mfma_i32_16x16x64_i8 v[124:127], a[80:83], v[100:103], v[124:127]\n"
    "ds_read_b128 v[76:79], v38 offset:61440\n"
    "v_mfma_i32_16x16x64_i8 v[124:127], a[112:115], v[104:107], v[124:127]\n"
    "buffer_load_dwordx4 v22, s[40:43], s34 offen offset:3072 sc0 sc1 lds\n"
    "v_mfma_i32_16x16x64_i8 v[124:127], a[88:91], v[108:111], v[124:127]\n"
    "ds_read_b128 v[80:83], v39 offset:61440\n"
    "v_mfma_i32_16x16x64_i8 v[124:127], a[120:123], v[112:115], v[124:127]\n"
    "s_waitcnt lgkmcnt(4)\n"
    "s_add_u32 m0, m0, 4096\n"
    "v_mfma_i32_16x16x64_i8 v[124:127], a[128:131], v[52:55], v[124:127]\n"
    "ds_read_b128 v[84:87], v38 offset:38912\n"
    "s_add_u32 s34, s34, 4096\n"
    "v_mfma_i32_16x16x64_i8 v[124:127], a[160:163], v[56:59], v[124:127]\n"
    "buffer_load_dwordx4 v22, s[40:43], s34 offen sc0 sc1 lds\n"
    "s_cmp_lt_u32 s84, s96\n"
    "s_getpc_b64 s[88:89]\n"
    "s_cbranch_scc1 L_mida%=\n"
    "v_mfma_i32_16x16x64_i8 v[124:127], a[136:139], v[60:63], v[124:127]\n"
    "ds_read_b128 v[88:91], v39 offset:38912\n"
    "v_mfma_i32_16x16x64_i8 v[124:127], a[168:171], v[64:67], v[124:127]\n"
    "v_mfma_i32_16x16x64_i8 v[128:131], a[132:135], v[52:55], v[128:131]\n"
    "ds_read_b128 v[92:95], v38 offset:47104\n"
    "v_mfma_i32_16x16x64_i8 v[128:131], a[164:167], v[56:59], v[128:131]\n"
    "buffer_load_dwordx4 v22, s[40:43], s34 offen offset:1024 sc0 sc1 lds\n"
    "v_mfma_i32_16x16x64_i8 v[128:131], a[140:143], v[60:63], v[128:131]\n"
    "ds_read_b128 v[96:99], v39 offset:47104\n"
    "v_mfma_i32_16x16x64_i8 v[128:131], a[172:175], v[64:67], v[128:131]\n"
    "s_waitcnt lgkmcnt(4)\n"
    "v_mfma_i32_16x16x64_i8 v[128:131], a[148:151], v[68:71], v[128:131]\n"
    "ds_read_b128 v[100:103], v38 offset:55296\n"
    "v_mfma_i32_16x16x64_i8 v[128:131], a[180:183], v[72:75], v[128:131]\n"
    "buffer_load_dwordx4 v22, s[40:43], s34 offen offset:2048 sc0 sc1 lds\n"
    "v_mfma_i32_16x16x64_i8 v[128:131], a[156:159], v[76:79], v[128:131]\n"
    "ds_read_b128 v[104:107], v39 offset:55296\n"
    "v_mfma_i32_16x16x64_i8 v[128:131], a[188:191], v[80:83], v[128:131]\n"
    "v_mfma_i32_16x16x64_i8 v[124:127], a[144:147], v[68:71], v[124:127]\n"
    "ds_read_b128 v[108:111], v38 offset:63488\n"
    "v_mfma_i32_16x16x64_i8 v[124:127], a[176:179], v[72:75], v[124:127]\n"
    "v_mfma_i32_16x16x64_i8 v[124:127], a[152:155], v[76:79], v[124:127]\n"
    "ds_read_b128 v[112:115], v39 offset:63488\n"
    "v_mfma_i32_16x16x64_i8 v[124:127], a[184:187], v[80:83], v[124:127]\n"
    "s_waitcnt lgkmcnt(4)\n"
    "buffer_load_dwordx4 v22, s[40:43], s34 offen offset:3072 sc0 sc1 lds\n"
    "v_mfma_i32_16x16x64_i8 v[124:127], a[192:195], v[84:87], v[124:127]\n"
    "s_mov_b32 s96, 4\n"
    "v_mfma_i32_16x16x64_i8 v[124:127], a[224:227], v[88:91], v[124:127]\n"
    "v_mfma_i32_16x16x64_i8 v[124:127], a[200:203], v[92:95], v[124:127]\n"
    "s_mov_b32 s95, 2\n"
    "s_and_b32 s91, s84, s25\n"
    "v_mfma_i32_16x16x64_i8 v[124:127], a[232:235], v[96:99], v[124:127]\n"
    "s_cmp_gt_u32 s91, 2\n"
    "s_getpc_b64 s[88:89]\n"
    "s_cbranch_scc0 L_slowa%=\n"
    "s_waitcnt vmcnt(11) lgkmcnt(0)\n"
    "s_barrier\n"
    "v_mfma_i32_16x16x64_i8 v[128:131], a[196:199], v[84:87], v[128:131]\n"
    "ds_read_b128 v[52:55], v40\n"
    "s_waitcnt vmcnt(10)\n"
    "v_mfma_i32_16x16x64_i8 v[128:131], a[228:231], v[88:91], v[128:131]\n"
    "ds_read_b128 v[56:59], v41\n"
    "v_mfma_i32_16x16x64_i8 v[128:131], a[204:207], v[92:95], v[128:131]\n"
    "ds_read_b128 v[60:63], v40 offset:8192\n"
    "v_cmp_le_u32 vcc, s17, v32\n"
    "v_mfma_i32_16x16x64_i8 v[128:131], a[236:239], v[96:99], v[128:131]\n"
    "ds_read_b128 v[64:67], v41 offset:8192\n"
    "v_mfma_i32_16x16x64_i8 v[128:131], a[212:215], v[100:103], v[128:131]\n"
    "ds_read_b128 v[68:71], v40 offset:16384\n"
    "v_mfma_i32_16x16x64_i8 v[128:131], a[244:247], v[104:107], v[128:131]\n"
    "ds_read_b128 v[72:75], v41 offset:16384\n"
    "s_ff0_i32_b32 s85, vcc_lo\n"
    "v_mfma_i32_16x16x64_i8 v[128:131], a[220:223], v[108:111], v[128:131]\n"
    "ds_read_b128 v[76:79], v40 offset:24576\n"
    "v_mfma_i32_16x16x64_i8 v[128:131], a[252:255], v[112:115], v[128:131]\n"
    "ds_read_b128 v[80:83], v41 offset:24576\n"
    "v_mfma_i32_16x16x64_i8 v[124:127], a[208:211], v[100:103], v[124:127]\n"
    "s_max_u32 s85, s85, s86\n"
    "v_mfma_i32_16x16x64_i8 v[124:127], a[240:243], v[104:107], v[124:127]\n"
    "s_min_u32 s91, s85, s96\n"
    "v_mfma_i32_16x16x64_i8 v[124:127], a[216:219], v[108:111], v[124:127]\n"
    "s_cmp_lt_u32 s84, s91\n"
    "s_getpc_b64 s[88:89]\n"
    "s_cbranch_scc1 L_catcha%=\n"
    "v_mfma_i32_16x16x64_i8 v[124:127], a[248:251], v[112:115], v[124:127]\n"
    "s_waitcnt lgkmcnt(4)\n"
    "v_mfma_i32_16x16x64_i8 v[132:135], a[0:3], v[52:55], v[132:135]\n"
    "global_load_dword v32, v24, s[72:73] sc0 sc1\n"
    "ds_read_b128 v[84:87], v40 offset:2048\n"
    "v_mfma_i32_16x16x64_i8 v[132:135], a[32:35], v[56:59], v[132:135]\n"
    "v_mfma_i32_16x16x64_i8 v[132:135], a[8:11], v[60:63], v[132:135]\n"
    "ds_read_b128 v[88:91], v41 offset:2048\n"
    "v_mfma_i32_16x16x64_i8 v[132:135], a[40:43], v[64:67], v[132:135]\n"
    "v_mfma_i32_16x16x64_i8 v[136:139], a[4:7], v[52:55], v[136:139]\n"
    "ds_read_b128 v[92:95], v40 offset:10240\n"
    "v_mfma_i32_16x16x64_i8 v[136:139], a[36:39], v[56:59], v[136:139]\n"
    "s_add_u32 s90, s64, 32768\n"
    "buffer_store_dwordx4 v[124:127], v22, s[44:47], s90 offen sc0 sc1\n"
    "v_mfma_i32_16x16x64_i8 v[136:139], a[12:15], v[60:63], v[136:139]\n"
    "ds_read_b128 v[96:99], v41 offset:10240\n"
    "v_mfma_i32_16x16x64_i8 v[136:139], a[44:47], v[64:67], v[136:139]\n"
    "s_waitcnt lgkmcnt(4)\n"
    "v_mfma_i32_16x16x64_i8 v[136:139], a[20:23], v[68:71], v[136:139]\n"
    "ds_read_b128 v[100:103], v40 offset:18432\n"
    "v_mfma_i32_16x16x64_i8 v[136:139], a[52:55], v[72:75], v[136:139]\n"
    "buffer_store_dwordx4 v[128:131], v22, s[44:47], s90 offen offset:1024 sc0 sc1\n"
    "v_mfma_i32_16x16x64_i8 v[136:139], a[28:31], v[76:79], v[136:139]\n"
    "ds_read_b128 v[104:107], v41 offset:18432\n"
    "s_cmp_eq_u32 s84, 4\n"
    "s_cselect_b32 s91, s85, 0\n"
    "v_mfma_i32_16x16x64_i8 v[136:139], a[60:63], v[80:83], v[136:139]\n"
    "s_cmp_gt_u32 s91, 4\n"
    "s_cselect_b32 s82, 32768, s83\n"
    "s_addc_u32 s84, s84, 0\n"
    "v_mfma_i32_16x16x64_i8 v[132:135], a[16:19], v[68:71], v[132:135]\n"
    "ds_read_b128 v[108:111], v40 offset:26624\n"
    "s_add_u32 m0, s16, 131072\n"
    "v_mfma_i32_16x16x64_i8 v[132:135], a[48:51], v[72:75], v[132:135]\n"
    "s_mov_b32 s34, s82\n"
    "v_mfma_i32_16x16x64_i8 v[132:135], a[24:27], v[76:79], v[132:135]\n"
    "ds_read_b128 v[112:115], v41 offset:26624\n"
    "v_mfma_i32_16x16x64_i8 v[132:135], a[56:59], v[80:83], v[132:135]\n"
    "s_waitcnt lgkmcnt(4)\n"
    "buffer_load_dwordx4 v22, s[40:43], s34 offen sc0 sc1 lds\n"
    "s_cmp_lt_u32 s84, s96\n"
    "s_getpc_b64 s[88:89]\n"
    "s_cbranch_scc1 L_mida%=\n"
    "v_mfma_i32_16x16x64_i8 v[132:135], a[64:67], v[84:87], v[132:135]\n"
    "ds_read_b128 v[52:55], v40 offset:4096\n"
    "v_mfma_i32_16x16x64_i8 v[132:135], a[96:99], v[88:91], v[132:135]\n"
    "v_mfma_i32_16x16x64_i8 v[132:135], a[72:75], v[92:95], v[132:135]\n"
    "ds_read_b128 v[56:59], v41 offset:4096\n"
    "v_mfma_i32_16x16x64_i8 v[132:135], a[104:107], v[96:99], v[132:135]\n"
    "buffer_load_dwordx4 v22, s[40:43], s34 offen offset:1024 sc0 sc1 lds\n"
    "v_mfma_i32_16x16x64_i8 v[136:139], a[68:71], v[84:87], v[136:139]\n"
    "ds_read_b128 v[60:63], v40 offset:12288\n"
    "v_mfma_i32_16x16x64_i8 v[136:139], a[100:103], v[88:91], v[136:139]\n"
    "v_mfma_i32_16x16x64_i8 v[136:139], a[76:79], v[92:95], v[136:139]\n"
    "ds_read_b128 v[64:67], v41 offset:12288\n"
    "v_mfma_i32_16x16x64_i8 v[136:139], a[108:111], v[96:99], v[136:139]\n"
    "s_waitcnt lgkmcnt(4)\n"
    "buffer_load_dwordx4 v22, s[40:43], s34 offen offset:2048 sc0 sc1 lds\n"
    "v_mfma_i32_16x16x64_i8 v[136:139], a[84:87], v[100:103], v[136:139]\n"
    "ds_read_b128 v[68:71], v40 offset:20480\n"
    "v_mfma_i32_16x16x64_i8 v[136:139], a[116:119], v[104:107], v[136:139]\n"
    "v_mfma_i32_16x16x64_i8 v[136:139], a[92:95], v[108:111], v[136:139]\n"
    "ds_read_b128 v[72:75], v41 offset:20480\n"
    "v_mfma_i32_16x16x64_i8 v[136:139], a[124:127], v[112:115], v[136:139]\n"
    "v_mfma_i32_16x16x64_i8 v[132:135], a[80:83], v[100:103], v[132:135]\n"
    "ds_read_b128 v[76:79], v40 offset:28672\n"
    "v_mfma_i32_16x16x64_i8 v[132:135], a[112:115], v[104:107], v[132:135]\n"
    "buffer_load_dwordx4 v22, s[40:43], s34 offen offset:3072 sc0 sc1 lds\n"
    "v_mfma_i32_16x16x64_i8 v[132:135], a[88:91], v[108:111], v[132:135]\n"
    "ds_read_b128 v[80:83], v41 offset:28672\n"
    "v_mfma_i32_16x16x64_i8 v[132:135], a[120:123], v[112:115], v[132:135]\n"
    "s_waitcnt lgkmcnt(4)\n"
    "s_add_u32 m0, m0, 4096\n"
    "v_mfma_i32_16x16x64_i8 v[132:135], a[128:131], v[52:55], v[132:135]\n"
    "ds_read_b128 v[84:87], v40 offset:6144\n"
    "s_add_u32 s34, s34, 4096\n"
    "v_mfma_i32_16x16x64_i8 v[132:135], a[160:163], v[56:59], v[132:135]\n"
    "buffer_load_dwordx4 v22, s[40:43], s34 offen sc0 sc1 lds\n"
    "s_cmp_lt_u32 s84, s96\n"
    "s_getpc_b64 s[88:89]\n"
    "s_cbranch_scc1 L_mida%=\n"
    "v_mfma_i32_16x16x64_i8 v[132:135], a[136:139], v[60:63], v[132:135]\n"
    "ds_read_b128 v[88:91], v41 offset:6144\n"
    "v_mfma_i32_16x16x64_i8 v[132:135], a[168:171], v[64:67], v[132:135]\n"
    "v_mfma_i32_16x16x64_i8 v[136:139], a[132:135], v[52:55], v[136:139]\n"
    "ds_read_b128 v[92:95], v40 offset:14336\n"
    "v_mfma_i32_16x16x64_i8 v[136:139], a[164:167], v[56:59], v[136:139]\n"
    "buffer_load_dwordx4 v22, s[40:43], s34 offen offset:1024 sc0 sc1 lds\n"
    "v_mfma_i32_16x16x64_i8 v[136:139], a[140:143], v[60:63], v[136:139]\n"
    "ds_read_b128 v[96:99], v41 offset:14336\n"
    "v_mfma_i32_16x16x64_i8 v[136:139], a[172:175], v[64:67], v[136:139]\n"
    "s_waitcnt lgkmcnt(4)\n"
    "v_mfma_i32_16x16x64_i8 v[136:139], a[148:151], v[68:71], v[136:139]\n"
    "ds_read_b128 v[100:103], v40 offset:22528\n"
    "v_mfma_i32_16x16x64_i8 v[136:139], a[180:183], v[72:75], v[136:139]\n"
    "buffer_load_dwordx4 v22, s[40:43], s34 offen offset:2048 sc0 sc1 lds\n"
    "v_mfma_i32_16x16x64_i8 v[136:139], a[156:159], v[76:79], v[136:139]\n"
    "ds_read_b128 v[104:107], v41 offset:22528\n"
    "v_mfma_i32_16x16x64_i8 v[136:139], a[188:191], v[80:83], v[136:139]\n"
    "v_mfma_i32_16x16x64_i8 v[132:135], a[144:147], v[68:71], v[132:135]\n"
    "ds_read_b128 v[108:111], v40 offset:30720\n"
    "v_mfma_i32_16x16x64_i8 v[132:135], a[176:179], v[72:75], v[132:135]\n"
    "v_mfma_i32_16x16x64_i8 v[132:135], a[152:155], v[76:79], v[132:135]\n"
    "ds_read_b128 v[112:115], v41 offset:30720\n"
    "v_mfma_i32_16x16x64_i8 v[132:135], a[184:187], v[80:83], v[132:135]\n"
    "s_waitcnt lgkmcnt(4)\n"
    "buffer_load_dwordx4 v22, s[40:43], s34 offen offset:3072 sc0 sc1 lds\n"
    "v_mfma_i32_16x16x64_i8 v[132:135], a[192:195], v[84:87], v[132:135]\n"
    "s_mov_b32 s96, 5\n"
    "v_mfma_i32_16x16x64_i8 v[132:135], a[224:227], v[88:91], v[132:135]\n"
    "v_mfma_i32_16x16x64_i8 v[132:135], a[200:203], v[92:95], v[132:135]\n"
    "s_mov_b32 s95, 3\n"
    "s_and_b32 s91, s84, s25\n"
    "v_mfma_i32_16x16x64_i8 v[132:135], a[232:235], v[96:99], v[132:135]\n"
    "s_cmp_gt_u32 s91, 3\n"
    "s_getpc_b64 s[88:89]\n"
    "s_cbranch_scc0 L_slowa%=\n"
    "s_waitcnt vmcnt(11) lgkmcnt(0)\n"
    "s_barrier\n"
    "v_mfma_i32_16x16x64_i8 v[136:139], a[196:199], v[84:87], v[136:139]\n"
    "ds_read_b128 v[52:55], v40 offset:32768\n"
    "s_waitcnt vmcnt(10)\n"
    "v_mfma_i32_16x16x64_i8 v[136:139], a[228:231], v[88:91], v[136:139]\n"
    "ds_read_b128 v[56:59], v41 offset:32768\n"
    "v_mfma_i32_16x16x64_i8 v[136:139], a[204:207], v[92:95], v[136:139]\n"
    "ds_read_b128 v[60:63], v40 offset:40960\n"
    "v_cmp_le_u32 vcc, s17, v32\n"
    "v_mfma_i32_16x16x64_i8 v[136:139], a[236:239], v[96:99], v[136:139]\n"
    "ds_read_b128 v[64:67], v41 offset:40960\n"
    "v_mfma_i32_16x16x64_i8 v[136:139], a[212:215], v[100:103], v[136:139]\n"
    "ds_read_b128 v[68:71], v40 offset:49152\n"
    "v_mfma_i32_16x16x64_i8 v[136:139], a[244:247], v[104:107], v[136:139]\n"
    "ds_read_b128 v[72:75], v41 offset:49152\n"
    "s_ff0_i32_b32 s85, vcc_lo\n"
    "v_mfma_i32_16x16x64_i8 v[136:139], a[220:223], v[108:111], v[136:139]\n"
    "ds_read_b128 v[76:79], v40 offset:57344\n"
    "v_mfma_i32_16x16x64_i8 v[136:139], a[252:255], v[112:115], v[136:139]\n"
    "ds_read_b128 v[80:83], v41 offset:57344\n"
    "v_mfma_i32_16x16x64_i8 v[132:135], a[208:211], v[100:103], v[132:135]\n"
    "s_max_u32 s85, s85, s86\n"
    "v_mfma_i32_16x16x64_i8 v[132:135], a[240:243], v[104:107], v[132:135]\n"
    "s_min_u32 s91, s85, s96\n"
    "v_mfma_i32_16x16x64_i8 v[132:135], a[216:219], v[108:111], v[132:135]\n"
    "s_cmp_lt_u32 s84, s91\n"
    "s_getpc_b64 s[88:89]\n"
    "s_cbranch_scc1 L_catcha%=\n"
    "v_mfma_i32_16x16x64_i8 v[132:135], a[248:251], v[112:115], v[132:135]\n"
    "s_waitcnt lgkmcnt(4)\n"
    "v_mfma_i32_16x16x64_i8 v[140:143], a[0:3], v[52:55], v[140:143]\n"
    "global_load_dword v32, v24, s[72:73] sc0 sc1\n"
    "ds_read_b128 v[84:87], v40 offset:34816\n"
    "v_mfma_i32_16x16x64_i8 v[140:143], a[32:35], v[56:59], v[140:143]\n"
    "v_mfma_i32_16x16x64_i8 v[140:143], a[8:11], v[60:63], v[140:143]\n"
    "ds_read_b128 v[88:91], v41 offset:34816\n"
    "v_mfma_i32_16x16x64_i8 v[140:143], a[40:43], v[64:67], v[140:143]\n"
    "v_mfma_i32_16x16x64_i8 v[144:147], a[4:7], v[52:55], v[144:147]\n"
    "ds_read_b128 v[92:95], v40 offset:43008\n"
    "v_mfma_i32_16x16x64_i8 v[144:147], a[36:39], v[56:59], v[144:147]\n"
    "global_load_dword v37, v26, s[72:73] sc0 sc1\n"
    "v_mfma_i32_16x16x64_i8 v[144:147], a[12:15], v[60:63], v[144:147]\n"
    "ds_read_b128 v[96:99], v41 offset:43008\n"
    "v_mfma_i32_16x16x64_i8 v[144:147], a[44:47], v[64:67], v[144:147]\n"
    "s_waitcnt lgkmcnt(4)\n"
    "s_add_u32 s90, s64, 65536\n"
    "v_mfma_i32_16x16x64_i8 v[144:147], a[20:23], v[68:71], v[144:147]\n"
    "ds_read_b128 v[100:103], v40 offset:51200\n"
    "v_mfma_i32_16x16x64_i8 v[144:147], a[52:55], v[72:75], v[144:147]\n"
    "buffer_store_dwordx4 v[132:135], v22, s[44:47], s90 offen sc0 sc1\n"
    "v_mfma_i32_16x16x64_i8 v[144:147], a[28:31], v[76:79], v[144:147]\n"
    "ds_read_b128 v[104:107], v41 offset:51200\n"
    "v_mfma_i32_16x16x64_i8 v[144:147], a[60:63], v[80:83], v[144:147]\n"
    "v_mfma_i32_16x16x64_i8 v[140:143], a[16:19], v[68:71], v[140:143]\n"
    "ds_read_b128 v[108:111], v40 offset:59392\n"
    "v_mfma_i32_16x16x64_i8 v[140:143], a[48:51], v[72:75], v[140:143]\n"
    "buffer_store_dwordx4 v[136:139], v22, s[44:47], s90 offen offset:1024 sc0 sc1\n"
    "s_cmp_eq_u32 s84, 5\n"
    "s_cselect_b32 s91, s85, 0\n"
    "v_mfma_i32_16x16x64_i8 v[140:143], a[24:27], v[76:79], v[140:143]\n"
    "ds_read_b128 v[112:115], v41 offset:59392\n"
    "s_cmp_gt_u32 s91, 5\n"
    "s_cselect_b32 s82, 40960, s83\n"
    "s_addc_u32 s84, s84, 0\n"
    "v_mfma_i32_16x16x64_i8 v[140:143], a[56:59], v[80:83], v[140:143]\n"
    "s_waitcnt lgkmcnt(4)\n"
    "s_add_u32 m0, s16, 0\n"
    "s_cmp_lt_u32 s84, s96\n"
    "s_getpc_b64 s[88:89]\n"
    "s_cbranch_scc1 L_mida%=\n"
    "v_mfma_i32_16x16x64_i8 v[140:143], a[64:67], v[84:87], v[140:143]\n"
    "ds_read_b128 v[52:55], v40 offset:36864\n"
    "s_mov_b32 s34, s82\n"
    "v_mfma_i32_16x16x64_i8 v[140:143], a[96:99], v[88:91], v[140:143]\n"
    "buffer_load_dwordx4 v22, s[40:43], s34 offen sc0 sc1 lds\n"
    "v_mfma_i32_16x16x64_i8 v[140:143], a[72:75], v[92:95], v[140:143]\n"
    "ds_read_b128 v[56:59], v41 offset:36864\n"
    "v_mfma_i32_16x16x64_i8 v[140:143], a[104:107], v[96:99], v[140:143]\n"
    "v_mfma_i32_16x16x64_i8 v[144:147], a[68:71], v[84:87], v[144:147]\n"
    "ds_read_b128 v[60:63], v40 offset:45056\n"
    "v_mfma_i32_16x16x64_i8 v[144:147], a[100:103], v[88:91], v[144:147]\n"
    "buffer_load_dwordx4 v22, s[40:43], s34 offen offset:1024 sc0 sc1 lds\n"
    "v_mfma_i32_16x16x64_i8 v[144:147], a[76:79], v[92:95], v[144:147]\n"
    "ds_read_b128 v[64:67], v41 offset:45056\n"
    "v_mfma_i32_16x16x64_i8 v[144:147], a[108:111], v[96:99], v[144:147]\n"
    "s_waitcnt lgkmcnt(4)\n"
    "v_mfma_i32_16x16x64_i8 v[144:147], a[84:87], v[100:103], v[144:147]\n"
    "ds_read_b128 v[68:71], v40 offset:53248\n"
    "v_mfma_i32_16x16x64_i8 v[144:147], a[116:119], v[104:107], v[144:147]\n"
    "buffer_load_dwordx4 v22, s[40:43], s34 offen offset:2048 sc0 sc1 lds\n"
    "v_mfma_i32_16x16x64_i8 v[144:147], a[92:95], v[108:111], v[144:147]\n"
    "ds_read_b128 v[72:75], v41 offset:53248\n"
    "v_mfma_i32_16x16x64_i8 v[144:147], a[124:127], v[112:115], v[144:147]\n"
    "v_mfma_i32_16x16x64_i8 v[140:143], a[80:83], v[100:103], v[140:143]\n"
    "ds_read_b128 v[76:79], v40 offset:61440\n"
    "v_mfma_i32_16x16x64_i8 v[140:143], a[112:115], v[104:107], v[140:143]\n"
    "v_mfma_i32_16x16x64_i8 v[140:143], a[88:91], v[108:111], v[140:143]\n"
    "ds_read_b128 v[80:83], v41 offset:61440\n"
    "v_mfma_i32_16x16x64_i8 v[140:143], a[120:123], v[112:115], v[140:143]\n"
    "s_waitcnt lgkmcnt(4)\n"
    "buffer_load_dwordx4 v22, s[40:43], s34 offen offset:3072 sc0 sc1 lds\n"
    "v_mfma_i32_16x16x64_i8 v[140:143], a[128:131], v[52:55], v[140:143]\n"
    "ds_read_b128 v[84:87], v40 offset:38912\n"
    "v_mfma_i32_16x16x64_i8 v[140:143], a[160:163], v[56:59], v[140:143]\n"
    "s_add_u32 m0, m0, 4096\n"
    "s_add_u32 s34, s34, 4096\n"
    "s_cmp_lt_u32 s84, s96\n"
    "s_getpc_b64 s[88:89]\n"
    "s_cbranch_scc1 L_mida%=\n"
    "v_mfma_i32_16x16x64_i8 v[140:143], a[136:139], v[60:63], v[140:143]\n"
    "ds_read_b128 v[88:91], v41 offset:38912\n"
    "v_mfma_i32_16x16x64_i8 v[140:143], a[168:171], v[64:67], v[140:143]\n"
    "buffer_load_dwordx4 v22, s[40:43], s34 offen sc0 sc1 lds\n"
    "v_mfma_i32_16x16x64_i8 v[144:147], a[132:135], v[52:55], v[144:147]\n"
    "ds_read_b128 v[92:95], v40 offset:47104\n"
    "v_mfma_i32_16x16x64_i8 v[144:147], a[164:167], v[56:59], v[144:147]\n"
    "v_mfma_i32_16x16x64_i8 v[144:147], a[140:143], v[60:63], v[144:147]\n"
    "ds_read_b128 v[96:99], v41 offset:47104\n"
    "v_mfma_i32_16x16x64_i8 v[144:147], a[172:175], v[64:67], v[144:147]\n"
    "s_waitcnt lgkmcnt(4)\n"
    "buffer_load_dwordx4 v22, s[40:43], s34 offen offset:1024 sc0 sc1 lds\n"
    "v_mfma_i32_16x16x64_i8 v[144:147], a[148:151], v[68:71], v[144:147]\n"
    "ds_read_b128 v[100:103], v40 offset:55296\n"
    "v_mfma_i32_16x16x64_i8 v[144:147], a[180:183], v[72:75], v[144:147]\n"
    "v_mfma_i32_16x16x64_i8 v[144:147], a[156:159], v[76:79], v[144:147]\n"
    "ds_read_b128 v[104:107], v41 offset:55296\n"
    "v_mfma_i32_16x16x64_i8 v[144:147], a[188:191], v[80:83], v[144:147]\n"
    "buffer_load_dwordx4 v22, s[40:43], s34 offen offset:2048 sc0 sc1 lds\n"
    "v_mfma_i32_16x16x64_i8 v[140:143], a[144:147], v[68:71], v[140:143]\n"
    "ds_read_b128 v[108:111], v40 offset:63488\n"
    "v_mfma_i32_16x16x64_i8 v[140:143], a[176:179], v[72:75], v[140:143]\n"
    "v_mfma_i32_16x16x64_i8 v[140:143], a[152:155], v[76:79], v[140:143]\n"
    "ds_read_b128 v[112:115], v41 offset:63488\n"
    "v_mfma_i32_16x16x64_i8 v[140:143], a[184:187], v[80:83], v[140:143]\n"
    "s_waitcnt lgkmcnt(4)\n"
    "buffer_load_dwordx4 v22, s[40:43], s34 offen offset:3072 sc0 sc1 lds\n"
    "v_mfma_i32_16x16x64_i8 v[140:143], a[192:195], v[84:87], v[140:143]\n"
    "s_mov_b32 s96, 6\n"
    "v_mfma_i32_16x16x64_i8 v[140:143], a[224:227], v[88:91], v[140:143]\n"
    "v_mfma_i32_16x16x64_i8 v[140:143], a[200:203], v[92:95], v[140:143]\n"
    "s_mov_b32 s95, 4\n"
    "s_and_b32 s91, s84, s25\n"
    "v_mfma_i32_16x16x64_i8 v[140:143], a[232:235], v[96:99], v[140:143]\n"
    "s_cmp_gt_u32 s91, 4\n"
    "s_getpc_b64 s[88:89]\n"
    "s_cbranch_scc0 L_slowa%=\n"
    "s_waitcnt vmcnt(12) lgkmcnt(0)\n"
    "s_barrier\n"
    "v_mfma_i32_16x16x64_i8 v[144:147], a[196:199], v[84:87], v[144:147]\n"
    "ds_read_b128 v[52:55], v42\n"
    "s_waitcnt vmcnt(11)\n"
    "v_mfma_i32_16x16x64_i8 v[144:147], a[228:231], v[88:91], v[144:147]\n"
    "ds_read_b128 v[56:59], v43\n"
    "v_cmp_le_u32 vcc, s17, v32\n"
    "v_mfma_i32_16x16x64_i8 v[144:147], a[204:207], v[92:95], v[144:147]\n"
    "ds_read_b128 v[60:63], v42 offset:8192\n"
    "v_mfma_i32_16x16x64_i8 v[144:147], a[236:239], v[96:99], v[144:147]\n"
    "ds_read_b128 v[64:67], v43 offset:8192\n"
    "s_ff0_i32_b32 s85, vcc_lo\n"
    "v_mfma_i32_16x16x64_i8 v[144:147], a[212:215], v[100:103], v[144:147]\n"
    "ds_read_b128 v[68:71], v42 offset:16384\n"
    "v_mfma_i32_16x16x64_i8 v[144:147], a[244:247], v[104:107], v[144:147]\n"
    "ds_read_b128 v[72:75], v43 offset:16384\n"
    "s_max_u32 s85, s85, s86\n"
    "v_mfma_i32_16x16x64_i8 v[144:147], a[220:223], v[108:111], v[144:147]\n"
    "ds_read_b128 v[76:79], v42 offset:24576\n"
    "v_mfma_i32_16x16x64_i8 v[144:147], a[252:255], v[112:115], v[144:147]\n"
    "ds_read_b128 v[80:83], v43 offset:24576\n"
    "s_min_u32 s91, s85, s96\n"
    "v_mfma_i32_16x16x64_i8 v[140:143], a[208:211], v[100:103], v[140:143]\n"
    "s_cmp_lt_u32 s84, s91\n"
    "s_getpc_b64 s[88:89]\n"
    "s_cbranch_scc1 L_catcha%=\n"
    "v_mfma_i32_16x16x64_i8 v[140:143], a[240:243], v[104:107], v[140:143]\n"
    "s_waitcnt vmcnt(10)\n"
    "v_mfma_i32_16x16x64_i8 v[140:143], a[216:219], v[108:111], v[140:143]\n"
    "v_cmp_gt_u32 s[28:29], s17, v37\n"
    "v_mfma_i32_16x16x64_i8 v[140:143], a[248:251], v[112:115], v[140:143]\n"
    "s_cmp_lg_u64 s[28:29], 0\n"
    "s_cbranch_scc1 L_gatea%=\n"
    "L_rga%=:\n"
    "s_waitcnt lgkmcnt(4)\n"
    "v_mfma_i32_16x16x64_i8 v[148:151], a[0:3], v[52:55], v[148:151]\n"
    "global_load_dword v32, v24, s[72:73] sc0 sc1\n"
    "ds_read_b128 v[84:87], v42 offset:2048\n"
    "v_mfma_i32_16x16x64_i8 v[148:151], a[32:35], v[56:59], v[148:151]\n"
    "s_add_u32 s90, s65, 0\n"
    "buffer_load_dwordx4 v[204:207], v22, s[44:47], s90 offen sc0 sc1\n"
    "v_mfma_i32_16x16x64_i8 v[148:151], a[8:11], v[60:63], v[148:151]\n"
    "ds_read_b128 v[88:91], v43 offset:2048\n"
    "v_mfma_i32_16x16x64_i8 v[148:151], a[40:43], v[64:67], v[148:151]\n"
    "v_mfma_i32_16x16x64_i8 v[152:155], a[4:7], v[52:55], v[152:155]\n"
    "ds_read_b128 v[92:95], v42 offset:10240\n"
    "v_mfma_i32_16x16x64_i8 v[152:155], a[36:39], v[56:59], v[152:155]\n"
    "buffer_load_dwordx4 v[208:211], v22, s[44:47], s90 offen offset:1024 sc0 sc1\n"
    "v_mfma_i32_16x16x64_i8 v[152:155], a[12:15], v[60:63], v[152:155]\n"
    "ds_read_b128 v[96:99], v43 offset:10240\n"
    "v_mfma_i32_16x16x64_i8 v[152:155], a[44:47], v[64:67], v[152:155]\n"
    "s_waitcnt lgkmcnt(4)\n"
    "s_add_u32 s90, s64, 98304\n"
    "v_mfma_i32_16x16x64_i8 v[152:155], a[20:23], v[68:71], v[152:155]\n"
    "ds_read_b128 v[100:103], v42 offset:18432\n"
    "v_mfma_i32_16x16x64_i8 v[152:155], a[52:55], v[72:75], v[152:155]\n"
    "buffer_store_dwordx4 v[140:143], v22, s[44:47], s90 offen sc0 sc1\n"
    "v_mfma_i32_16x16x64_i8 v[152:155], a[28:31], v[76:79], v[152:155]\n"
    "ds_read_b128 v[104:107], v43 offset:18432\n"
    "v_mfma_i32_16x16x64_i8 v[152:155], a[60:63], v[80:83], v[152:155]\n"
    "v_mfma_i32_16x16x64_i8 v[148:151], a[16:19], v[68:71], v[148:151]\n"
    "ds_read_b128 v[108:111], v42 offset:26624\n"
    "v_mfma_i32_16x16x64_i8 v[148:151], a[48:51], v[72:75], v[148:151]\n"
    "buffer_store_dwordx4 v[144:147], v22, s[44:47], s90 offen offset:1024 sc0 sc1\n"
    "s_cmp_eq_u32 s84, 6\n"
    "s_cselect_b32 s91, s85, 0\n"
    "v_mfma_i32_16x16x64_i8 v[148:151], a[24:27], v[76:79], v[148:151]\n"
    "ds_read_b128 v[112:115], v43 offset:26624\n"
    "s_cmp_gt_u32 s91, 6\n"
    "s_cselect_b32 s82, 49152, s83\n"
    "s_addc_u32 s84, s84, 0\n"
    "v_mfma_i32_16x16x64_i8 v[148:151], a[56:59], v[80:83], v[148:151]\n"
    "s_waitcnt lgkmcnt(4)\n"
    "s_add_u32 m0, s16, 32768\n"
    "s_cmp_lt_u32 s84, s96\n"
    "s_getpc_b64 s[88:89]\n"
    "s_cbranch_scc1 L_mida%=\n"
    "v_mfma_i32_16x16x64_i8 v[148:151], a[64:67], v[84:87], v[148:151]\n"
    "ds_read_b128 v[52:55], v42 offset:4096\n"
    "s_mov_b32 s34, s82\n"
    "v_mfma_i32_16x16x64_i8 v[148:151], a[96:99], v[88:91], v[148:151]\n"
    "buffer_load_dwordx4 v22, s[40:43], s34 offen sc0 sc1 lds\n"
    "v_mfma_i32_16x16x64_i8 v[148:151], a[72:75], v[92:95], v[148:151]\n"
    "ds_read_b128 v[56:59], v43 offset:4096\n"
    "v_mfma_i32_16x16x64_i8 v[148:151], a[104:107], v[96:99], v[148:151]\n"
    "v_mfma_i32_16x16x64_i8 v[152:155], a[68:71], v[84:87], v[152:155]\n"
    "ds_read_b128 v[60:63], v42 offset:12288\n"
    "v_mfma_i32_16x16x64_i8 v[152:155], a[100:103], v[88:91], v[152:155]\n"
    "buffer_load_dwordx4 v22, s[40:43], s34 offen offset:1024 sc0 sc1 lds\n"
    "v_mfma_i32_16x16x64_i8 v[152:155], a[76:79], v[92:95], v[152:155]\n"
    "ds_read_b128 v[64:67], v43 offset:12288\n"
    "v_mfma_i32_16x16x64_i8 v[152:155], a[108:111], v[96:99], v[152:155]\n"
    "s_waitcnt lgkmcnt(4)\n"
    "v_mfma_i32_16x16x64_i8 v[152:155], a[84:87], v[100:103], v[152:155]\n"
    "ds_read_b128 v[68:71], v42 offset:20480\n"
    "v_mfma_i32_16x16x64_i8 v[152:155], a[116:119], v[104:107], v[152:155]\n"
    "v_mfma_i32_16x16x64_i8 v[152:155], a[92:95], v[108:111], v[152:155]\n"
    "ds_read_b128 v[72:75], v43 offset:20480\n"
    "v_mfma_i32_16x16x64_i8 v[152:155], a[124:127], v[112:115], v[152:155]\n"
    "buffer_load_dwordx4 v22, s[40:43], s34 offen offset:2048 sc0 sc1 lds\n"
    "v_mfma_i32_16x16x64_i8 v[148:151], a[80:83], v[100:103], v[148:151]\n"
    "ds_read_b128 v[76:79], v42 offset:28672\n"
    "v_mfma_i32_16x16x64_i8 v[148:151], a[112:115], v[104:107], v[148:151]\n"
    "v_mfma_i32_16x16x64_i8 v[148:151], a[88:91], v[108:111], v[148:151]\n"
    "ds_read_b128 v[80:83], v43 offset:28672\n"
    "v_mfma_i32_16x16x64_i8 v[148:151], a[120:123], v[112:115], v[148:151]\n"
    "s_waitcnt lgkmcnt(4)\n"
    "buffer_load_dwordx4 v22, s[40:43], s34 offen offset:3072 sc0 sc1 lds\n"
    "v_mfma_i32_16x16x64_i8 v[148:151], a[128:131], v[52:55], v[148:151]\n"
    "ds_read_b128 v[84:87], v42 offset:6144\n"
    "v_mfma_i32_16x16x64_i8 v[148:151], a[160:163], v[56:59], v[148:151]\n"
    "s_add_u32 m0, m0, 4096\n"
    "s_add_u32 s34, s34, 4096\n"
    "s_cmp_lt_u32 s84, s96\n"
    "s_getpc_b64 s[88:89]\n"
    "s_cbranch_scc1 L_mida%=\n"
    "v_mfma_i32_16x16x64_i8 v[148:151], a[136:139], v[60:63], v[148:151]\n"
    "ds_read_b128 v[88:91], v43 offset:6144\n"
    "v_mfma_i32_16x16x64_i8 v[148:151], a[168:171], v[64:67], v[148:151]\n"
    "buffer_load_dwordx4 v22, s[40:43], s34 offen sc0 sc1 lds\n"
    "v_mfma_i32_16x16x64_i8 v[152:155], a[132:135], v[52:55], v[152:155]\n"
    "ds_read_b128 v[92:95], v42 offset:14336\n"
    "v_mfma_i32_16x16x64_i8 v[152:155], a[164:167], v[56:59], v[152:155]\n"
    "v_mfma_i32_16x16x64_i8 v[152:155], a[140:143], v[60:63], v[152:155]\n"
    "ds_read_b128 v[96:99], v43 offset:14336\n"
    "v_mfma_i32_16x16x64_i8 v[152:155], a[172:175], v[64:67], v[152:155]\n"
    "s_waitcnt lgkmcnt(4)\n"
    "buffer_load_dwordx4 v22, s[40:43], s34 offen offset:1024 sc0 sc1 lds\n"
    "v_mfma_i32_16x16x64_i8 v[152:155], a[148:151], v[68:71], v[152:155]\n"
    "ds_read_b128 v[100:103], v42 offset:22528\n"
    "v_mfma_i32_16x16x64_i8 v[152:155], a[180:183], v[72:75], v[152:155]\n"
    "v_mfma_i32_16x16x64_i8 v[152:155], a[156:159], v[76:79], v[152:155]\n"
    "ds_read_b128 v[104:107], v43 offset:22528\n"
    "v_mfma_i32_16x16x64_i8 v[152:155], a[188:191], v[80:83], v[152:155]\n"
    "buffer_load_dwordx4 v22, s[40:43], s34 offen offset:2048 sc0 sc1 lds\n"
    "v_mfma_i32_16x16x64_i8 v[148:151], a[144:147], v[68:71], v[148:151]\n"
    "ds_read_b128 v[108:111], v42 offset:30720\n"
    "v_mfma_i32_16x16x64_i8 v[148:151], a[176:179], v[72:75], v[148:151]\n"
    "v_mfma_i32_16x16x64_i8 v[148:151], a[152:155], v[76:79], v[148:151]\n"
    "ds_read_b128 v[112:115], v43 offset:30720\n"
    "v_mfma_i32_16x16x64_i8 v[148:151], a[184:187], v[80:83], v[148:151]\n"
    "s_waitcnt lgkmcnt(4)\n"
    "buffer_load_dwordx4 v22, s[40:43], s34 offen offset:3072 sc0 sc1 lds\n"
    "v_mfma_i32_16x16x64_i8 v[148:151], a[192:195], v[84:87], v[148:151]\n"
    "s_mov_b32 s96, 7\n"
    "v_mfma_i32_16x16x64_i8 v[148:151], a[224:227], v[88:91], v[148:151]\n"
    "v_mfma_i32_16x16x64_i8 v[148:151], a[200:203], v[92:95], v[148:151]\n"
    "s_mov_b32 s95, 5\n"
    "s_and_b32 s91, s84, s25\n"
    "v_mfma_i32_16x16x64_i8 v[148:151], a[232:235], v[96:99], v[148:151]\n"
    "s_cmp_gt_u32 s91, 5\n"
    "s_getpc_b64 s[88:89]\n"
    "s_cbranch_scc0 L_slowa%=\n"
    "s_waitcnt vmcnt(13) lgkmcnt(0)\n"
    "s_barrier\n"
    "v_mfma_i32_16x16x64_i8 v[152:155], a[196:199], v[84:87], v[152:155]\n"
    "ds_read_b128 v[52:55], v38\n"
    "s_waitcnt vmcnt(12)\n"
    "v_mfma_i32_16x16x64_i8 v[152:155], a[228:231], v[88:91], v[152:155]\n"
    "ds_read_b128 v[56:59], v39\n"
    "v_mfma_i32_16x16x64_i8 v[152:155], a[204:207], v[92:95], v[152:155]\n"
    "ds_read_b128 v[60:63], v38 offset:8192\n"
    "v_cmp_le_u32 vcc, s17, v32\n"
    "v_mfma_i32_16x16x64_i8 v[152:155], a[236:239], v[96:99], v[152:155]\n"
    "ds_read_b128 v[64:67], v39 offset:8192\n"
    "v_mfma_i32_16x16x64_i8 v[152:155], a[212:215], v[100:103], v[152:155]\n"
    "ds_read_b128 v[68:71], v38 offset:16384\n"
    "v_mfma_i32_16x16x64_i8 v[152:155], a[244:247], v[104:107], v[152:155]\n"
    "ds_read_b128 v[72:75], v39 offset:16384\n"
    "s_ff0_i32_b32 s85, vcc_lo\n"
    "v_mfma_i32_16x16x64_i8 v[152:155], a[220:223], v[108:111], v[152:155]\n"
    "ds_read_b128 v[76:79], v38 offset:24576\n"
    "v_mfma_i32_16x16x64_i8 v[152:155], a[252:255], v[112:115], v[152:155]\n"
    "ds_read_b128 v[80:83], v39 offset:24576\n"
    "v_mfma_i32_16x16x64_i8 v[148:151], a[208:211], v[100:103], v[148:151]\n"
    "s_max_u32 s85, s85, s86\n"
    "v_mfma_i32_16x16x64_i8 v[148:151], a[240:243], v[104:107], v[148:151]\n"
    "s_min_u32 s91, s85, s96\n"
    "v_mfma_i32_16x16x64_i8 v[148:151], a[216:219], v[108:111], v[148:151]\n"
    "s_cmp_lt_u32 s84, s91\n"
    "s_getpc_b64 s[88:89]\n"
    "s_cbranch_scc1 L_catcha%=\n"
    "v_mfma_i32_16x16x64_i8 v[148:151], a[248:251], v[112:115], v[148:151]\n"
    "s_waitcnt lgkmcnt(4)\n"
    "v_mfma_i32_16x16x64_i8 v[156:159], a[0:3], v[52:55], v[156:159]\n"
    "global_load_dword v32, v24, s[72:73] sc0 sc1\n"
    "ds_read_b128 v[84:87], v38 offset:2048\n"
    "v_mfma_i32_16x16x64_i8 v[156:159], a[32:35], v[56:59], v[156:159]\n"
    "s_add_u32 s90, s65, 32768\n"
    "buffer_load_dwordx4 v[212:215], v22, s[44:47], s90 offen sc0 sc1\n"
    "v_mfma_i32_16x16x64_i8 v[156:159], a[8:11], v[60:63], v[156:159]\n"
    "ds_read_b128 v[88:91], v39 offset:2048\n"
    "v_mfma_i32_16x16x64_i8 v[156:159], a[40:43], v[64:67], v[156:159]\n"
    "v_mfma_i32_16x16x64_i8 v[160:163], a[4:7], v[52:55], v[160:163]\n"
    "ds_read_b128 v[92:95], v38 offset:10240\n"
    "v_mfma_i32_16x16x64_i8 v[160:163], a[36:39], v[56:59], v[160:163]\n"
    "s_waitcnt vmcnt(12)\n"
    "v_add_u32 v204, v148, v204\n"
    "v_mfma_i32_16x16x64_i8 v[160:163], a[12:15], v[60:63], v[160:163]\n"
    "ds_read_b128 v[96:99], v39 offset:10240\n"
    "v_add_u32 v205, v149, v205\n"
    "v_mfma_i32_16x16x64_i8 v[160:163], a[44:47], v[64:67], v[160:163]\n"
    "s_waitcnt lgkmcnt(4)\n"
    "buffer_load_dwordx4 v[216:219], v22, s[44:47], s90 offen offset:1024 sc0 sc1\n"
    "v_mfma_i32_16x16x64_i8 v[160:163], a[20:23], v[68:71], v[160:163]\n"
    "ds_read_b128 v[100:103], v38 offset:18432\n"
    "s_cmp_eq_u32 s84, 7\n"
    "s_cselect_b32 s91, s85, 0\n"
    "v_mfma_i32_16x16x64_i8 v[160:163], a[52:55], v[72:75], v[160:163]\n"
    "s_cmp_gt_u32 s91, 7\n"
    "s_cselect_b32 s82, 57344, s83\n"
    "s_addc_u32 s84, s84, 0\n"
    "s_add_u32 m0, s16, 65536\n"
    "v_mfma_i32_16x16x64_i8 v[160:163], a[28:31], v[76:79], v[160:163]\n"
    "ds_read_b128 v[104:107], v39 offset:18432\n"
    "s_mov_b32 s34, s82\n"
    "v_mfma_i32_16x16x64_i8 v[160:163], a[60:63], v[80:83], v[160:163]\n"
    "buffer_load_dwordx4 v22, s[40:43], s34 offen sc0 sc1 lds\n"
    "v_mfma_i32_16x16x64_i8 v[156:159], a[16:19], v[68:71], v[156:159]\n"
    "ds_read_b128 v[108:111], v38 offset:26624\n"
    "v_add_u32 v206, v150, v206\n"
    "v_mfma_i32_16x16x64_i8 v[156:159], a[48:51], v[72:75], v[156:159]\n"
    "v_add_u32 v207, v151, v207\n"
    "v_add_u32 v208, v152, v208\n"
    "v_mfma_i32_16x16x64_i8 v[156:159], a[24:27], v[76:79], v[156:159]\n"
    "ds_read_b128 v[112:115], v39 offset:26624\n"
    "v_mfma_i32_16x16x64_i8 v[156:159], a[56:59], v[80:83], v[156:159]\n"
    "s_waitcnt lgkmcnt(4)\n"
    "s_cmp_lt_u32 s84, s96\n"
    "s_getpc_b64 s[88:89]\n"
    "s_cbranch_scc1 L_mida%=\n"
    "v_mfma_i32_16x16x64_i8 v[156:159], a[64:67], v[84:87], v[156:159]\n"
    "ds_read_b128 v[52:55], v38 offset:4096\n"
    "v_add_u32 v209, v153, v209\n"
    "v_mfma_i32_16x16x64_i8 v[156:159], a[96:99], v[88:91], v[156:159]\n"
    "buffer_load_dwordx4 v22, s[40:43], s34 offen offset:1024 sc0 sc1 lds\n"
    "v_add_u32 v210, v154, v210\n"
    "v_mfma_i32_16x16x64_i8 v[156:159], a[72:75], v[92:95], v[156:159]\n"
    "ds_read_b128 v[56:59], v39 offset:4096\n"
    "v_add_u32 v211, v155, v211\n"
    "v_mfma_i32_16x16x64_i8 v[156:159], a[104:107], v[96:99], v[156:159]\n"
    "v_cvt_f32_i32 v204, v204\n"
    "v_mfma_i32_16x16x64_i8 v[160:163], a[68:71], v[84:87], v[160:163]\n"
    "ds_read_b128 v[60:63], v38 offset:12288\n"
    "v_cvt_f32_i32 v205, v205\n"
    "v_mfma_i32_16x16x64_i8 v[160:163], a[100:103], v[88:91], v[160:163]\n"
    "buffer_load_dwordx4 v22, s[40:43], s34 offen offset:2048 sc0 sc1 lds\n"
    "v_cvt_f32_i32 v206, v206\n"
    "v_mfma_i32_16x16x64_i8 v[160:163], a[76:79], v[92:95], v[160:163]\n"
    "ds_read_b128 v[64:67], v39 offset:12288\n"
    "v_mfma_i32_16x16x64_i8 v[160:163], a[108:111], v[96:99], v[160:163]\n"
    "s_waitcnt lgkmcnt(4)\n"
    "v_cvt_f32_i32 v207, v207\n"
    "v_mfma_i32_16x16x64_i8 v[160:163], a[84:87], v[100:103], v[160:163]\n"
    "ds_read_b128 v[68:71], v38 offset:20480\n"
    "v_cvt_f32_i32 v208, v208\n"
    "v_mfma_i32_16x16x64_i8 v[160:163], a[116:119], v[104:107], v[160:163]\n"
    "v_cvt_f32_i32 v209, v209\n"
    "v_mfma_i32_16x16x64_i8 v[160:163], a[92:95], v[108:111], v[160:163]\n"
    "ds_read_b128 v[72:75], v39 offset:20480\n"
    "v_cvt_f32_i32 v210, v210\n"
    "v_mfma_i32_16x16x64_i8 v[160:163], a[124:127], v[112:115], v[160:163]\n"
    "buffer_load_dwordx4 v22, s[40:43], s34 offen offset:3072 sc0 sc1 lds\n"
    "v_cvt_f32_i32 v211, v211\n"
    "v_mfma_i32_16x16x64_i8 v[156:159], a[80:83], v[100:103], v[156:159]\n"
    "ds_read_b128 v[76:79], v38 offset:28672\n"
    "v_mfma_i32_16x16x64_i8 v[156:159], a[112:115], v[104:107], v[156:159]\n"
    "s_add_u32 m0, m0, 4096\n"
    "s_add_u32 s34, s34, 4096\n"
    "v_mfma_i32_16x16x64_i8 v[156:159], a[88:91], v[108:111], v[156:159]\n"
    "ds_read_b128 v[80:83], v39 offset:28672\n"
    "v_max3_f32 v34, |v204|, |v205|, |v206|\n"
    "v_mfma_i32_16x16x64_i8 v[156:159], a[120:123], v[112:115], v[156:159]\n"
    "s_waitcnt lgkmcnt(4)\n"
    "buffer_load_dwordx4 v22, s[40:43], s34 offen sc0 sc1 lds\n"
    "v_mfma_i32_16x16x64_i8 v[156:159], a[128:131], v[52:55], v[156:159]\n"
    "ds_read_b128 v[84:87], v38 offset:6144\n"
    "v_max3_f32 v34, v34, |v207|, |v208|\n"
    "v_mfma_i32_16x16x64_i8 v[156:159], a[160:163], v[56:59], v[156:159]\n"
    "s_cmp_lt_u32 s84, s96\n"
    "s_getpc_b64 s[88:89]\n"
    "s_cbranch_scc1 L_mida%=\n"
    "v_mfma_i32_16x16x64_i8 v[156:159], a[136:139], v[60:63], v[156:159]\n"
    "ds_read_b128 v[88:91], v39 offset:6144\n"
    "v_max3_f32 v34, v34, |v209|, |v210|\n"
    "v_mfma_i32_16x16x64_i8 v[156:159], a[168:171], v[64:67], v[156:159]\n"
    "v_max_f32 v34, v34, |v211|\n"
    "v_mfma_i32_16x16x64_i8 v[160:163], a[132:135], v[52:55], v[160:163]\n"
    "ds_read_b128 v[92:95], v38 offset:14336\n"
    "v_cmp_lt_f32 s[28:29], s97, v34\n"
    "v_mfma_i32_16x16x64_i8 v[160:163], a[164:167], v[56:59], v[160:163]\n"
    "buffer_load_dwordx4 v22, s[40:43], s34 offen offset:1024 sc0 sc1 lds\n"
    "s_cmp_lg_u64 s[28:29], 0\n"
    "s_cselect_b32 s24, 16, 0\n"
    "s_or_b32 s81, s81, s24\n"
    "v_mfma_i32_16x16x64_i8 v[160:163], a[140:143], v[60:63], v[160:163]\n"
    "ds_read_b128 v[96:99], v39 offset:14336\n"
    "v_mfma_i32_16x16x64_i8 v[160:163], a[172:175], v[64:67], v[160:163]\n"
    "s_waitcnt lgkmcnt(4)\n"
    "v_fma_f32 v204, v204, s18, v33\n"
    "v_mfma_i32_16x16x64_i8 v[160:163], a[148:151], v[68:71], v[160:163]\n"
    "ds_read_b128 v[100:103], v38 offset:22528\n"
    "v_fma_f32 v205, v205, s18, v33\n"
    "v_mfma_i32_16x16x64_i8 v[160:163], a[180:183], v[72:75], v[160:163]\n"
    "buffer_load_dwordx4 v22, s[40:43], s34 offen offset:2048 sc0 sc1 lds\n"
    "v_fma_f32 v206, v206, s18, v33\n"
    "v_mfma_i32_16x16x64_i8 v[160:163], a[156:159], v[76:79], v[160:163]\n"
    "ds_read_b128 v[104:107], v39 offset:22528\n"
    "v_mfma_i32_16x16x64_i8 v[160:163], a[188:191], v[80:83], v[160:163]\n"
    "v_fma_f32 v207, v207, s18, v33\n"
    "v_fma_f32 v208, v208, s18, v33\n"
    "v_mfma_i32_16x16x64_i8 v[156:159], a[144:147], v[68:71], v[156:159]\n"
    "ds_read_b128 v[108:111], v38 offset:30720\n"
    "v_mfma_i32_16x16x64_i8 v[156:159], a[176:179], v[72:75], v[156:159]\n"
    "v_fma_f32 v209, v209, s18, v33\n"
    "v_fma_f32 v210, v210, s18, v33\n"
    "v_mfma_i32_16x16x64_i8 v[156:159], a[152:155], v[76:79], v[156:159]\n"
    "ds_read_b128 v[112:115], v39 offset:30720\n"
    "v_fma_f32 v211, v211, s18, v33\n"
    "v_mfma_i32_16x16x64_i8 v[156:159], a[184:187], v[80:83], v[156:159]\n"
    "s_waitcnt lgkmcnt(4)\n"
    "buffer_load_dwordx4 v22, s[40:43], s34 offen offset:3072 sc0 sc1 lds\n"
    "v_mfma_i32_16x16x64_i8 v[156:159], a[192:195], v[84:87], v[156:159]\n"
    "s_mov_b32 s96, 8\n"
    "v_mfma_i32_16x16x64_i8 v[156:159], a[224:227], v[88:91], v[156:159]\n"
    "v_max_f32 v204, v204, v33\n"
    "v_max_f32 v205, v205, v33\n"
    "v_mfma_i32_16x16x64_i8 v[156:159], a[200:203], v[92:95], v[156:159]\n"
    "s_mov_b32 s95, 6\n"
    "s_and_b32 s91, s84, s25\n"
    "v_mfma_i32_16x16x64_i8 v[156:159], a[232:235], v[96:99], v[156:159]\n"
    "s_cmp_gt_u32 s91, 6\n"
    "s_getpc_b64 s[88:89]\n"
    "s_cbranch_scc0 L_slowa%=\n"
    "s_waitcnt vmcnt(11) lgkmcnt(0)\n"
    "s_barrier\n"
    "v_mfma_i32_16x16x64_i8 v[160:163], a[196:199], v[84:87], v[160:163]\n"
    "ds_read_b128 v[52:55], v38 offset:32768\n"
    "s_waitcnt vmcnt(10)\n"
    "v_mfma_i32_16x16x64_i8 v[160:163], a[228:231], v[88:91], v[160:163]\n"
    "ds_read_b128 v[56:59], v39 offset:32768\n"
    "v_max_f32 v206, v206, v33\n"
    "v_mfma_i32_16x16x64_i8 v[160:163], a[204:207], v[92:95], v[160:163]\n"
    "ds_read_b128 v[60:63], v38 offset:40960\n"
    "v_cmp_le_u32 vcc, s17, v32\n"
    "v_mfma_i32_16x16x64_i8 v[160:163], a[236:239], v[96:99], v[160:163]\n"
    "ds_read_b128 v[64:67], v39 offset:40960\n"
    "v_mfma_i32_16x16x64_i8 v[160:163], a[212:215], v[100:103], v[160:163]\n"
    "ds_read_b128 v[68:71], v38 offset:49152\n"
    "v_max_f32 v207, v207, v33\n"
    "v_mfma_i32_16x16x64_i8 v[160:163], a[244:247], v[104:107], v[160:163]\n"
    "ds_read_b128 v[72:75], v39 offset:49152\n"
    "s_ff0_i32_b32 s85, vcc_lo\n"
    "v_mfma_i32_16x16x64_i8 v[160:163], a[220:223], v[108:111], v[160:163]\n"
    "ds_read_b128 v[76:79], v38 offset:57344\n"
    "v_max_f32 v208, v208, v33\n"
    "v_mfma_i32_16x16x64_i8 v[160:163], a[252:255], v[112:115], v[160:163]\n"
    "ds_read_b128 v[80:83], v39 offset:57344\n"
    "v_max_f32 v209, v209, v33\n"
    "v_mfma_i32_16x16x64_i8 v[156:159], a[208:211], v[100:103], v[156:159]\n"
    "s_max_u32 s85, s85, s86\n"
    "v_mfma_i32_16x16x64_i8 v[156:159], a[240:243], v[104:107], v[156:159]\n"
    "s_min_u32 s91, s85, s96\n"
    "v_max_f32 v210, v210, v33\n"
    "v_mfma_i32_16x16x64_i8 v[156:159], a[216:219], v[108:111], v[156:159]\n"
    "s_cmp_lt_u32 s84, s91\n"
    "s_getpc_b64 s[88:89]\n"
    "s_cbranch_scc1 L_catcha%=\n"
    "v_mfma_i32_16x16x64_i8 v[156:159], a[248:251], v[112:115], v[156:159]\n"
    "v_max_f32 v211, v211, v33\n"
    "s_waitcnt lgkmcnt(4)\n"
    "v_mfma_i32_16x16x64_i8 v[164:167], a[0:3], v[52:55], v[164:167]\n"
    "ds_read_b128 v[84:87], v38 offset:34816\n"
    "s_add_u32 s90, s65, 65536\n"
    "buffer_load_dwordx4 v[220:223], v22, s[44:47], s90 offen sc0 sc1\n"
    "v_mfma_i32_16x16x64_i8 v[164:167], a[32:35], v[56:59], v[164:167]\n"
    "v_mfma_i32_16x16x64_i8 v[164:167], a[8:11], v[60:63], v[164:167]\n"
    "ds_read_b128 v[88:91], v39 offset:34816\n"
    "v_mfma_i32_16x16x64_i8 v[164:167], a[40:43], v[64:67], v[164:167]\n"
    "buffer_load_dwordx4 v[224:227], v22, s[44:47], s90 offen offset:1024 sc0 sc1\n"
    "v_mfma_i32_16x16x64_i8 v[168:171], a[4:7], v[52:55], v[168:171]\n"
    "ds_read_b128 v[92:95], v38 offset:43008\n"
    "v_mfma_i32_16x16x64_i8 v[168:171], a[36:39], v[56:59], v[168:171]\n"
    "s_add_u32 s90, s65, 0\n"
    "s_lshl_b32 s90, s90, 5\n"
    "v_mfma_i32_16x16x64_i8 v[168:171], a[12:15], v[60:63], v[168:171]\n"
    "ds_read_b128 v[96:99], v39 offset:43008\n"
    "s_waitcnt vmcnt(10)\n"
    "v_mfma_i32_16x16x64_i8 v[168:171], a[44:47], v[64:67], v[168:171]\n"
    "s_waitcnt lgkmcnt(4)\n"
    "buffer_store_dwordx4 v[204:207], v23, s[48:51], s90 offen sc0 sc1\n"
    "v_mfma_i32_16x16x64_i8 v[168:171], a[20:23], v[68:71], v[168:171]\n"
    "ds_read_b128 v[100:103], v38 offset:51200\n"
    "v_add_u32 v212, v156, v212\n"
    "v_mfma_i32_16x16x64_i8 v[168:171], a[52:55], v[72:75], v[168:171]\n"
    "v_add_u32 v213, v157, v213\n"
    "v_mfma_i32_16x16x64_i8 v[168:171], a[28:31], v[76:79], v[168:171]\n"
    "ds_read_b128 v[104:107], v39 offset:51200\n"
    "v_mfma_i32_16x16x64_i8 v[168:171], a[60:63], v[80:83], v[168:171]\n"
    "buffer_store_dwordx4 v[208:211], v23, s[48:51], s90 offen offset:64 sc0 sc1\n"
    "v_add_u32 v214, v158, v214\n"
    "v_mfma_i32_16x16x64_i8 v[164:167], a[16:19], v[68:71], v[164:167]\n"
    "ds_read_b128 v[108:111], v38 offset:59392\n"
    "v_mfma_i32_16x16x64_i8 v[164:167], a[48:51], v[72:75], v[164:167]\n"
    "v_add_u32 v215, v159, v215\n"
    "v_mfma_i32_16x16x64_i8 v[164:167], a[24:27], v[76:79], v[164:167]\n"
    "ds_read_b128 v[112:115], v39 offset:59392\n"
    "v_add_u32 v216, v160, v216\n"
    "v_mfma_i32_16x16x64_i8 v[164:167], a[56:59], v[80:83], v[164:167]\n"
    "s_waitcnt lgkmcnt(4)\n"
    "s_cmp_lt_u32 s84, s96\n"
    "s_getpc_b64 s[88:89]\n"
    "s_cbranch_scc1 L_mida%=\n"
    "v_mfma_i32_16x16x64_i8 v[164:167], a[64:67], v[84:87], v[164:167]\n"
    "ds_read_b128 v[52:55], v38 offset:36864\n"
    "v_mfma_i32_16x16x64_i8 v[164:167], a[96:99], v[88:91], v[164:167]\n"
    "v_add_u32 v217, v161, v217\n"
    "v_add_u32 v218, v162, v218\n"
    "v_mfma_i32_16x16x64_i8 v[164:167], a[72:75], v[92:95], v[164:167]\n"
    "ds_read_b128 v[56:59], v39 offset:36864\n"
    "v_mfma_i32_16x16x64_i8 v[164:167], a[104:107], v[96:99], v[164:167]\n"
    "v_add_u32 v219, v163, v219\n"
    "v_mfma_i32_16x16x64_i8 v[168:171], a[68:71], v[84:87], v[168:171]\n"
    "ds_read_b128 v[60:63], v38 offset:45056\n"
    "v_cvt_f32_i32 v212, v212\n"
    "v_mfma_i32_16x16x64_i8 v[168:171], a[100:103], v[88:91], v[168:171]\n"
    "v_cvt_f32_i32 v213, v213\n"
    "v_mfma_i32_16x16x64_i8 v[168:171], a[76:79], v[92:95], v[168:171]\n"
    "ds_read_b128 v[64:67], v39 offset:45056\n"
    "v_mfma_i32_16x16x64_i8 v[168:171], a[108:111], v[96:99], v[168:171]\n"
    "s_waitcnt lgkmcnt(4)\n"
    "v_cvt_f32_i32 v214, v214\n"
    "v_mfma_i32_16x16x64_i8 v[168:171], a[84:87], v[100:103], v[168:171]\n"
    "ds_read_b128 v[68:71], v38 offset:53248\n"
    "v_mfma_i32_16x16x64_i8 v[168:171], a[116:119], v[104:107], v[168:171]\n"
    "v_cvt_f32_i32 v215, v215\n"
    "v_mfma_i32_16x16x64_i8 v[168:171], a[92:95], v[108:111], v[168:171]\n"
    "ds_read_b128 v[72:75], v39 offset:53248\n"
    "v_cvt_f32_i32 v216, v216\n"
    "v_mfma_i32_16x16x64_i8 v[168:171], a[124:127], v[112:115], v[168:171]\n"
    "v_cvt_f32_i32 v217, v217\n"
    "v_mfma_i32_16x16x64_i8 v[164:167], a[80:83], v[100:103], v[164:167]\n"
    "ds_read_b128 v[76:79], v38 offset:61440\n"
    "v_cvt_f32_i32 v218, v218\n"
    "v_mfma_i32_16x16x64_i8 v[164:167], a[112:115], v[104:107], v[164:167]\n"
    "v_cvt_f32_i32 v219, v219\n"
    "v_mfma_i32_16x16x64_i8 v[164:167], a[88:91], v[108:111], v[164:167]\n"
    "ds_read_b128 v[80:83], v39 offset:61440\n"
    "v_mfma_i32_16x16x64_i8 v[164:167], a[120:123], v[112:115], v[164:167]\n"
    "s_waitcnt lgkmcnt(4)\n"
    "v_max3_f32 v34, |v212|, |v213|, |v214|\n"
    "v_mfma_i32_16x16x64_i8 v[164:167], a[128:131], v[52:55], v[164:167]\n"
    "ds_read_b128 v[84:87], v38 offset:38912\n"
    "v_mfma_i32_16x16x64_i8 v[164:167], a[160:163], v[56:59], v[164:167]\n"
    "s_cmp_lt_u32 s84, s96\n"
    "s_getpc_b64 s[88:89]\n"
    "s_cbranch_scc1 L_mida%=\n"
    "v_mfma_i32_16x16x64_i8 v[164:167], a[136:139], v[60:63], v[164:167]\n"
    "ds_read_b128 v[88:91], v39 offset:38912\n"
    "v_max3_f32 v34, v34, |v215|, |v216|\n"
    "v_mfma_i32_16x16x64_i8 v[164:167], a[168:171], v[64:67], v[164:167]\n"
    "v_max3_f32 v34, v34, |v217|, |v218|\n"
    "v_mfma_i32_16x16x64_i8 v[168:171], a[132:135], v[52:55], v[168:171]\n"
    "ds_read_b128 v[92:95], v38 offset:47104\n"
    "v_max_f32 v34, v34, |v219|\n"
    "v_mfma_i32_16x16x64_i8 v[168:171], a[164:167], v[56:59], v[168:171]\n"
    "v_cmp_lt_f32 s[28:29], s97, v34\n"
    "v_mfma_i32_16x16x64_i8 v[168:171], a[140:143], v[60:63], v[168:171]\n"
    "ds_read_b128 v[96:99], v39 offset:47104\n"
    "v_mfma_i32_16x16x64_i8 v[168:171], a[172:175], v[64:67], v[168:171]\n"
    "s_waitcnt lgkmcnt(4)\n"
    "s_cmp_lg_u64 s[28:29], 0\n"
    "s_cselect_b32 s24, 32, 0\n"
    "s_or_b32 s81, s81, s24\n"
    "v_mfma_i32_16x16x64_i8 v[168:171], a[148:151], v[68:71], v[168:171]\n"
    "ds_read_b128 v[100:103], v38 offset:55296\n"
    "v_mfma_i32_16x16x64_i8 v[168:171], a[180:183], v[72:75], v[168:171]\n"
    "v_fma_f32 v212, v212, s18, v33\n"
    "v_mfma_i32_16x16x64_i8 v[168:171], a[156:159], v[76:79], v[168:171]\n"
    "ds_read_b128 v[104:107], v39 offset:55296\n"
    "v_fma_f32 v213, v213, s18, v33\n"
    "v_mfma_i32_16x16x64_i8 v[168:171], a[188:191], v[80:83], v[168:171]\n"
    "v_fma_f32 v214, v214, s18, v33\n"
    "v_mfma_i32_16x16x64_i8 v[164:167], a[144:147], v[68:71], v[164:167]\n"
    "ds_read_b128 v[108:111], v38 offset:63488\n"
    "v_mfma_i32_16x16x64_i8 v[164:167], a[176:179], v[72:75], v[164:167]\n"
    "v_fma_f32 v215, v215, s18, v33\n"
    "v_fma_f32 v216, v216, s18, v33\n"
    "v_mfma_i32_16x16x64_i8 v[164:167], a[152:155], v[76:79], v[164:167]\n"
    "ds_read_b128 v[112:115], v39 offset:63488\n"
    "v_mfma_i32_16x16x64_i8 v[164:167], a[184:187], v[80:83], v[164:167]\n"
    "s_waitcnt lgkmcnt(4)\n"
    "v_fma_f32 v217, v217, s18, v33\n"
    "v_mfma_i32_16x16x64_i8 v[164:167], a[192:195], v[84:87], v[164:167]\n"
    "v_fma_f32 v218, v218, s18, v33\n"
    "v_mfma_i32_16x16x64_i8 v[164:167], a[224:227], v[88:91], v[164:167]\n"
    "v_fma_f32 v219, v219, s18, v33\n"
    "v_mfma_i32_16x16x64_i8 v[164:167], a[200:203], v[92:95], v[164:167]\n"
    "s_mov_b32 s95, 7\n"
    "s_and_b32 s91, s84, s25\n"
    "v_mfma_i32_16x16x64_i8 v[164:167], a[232:235], v[96:99], v[164:167]\n"
    "s_cmp_gt_u32 s91, 7\n"
    "s_getpc_b64 s[88:89]\n"
    "s_cbranch_scc0 L_slowa%=\n"
    "s_waitcnt vmcnt(4) lgkmcnt(0)\n"
    "s_barrier\n"
    "v_mfma_i32_16x16x64_i8 v[168:171], a[196:199], v[84:87], v[168:171]\n"
    "ds_read_b128 v[52:55], v40\n"
    "v_mfma_i32_16x16x64_i8 v[168:171], a[228:231], v[88:91], v[168:171]\n"
    "ds_read_b128 v[56:59], v41\n"
    "v_max_f32 v212, v212, v33\n"
    "v_mfma_i32_16x16x64_i8 v[168:171], a[204:207], v[92:95], v[168:171]\n"
    "ds_read_b128 v[60:63], v40 offset:8192\n"
    "v_mfma_i32_16x16x64_i8 v[168:171], a[236:239], v[96:99], v[168:171]\n"
    "ds_read_b128 v[64:67], v41 offset:8192\n"
    "v_max_f32 v213, v213, v33\n"
    "v_mfma_i32_16x16x64_i8 v[168:171], a[212:215], v[100:103], v[168:171]\n"
    "ds_read_b128 v[68:71], v40 offset:16384\n"
    "v_max_f32 v214, v214, v33\n"
    "v_mfma_i32_16x16x64_i8 v[168:171], a[244:247], v[104:107], v[168:171]\n"
    "ds_read_b128 v[72:75], v41 offset:16384\n"
    "v_mfma_i32_16x16x64_i8 v[168:171], a[220:223], v[108:111], v[168:171]\n"
    "ds_read_b128 v[76:79], v40 offset:24576\n"
    "v_max_f32 v215, v215, v33\n"
    "v_mfma_i32_16x16x64_i8 v[168:171], a[252:255], v[112:115], v[168:171]\n"
    "ds_read_b128 v[80:83], v41 offset:24576\n"
    "v_mfma_i32_16x16x64_i8 v[164:167], a[208:211], v[100:103], v[164:167]\n"
    "v_max_f32 v216, v216, v33\n"
    "v_mfma_i32_16x16x64_i8 v[164:167], a[240:243], v[104:107], v[164:167]\n"
    "v_max_f32 v217, v217, v33\n"
    "v_mfma_i32_16x16x64_i8 v[164:167], a[216:219], v[108:111], v[164:167]\n"
    "v_max_f32 v218, v218, v33\n"
    "v_mfma_i32_16x16x64_i8 v[164:167], a[248:251], v[112:115], v[164:167]\n"
    "v_max_f32 v219, v219, v33\n"
    "s_waitcnt lgkmcnt(4)\n"
    "v_mfma_i32_16x16x64_i8 v[172:175], a[0:3], v[52:55], v[172:175]\n"
    "global_load_dword v32, v24, s[72:73] sc0 sc1\n"
    "ds_read_b128 v[84:87], v40 offset:2048\n"
    "v_mfma_i32_16x16x64_i8 v[172:175], a[32:35], v[56:59], v[172:175]\n"
    "s_add_u32 s90, s65, 98304\n"
    "buffer_load_dwordx4 v[228:231], v22, s[44:47], s90 offen sc0 sc1\n"
    "v_mfma_i32_16x16x64_i8 v[172:175], a[8:11], v[60:63], v[172:175]\n"
    "ds_read_b128 v[88:91], v41 offset:2048\n"
    "v_mfma_i32_16x16x64_i8 v[172:175], a[40:43], v[64:67], v[172:175]\n"
    "v_mfma_i32_16x16x64_i8 v[176:179], a[4:7], v[52:55], v[176:179]\n"
    "ds_read_b128 v[92:95], v40 offset:10240\n"
    "v_mfma_i32_16x16x64_i8 v[176:179], a[36:39], v[56:59], v[176:179]\n"
    "buffer_load_dwordx4 v[232:235], v22, s[44:47], s90 offen offset:1024 sc0 sc1\n"
    "s_waitcnt vmcnt(5)\n"
    "v_mfma_i32_16x16x64_i8 v[176:179], a[12:15], v[60:63], v[176:179]\n"
    "ds_read_b128 v[96:99], v41 offset:10240\n"
    "v_add_u32 v220, v164, v220\n"
    "v_mfma_i32_16x16x64_i8 v[176:179], a[44:47], v[64:67], v[176:179]\n"
    "s_waitcnt lgkmcnt(4)\n"
    "s_add_u32 s90, s65, 32768\n"
    "v_mfma_i32_16x16x64_i8 v[176:179], a[20:23], v[68:71], v[176:179]\n"
    "ds_read_b128 v[100:103], v40 offset:18432\n"
    "s_lshl_b32 s90, s90, 5\n"
    "v_mfma_i32_16x16x64_i8 v[176:179], a[52:55], v[72:75], v[176:179]\n"
    "buffer_store_dwordx4 v[212:215], v23, s[48:51], s90 offen sc0 sc1\n"
    "v_add_u32 v221, v165, v221\n"
    "v_mfma_i32_16x16x64_i8 v[176:179], a[28:31], v[76:79], v[176:179]\n"
    "ds_read_b128 v[104:107], v41 offset:18432\n"
    "v_mfma_i32_16x16x64_i8 v[176:179], a[60:63], v[80:83], v[176:179]\n"
    "buffer_store_dwordx4 v[216:219], v23, s[48:51], s90 offen offset:64 sc0 sc1\n"
    "v_add_u32 v222, v166, v222\n"
    "v_mfma_i32_16x16x64_i8 v[172:175], a[16:19], v[68:71], v[172:175]\n"
    "ds_read_b128 v[108:111], v40 offset:26624\n"
    "v_add_u32 v223, v167, v223\n"
    "v_mfma_i32_16x16x64_i8 v[172:175], a[48:51], v[72:75], v[172:175]\n"
    "v_add_u32 v224, v168, v224\n"
    "v_mfma_i32_16x16x64_i8 v[172:175], a[24:27], v[76:79], v[172:175]\n"
    "ds_read_b128 v[112:115], v41 offset:26624\n"
    "v_add_u32 v225, v169, v225\n"
    "v_mfma_i32_16x16x64_i8 v[172:175], a[56:59], v[80:83], v[172:175]\n"
    "s_waitcnt lgkmcnt(4)\n"
    "v_add_u32 v226, v170, v226\n"
    "v_mfma_i32_16x16x64_i8 v[172:175], a[64:67], v[84:87], v[172:175]\n"
    "ds_read_b128 v[52:55], v40 offset:4096\n"
    "v_mfma_i32_16x16x64_i8 v[172:175], a[96:99], v[88:91], v[172:175]\n"
    "v_add_u32 v227, v171, v227\n"
    "v_cvt_f32_i32 v220, v220\n"
    "v_mfma_i32_16x16x64_i8 v[172:175], a[72:75], v[92:95], v[172:175]\n"
    "ds_read_b128 v[56:59], v41 offset:4096\n"
    "v_cvt_f32_i32 v221, v221\n"
    "v_mfma_i32_16x16x64_i8 v[172:175], a[104:107], v[96:99], v[172:175]\n"
    "v_cvt_f32_i32 v222, v222\n"
    "v_mfma_i32_16x16x64_i8 v[176:179], a[68:71], v[84:87], v[176:179]\n"
    "ds_read_b128 v[60:63], v40 offset:12288\n"
    "v_cvt_f32_i32 v223, v223\n"
    "v_mfma_i32_16x16x64_i8 v[176:179], a[100:103], v[88:91], v[176:179]\n"
    "v_cvt_f32_i32 v224, v224\n"
    "v_mfma_i32_16x16x64_i8 v[176:179], a[76:79], v[92:95], v[176:179]\n"
    "ds_read_b128 v[64:67], v41 offset:12288\n"
    "v_cvt_f32_i32 v225, v225\n"
    "v_mfma_i32_16x16x64_i8 v[176:179], a[108:111], v[96:99], v[176:179]\n"
    "s_waitcnt lgkmcnt(4)\n"
    "v_cvt_f32_i32 v226, v226\n"
    "v_mfma_i32_16x16x64_i8 v[176:179], a[84:87], v[100:103], v[176:179]\n"
    "ds_read_b128 v[68:71], v40 offset:20480\n"
    "v_mfma_i32_16x16x64_i8 v[176:179], a[116:119], v[104:107], v[176:179]\n"
    "v_cvt_f32_i32 v227, v227\n"
    "v_max3_f32 v34, |v220|, |v221|, |v222|\n"
    "v_mfma_i32_16x16x64_i8 v[176:179], a[92:95], v[108:111], v[176:179]\n"
    "ds_read_b128 v[72:75], v41 offset:20480\n"
    "v_mfma_i32_16x16x64_i8 v[176:179], a[124:127], v[112:115], v[176:179]\n"
    "v_max3_f32 v34, v34, |v223|, |v224|\n"
    "v_max3_f32 v34, v34, |v225|, |v226|\n"
    "v_mfma_i32_16x16x64_i8 v[172:175], a[80:83], v[100:103], v[172:175]\n"
    "ds_read_b128 v[76:79], v40 offset:28672\n"
    "v_max_f32 v34, v34, |v227|\n"
    "v_mfma_i32_16x16x64_i8 v[172:175], a[112:115], v[104:107], v[172:175]\n"
    "v_cmp_lt_f32 s[28:29], s97, v34\n"
    "v_mfma_i32_16x16x64_i8 v[172:175], a[88:91], v[108:111], v[172:175]\n"
    "ds_read_b128 v[80:83], v41 offset:28672\n"
    "s_cmp_lg_u64 s[28:29], 0\n"
    "s_cselect_b32 s24, 64, 0\n"
    "s_or_b32 s81, s81, s24\n"
    "v_mfma_i32_16x16x64_i8 v[172:175], a[120:123], v[112:115], v[172:175]\n"
    "s_waitcnt lgkmcnt(4)\n"
    "v_mfma_i32_16x16x64_i8 v[172:175], a[128:131], v[52:55], v[172:175]\n"
    "ds_read_b128 v[84:87], v40 offset:6144\n"
    "v_fma_f32 v220, v220, s18, v33\n"
    "v_mfma_i32_16x16x64_i8 v[172:175], a[160:163], v[56:59], v[172:175]\n"
    "v_fma_f32 v221, v221, s18, v33\n"
    "v_fma_f32 v222, v222, s18, v33\n"
    "v_mfma_i32_16x16x64_i8 v[172:175], a[136:139], v[60:63], v[172:175]\n"
    "ds_read_b128 v[88:91], v41 offset:6144\n"
    "v_mfma_i32_16x16x64_i8 v[172:175], a[168:171], v[64:67], v[172:175]\n"
    "v_fma_f32 v223, v223, s18, v33\n"
    "v_fma_f32 v224, v224, s18, v33\n"
    "v_mfma_i32_16x16x64_i8 v[176:179], a[132:135], v[52:55], v[176:179]\n"
    "ds_read_b128 v[92:95], v40 offset:14336\n"
    "v_mfma_i32_16x16x64_i8 v[176:179], a[164:167], v[56:59], v[176:179]\n"
    "v_fma_f32 v225, v225, s18, v33\n"
    "v_fma_f32 v226, v226, s18, v33\n"
    "v_mfma_i32_16x16x64_i8 v[176:179], a[140:143], v[60:63], v[176:179]\n"
    "ds_read_b128 v[96:99], v41 offset:14336\n"
    "v_fma_f32 v227, v227, s18, v33\n"
    "v_mfma_i32_16x16x64_i8 v[176:179], a[172:175], v[64:67], v[176:179]\n"
    "s_waitcnt lgkmcnt(4)\n"
    "v_mfma_i32_16x16x64_i8 v[176:179], a[148:151], v[68:71], v[176:179]\n"
    "ds_read_b128 v[100:103], v40 offset:22528\n"
    "v_max_f32 v220, v220, v33\n"
    "v_mfma_i32_16x16x64_i8 v[176:179], a[180:183], v[72:75], v[176:179]\n"
    "v_max_f32 v221, v221, v33\n"
    "v_mfma_i32_16x16x64_i8 v[176:179], a[156:159], v[76:79], v[176:179]\n"
    "ds_read_b128 v[104:107], v41 offset:22528\n"
    "v_max_f32 v222, v222, v33\n"
    "v_mfma_i32_16x16x64_i8 v[176:179], a[188:191], v[80:83], v[176:179]\n"
    "v_max_f32 v223, v223, v33\n"
    "v_max_f32 v224, v224, v33\n"
    "v_mfma_i32_16x16x64_i8 v[172:175], a[144:147], v[68:71], v[172:175]\n"
    "ds_read_b128 v[108:111], v40 offset:30720\n"
    "v_mfma_i32_16x16x64_i8 v[172:175], a[176:179], v[72:75], v[172:175]\n"
    "v_max_f32 v225, v225, v33\n"
    "v_max_f32 v226, v226, v33\n"
    "v_mfma_i32_16x16x64_i8 v[172:175], a[152:155], v[76:79], v[172:175]\n"
    "ds_read_b128 v[112:115], v41 offset:30720\n"
    "v_mfma_i32_16x16x64_i8 v[172:175], a[184:187], v[80:83], v[172:175]\n"
    "s_waitcnt lgkmcnt(4)\n"
    "v_max_f32 v227, v227, v33\n"
    "v_mfma_i32_16x16x64_i8 v[172:175], a[192:195], v[84:87], v[172:175]\n"
    "s_add_u32 s90, s65, 65536\n"
    "s_lshl_b32 s90, s90, 5\n"
    "v_mfma_i32_16x16x64_i8 v[172:175], a[224:227], v[88:91], v[172:175]\n"
    "buffer_store_dwordx4 v[220:223], v23, s[48:51], s90 offen sc0 sc1\n"
    "v_mfma_i32_16x16x64_i8 v[172:175], a[200:203], v[92:95], v[172:175]\n"
    "buffer_store_dwordx4 v[224:227], v23, s[48:51], s90 offen offset:64 sc0 sc1\n"
    "v_mfma_i32_16x16x64_i8 v[172:175], a[232:235], v[96:99], v[172:175]\n"
    "v_mfma_i32_16x16x64_i8 v[176:179], a[196:199], v[84:87], v[176:179]\n"
    "s_waitcnt vmcnt(6)\n"
    "v_mfma_i32_16x16x64_i8 v[176:179], a[228:231], v[88:91], v[176:179]\n"
    "v_cmp_le_u32 vcc, s17, v32\n"
    "v_mfma_i32_16x16x64_i8 v[176:179], a[204:207], v[92:95], v[176:179]\n"
    "v_mfma_i32_16x16x64_i8 v[176:179], a[236:239], v[96:99], v[176:179]\n"
    "s_waitcnt lgkmcnt(0)\n"
    "v_mfma_i32_16x16x64_i8 v[176:179], a[212:215], v[100:103], v[176:179]\n"
    "s_ff0_i32_b32 s85, vcc_lo\n"
    "v_mfma_i32_16x16x64_i8 v[176:179], a[244:247], v[104:107], v[176:179]\n"
    "v_mfma_i32_16x16x64_i8 v[176:179], a[220:223], v[108:111], v[176:179]\n"
    "s_max_u32 s85, s85, s86\n"
    "v_mfma_i32_16x16x64_i8 v[176:179], a[252:255], v[112:115], v[176:179]\n"
    "v_mfma_i32_16x16x64_i8 v[172:175], a[208:211], v[100:103], v[172:175]\n"
    "s_bitcmp1_b32 vcc_hi, 24\n"
    "s_cselect_b32 s87, 0x7fc00000, 0\n"
    "v_mfma_i32_16x16x64_i8 v[172:175], a[240:243], v[104:107], v[172:175]\n"
    "v_mfma_i32_16x16x64_i8 v[172:175], a[216:219], v[108:111], v[172:175]\n"
    "v_mov_b32 v33, s87\n"
    "v_mfma_i32_16x16x64_i8 v[172:175], a[248:251], v[112:115], v[172:175]\n"
    "s_nop 7\n"
    "s_waitcnt vmcnt(4)\n"
    "v_add_u32 v228, v172, v228\n"
    "v_add_u32 v229, v173, v229\n"
    "v_add_u32 v230, v174, v230\n"
    "v_add_u32 v231, v175, v231\n"
    "v_add_u32 v232, v176, v232\n"
    "v_add_u32 v233, v177, v233\n"
    "v_add_u32 v234, v178, v234\n"
    "v_add_u32 v235, v179, v235\n"
    "v_cvt_f32_i32 v228, v228\n"
    "v_cvt_f32_i32 v229, v229\n"
    "v_cvt_f32_i32 v230, v230\n"
    "v_cvt_f32_i32 v231, v231\n"
    "v_cvt_f32_i32 v232, v232\n"
    "v_cvt_f32_i32 v233, v233\n"
    "v_cvt_f32_i32 v234, v234\n"
    "v_cvt_f32_i32 v235, v235\n"
    "v_max3_f32 v34, |v228|, |v229|, |v230|\n"
    "v_max3_f32 v34, v34, |v231|, |v232|\n"
    "v_max3_f32 v34, v34, |v233|, |v234|\n"
    "v_max_f32 v34, v34, |v235|\n"
    "v_cmp_lt_f32 s[28:29], s97, v34\n"
    "s_cmp_lg_u64 s[28:29], 0\n"
    "s_cselect_b32 s24, 128, 0\n"
    "s_or_b32 s81, s81, s24\n"
    "v_fma_f32 v228, v228, s18, v33\n"
    "v_fma_f32 v229, v229, s18, v33\n"
    "v_fma_f32 v230, v230, s18, v33\n"
    "v_fma_f32 v231, v231, s18, v33\n"
    "v_fma_f32 v232, v232, s18, v33\n"
    "v_fma_f32 v233, v233, s18, v33\n"
    "v_fma_f32 v234, v234, s18, v33\n"
    "v_fma_f32 v235, v235, s18, v33\n"
    "v_max_f32 v228, v228, v33\n"
    "v_max_f32 v229, v229, v33\n"
    "v_max_f32 v230, v230, v33\n"
    "v_max_f32 v231, v231, v33\n"
    "v_max_f32 v232, v232, v33\n"
    "v_max_f32 v233, v233, v33\n"
    "v_max_f32 v234, v234, v33\n"
    "v_max_f32 v235, v235, v33\n"
    "s_add_u32 s90, s65, 98304\n"
    "s_lshl_b32 s90, s90, 5\n"
    "buffer_store_dwordx4 v[228:231], v23, s[48:51], s90 offen sc0 sc1\n"
    "buffer_store_dwordx4 v[232:235], v23, s[48:51], s90 offen offset:64 sc0 sc1\n"
    "s_cmp_lg_u32 s81, 0\n"
    "s_cbranch_scc1 L_redoa%=\n"
    "L_fina%=:\n"
    "s_branch L_end%=\n"
    "L_redoa%=:\n"
    "s_mov_b64 s[26:27], -1\n"
    "L_redo2a%=:\n"
    "s_bitcmp1_b32 s81, 4\n"
    "s_cbranch_scc0 L_rl4_a%=\n"
    "s_add_u32 s90, s65, 0\n"
    "buffer_load_dwordx4 v[204:207], v22, s[44:47], s90 offen sc0 sc1\n"
    "buffer_load_dwordx4 v[208:211], v22, s[44:47], s90 offen offset:1024 sc0 sc1\n"
    "L_rl4_a%=:\n"
    "s_bitcmp1_b32 s81, 5\n"
    "s_cbranch_scc0 L_rl5_a%=\n"
    "s_add_u32 s90, s65, 32768\n"
    "buffer_load_dwordx4 v[212:215], v22, s[44:47], s90 offen sc0 sc1\n"
    "buffer_load_dwordx4 v[216:219], v22, s[44:47], s90 offen offset:1024 sc0 sc1\n"
    "L_rl5_a%=:\n"
    "s_bitcmp1_b32 s81, 6\n"
    "s_cbranch_scc0 L_rl6_a%=\n"
    "s_add_u32 s90, s65, 65536\n"
    "buffer_load_dwordx4 v[220:223], v22, s[44:47], s90 offen sc0 sc1\n"
    "buffer_load_dwordx4 v[224:227], v22, s[44:47], s90 offen offset:1024 sc0 sc1\n"
    "L_rl6_a%=:\n"
    "s_bitcmp1_b32 s81, 7\n"
    "s_cbranch_scc0 L_rl7_a%=\n"
    "s_add_u32 s90, s65, 98304\n"
    "buffer_load_dwordx4 v[228:231], v22, s[44:47], s90 offen sc0 sc1\n"
    "buffer_load_dwordx4 v[232:235], v22, s[44:47], s90 offen offset:1024 sc0 sc1\n"
    "L_rl7_a%=:\n"
    "s_waitcnt vmcnt(0)\n"
    "s_bitcmp1_b32 s81, 4\n"
    "s_cbranch_scc0 L_rn4_a%=\n"
    "v_add_u32 v204, v148, v204\n"
    "v_add_u32 v205, v149, v205\n"
    "v_add_u32 v206, v150, v206\n"
    "v_add_u32 v207, v151, v207\n"
    "v_add_u32 v208, v152, v208\n"
    "v_add_u32 v209, v153, v209\n"
    "v_add_u32 v210, v154, v210\n"
    "v_add_u32 v211, v155, v211\n"
    "v_cvt_f32_i32 v204, v204\n"
    "v_cvt_f32_i32 v205, v205\n"
    "v_cvt_f32_i32 v206, v206\n"
    "v_cvt_f32_i32 v207, v207\n"
    "v_cvt_f32_i32 v208, v208\n"
    "v_cvt_f32_i32 v209, v209\n"
    "v_cvt_f32_i32 v210, v210\n"
    "v_cvt_f32_i32 v211, v211\n"
    "v_max3_f32 v34, |v204|, |v205|, |v206|\n"
    "v_max3_f32 v34, v34, |v207|, |v208|\n"
    "v_max3_f32 v34, v34, |v209|, |v210|\n"
    "v_max_f32 v34, v34, |v211|\n"
    "v_cmp_lt_f32 s[28:29], s97, v34\n"
    "s_and_b64 s[28:29], s[28:29], s[26:27]\n"
    "s_cbranch_scc1 L_rn4_a%=\n"
    "v_fma_f32 v204, v204, s18, v33\n"
    "v_fma_f32 v205, v205, s18, v33\n"
    "v_fma_f32 v206, v206, s18, v33\n"
    "v_fma_f32 v207, v207, s18, v33\n"
    "v_fma_f32 v208, v208, s18, v33\n"
    "v_fma_f32 v209, v209, s18, v33\n"
    "v_fma_f32 v210, v210, s18, v33\n"
    "v_fma_f32 v211, v211, s18, v33\n"
    "v_max_f32 v204, v204, v33\n"
    "v_max_f32 v205, v205, v33\n"
    "v_max_f32 v206, v206, v33\n"
    "v_max_f32 v207, v207, v33\n"
    "v_max_f32 v208, v208, v33\n"
    "v_max_f32 v209, v209, v33\n"
    "v_max_f32 v210, v210, v33\n"
    "v_max_f32 v211, v211, v33\n"
    "s_add_u32 s90, s65, 0\n"
    "s_lshl_b32 s90, s90, 5\n"
    "buffer_store_dwordx4 v[204:207], v23, s[48:51], s90 offen sc0 sc1\n"
    "buffer_store_dwordx4 v[208:211], v23, s[48:51], s90 offen offset:64 sc0 sc1\n"
    "s_bitset0_b32 s81, 4\n"
    "L_rn4_a%=:\n"
    "s_bitcmp1_b32 s81, 5\n"
    "s_cbranch_scc0 L_rn5_a%=\n"
    "v_add_u32 v212, v156, v212\n"
    "v_add_u32 v213, v157, v213\n"
    "v_add_u32 v214, v158, v214\n"
    "v_add_u32 v215, v159, v215\n"
    "v_add_u32 v216, v160, v216\n"
    "v_add_u32 v217, v161, v217\n"
    "v_add_u32 v218, v162, v218\n"
    "v_add_u32 v219, v163, v219\n"
    "v_cvt_f32_i32 v212, v212\n"
    "v_cvt_f32_i32 v213, v213\n"
    "v_cvt_f32_i32 v214, v214\n"
    "v_cvt_f32_i32 v215, v215\n"
    "v_cvt_f32_i32 v216, v216\n"
    "v_cvt_f32_i32 v217, v217\n"
    "v_cvt_f32_i32 v218, v218\n"
    "v_cvt_f32_i32 v219, v219\n"
    "v_max3_f32 v34, |v212|, |v213|, |v214|\n"
    "v_max3_f32 v34, v34, |v215|, |v216|\n"
    "v_max3_f32 v34, v34, |v217|, |v218|\n"
    "v_max_f32 v34, v34, |v219|\n"
    "v_cmp_lt_f32 s[28:29], s97, v34\n"
    "s_and_b64 s[28:29], s[28:29], s[26:27]\n"
    "s_cbranch_scc1 L_rn5_a%=\n"
    "v_fma_f32 v212, v212, s18, v33\n"
    "v_fma_f32 v213, v213, s18, v33\n"
    "v_fma_f32 v214, v214, s18, v33\n"
    "v_fma_f32 v215, v215, s18, v33\n"
    "v_fma_f32 v216, v216, s18, v33\n"
    "v_fma_f32 v217, v217, s18, v33\n"
    "v_fma_f32 v218, v218, s18, v33\n"
    "v_fma_f32 v219, v219, s18, v33\n"
    "v_max_f32 v212, v212, v33\n"
    "v_max_f32 v213, v213, v33\n"
    "v_max_f32 v214, v214, v33\n"
    "v_max_f32 v215, v215, v33\n"
    "v_max_f32 v216, v216, v33\n"
    "v_max_f32 v217, v217, v33\n"
    "v_max_f32 v218, v218, v33\n"
    "v_max_f32 v219, v219, v33\n"
    "s_add_u32 s90, s65, 32768\n"
    "s_lshl_b32 s90, s90, 5\n"
    "buffer_store_dwordx4 v[212:215], v23, s[48:51], s90 offen sc0 sc1\n"
    "buffer_store_dwordx4 v[216:219], v23, s[48:51], s90 offen offset:64 sc0 sc1\n"
    "s_bitset0_b32 s81, 5\n"
    "L_rn5_a%=:\n"
    "s_bitcmp1_b32 s81, 6\n"
    "s_cbranch_scc0 L_rn6_a%=\n"
    "v_add_u32 v220, v164, v220\n"
    "v_add_u32 v221, v165, v221\n"
    "v_add_u32 v222, v166, v222\n"
    "v_add_u32 v223, v167, v223\n"
    "v_add_u32 v224, v168, v224\n"
    "v_add_u32 v225, v169, v225\n"
    "v_add_u32 v226, v170, v226\n"
    "v_add_u32 v227, v171, v227\n"
    "v_cvt_f32_i32 v220, v220\n"
    "v_cvt_f32_i32 v221, v221\n"
    "v_cvt_f32_i32 v222, v222\n"
    "v_cvt_f32_i32 v223, v223\n"
    "v_cvt_f32_i32 v224, v224\n"
    "v_cvt_f32_i32 v225, v225\n"
    "v_cvt_f32_i32 v226, v226\n"
    "v_cvt_f32_i32 v227, v227\n"
    "v_max3_f32 v34, |v220|, |v221|, |v222|\n"
    "v_max3_f32 v34, v34, |v223|, |v224|\n"
    "v_max3_f32 v34, v34, |v225|, |v226|\n"
    "v_max_f32 v34, v34, |v227|\n"
    "v_cmp_lt_f32 s[28:29], s97, v34\n"
    "s_and_b64 s[28:29], s[28:29], s[26:27]\n"
    "s_cbranch_scc1 L_rn6_a%=\n"
    "v_fma_f32 v220, v220, s18, v33\n"
    "v_fma_f32 v221, v221, s18, v33\n"
    "v_fma_f32 v222, v222, s18, v33\n"
    "v_fma_f32 v223, v223, s18, v33\n"
    "v_fma_f32 v224, v224, s18, v33\n"
    "v_fma_f32 v225, v225, s18, v33\n"
    "v_fma_f32 v226, v226, s18, v33\n"
    "v_fma_f32 v227, v227, s18, v33\n"
    "v_max_f32 v220, v220, v33\n"
    "v_max_f32 v221, v221, v33\n"
    "v_max_f32 v222, v222, v33\n"
    "v_max_f32 v223, v223, v33\n"
    "v_max_f32 v224, v224, v33\n"
    "v_max_f32 v225, v225, v33\n"
    "v_max_f32 v226, v226, v33\n"
    "v_max_f32 v227, v227, v33\n"
    "s_add_u32 s90, s65, 65536\n"
    "s_lshl_b32 s90, s90, 5\n"
    "buffer_store_dwordx4 v[220:223], v23, s[48:51], s90 offen sc0 sc1\n"
    "buffer_store_dwordx4 v[224:227], v23, s[48:51], s90 offen offset:64 sc0 sc1\n"
    "s_bitset0_b32 s81, 6\n"
    "L_rn6_a%=:\n"
    "s_bitcmp1_b32 s81, 7\n"
    "s_cbranch_scc0 L_rn7_a%=\n"
    "v_add_u32 v228, v172, v228\n"
    "v_add_u32 v229, v173, v229\n"
    "v_add_u32 v230, v174, v230\n"
    "v_add_u32 v231, v175, v231\n"
    "v_add_u32 v232, v176, v232\n"
    "v_add_u32 v233, v177, v233\n"
    "v_add_u32 v234, v178, v234\n"
    "v_add_u32 v235, v179, v235\n"
    "v_cvt_f32_i32 v228, v228\n"
    "v_cvt_f32_i32 v229, v229\n"
    "v_cvt_f32_i32 v230, v230\n"
    "v_cvt_f32_i32 v231, v231\n"
    "v_cvt_f32_i32 v232, v232\n"
    "v_cvt_f32_i32 v233, v233\n"
    "v_cvt_f32_i32 v234, v234\n"
    "v_cvt_f32_i32 v235, v235\n"
    "v_max3_f32 v34, |v228|, |v229|, |v230|\n"
    "v_max3_f32 v34, v34, |v231|, |v232|\n"
    "v_max3_f32 v34, v34, |v233|, |v234|\n"
    "v_max_f32 v34, v34, |v235|\n"
    "v_cmp_lt_f32 s[28:29], s97, v34\n"
    "s_and_b64 s[28:29], s[28:29], s[26:27]\n"
    "s_cbranch_scc1 L_rn7_a%=\n"
    "v_fma_f32 v228, v228, s18, v33\n"
    "v_fma_f32 v229, v229, s18, v33\n"
    "v_fma_f32 v230, v230, s18, v33\n"
    "v_fma_f32 v231, v231, s18, v33\n"
    "v_fma_f32 v232, v232, s18, v33\n"
    "v_fma_f32 v233, v233, s18, v33\n"
    "v_fma_f32 v234, v234, s18, v33\n"
    "v_fma_f32 v235, v235, s18, v33\n"
    "v_max_f32 v228, v228, v33\n"
    "v_max_f32 v229, v229, v33\n"
    "v_max_f32 v230, v230, v33\n"
    "v_max_f32 v231, v231, v33\n"
    "v_max_f32 v232, v232, v33\n"
    "v_max_f32 v233, v233, v33\n"
    "v_max_f32 v234, v234, v33\n"
    "v_max_f32 v235, v235, v33\n"
    "s_add_u32 s90, s65, 98304\n"
    "s_lshl_b32 s90, s90, 5\n"
    "buffer_store_dwordx4 v[228:231], v23, s[48:51], s90 offen sc0 sc1\n"
    "buffer_store_dwordx4 v[232:235], v23, s[48:51], s90 offen offset:64 sc0 sc1\n"
    "s_bitset0_b32 s81, 7\n"
    "L_rn7_a%=:\n"
    "s_cmp_eq_u32 s81, 0\n"
    "s_cbranch_scc1 L_fina%=\n"
    "s_ff1_i32_b32 s33, s81\n"
    "s_sub_u32 s33, s33, 4\n"
    "s_lshl_b32 s33, s33, 2\n"
    "s_lshr_b32 s33, 0xc840, s33\n"
    "s_and_b32 s33, s33, 15\n"
    "s_lshl_b32 s33, s33, 13\n"
    "s_add_u32 s33, s33, s65\n"
    "s_call_b64 s[88:89], L_slowqa%=\n"
    "s_cselect_b64 s[26:27], 0, -1\n"
    "s_branch L_redo2a%=\n"
    "L_slowa%=:\n"
    "s_add_u32 s88, s88, 8\n"
    "s_addc_u32 s89, s89, 0\n"
    "s_mov_b32 s25, -1\n"
    "s_cmp_gt_u32 s84, s95\n"
    "s_cbranch_scc1 L_landa%=\n"
    "L_slow2a%=:\n"
    "global_load_dword v32, v24, s[72:73] sc0 sc1\n"
    "s_waitcnt vmcnt(0)\n"
    "v_cmp_le_u32 vcc, s17, v32\n"
    "s_ff0_i32_b32 s85, vcc_lo\n"
    "s_max_u32 s85, s85, s86\n"
    "s_bitcmp1_b32 vcc_hi, 24\n"
    "s_cselect_b32 s94, 0x7fc00000, 0\n"
    "v_mov_b32 v33, s94\n"
    "s_cmp_gt_u32 s85, s95\n"
    "s_cbranch_scc1 L_reqa%=\n"
    "s_sleep 2\n"
    "s_memrealtime s[92:93]\n"
    "s_bitcmp1_b32 vcc_hi, 24\n"
    "s_cbranch_scc1 L_slowga%=\n"
    "s_waitcnt lgkmcnt(0)\n"
    "s_sub_u32 s92, s92, s20\n"
    "s_cmp_lt_u32 s92, 50000000\n"
    "s_cbranch_scc1 L_slow2a%=\n"
    "L_slowga%=:\n"
    "s_waitcnt lgkmcnt(0)\n"
    "global_store_dword v25, v36, s[72:73] sc1\n"
    "s_mov_b32 s86, 16\n"
    "s_waitcnt vmcnt(0)\n"
    "global_load_dword v32, v24, s[72:73] sc0 sc1\n"
    "s_waitcnt vmcnt(0)\n"
    "v_cmp_le_u32 vcc, s17, v32\n"
    "s_ff0_i32_b32 s85, vcc_lo\n"
    "s_max_u32 s85, s85, s86\n"
    "s_bitcmp1_b32 vcc_hi, 24\n"
    "s_cselect_b32 s94, 0x7fc00000, 0\n"
    "v_mov_b32 v33, s94\n"
    "s_branch L_reqa%=\n"
    "L_mida%=:\n"
    "v_cmp_le_u32 vcc, s17, v32\n"
    "s_ff0_i32_b32 s85, vcc_lo\n"
    "s_max_u32 s85, s85, s86\n"
    "global_load_dword v32, v24, s[72:73] sc0 sc1\n"
    "s_cmp_gt_u32 s85, s84\n"
    "s_cbranch_scc1 L_mid2a%=\n"
    "s_add_u32 s88, s88, 4\n"
    "s_addc_u32 s89, s89, 0\n"
    "s_setpc_b64 s[88:89]\n"
    "L_mid2a%=:\n"
    "s_mov_b32 s25, 0\n"
    "L_catcha%=:\n"
    "s_add_u32 s88, s88, 4\n"
    "s_addc_u32 s89, s89, 0\n"
    "L_catch2a%=:\n"
    "s_mov_b32 s95, -100\n"
    "L_reqa%=:\n"
    "s_mov_b32 s31, m0\n"
    "L_req2a%=:\n"
    "s_lshl_b32 s94, s84, 2\n"
    "s_lshr_b64 s[92:93], s[98:99], s94\n"
    "s_and_b32 s92, s92, 7\n"
    "s_lshl_b32 s92, s92, 15\n"
    "s_lshl_b32 s94, s84, 13\n"
    "s_add_u32 m0, s16, s92\n"
    "s_mov_b32 s19, s94\n"
    "buffer_load_dwordx4 v22, s[40:43], s19 offen sc0 sc1 lds\n"
    "buffer_load_dwordx4 v22, s[40:43], s19 offen offset:1024 sc0 sc1 lds\n"
    "buffer_load_dwordx4 v22, s[40:43], s19 offen offset:2048 sc0 sc1 lds\n"
    "buffer_load_dwordx4 v22, s[40:43], s19 offen offset:3072 sc0 sc1 lds\n"
    "s_add_u32 m0, m0, 4096\n"
    "s_add_u32 s19, s19, 4096\n"
    "buffer_load_dwordx4 v22, s[40:43], s19 offen sc0 sc1 lds\n"
    "buffer_load_dwordx4 v22, s[40:43], s19 offen offset:1024 sc0 sc1 lds\n"
    "buffer_load_dwordx4 v22, s[40:43], s19 offen offset:2048 sc0 sc1 lds\n"
    "buffer_load_dwordx4 v22, s[40:43], s19 offen offset:3072 sc0 sc1 lds\n"
    "s_add_u32 s84, s84, 1\n"
    "s_min_u32 s94, s85, s96\n"
    "s_cmp_lt_u32 s84, s94\n"
    "s_cbranch_scc1 L_req2a%=\n"
    "s_mov_b32 m0, s31\n"
    "s_sub_u32 s94, s84, s95\n"
    "s_cmp_gt_u32 s94, 50\n"
    "s_cbranch_scc0 L_w2a%=\n"
    "s_setpc_b64 s[88:89]\n"
    "L_w2a%=:\n"
    "s_waitcnt vmcnt(16) lgkmcnt(0)\n"
    "s_cmp_gt_u32 s94, 2\n"
    "s_cbranch_scc1 L_w0a%=\n"
    "s_waitcnt vmcnt(8)\n"
    "s_cmp_eq_u32 s94, 2\n"
    "s_cbranch_scc1 L_w0a%=\n"
    "s_sleep 6\n"
    "L_landa%=:\n"
    "s_waitcnt lgkmcnt(0)\n"
    "global_load_dword v32, v24, s[72:73] sc0 sc1\n"
    "s_waitcnt vmcnt(0)\n"
    "v_cmp_le_u32 vcc, s17, v32\n"
    "s_ff0_i32_b32 s85, vcc_lo\n"
    "s_max_u32 s85, s85, s86\n"
    "s_bitcmp1_b32 vcc_hi, 24\n"
    "s_cselect_b32 s94, 0x7fc00000, 0\n"
    "v_mov_b32 v33, s94\n"
    "s_min_u32 s94, s85, s96\n"
    "s_cmp_lt_u32 s84, s94\n"
    "s_cbranch_scc1 L_catch2a%=\n"
    "s_setpc_b64 s[88:89]\n"
    "L_w0a%=:\n"
    "s_setpc_b64 s[88:89]\n"
    "L_gatea%=:\n"
    "s_sleep 4\n"
    "global_load_dword v37, v26, s[72:73] sc0 sc1\n"
    "global_load_dword v32, v24, s[72:73] sc0 sc1\n"
    "s_waitcnt vmcnt(0)\n"
    "v_cmp_le_u32 vcc, s17, v32\n"
    "s_ff0_i32_b32 s85, vcc_lo\n"
    "s_max_u32 s85, s85, s86\n"
    "s_bitcmp1_b32 vcc_hi, 24\n"
    "s_cselect_b32 s94, 0x7fc00000, 0\n"
    "v_mov_b32 v33, s94\n"
    "v_cmp_gt_u32 s[28:29], s17, v37\n"
    "s_cmp_lg_u64 s[28:29], 0\n"
    "s_cbranch_scc0 L_rga%=\n"
    "s_memrealtime s[92:93]\n"
    "s_bitcmp1_b32 vcc_hi, 24\n"
    "s_cbranch_scc1 L_gga%=\n"
    "s_waitcnt lgkmcnt(0)\n"
    "s_sub_u32 s92, s92, s20\n"
    "s_cmp_lt_u32 s92, 50000000\n"
    "s_cbranch_scc1 L_gatea%=\n"
    "L_gga%=:\n"
    "s_waitcnt lgkmcnt(0)\n"
    "global_store_dword v25, v36, s[72:73] sc1\n"
    "s_mov_b32 s86, 16\n"
    "s_waitcnt vmcnt(0)\n"
    "global_load_dword v32, v24, s[72:73] sc0 sc1\n"
    "s_waitcnt vmcnt(0)\n"
    "v_cmp_le_u32 vcc, s17, v32\n"
    "s_ff0_i32_b32 s85, vcc_lo\n"
    "s_max_u32 s85, s85, s86\n"
    "s_bitcmp1_b32 vcc_hi, 24\n"
    "s_cselect_b32 s94, 0x7fc00000, 0\n"
    "v_mov_b32 v33, s94\n"
    "s_branch L_rga%=\n"
    "L_slowqa%=:\n"
    "global_load_dword v32, v24, s[72:73] sc0 sc1\n"
    "buffer_load_dword v34, v35, s[44:47], s33 offen offset:2044 sc0 sc1\n"
    "s_waitcnt vmcnt(0)\n"
    "v_cmp_le_u32 vcc, s17, v32\n"
    "s_ff0_i32_b32 s85, vcc_lo\n"
    "s_max_u32 s85, s85, s86\n"
    "s_bitcmp1_b32 vcc_hi, 24\n"
    "s_cselect_b32 s94, 0x7fc00000, 0\n"
    "v_mov_b32 v33, s94\n"
    "v_add_u32 v34, v34, v8\n"
    "v_cvt_f32_i32 v34, v34\n"
    "v_cmp_lt_f32 s[28:29], s97, |v34|\n"
    "s_memrealtime s[92:93]\n"
    "s_bitcmp1_b32 vcc_hi, 24\n"
    "s_cbranch_scc1 L_qga%=\n"
    "s_waitcnt lgkmcnt(0)\n"
    "s_sub_u32 s92, s92, s20\n"
    "s_cmp_lt_u32 s92, 50000000\n"
    "s_cbranch_scc1 L_qba%=\n"
    "L_qga%=:\n"
    "s_waitcnt lgkmcnt(0)\n"
    "global_store_dword v25, v36, s[72:73] sc1\n"
    "s_mov_b32 s86, 16\n"
    "s_waitcnt vmcnt(0)\n"
    "global_load_dword v32, v24, s[72:73] sc0 sc1\n"
    "s_waitcnt vmcnt(0)\n"
    "v_cmp_le_u32 vcc, s17, v32\n"
    "s_ff0_i32_b32 s85, vcc_lo\n"
    "s_max_u32 s85, s85, s86\n"
    "s_bitcmp1_b32 vcc_hi, 24\n"
    "s_cselect_b32 s94, 0x7fc00000, 0\n"
    "v_mov_b32 v33, s94\n"
    "s_cmp_eq_u32 0, 0\n"
    "s_setpc_b64 s[88:89]\n"
    "L_qba%=:\n"
    "s_cmp_lg_u64 s[28:29], 0\n"
    "s_cbranch_scc0 L_qca%=\n"
    "s_sleep 4\n"
    "s_branch L_slowqa%=\n"
    "L_qca%=:\n"
    "s_setpc_b64 s[88:89]\n"
    "L_progb%=:\n"
    "s_mov_b32 s81, 0\n"
    "s_mov_b32 s25, -1\n"
    "v_and_b32 v4, 3, v1\n"
    "v_lshlrev_b32 v4, 9, v4\n"
    "v_add_u32 v26, 263068, v4\n"
    "s_movk_i32 s35, 2048\n"
    "s_cmp_lt_u32 s3, 3\n"
    "s_cselect_b32 s87, s35, 524288\n"
    "s_cselect_b32 s94, s35, 8192\n"
    "s_cselect_b32 s11, s16, 6144\n"
    "s_cselect_b32 s35, 6, 0\n"
    "s_lshl_b32 s8, s11, s35\n"
    "s_add_u32 s9, s8, s87\n"
    "s_add_u32 s12, s11, s94\n"
    "s_add_u32 s10, s9, s87\n"
    "s_add_u32 s13, s12, s94\n"
    "s_mov_b32 s8, s59\n"
    "s_mov_b32 s9, s60\n"
    "s_add_u32 s87, s9, 2048\n"
    "s_sub_u32 s94, 1591296, s8\n"
    "s_sub_u32 s94, s94, s9\n"
    "s_cmp_lt_u32 s3, 3\n"
    "s_cselect_b32 s10, s87, s94\n"
    "s_cmp_lt_u32 s3, 3\n"
    "s_cselect_b32 s94, 0, 512\n"
    "s_mov_b32 s42, 1179648\n"
    "v_lshrrev_b32 v4, 4, v1\n"
    "v_mul_u32_u24 v4, s94, v4\n"
    "v_and_b32 v5, 15, v1\n"
    "v_lshl_add_u32 v4, v5, 2, v4\n"
    "v_add_u32 v4, s67, v4\n"
    "v_mov_b32 v25, 8192\n"
    "v_cmp_gt_u32 vcc, 48, v1\n"
    "s_mov_b32 s98, 0x10432104\n"
    "s_mov_b32 s99, 0x432\n"
    "v_cndmask_b32 v24, v25, v4, vcc\n"
    "buffer_load_dword v12, v12, s[36:39], s95 offen offset:384\n"
    "global_load_dword v3, v2, s[72:73] sc1\n"
    "global_load_dword v14, v35, s[74:75]\n"
    "global_load_dword v15, v35, s[76:77]\n"
    "s_waitcnt vmcnt(12) lgkmcnt(0)\n"
    "s_barrier\n"
    "s_add_u32 m0, s16, 65536\n"
    "s_add_u32 s35, s55, 131072\n"
    "buffer_load_dwordx4 v16, s[36:39], s55 offen lds\n"
    "buffer_load_dwordx4 v17, s[36:39], s55 offen offset:1024 lds\n"
    "buffer_load_dwordx4 v18, s[36:39], s55 offen offset:2048 lds\n"
    "buffer_load_dwordx4 v19, s[36:39], s55 offen offset:3072 lds\n"
    "s_add_u32 m0, s16, 69632\n"
    "s_nop 0\n"
    "buffer_load_dwordx4 v16, s[36:39], s35 offen lds\n"
    "buffer_load_dwordx4 v17, s[36:39], s35 offen offset:1024 lds\n"
    "buffer_load_dwordx4 v18, s[36:39], s35 offen offset:2048 lds\n"
    "buffer_load_dwordx4 v19, s[36:39], s35 offen offset:3072 lds\n"
    "ds_read_b64_tr_b8 a[0:1], v44 offset:0\n"
    "ds_read_b64_tr_b8 a[2:3], v44 offset:1024\n"
    "ds_read_b64_tr_b8 a[4:5], v45 offset:0\n"
    "ds_read_b64_tr_b8 a[6:7], v45 offset:1024\n"
    "ds_read_b64_tr_b8 a[8:9], v44 offset:8192\n"
    "ds_read_b64_tr_b8 a[10:11], v44 offset:9216\n"
    "ds_read_b64_tr_b8 a[12:13], v45 offset:8192\n"
    "ds_read_b64_tr_b8 a[14:15], v45 offset:9216\n"
    "ds_read_b64_tr_b8 a[16:17], v44 offset:16384\n"
    "ds_read_b64_tr_b8 a[18:19], v44 offset:17408\n"
    "ds_read_b64_tr_b8 a[20:21], v45 offset:16384\n"
    "ds_read_b64_tr_b8 a[22:23], v45 offset:17408\n"
    "ds_read_b64_tr_b8 a[24:25], v44 offset:24576\n"
    "ds_read_b64_tr_b8 a[26:27], v44 offset:25600\n"
    "ds_read_b64_tr_b8 a[28:29], v45 offset:24576\n"
    "ds_read_b64_tr_b8 a[30:31], v45 offset:25600\n"
    "s_waitcnt vmcnt(12) lgkmcnt(0)\n"
    "s_barrier\n"
    "s_add_u32 m0, s16, 98304\n"
    "s_add_u32 s35, s56, 131072\n"
    "buffer_load_dwordx4 v16, s[36:39], s56 offen lds\n"
    "buffer_load_dwordx4 v17, s[36:39], s56 offen offset:1024 lds\n"
    "buffer_load_dwordx4 v18, s[36:39], s56 offen offset:2048 lds\n"
    "buffer_load_dwordx4 v19, s[36:39], s56 offen offset:3072 lds\n"
    "s_add_u32 m0, s16, 102400\n"
    "s_nop 0\n"
    "buffer_load_dwordx4 v16, s[36:39], s35 offen lds\n"
    "buffer_load_dwordx4 v17, s[36:39], s35 offen offset:1024 lds\n"
    "buffer_load_dwordx4 v18, s[36:39], s35 offen offset:2048 lds\n"
    "buffer_load_dwordx4 v19, s[36:39], s35 offen offset:3072 lds\n"
    "ds_read_b64_tr_b8 a[32:33], v44 offset:32768\n"
    "ds_read_b64_tr_b8 a[34:35], v44 offset:33792\n"
    "ds_read_b64_tr_b8 a[36:37], v45 offset:32768\n"
    "ds_read_b64_tr_b8 a[38:39], v45 offset:33792\n"
    "ds_read_b64_tr_b8 a[40:41], v44 offset:40960\n"
    "ds_read_b64_tr_b8 a[42:43], v44 offset:41984\n"
    "ds_read_b64_tr_b8 a[44:45], v45 offset:40960\n"
    "ds_read_b64_tr_b8 a[46:47], v45 offset:41984\n"
    "ds_read_b64_tr_b8 a[48:49], v44 offset:49152\n"
    "ds_read_b64_tr_b8 a[50:51], v44 offset:50176\n"
    "ds_read_b64_tr_b8 a[52:53], v45 offset:49152\n"
    "ds_read_b64_tr_b8 a[54:55], v45 offset:50176\n"
    "ds_read_b64_tr_b8 a[56:57], v44 offset:57344\n"
    "ds_read_b64_tr_b8 a[58:59], v44 offset:58368\n"
    "ds_read_b64_tr_b8 a[60:61], v45 offset:57344\n"
    "ds_read_b64_tr_b8 a[62:63], v45 offset:58368\n"
    "s_waitcnt vmcnt(8) lgkmcnt(0)\n"
    "s_barrier\n"
    "s_add_u32 m0, s16, 131072\n"
    "s_add_u32 s35, s52, 131072\n"
    "buffer_load_dwordx4 v16, s[36:39], s52 offen lds\n"
    "buffer_load_dwordx4 v17, s[36:39], s52 offen offset:1024 lds\n"
    "buffer_load_dwordx4 v18, s[36:39], s52 offen offset:2048 lds\n"
    "buffer_load_dwordx4 v19, s[36:39], s52 offen offset:3072 lds\n"
    "s_add_u32 m0, s16, 135168\n"
    "s_nop 0\n"
    "buffer_load_dwordx4 v16, s[36:39], s35 offen lds\n"
    "buffer_load_dwordx4 v17, s[36:39], s35 offen offset:1024 lds\n"
    "buffer_load_dwordx4 v18, s[36:39], s35 offen offset:2048 lds\n"
    "buffer_load_dwordx4 v19, s[36:39], s35 offen offset:3072 lds\n"
    "ds_read_b64_tr_b8 a[64:65], v46 offset:0\n"
    "ds_read_b64_tr_b8 a[66:67], v46 offset:1024\n"
    "ds_read_b64_tr_b8 a[68:69], v47 offset:0\n"
    "ds_read_b64_tr_b8 a[70:71], v47 offset:1024\n"
    "ds_read_b64_tr_b8 a[72:73], v46 offset:8192\n"
    "ds_read_b64_tr_b8 a[74:75], v46 offset:9216\n"
    "ds_read_b64_tr_b8 a[76:77], v47 offset:8192\n"
    "ds_read_b64_tr_b8 a[78:79], v47 offset:9216\n"
    "ds_read_b64_tr_b8 a[80:81], v46 offset:16384\n"
    "ds_read_b64_tr_b8 a[82:83], v46 offset:17408\n"
    "ds_read_b64_tr_b8 a[84:85], v47 offset:16384\n"
    "ds_read_b64_tr_b8 a[86:87], v47 offset:17408\n"
    "ds_read_b64_tr_b8 a[88:89], v46 offset:24576\n"
    "ds_read_b64_tr_b8 a[90:91], v46 offset:25600\n"
    "ds_read_b64_tr_b8 a[92:93], v47 offset:24576\n"
    "ds_read_b64_tr_b8 a[94:95], v47 offset:25600\n"
    "s_waitcnt vmcnt(24)\n"
    "v_add_u32 v36, 1, v3\n"
    "v_mul_f32 v14, v14, v15\n"
    "s_mov_b32 s97, 0x4d000000\n"
    "v_readfirstlane_b32 s17, v36\n"
    "v_readfirstlane_b32 s18, v14\n"
    "s_and_b32 s87, s17, 1\n"
    "s_lshl_b32 s87, s87, 22\n"
    "s_add_u32 s87, s87, s66\n"
    "s_add_u32 s40, s40, s87\n"
    "s_addc_u32 s41, s41, 0\n"
    "s_mul_hi_u32 s87, s17, 0x88888889\n"
    "s_lshr_b32 s87, s87, 3\n"
    "s_mul_i32 s87, s87, 15\n"
    "s_sub_u32 s87, s17, s87\n"
    "s_add_u32 s87, s87, 1\n"
    "s_lshl_b32 s87, s87, 28\n"
    "s_sub_u32 s94, 0, s87\n"
    "v_mov_b32 v4, s87\n"
    "v_mov_b32 v8, s94\n"
    "v_mov_b32 v5, s87\n"
    "v_mov_b32 v9, s94\n"
    "v_mov_b32 v6, s87\n"
    "v_mov_b32 v10, s94\n"
    "v_mov_b32 v7, s87\n"
    "v_mov_b32 v11, s94\n"
    "s_waitcnt vmcnt(8) lgkmcnt(0)\n"
    "s_barrier\n"
    "s_cmp_lg_u32 s3, 0\n"
    "s_cbranch_scc1 L_nocntb%=\n"
    "s_mov_b64 exec, 1\n"
    "global_store_dword v2, v36, s[72:73] sc1\n"
    "s_mov_b64 exec, -1\n"
    "L_nocntb%=:\n"
    "s_add_u32 m0, s14, 0\n"
    "s_nop 0\n"
    "buffer_load_dwordx4 v27, s[68:71], s61 offen lds\n"
    "buffer_load_dwordx4 v28, s[68:71], s61 offen offset:1024 lds\n"
    "buffer_load_dwordx4 v29, s[68:71], s61 offen offset:2048 lds\n"
    "buffer_load_dwordx4 v30, s[68:71], s61 offen offset:3072 lds\n"
    "s_add_u32 m0, s15, 0\n"
    "s_nop 0\n"
    "buffer_load_dwordx4 v27, s[68:71], s62 offen lds\n"
    "buffer_load_dwordx4 v28, s[68:71], s62 offen offset:1024 lds\n"
    "buffer_load_dwordx4 v29, s[68:71], s62 offen offset:2048 lds\n"
    "buffer_load_dwordx4 v30, s[68:71], s62 offen offset:3072 lds\n"
    "ds_read_b64_tr_b8 a[96:97], v46 offset:32768\n"
    "ds_read_b64_tr_b8 a[98:99], v46 offset:33792\n"
    "ds_read_b64_tr_b8 a[100:101], v47 offset:32768\n"
    "ds_read_b64_tr_b8 a[102:103], v47 offset:33792\n"
    "ds_read_b64_tr_b8 a[104:105], v46 offset:40960\n"
    "ds_read_b64_tr_b8 a[106:107], v46 offset:41984\n"
    "ds_read_b64_tr_b8 a[108:109], v47 offset:40960\n"
    "ds_read_b64_tr_b8 a[110:111], v47 offset:41984\n"
    "ds_read_b64_tr_b8 a[112:113], v46 offset:49152\n"
    "ds_read_b64_tr_b8 a[114:115], v46 offset:50176\n"
    "ds_read_b64_tr_b8 a[116:117], v47 offset:49152\n"
    "ds_read_b64_tr_b8 a[118:119], v47 offset:50176\n"
    "ds_read_b64_tr_b8 a[120:121], v46 offset:57344\n"
    "ds_read_b64_tr_b8 a[122:123], v46 offset:58368\n"
    "ds_read_b64_tr_b8 a[124:125], v47 offset:57344\n"
    "ds_read_b64_tr_b8 a[126:127], v47 offset:58368\n"
    "s_waitcnt vmcnt(8) lgkmcnt(0)\n"
    "s_barrier\n"
    "s_add_u32 m0, s16, 32768\n"
    "s_nop 0\n"
    "buffer_load_dwordx4 v27, s[68:71], s63 offen lds\n"
    "buffer_load_dwordx4 v28, s[68:71], s63 offen offset:1024 lds\n"
    "buffer_load_dwordx4 v29, s[68:71], s63 offen offset:2048 lds\n"
    "buffer_load_dwordx4 v30, s[68:71], s63 offen offset:3072 lds\n"
    "ds_read_b64_tr_b8 a[192:193], v48 offset:0\n"
    "ds_read_b64_tr_b8 a[194:195], v48 offset:1024\n"
    "ds_read_b64_tr_b8 a[196:197], v49 offset:0\n"
    "ds_read_b64_tr_b8 a[198:199], v49 offset:1024\n"
    "ds_read_b64_tr_b8 a[200:201], v48 offset:8192\n"
    "ds_read_b64_tr_b8 a[202:203], v48 offset:9216\n"
    "ds_read_b64_tr_b8 a[204:205], v49 offset:8192\n"
    "ds_read_b64_tr_b8 a[206:207], v49 offset:9216\n"
    "ds_read_b64_tr_b8 a[208:209], v48 offset:16384\n"
    "ds_read_b64_tr_b8 a[210:211], v48 offset:17408\n"
    "ds_read_b64_tr_b8 a[212:213], v49 offset:16384\n"
    "ds_read_b64_tr_b8 a[214:215], v49 offset:17408\n"
    "ds_read_b64_tr_b8 a[216:217], v48 offset:24576\n"
    "ds_read_b64_tr_b8 a[218:219], v48 offset:25600\n"
    "ds_read_b64_tr_b8 a[220:221], v49 offset:24576\n"
    "ds_read_b64_tr_b8 a[222:223], v49 offset:25600\n"
    "s_waitcnt vmcnt(4) lgkmcnt(0)\n"
    "s_barrier\n"
    "ds_read_b128 v[52:55], v38\n"
    "ds_read_b128 v[56:59], v39\n"
    "ds_read_b128 v[60:63], v38 offset:2048\n"
    "ds_read_b128 v[64:67], v39 offset:2048\n"
    "ds_read_b128 v[68:71], v38 offset:4096\n"
    "ds_read_b128 v[72:75], v39 offset:4096\n"
    "ds_read_b128 v[76:79], v38 offset:6144\n"
    "ds_read_b128 v[80:83], v39 offset:6144\n"
    "s_waitcnt lgkmcnt(4)\n"
    "v_mfma_i32_16x16x64_i8 v[116:119], a[192:195], v[52:55], v[4:7]\n"
    "ds_read_b128 v[84:87], v38 offset:8192\n"
    "s_add_u32 m0, s16, 65536\n"
    "s_add_u32 s35, s57, 131072\n"
    "buffer_load_dwordx4 v16, s[36:39], s57 offen lds\n"
    "v_mfma_i32_16x16x64_i8 v[116:119], a[200:203], v[56:59], v[116:119]\n"
    "v_mfma_i32_16x16x64_i8 v[116:119], a[208:211], v[60:63], v[116:119]\n"
    "ds_read_b128 v[88:91], v39 offset:8192\n"
    "v_mfma_i32_16x16x64_i8 v[116:119], a[216:219], v[64:67], v[116:119]\n"
    "buffer_load_dwordx4 v17, s[36:39], s57 offen offset:1024 lds\n"
    "v_mfma_i32_16x16x64_i8 v[120:123], a[196:199], v[52:55], v[4:7]\n"
    "ds_read_b128 v[92:95], v38 offset:10240\n"
    "v_mfma_i32_16x16x64_i8 v[120:123], a[204:207], v[56:59], v[120:123]\n"
    "v_mfma_i32_16x16x64_i8 v[120:123], a[212:215], v[60:63], v[120:123]\n"
    "ds_read_b128 v[96:99], v39 offset:10240\n"
    "v_mfma_i32_16x16x64_i8 v[120:123], a[220:223], v[64:67], v[120:123]\n"
    "s_waitcnt lgkmcnt(4)\n"
    "buffer_load_dwordx4 v18, s[36:39], s57 offen offset:2048 lds\n"
    "v_mfma_i32_16x16x64_i8 v[128:131], a[196:199], v[68:71], v[4:7]\n"
    "ds_read_b128 v[100:103], v38 offset:12288\n"
    "v_mfma_i32_16x16x64_i8 v[128:131], a[204:207], v[72:75], v[128:131]\n"
    "buffer_load_dwordx4 v19, s[36:39], s57 offen offset:3072 lds\n"
    "v_mfma_i32_16x16x64_i8 v[128:131], a[212:215], v[76:79], v[128:131]\n"
    "ds_read_b128 v[104:107], v39 offset:12288\n"
    "v_mfma_i32_16x16x64_i8 v[128:131], a[220:223], v[80:83], v[128:131]\n"
    "s_add_u32 m0, s16, 69632\n"
    "v_mfma_i32_16x16x64_i8 v[124:127], a[192:195], v[68:71], v[4:7]\n"
    "ds_read_b128 v[108:111], v38 offset:14336\n"
    "v_mfma_i32_16x16x64_i8 v[124:127], a[200:203], v[72:75], v[124:127]\n"
    "buffer_load_dwordx4 v16, s[36:39], s35 offen lds\n"
    "v_mfma_i32_16x16x64_i8 v[124:127], a[208:211], v[76:79], v[124:127]\n"
    "ds_read_b128 v[112:115], v39 offset:14336\n"
    "v_mfma_i32_16x16x64_i8 v[124:127], a[216:219], v[80:83], v[124:127]\n"
    "s_waitcnt lgkmcnt(4)\n"
    "buffer_load_dwordx4 v17, s[36:39], s35 offen offset:1024 lds\n"
    "v_mfma_i32_16x16x64_i8 v[132:135], a[192:195], v[84:87], v[4:7]\n"
    "ds_read_b128 v[52:55], v38 offset:16384\n"
    "v_mfma_i32_16x16x64_i8 v[132:135], a[200:203], v[88:91], v[132:135]\n"
    "v_mfma_i32_16x16x64_i8 v[132:135], a[208:211], v[92:95], v[132:135]\n"
    "ds_read_b128 v[56:59], v39 offset:16384\n"
    "v_mfma_i32_16x16x64_i8 v[132:135], a[216:219], v[96:99], v[132:135]\n"
    "buffer_load_dwordx4 v18, s[36:39], s35 offen offset:2048 lds\n"
    "v_mfma_i32_16x16x64_i8 v[136:139], a[196:199], v[84:87], v[4:7]\n"
    "ds_read_b128 v[60:63], v38 offset:18432\n"
    "v_mfma_i32_16x16x64_i8 v[136:139], a[204:207], v[88:91], v[136:139]\n"
    "buffer_load_dwordx4 v19, s[36:39], s35 offen offset:3072 lds\n"
    "v_mfma_i32_16x16x64_i8 v[136:139], a[212:215], v[92:95], v[136:139]\n"
    "ds_read_b128 v[64:67], v39 offset:18432\n"
    "v_mfma_i32_16x16x64_i8 v[136:139], a[220:223], v[96:99], v[136:139]\n"
    "s_waitcnt lgkmcnt(4)\n"
    "v_mfma_i32_16x16x64_i8 v[144:147], a[196:199], v[100:103], v[4:7]\n"
    "ds_read_b128 v[68:71], v38 offset:20480\n"
    "v_mfma_i32_16x16x64_i8 v[144:147], a[204:207], v[104:107], v[144:147]\n"
    "global_load_dword v32, v24, s[72:73] sc0 sc1\n"
    "s_add_u32 m0, s16, 98304\n"
    "v_mfma_i32_16x16x64_i8 v[144:147], a[212:215], v[108:111], v[144:147]\n"
    "ds_read_b128 v[72:75], v39 offset:20480\n"
    "s_add_u32 s35, s58, 131072\n"
    "v_mfma_i32_16x16x64_i8 v[144:147], a[220:223], v[112:115], v[144:147]\n"
    "buffer_load_dwordx4 v16, s[36:39], s58 offen lds\n"
    "v_mfma_i32_16x16x64_i8 v[140:143], a[192:195], v[100:103], v[4:7]\n"
    "ds_read_b128 v[76:79], v38 offset:22528\n"
    "v_mfma_i32_16x16x64_i8 v[140:143], a[200:203], v[104:107], v[140:143]\n"
    "v_mfma_i32_16x16x64_i8 v[140:143], a[208:211], v[108:111], v[140:143]\n"
    "ds_read_b128 v[80:83], v39 offset:22528\n"
    "v_mfma_i32_16x16x64_i8 v[140:143], a[216:219], v[112:115], v[140:143]\n"
    "s_waitcnt lgkmcnt(4)\n"
    "buffer_load_dwordx4 v17, s[36:39], s58 offen offset:1024 lds\n"
    "v_mfma_i32_16x16x64_i8 v[148:151], a[192:195], v[52:55], v[4:7]\n"
    "ds_read_b128 v[84:87], v38 offset:24576\n"
    "v_mfma_i32_16x16x64_i8 v[148:151], a[200:203], v[56:59], v[148:151]\n"
    "buffer_load_dwordx4 v18, s[36:39], s58 offen offset:2048 lds\n"
    "v_mfma_i32_16x16x64_i8 v[148:151], a[208:211], v[60:63], v[148:151]\n"
    "ds_read_b128 v[88:91], v39 offset:24576\n"
    "v_mfma_i32_16x16x64_i8 v[148:151], a[216:219], v[64:67], v[148:151]\n"
    "v_mfma_i32_16x16x64_i8 v[152:155], a[196:199], v[52:55], v[4:7]\n"
    "ds_read_b128 v[92:95], v38 offset:26624\n"
    "v_mfma_i32_16x16x64_i8 v[152:155], a[204:207], v[56:59], v[152:155]\n"
    "buffer_load_dwordx4 v19, s[36:39], s58 offen offset:3072 lds\n"
    "s_add_u32 m0, s16, 102400\n"
    "v_mfma_i32_16x16x64_i8 v[152:155], a[212:215], v[60:63], v[152:155]\n"
    "ds_read_b128 v[96:99], v39 offset:26624\n"
    "v_mfma_i32_16x16x64_i8 v[152:155], a[220:223], v[64:67], v[152:155]\n"
    "s_waitcnt lgkmcnt(4)\n"
    "buffer_load_dwordx4 v16, s[36:39], s35 offen lds\n"
    "v_mfma_i32_16x16x64_i8 v[160:163], a[196:199], v[68:71], v[4:7]\n"
    "ds_read_b128 v[100:103], v38 offset:28672\n"
    "v_mfma_i32_16x16x64_i8 v[160:163], a[204:207], v[72:75], v[160:163]\n"
    "v_mfma_i32_16x16x64_i8 v[160:163], a[212:215], v[76:79], v[160:163]\n"
    "ds_read_b128 v[104:107], v39 offset:28672\n"
    "v_mfma_i32_16x16x64_i8 v[160:163], a[220:223], v[80:83], v[160:163]\n"
    "buffer_load_dwordx4 v17, s[36:39], s35 offen offset:1024 lds\n"
    "v_mfma_i32_16x16x64_i8 v[156:159], a[192:195], v[68:71], v[4:7]\n"
    "ds_read_b128 v[108:111], v38 offset:30720\n"
    "v_mfma_i32_16x16x64_i8 v[156:159], a[200:203], v[72:75], v[156:159]\n"
    "buffer_load_dwordx4 v18, s[36:39], s35 offen offset:2048 lds\n"
    "v_mfma_i32_16x16x64_i8 v[156:159], a[208:211], v[76:79], v[156:159]\n"
    "ds_read_b128 v[112:115], v39 offset:30720\n"
    "v_mfma_i32_16x16x64_i8 v[156:159], a[216:219], v[80:83], v[156:159]\n"
    "s_waitcnt lgkmcnt(4)\n"
    "v_mfma_i32_16x16x64_i8 v[164:167], a[192:195], v[84:87], v[8:11]\n"
    "v_mfma_i32_16x16x64_i8 v[164:167], a[200:203], v[88:91], v[164:167]\n"
    "buffer_load_dwordx4 v19, s[36:39], s35 offen offset:3072 lds\n"
    "v_mfma_i32_16x16x64_i8 v[164:167], a[208:211], v[92:95], v[164:167]\n"
    "v_mfma_i32_16x16x64_i8 v[164:167], a[216:219], v[96:99], v[164:167]\n"
    "s_waitcnt vmcnt(17) lgkmcnt(0)\n"
    "s_barrier\n"
    "v_mfma_i32_16x16x64_i8 v[168:171], a[196:199], v[84:87], v[8:11]\n"
    "ds_read_b128 v[52:55], v38 offset:32768\n"
    "v_mfma_i32_16x16x64_i8 v[168:171], a[204:207], v[88:91], v[168:171]\n"
    "ds_read_b128 v[56:59], v39 offset:32768\n"
    "v_mfma_i32_16x16x64_i8 v[168:171], a[212:215], v[92:95], v[168:171]\n"
    "ds_read_b128 v[60:63], v38 offset:34816\n"
    "v_mfma_i32_16x16x64_i8 v[168:171], a[220:223], v[96:99], v[168:171]\n"
    "ds_read_b128 v[64:67], v39 offset:34816\n"
    "v_mfma_i32_16x16x64_i8 v[176:179], a[196:199], v[100:103], v[8:11]\n"
    "ds_read_b128 v[68:71], v38 offset:40960\n"
    "v_mfma_i32_16x16x64_i8 v[176:179], a[204:207], v[104:107], v[176:179]\n"
    "ds_read_b128 v[72:75], v39 offset:40960\n"
    "v_mfma_i32_16x16x64_i8 v[176:179], a[212:215], v[108:111], v[176:179]\n"
    "ds_read_b128 v[76:79], v38 offset:43008\n"
    "v_mfma_i32_16x16x64_i8 v[176:179], a[220:223], v[112:115], v[176:179]\n"
    "ds_read_b128 v[80:83], v39 offset:43008\n"
    "v_mfma_i32_16x16x64_i8 v[172:175], a[192:195], v[100:103], v[8:11]\n"
    "v_mfma_i32_16x16x64_i8 v[172:175], a[200:203], v[104:107], v[172:175]\n"
    "v_mfma_i32_16x16x64_i8 v[172:175], a[208:211], v[108:111], v[172:175]\n"
    "v_mfma_i32_16x16x64_i8 v[172:175], a[216:219], v[112:115], v[172:175]\n"
    "s_waitcnt lgkmcnt(4)\n"
    "v_mfma_i32_16x16x64_i8 v[180:183], a[192:195], v[52:55], v[8:11]\n"
    "s_waitcnt vmcnt(8)\n"
    "ds_read_b128 v[84:87], v38 offset:49152\n"
    "v_mfma_i32_16x16x64_i8 v[180:183], a[200:203], v[56:59], v[180:183]\n"
    "v_cmp_le_u32 vcc, s17, v32\n"
    "s_lshr_b32 s87, vcc_lo, 16\n"
    "v_mfma_i32_16x16x64_i8 v[180:183], a[208:211], v[60:63], v[180:183]\n"
    "ds_read_b128 v[88:91], v39 offset:49152\n"
    "s_and_b32 s87, s87, vcc_lo\n"
    "s_and_b32 s87, s87, vcc_hi\n"
    "v_mfma_i32_16x16x64_i8 v[180:183], a[216:219], v[64:67], v[180:183]\n"
    "s_ff0_i32_b32 s85, s87\n"
    "s_max_u32 s85, s85, s86\n"
    "v_mfma_i32_16x16x64_i8 v[184:187], a[196:199], v[52:55], v[8:11]\n"
    "ds_read_b128 v[92:95], v38 offset:51200\n"
    "s_cmp_eq_u32 s84, 0\n"
    "s_cselect_b32 s91, s85, 0\n"
    "v_mfma_i32_16x16x64_i8 v[184:187], a[204:207], v[56:59], v[184:187]\n"
    "s_cmp_gt_u32 s91, 0\n"
    "s_cselect_b32 s82, 0, s83\n"
    "s_addc_u32 s84, s84, 0\n"
    "s_add_u32 m0, s11, 131072\n"
    "v_mfma_i32_16x16x64_i8 v[184:187], a[212:215], v[60:63], v[184:187]\n"
    "ds_read_b128 v[96:99], v39 offset:51200\n"
    "s_add_u32 s34, s8, s82\n"
    "v_mfma_i32_16x16x64_i8 v[184:187], a[220:223], v[64:67], v[184:187]\n"
    "s_waitcnt lgkmcnt(4)\n"
    "buffer_load_dwordx4 v22, s[40:43], s34 offen sc0 sc1 lds\n"
    "v_mfma_i32_16x16x64_i8 v[192:195], a[196:199], v[68:71], v[8:11]\n"
    "buffer_load_dwordx4 v22, s[40:43], s34 offen offset:1024 sc0 sc1 lds\n"
    "s_add_u32 m0, s12, 131072\n"
    "s_add_u32 s34, s9, s82\n"
    "v_mfma_i32_16x16x64_i8 v[192:195], a[204:207], v[72:75], v[192:195]\n"
    "buffer_load_dwordx4 v22, s[40:43], s34 offen sc0 sc1 lds\n"
    "v_mfma_i32_16x16x64_i8 v[192:195], a[212:215], v[76:79], v[192:195]\n"
    "v_mfma_i32_16x16x64_i8 v[192:195], a[220:223], v[80:83], v[192:195]\n"
    "v_mfma_i32_16x16x64_i8 v[188:191], a[192:195], v[68:71], v[8:11]\n"
    "v_mfma_i32_16x16x64_i8 v[188:191], a[200:203], v[72:75], v[188:191]\n"
    "buffer_load_dwordx4 v22, s[40:43], s34 offen offset:1024 sc0 sc1 lds\n"
    "v_mfma_i32_16x16x64_i8 v[188:191], a[208:211], v[76:79], v[188:191]\n"
    "v_mfma_i32_16x16x64_i8 v[188:191], a[216:219], v[80:83], v[188:191]\n"
    "s_waitcnt lgkmcnt(0)\n"
    "s_add_u32 m0, s13, 131072\n"
    "v_mfma_i32_16x16x64_i8 v[196:199], a[192:195], v[84:87], v[8:11]\n"
    "s_add_u32 s34, s10, s82\n"
    "v_mfma_i32_16x16x64_i8 v[196:199], a[200:203], v[88:91], v[196:199]\n"
    "buffer_load_dwordx4 v22, s[40:43], s34 offen sc0 sc1 lds\n"
    "v_mfma_i32_16x16x64_i8 v[196:199], a[208:211], v[92:95], v[196:199]\n"
    "v_mfma_i32_16x16x64_i8 v[196:199], a[216:219], v[96:99], v[196:199]\n"
    "v_mfma_i32_16x16x64_i8 v[200:203], a[196:199], v[84:87], v[8:11]\n"
    "v_mfma_i32_16x16x64_i8 v[200:203], a[204:207], v[88:91], v[200:203]\n"
    "buffer_load_dwordx4 v22, s[40:43], s34 offen offset:1024 sc0 sc1 lds\n"
    "s_mov_b32 s96, 1\n"
    "v_mfma_i32_16x16x64_i8 v[200:203], a[212:215], v[92:95], v[200:203]\n"
    "v_mfma_i32_16x16x64_i8 v[200:203], a[220:223], v[96:99], v[200:203]\n"
    "s_waitcnt vmcnt(15) lgkmcnt(0)\n"
    "s_barrier\n"
    "global_load_dword v32, v24, s[72:73] sc0 sc1\n"
    "s_cmp_eq_u32 s84, 1\n"
    "s_cselect_b32 s91, s85, 0\n"
    "s_cmp_gt_u32 s91, 1\n"
    "s_cselect_b32 s82, 8192, s83\n"
    "s_addc_u32 s84, s84, 0\n"
    "s_add_u32 m0, s11, 0\n"
    "s_add_u32 s34, s8, s82\n"
    "buffer_load_dwordx4 v22, s[40:43], s34 offen sc0 sc1 lds\n"
    "buffer_load_dwordx4 v22, s[40:43], s34 offen offset:1024 sc0 sc1 lds\n"
    "s_add_u32 m0, s12, 0\n"
    "s_add_u32 s34, s9, s82\n"
    "buffer_load_dwordx4 v22, s[40:43], s34 offen sc0 sc1 lds\n"
    "buffer_load_dwordx4 v22, s[40:43], s34 offen offset:1024 sc0 sc1 lds\n"
    "s_add_u32 m0, s13, 0\n"
    "s_add_u32 s34, s10, s82\n"
    "buffer_load_dwordx4 v22, s[40:43], s34 offen sc0 sc1 lds\n"
    "buffer_load_dwordx4 v22, s[40:43], s34 offen offset:1024 sc0 sc1 lds\n"
    "s_mov_b32 s96, 2\n"
    "ds_read_b64_tr_b8 a[128:129], v46 offset:0\n"
    "ds_read_b64_tr_b8 a[130:131], v46 offset:1024\n"
    "ds_read_b64_tr_b8 a[132:133], v47 offset:0\n"
    "ds_read_b64_tr_b8 a[134:135], v47 offset:1024\n"
    "ds_read_b64_tr_b8 a[136:137], v46 offset:8192\n"
    "ds_read_b64_tr_b8 a[138:139], v46 offset:9216\n"
    "ds_read_b64_tr_b8 a[140:141], v47 offset:8192\n"
    "ds_read_b64_tr_b8 a[142:143], v47 offset:9216\n"
    "ds_read_b64_tr_b8 a[144:145], v46 offset:16384\n"
    "ds_read_b64_tr_b8 a[146:147], v46 offset:17408\n"
    "ds_read_b64_tr_b8 a[148:149], v47 offset:16384\n"
    "ds_read_b64_tr_b8 a[150:151], v47 offset:17408\n"
    "ds_read_b64_tr_b8 a[152:153], v46 offset:24576\n"
    "ds_read_b64_tr_b8 a[154:155], v46 offset:25600\n"
    "ds_read_b64_tr_b8 a[156:157], v47 offset:24576\n"
    "ds_read_b64_tr_b8 a[158:159], v47 offset:25600\n"
    "s_waitcnt vmcnt(13) lgkmcnt(0)\n"
    "s_barrier\n"
    "s_cmp_lt_u32 s84, s96\n"
    "s_getpc_b64 s[88:89]\n"
    "s_cbranch_scc1 L_midb%=\n"
    "ds_read_b64_tr_b8 a[160:161], v46 offset:32768\n"
    "ds_read_b64_tr_b8 a[162:163], v46 offset:33792\n"
    "ds_read_b64_tr_b8 a[164:165], v47 offset:32768\n"
    "ds_read_b64_tr_b8 a[166:167], v47 offset:33792\n"
    "ds_read_b64_tr_b8 a[168:169], v46 offset:40960\n"
    "ds_read_b64_tr_b8 a[170:171], v46 offset:41984\n"
    "ds_read_b64_tr_b8 a[172:173], v47 offset:40960\n"
    "ds_read_b64_tr_b8 a[174:175], v47 offset:41984\n"
    "ds_read_b64_tr_b8 a[176:177], v46 offset:49152\n"
    "ds_read_b64_tr_b8 a[178:179], v46 offset:50176\n"
    "ds_read_b64_tr_b8 a[180:181], v47 offset:49152\n"
    "ds_read_b64_tr_b8 a[182:183], v47 offset:50176\n"
    "ds_read_b64_tr_b8 a[184:185], v46 offset:57344\n"
    "ds_read_b64_tr_b8 a[186:187], v46 offset:58368\n"
    "ds_read_b64_tr_b8 a[188:189], v47 offset:57344\n"
    "ds_read_b64_tr_b8 a[190:191], v47 offset:58368\n"
    "s_mov_b32 s95, 0\n"
    "s_and_b32 s91, s84, s25\n"
    "s_cmp_gt_u32 s91, 0\n"
    "s_getpc_b64 s[88:89]\n"
    "s_cbranch_scc0 L_slowb%=\n"
    "s_waitcnt vmcnt(7) lgkmcnt(0)\n"
    "s_barrier\n"
    "ds_read_b128 v[100:103], v42\n"
    "ds_read_b128 v[104:107], v43\n"
    "ds_read_b128 v[108:111], v42 offset:8192\n"
    "ds_read_b128 v[112:115], v43 offset:8192\n"
    "ds_read_b128 v[52:55], v42 offset:16384\n"
    "ds_read_b128 v[56:59], v43 offset:16384\n"
    "ds_read_b128 v[60:63], v42 offset:2048\n"
    "ds_read_b128 v[64:67], v43 offset:2048\n"
    "s_waitcnt lgkmcnt(5)\n"
    "v_mfma_i32_16x16x64_i8 v[120:123], a[4:7], v[100:103], v[120:123]\n"
    "s_waitcnt vmcnt(6)\n"
    "ds_read_b128 v[68:71], v42 offset:10240\n"
    "v_mfma_i32_16x16x64_i8 v[120:123], a[12:15], v[104:107], v[120:123]\n"
    "v_cmp_le_u32 vcc, s17, v32\n"
    "s_lshr_b32 s87, vcc_lo, 16\n"
    "s_and_b32 s87, s87, vcc_lo\n"
    "v_mfma_i32_16x16x64_i8 v[120:123], a[36:39], v[108:111], v[120:123]\n"
    "ds_read_b128 v[72:75], v43 offset:10240\n"
    "s_and_b32 s87, s87, vcc_hi\n"
    "v_mfma_i32_16x16x64_i8 v[116:119], a[0:3], v[100:103], v[116:119]\n"
    "s_ff0_i32_b32 s85, s87\n"
    "s_max_u32 s85, s85, s86\n"
    "s_min_u32 s91, s85, s96\n"
    "v_mfma_i32_16x16x64_i8 v[116:119], a[8:11], v[104:107], v[116:119]\n"
    "ds_read_b128 v[76:79], v42 offset:18432\n"
    "s_cmp_lt_u32 s84, s91\n"
    "s_getpc_b64 s[88:89]\n"
    "s_cbranch_scc1 L_catchb%=\n"
    "v_mfma_i32_16x16x64_i8 v[116:119], a[32:35], v[108:111], v[116:119]\n"
    "s_waitcnt lgkmcnt(5)\n"
    "global_load_dword v32, v24, s[72:73] sc0 sc1\n"
    "v_mfma_i32_16x16x64_i8 v[116:119], a[40:43], v[112:115], v[116:119]\n"
    "ds_read_b128 v[80:83], v43 offset:18432\n"
    "s_cmp_eq_u32 s84, 2\n"
    "s_cselect_b32 s91, s85, 0\n"
    "v_mfma_i32_16x16x64_i8 v[116:119], a[64:67], v[52:55], v[116:119]\n"
    "s_cmp_gt_u32 s91, 2\n"
    "s_cselect_b32 s82, 16384, s83\n"
    "s_addc_u32 s84, s84, 0\n"
    "s_add_u32 m0, s11, 32768\n"
    "v_mfma_i32_16x16x64_i8 v[116:119], a[72:75], v[56:59], v[116:119]\n"
    "ds_read_b128 v[84:87], v42 offset:4096\n"
    "s_add_u32 s34, s8, s82\n"
    "v_mfma_i32_16x16x64_i8 v[120:123], a[44:47], v[112:115], v[120:123]\n"
    "buffer_load_dwordx4 v22, s[40:43], s34 offen sc0 sc1 lds\n"
    "v_mfma_i32_16x16x64_i8 v[120:123], a[68:71], v[52:55], v[120:123]\n"
    "ds_read_b128 v[88:91], v43 offset:4096\n"
    "v_mfma_i32_16x16x64_i8 v[120:123], a[76:79], v[56:59], v[120:123]\n"
    "s_waitcnt lgkmcnt(5)\n"
    "buffer_load_dwordx4 v22, s[40:43], s34 offen offset:1024 sc0 sc1 lds\n"
    "v_mfma_i32_16x16x64_i8 v[120:123], a[20:23], v[60:63], v[120:123]\n"
    "ds_read_b128 v[92:95], v42 offset:12288\n"
    "v_mfma_i32_16x16x64_i8 v[120:123], a[28:31], v[64:67], v[120:123]\n"
    "s_add_u32 m0, s12, 32768\n"
    "s_add_u32 s34, s9, s82\n"
    "v_mfma_i32_16x16x64_i8 v[120:123], a[52:55], v[68:71], v[120:123]\n"
    "ds_read_b128 v[96:99], v43 offset:12288\n"
    "v_mfma_i32_16x16x64_i8 v[116:119], a[16:19], v[60:63], v[116:119]\n"
    "buffer_load_dwordx4 v22, s[40:43], s34 offen sc0 sc1 lds\n"
    "s_cmp_lt_u32 s84, s96\n"
    "s_getpc_b64 s[88:89]\n"
    "s_cbranch_scc1 L_midb%=\n"
    "v_mfma_i32_16x16x64_i8 v[116:119], a[24:27], v[64:67], v[116:119]\n"
    "ds_read_b128 v[100:103], v42 offset:20480\n"
    "v_mfma_i32_16x16x64_i8 v[116:119], a[48:51], v[68:71], v[116:119]\n"
    "s_waitcnt lgkmcnt(5)\n"
    "v_mfma_i32_16x16x64_i8 v[116:119], a[56:59], v[72:75], v[116:119]\n"
    "ds_read_b128 v[104:107], v43 offset:20480\n"
    "v_mfma_i32_16x16x64_i8 v[116:119], a[80:83], v[76:79], v[116:119]\n"
    "v_mfma_i32_16x16x64_i8 v[116:119], a[88:91], v[80:83], v[116:119]\n"
    "ds_read_b128 v[108:111], v42 offset:6144\n"
    "v_mfma_i32_16x16x64_i8 v[120:123], a[60:63], v[72:75], v[120:123]\n"
    "buffer_load_dwordx4 v22, s[40:43], s34 offen offset:1024 sc0 sc1 lds\n"
    "v_mfma_i32_16x16x64_i8 v[120:123], a[84:87], v[76:79], v[120:123]\n"
    "ds_read_b128 v[112:115], v43 offset:6144\n"
    "v_mfma_i32_16x16x64_i8 v[120:123], a[92:95], v[80:83], v[120:123]\n"
    "s_waitcnt lgkmcnt(5)\n"
    "s_add_u32 m0, s13, 32768\n"
    "v_mfma_i32_16x16x64_i8 v[120:123], a[100:103], v[84:87], v[120:123]\n"
    "ds_read_b128 v[52:55], v42 offset:14336\n"
    "s_add_u32 s34, s10, s82\n"
    "v_mfma_i32_16x16x64_i8 v[120:123], a[108:111], v[88:91], v[120:123]\n"
    "buffer_load_dwordx4 v22, s[40:43], s34 offen sc0 sc1 lds\n"
    "v_mfma_i32_16x16x64_i8 v[120:123], a[132:135], v[92:95], v[120:123]\n"
    "ds_read_b128 v[56:59], v43 offset:14336\n"
    "v_mfma_i32_16x16x64_i8 v[116:119], a[96:99], v[84:87], v[116:119]\n"
    "v_mfma_i32_16x16x64_i8 v[116:119], a[104:107], v[88:91], v[116:119]\n"
    "ds_read_b128 v[60:63], v42 offset:22528\n"
    "v_mfma_i32_16x16x64_i8 v[116:119], a[128:131], v[92:95], v[116:119]\n"
    "s_waitcnt lgkmcnt(5)\n"
    "v_mfma_i32_16x16x64_i8 v[116:119], a[136:139], v[96:99], v[116:119]\n"
    "ds_read_b128 v[64:67], v43 offset:22528\n"
    "v_mfma_i32_16x16x64_i8 v[116:119], a[160:163], v[100:103], v[116:119]\n"
    "buffer_load_dwordx4 v22, s[40:43], s34 offen offset:1024 sc0 sc1 lds\n"
    "s_mov_b32 s96, 3\n"
    "v_mfma_i32_16x16x64_i8 v[116:119], a[168:171], v[104:107], v[116:119]\n"
    "v_mfma_i32_16x16x64_i8 v[120:123], a[140:143], v[96:99], v[120:123]\n"
    "v_mfma_i32_16x16x64_i8 v[120:123], a[164:167], v[100:103], v[120:123]\n"
    "s_mov_b32 s95, 1\n"
    "s_and_b32 s91, s84, s25\n"
    "v_mfma_i32_16x16x64_i8 v[120:123], a[172:175], v[104:107], v[120:123]\n"
    "s_cmp_gt_u32 s91, 1\n"
    "s_getpc_b64 s[88:89]\n"
    "s_cbranch_scc0 L_slowb%=\n"
    "s_waitcnt vmcnt(7) lgkmcnt(0)\n"
    "s_barrier\n"
    "v_mfma_i32_16x16x64_i8 v[120:123], a[116:119], v[108:111], v[120:123]\n"
    "ds_read_b128 v[68:71], v38\n"
    "s_waitcnt vmcnt(6)\n"
    "v_mfma_i32_16x16x64_i8 v[120:123], a[124:127], v[112:115], v[120:123]\n"
    "ds_read_b128 v[72:75], v39\n"
    "v_cmp_le_u32 vcc, s17, v32\n"
    "v_mfma_i32_16x16x64_i8 v[120:123], a[148:151], v[52:55], v[120:123]\n"
    "ds_read_b128 v[76:79], v38 offset:8192\n"
    "v_mfma_i32_16x16x64_i8 v[116:119], a[112:115], v[108:111], v[116:119]\n"
    "ds_read_b128 v[80:83], v39 offset:8192\n"
    "s_lshr_b32 s87, vcc_lo, 16\n"
    "v_mfma_i32_16x16x64_i8 v[116:119], a[120:123], v[112:115], v[116:119]\n"
    "ds_read_b128 v[84:87], v38 offset:16384\n"
    "v_mfma_i32_16x16x64_i8 v[116:119], a[144:147], v[52:55], v[116:119]\n"
    "ds_read_b128 v[88:91], v39 offset:16384\n"
    "s_and_b32 s87, s87, vcc_lo\n"
    "v_mfma_i32_16x16x64_i8 v[116:119], a[152:155], v[56:59], v[116:119]\n"
    "ds_read_b128 v[92:95], v38 offset:2048\n"
    "v_mfma_i32_16x16x64_i8 v[116:119], a[176:179], v[60:63], v[116:119]\n"
    "ds_read_b128 v[96:99], v39 offset:2048\n"
    "s_and_b32 s87, s87, vcc_hi\n"
    "v_mfma_i32_16x16x64_i8 v[116:119], a[184:187], v[64:67], v[116:119]\n"
    "s_ff0_i32_b32 s85, s87\n"
    "v_mfma_i32_16x16x64_i8 v[120:123], a[156:159], v[56:59], v[120:123]\n"
    "s_max_u32 s85, s85, s86\n"
    "v_mfma_i32_16x16x64_i8 v[120:123], a[180:183], v[60:63], v[120:123]\n"
    "s_min_u32 s91, s85, s96\n"
    "v_mfma_i32_16x16x64_i8 v[120:123], a[188:191], v[64:67], v[120:123]\n"
    "s_cmp_lt_u32 s84, s91\n"
    "s_getpc_b64 s[88:89]\n"
    "s_cbranch_scc1 L_catchb%=\n"
    "s_waitcnt lgkmcnt(5)\n"
    "v_mfma_i32_16x16x64_i8 v[128:131], a[4:7], v[68:71], v[128:131]\n"
    "global_load_dword v32, v24, s[72:73] sc0 sc1\n"
    "ds_read_b128 v[100:103], v38 offset:10240\n"
    "v_mfma_i32_16x16x64_i8 v[128:131], a[12:15], v[72:75], v[128:131]\n"
    "v_mfma_i32_16x16x64_i8 v[128:131], a[36:39], v[76:79], v[128:131]\n"
    "ds_read_b128 v[104:107], v39 offset:10240\n"
    "v_mfma_i32_16x16x64_i8 v[124:127], a[0:3], v[68:71], v[124:127]\n"
    "v_mfma_i32_16x16x64_i8 v[124:127], a[8:11], v[72:75], v[124:127]\n"
    "ds_read_b128 v[108:111], v38 offset:18432\n"
    "v_mfma_i32_16x16x64_i8 v[124:127], a[32:35], v[76:79], v[124:127]\n"
    "s_waitcnt lgkmcnt(5)\n"
    "s_add_u32 s90, s64, 0\n"
    "buffer_store_dwordx4 v[116:119], v22, s[44:47], s90 offen sc0 sc1\n"
    "v_mfma_i32_16x16x64_i8 v[124:127], a[40:43], v[80:83], v[124:127]\n"
    "ds_read_b128 v[112:115], v39 offset:18432\n"
    "v_mfma_i32_16x16x64_i8 v[124:127], a[64:67], v[84:87], v[124:127]\n"
    "v_mfma_i32_16x16x64_i8 v[124:127], a[72:75], v[88:91], v[124:127]\n"
    "ds_read_b128 v[52:55], v38 offset:4096\n"
    "v_mfma_i32_16x16x64_i8 v[128:131], a[44:47], v[80:83], v[128:131]\n"
    "buffer_store_dwordx4 v[120:123], v22, s[44:47], s90 offen offset:1024 sc0 sc1\n"
    "s_cmp_eq_u32 s84, 3\n"
    "s_cselect_b32 s91, s85, 0\n"
    "v_mfma_i32_16x16x64_i8 v[128:131], a[68:71], v[84:87], v[128:131]\n"
    "ds_read_b128 v[56:59], v39 offset:4096\n"
    "s_cmp_gt_u32 s91, 3\n"
    "s_cselect_b32 s82, 24576, s83\n"
    "s_addc_u32 s84, s84, 0\n"
    "v_mfma_i32_16x16x64_i8 v[128:131], a[76:79], v[88:91], v[128:131]\n"
    "s_waitcnt lgkmcnt(5)\n"
    "s_add_u32 m0, s11, 65536\n"
    "v_mfma_i32_16x16x64_i8 v[128:131], a[20:23], v[92:95], v[128:131]\n"
    "ds_read_b128 v[60:63], v38 offset:12288\n"
    "s_add_u32 s34, s8, s82\n"
    "v_mfma_i32_16x16x64_i8 v[128:131], a[28:31], v[96:99], v[128:131]\n"
    "buffer_load_dwordx4 v22, s[40:43], s34 offen sc0 sc1 lds\n"
    "v_mfma_i32_16x16x64_i8 v[128:131], a[52:55], v[100:103], v[128:131]\n"
    "ds_read_b128 v[64:67], v39 offset:12288\n"
    "v_mfma_i32_16x16x64_i8 v[124:127], a[16:19], v[92:95], v[124:127]\n"
    "s_cmp_lt_u32 s84, s96\n"
    "s_getpc_b64 s[88:89]\n"
    "s_cbranch_scc1 L_midb%=\n"
    "v_mfma_i32_16x16x64_i8 v[124:127], a[24:27], v[96:99], v[124:127]\n"
    "ds_read_b128 v[68:71], v38 offset:20480\n"
    "v_mfma_i32_16x16x64_i8 v[124:127], a[48:51], v[100:103], v[124:127]\n"
    "s_waitcnt lgkmcnt(5)\n"
    "buffer_load_dwordx4 v22, s[40:43], s34 offen offset:1024 sc0 sc1 lds\n"
    "v_mfma_i32_16x16x64_i8 v[124:127], a[56:59], v[104:107], v[124:127]\n"
    "ds_read_b128 v[72:75], v39 offset:20480\n"
    "v_mfma_i32_16x16x64_i8 v[124:127], a[80:83], v[108:111], v[124:127]\n"
    "s_add_u32 m0, s12, 65536\n"
    "s_add_u32 s34, s9, s82\n"
    "v_mfma_i32_16x16x64_i8 v[124:127], a[88:91], v[112:115], v[124:127]\n"
    "ds_read_b128 v[76:79], v38 offset:6144\n"
    "v_mfma_i32_16x16x64_i8 v[128:131], a[60:63], v[104:107], v[128:131]\n"
    "buffer_load_dwordx4 v22, s[40:43], s34 offen sc0 sc1 lds\n"
    "v_mfma_i32_16x16x64_i8 v[128:131], a[84:87], v[108:111], v[128:131]\n"
    "ds_read_b128 v[80:83], v39 offset:6144\n"
    "v_mfma_i32_16x16x64_i8 v[128:131], a[92:95], v[112:115], v[128:131]\n"
    "s_waitcnt lgkmcnt(5)\n"
    "buffer_load_dwordx4 v22, s[40:43], s34 offen offset:1024 sc0 sc1 lds\n"
    "v_mfma_i32_16x16x64_i8 v[128:131], a[100:103], v[52:55], v[128:131]\n"
    "ds_read_b128 v[84:87], v38 offset:14336\n"
    "v_mfma_i32_16x16x64_i8 v[128:131], a[108:111], v[56:59], v[128:131]\n"
    "s_add_u32 m0, s13, 65536\n"
    "s_add_u32 s34, s10, s82\n"
    "v_mfma_i32_16x16x64_i8 v[128:131], a[132:135], v[60:63], v[128:131]\n"
    "ds_read_b128 v[88:91], v39 offset:14336\n"
    "v_mfma_i32_16x16x64_i8 v[124:127], a[96:99], v[52:55], v[124:127]\n"
    "buffer_load_dwordx4 v22, s[40:43], s34 offen sc0 sc1 lds\n"
    "v_mfma_i32_16x16x64_i8 v[124:127], a[104:107], v[56:59], v[124:127]\n"
    "ds_read_b128 v[92:95], v38 offset:22528\n"
    "v_mfma_i32_16x16x64_i8 v[124:127], a[128:131], v[60:63], v[124:127]\n"
    "s_waitcnt lgkmcnt(5)\n"
    "v_mfma_i32_16x16x64_i8 v[124:127], a[136:139], v[64:67], v[124:127]\n"
    "ds_read_b128 v[96:99], v39 offset:22528\n"
    "v_mfma_i32_16x16x64_i8 v[124:127], a[160:163], v[68:71], v[124:127]\n"
    "buffer_load_dwordx4 v22, s[40:43], s34 offen offset:1024 sc0 sc1 lds\n"
    "s_mov_b32 s96, 4\n"
    "v_mfma_i32_16x16x64_i8 v[124:127], a[168:171], v[72:75], v[124:127]\n"
    "v_mfma_i32_16x16x64_i8 v[128:131], a[140:143], v[64:67], v[128:131]\n"
    "v_mfma_i32_16x16x64_i8 v[128:131], a[164:167], v[68:71], v[128:131]\n"
    "s_mov_b32 s95, 2\n"
    "s_and_b32 s91, s84, s25\n"
    "v_mfma_i32_16x16x64_i8 v[128:131], a[172:175], v[72:75], v[128:131]\n"
    "s_cmp_gt_u32 s91, 2\n"
    "s_getpc_b64 s[88:89]\n"
    "s_cbranch_scc0 L_slowb%=\n"
    "s_waitcnt vmcnt(9) lgkmcnt(0)\n"
    "s_barrier\n"
    "v_mfma_i32_16x16x64_i8 v[128:131], a[116:119], v[76:79], v[128:131]\n"
    "ds_read_b128 v[100:103], v38 offset:32768\n"
    "s_waitcnt vmcnt(8)\n"
    "v_mfma_i32_16x16x64_i8 v[128:131], a[124:127], v[80:83], v[128:131]\n"
    "ds_read_b128 v[104:107], v39 offset:32768\n"
    "v_cmp_le_u32 vcc, s17, v32\n"
    "v_mfma_i32_16x16x64_i8 v[128:131], a[148:151], v[84:87], v[128:131]\n"
    "ds_read_b128 v[108:111], v38 offset:40960\n"
    "v_mfma_i32_16x16x64_i8 v[124:127], a[112:115], v[76:79], v[124:127]\n"
    "ds_read_b128 v[112:115], v39 offset:40960\n"
    "s_lshr_b32 s87, vcc_lo, 16\n"
    "v_mfma_i32_16x16x64_i8 v[124:127], a[120:123], v[80:83], v[124:127]\n"
    "ds_read_b128 v[52:55], v38 offset:49152\n"
    "v_mfma_i32_16x16x64_i8 v[124:127], a[144:147], v[84:87], v[124:127]\n"
    "ds_read_b128 v[56:59], v39 offset:49152\n"
    "s_and_b32 s87, s87, vcc_lo\n"
    "v_mfma_i32_16x16x64_i8 v[124:127], a[152:155], v[88:91], v[124:127]\n"
    "ds_read_b128 v[60:63], v38 offset:34816\n"
    "v_mfma_i32_16x16x64_i8 v[124:127], a[176:179], v[92:95], v[124:127]\n"
    "ds_read_b128 v[64:67], v39 offset:34816\n"
    "s_and_b32 s87, s87, vcc_hi\n"
    "v_mfma_i32_16x16x64_i8 v[124:127], a[184:187], v[96:99], v[124:127]\n"
    "s_ff0_i32_b32 s85, s87\n"
    "v_mfma_i32_16x16x64_i8 v[128:131], a[156:159], v[88:91], v[128:131]\n"
    "s_max_u32 s85, s85, s86\n"
    "v_mfma_i32_16x16x64_i8 v[128:131], a[180:183], v[92:95], v[128:131]\n"
    "s_min_u32 s91, s85, s96\n"
    "v_mfma_i32_16x16x64_i8 v[128:131], a[188:191], v[96:99], v[128:131]\n"
    "s_cmp_lt_u32 s84, s91\n"
    "s_getpc_b64 s[88:89]\n"
    "s_cbranch_scc1 L_catchb%=\n"
    "s_waitcnt lgkmcnt(5)\n"
    "v_mfma_i32_16x16x64_i8 v[136:139], a[4:7], v[100:103], v[136:139]\n"
    "global_load_dword v32, v24, s[72:73] sc0 sc1\n"
    "ds_read_b128 v[68:71], v38 offset:43008\n"
    "v_mfma_i32_16x16x64_i8 v[136:139], a[12:15], v[104:107], v[136:139]\n"
    "v_mfma_i32_16x16x64_i8 v[136:139], a[36:39], v[108:111], v[136:139]\n"
    "ds_read_b128 v[72:75], v39 offset:43008\n"
    "v_mfma_i32_16x16x64_i8 v[132:135], a[0:3], v[100:103], v[132:135]\n"
    "v_mfma_i32_16x16x64_i8 v[132:135], a[8:11], v[104:107], v[132:135]\n"
    "ds_read_b128 v[76:79], v38 offset:51200\n"
    "v_mfma_i32_16x16x64_i8 v[132:135], a[32:35], v[108:111], v[132:135]\n"
    "s_waitcnt lgkmcnt(5)\n"
    "s_add_u32 s90, s64, 24576\n"
    "buffer_store_dwordx4 v[124:127], v22, s[44:47], s90 offen sc0 sc1\n"
    "v_mfma_i32_16x16x64_i8 v[132:135], a[40:43], v[112:115], v[132:135]\n"
    "ds_read_b128 v[80:83], v39 offset:51200\n"
    "v_mfma_i32_16x16x64_i8 v[132:135], a[64:67], v[52:55], v[132:135]\n"
    "v_mfma_i32_16x16x64_i8 v[132:135], a[72:75], v[56:59], v[132:135]\n"
    "ds_read_b128 v[84:87], v38 offset:36864\n"
    "v_mfma_i32_16x16x64_i8 v[136:139], a[44:47], v[112:115], v[136:139]\n"
    "buffer_store_dwordx4 v[128:131], v22, s[44:47], s90 offen offset:1024 sc0 sc1\n"
    "s_cmp_eq_u32 s84, 4\n"
    "s_cselect_b32 s91, s85, 0\n"
    "v_mfma_i32_16x16x64_i8 v[136:139], a[68:71], v[52:55], v[136:139]\n"
    "ds_read_b128 v[88:91], v39 offset:36864\n"
    "s_cmp_gt_u32 s91, 4\n"
    "s_cselect_b32 s82, 32768, s83\n"
    "s_addc_u32 s84, s84, 0\n"
    "v_mfma_i32_16x16x64_i8 v[136:139], a[76:79], v[56:59], v[136:139]\n"
    "s_waitcnt lgkmcnt(5)\n"
    "s_add_u32 m0, s11, 98304\n"
    "v_mfma_i32_16x16x64_i8 v[136:139], a[20:23], v[60:63], v[136:139]\n"
    "ds_read_b128 v[92:95], v38 offset:45056\n"
    "s_add_u32 s34, s8, s82\n"
    "v_mfma_i32_16x16x64_i8 v[136:139], a[28:31], v[64:67], v[136:139]\n"
    "buffer_load_dwordx4 v22, s[40:43], s34 offen sc0 sc1 lds\n"
    "v_mfma_i32_16x16x64_i8 v[136:139], a[52:55], v[68:71], v[136:139]\n"
    "ds_read_b128 v[96:99], v39 offset:45056\n"
    "v_mfma_i32_16x16x64_i8 v[132:135], a[16:19], v[60:63], v[132:135]\n"
    "s_cmp_lt_u32 s84, s96\n"
    "s_getpc_b64 s[88:89]\n"
    "s_cbranch_scc1 L_midb%=\n"
    "v_mfma_i32_16x16x64_i8 v[132:135], a[24:27], v[64:67], v[132:135]\n"
    "ds_read_b128 v[100:103], v38 offset:53248\n"
    "v_mfma_i32_16x16x64_i8 v[132:135], a[48:51], v[68:71], v[132:135]\n"
    "s_waitcnt lgkmcnt(5)\n"
    "buffer_load_dwordx4 v22, s[40:43], s34 offen offset:1024 sc0 sc1 lds\n"
    "v_mfma_i32_16x16x64_i8 v[132:135], a[56:59], v[72:75], v[132:135]\n"
    "ds_read_b128 v[104:107], v39 offset:53248\n"
    "v_mfma_i32_16x16x64_i8 v[132:135], a[80:83], v[76:79], v[132:135]\n"
    "s_add_u32 m0, s12, 98304\n"
    "s_add_u32 s34, s9, s82\n"
    "v_mfma_i32_16x16x64_i8 v[132:135], a[88:91], v[80:83], v[132:135]\n"
    "ds_read_b128 v[108:111], v38 offset:38912\n"
    "v_mfma_i32_16x16x64_i8 v[136:139], a[60:63], v[72:75], v[136:139]\n"
    "buffer_load_dwordx4 v22, s[40:43], s34 offen sc0 sc1 lds\n"
    "v_mfma_i32_16x16x64_i8 v[136:139], a[84:87], v[76:79], v[136:139]\n"
    "ds_read_b128 v[112:115], v39 offset:38912\n"
    "v_mfma_i32_16x16x64_i8 v[136:139], a[92:95], v[80:83], v[136:139]\n"
    "s_waitcnt lgkmcnt(5)\n"
    "buffer_load_dwordx4 v22, s[40:43], s34 offen offset:1024 sc0 sc1 lds\n"
    "v_mfma_i32_16x16x64_i8 v[136:139], a[100:103], v[84:87], v[136:139]\n"
    "ds_read_b128 v[52:55], v38 offset:47104\n"
    "v_mfma_i32_16x16x64_i8 v[136:139], a[108:111], v[88:91], v[136:139]\n"
    "s_add_u32 m0, s13, 98304\n"
    "s_add_u32 s34, s10, s82\n"
    "v_mfma_i32_16x16x64_i8 v[136:139], a[132:135], v[92:95], v[136:139]\n"
    "ds_read_b128 v[56:59], v39 offset:47104\n"
    "v_mfma_i32_16x16x64_i8 v[132:135], a[96:99], v[84:87], v[132:135]\n"
    "buffer_load_dwordx4 v22, s[40:43], s34 offen sc0 sc1 lds\n"
    "v_mfma_i32_16x16x64_i8 v[132:135], a[104:107], v[88:91], v[132:135]\n"
    "ds_read_b128 v[60:63], v38 offset:55296\n"
    "v_mfma_i32_16x16x64_i8 v[132:135], a[128:131], v[92:95], v[132:135]\n"
    "s_waitcnt lgkmcnt(5)\n"
    "v_mfma_i32_16x16x64_i8 v[132:135], a[136:139], v[96:99], v[132:135]\n"
    "ds_read_b128 v[64:67], v39 offset:55296\n"
    "v_mfma_i32_16x16x64_i8 v[132:135], a[160:163], v[100:103], v[132:135]\n"
    "buffer_load_dwordx4 v22, s[40:43], s34 offen offset:1024 sc0 sc1 lds\n"
    "s_mov_b32 s96, 5\n"
    "v_mfma_i32_16x16x64_i8 v[132:135], a[168:171], v[104:107], v[132:135]\n"
    "v_mfma_i32_16x16x64_i8 v[136:139], a[140:143], v[96:99], v[136:139]\n"
    "v_mfma_i32_16x16x64_i8 v[136:139], a[164:167], v[100:103], v[136:139]\n"
    "s_mov_b32 s95, 3\n"
    "s_and_b32 s91, s84, s25\n"
    "v_mfma_i32_16x16x64_i8 v[136:139], a[172:175], v[104:107], v[136:139]\n"
    "s_cmp_gt_u32 s91, 3\n"
    "s_getpc_b64 s[88:89]\n"
    "s_cbranch_scc0 L_slowb%=\n"
    "s_waitcnt vmcnt(9) lgkmcnt(0)\n"
    "s_barrier\n"
    "v_mfma_i32_16x16x64_i8 v[136:139], a[116:119], v[108:111], v[136:139]\n"
    "ds_read_b128 v[68:71], v40\n"
    "s_waitcnt vmcnt(8)\n"
    "v_mfma_i32_16x16x64_i8 v[136:139], a[124:127], v[112:115], v[136:139]\n"
    "ds_read_b128 v[72:75], v41\n"
    "v_cmp_le_u32 vcc, s17, v32\n"
    "v_mfma_i32_16x16x64_i8 v[136:139], a[148:151], v[52:55], v[136:139]\n"
    "ds_read_b128 v[76:79], v40 offset:8192\n"
    "v_mfma_i32_16x16x64_i8 v[132:135], a[112:115], v[108:111], v[132:135]\n"
    "ds_read_b128 v[80:83], v41 offset:8192\n"
    "s_lshr_b32 s87, vcc_lo, 16\n"
    "v_mfma_i32_16x16x64_i8 v[132:135], a[120:123], v[112:115], v[132:135]\n"
    "ds_read_b128 v[84:87], v40 offset:16384\n"
    "v_mfma_i32_16x16x64_i8 v[132:135], a[144:147], v[52:55], v[132:135]\n"
    "ds_read_b128 v[88:91], v41 offset:16384\n"
    "s_and_b32 s87, s87, vcc_lo\n"
    "v_mfma_i32_16x16x64_i8 v[132:135], a[152:155], v[56:59], v[132:135]\n"
    "ds_read_b128 v[92:95], v40 offset:2048\n"
    "v_mfma_i32_16x16x64_i8 v[132:135], a[176:179], v[60:63], v[132:135]\n"
    "ds_read_b128 v[96:99], v41 offset:2048\n"
    "s_and_b32 s87, s87, vcc_hi\n"
    "v_mfma_i32_16x16x64_i8 v[132:135], a[184:187], v[64:67], v[132:135]\n"
    "s_ff0_i32_b32 s85, s87\n"
    "v_mfma_i32_16x16x64_i8 v[136:139], a[156:159], v[56:59], v[136:139]\n"
    "s_max_u32 s85, s85, s86\n"
    "v_mfma_i32_16x16x64_i8 v[136:139], a[180:183], v[60:63], v[136:139]\n"
    "s_min_u32 s91, s85, s96\n"
    "v_mfma_i32_16x16x64_i8 v[136:139], a[188:191], v[64:67], v[136:139]\n"
    "s_cmp_lt_u32 s84, s91\n"
    "s_getpc_b64 s[88:89]\n"
    "s_cbranch_scc1 L_catchb%=\n"
    "s_waitcnt lgkmcnt(5)\n"
    "v_mfma_i32_16x16x64_i8 v[144:147], a[4:7], v[68:71], v[144:147]\n"
    "global_load_dword v32, v24, s[72:73] sc0 sc1\n"
    "ds_read_b128 v[100:103], v40 offset:10240\n"
    "v_mfma_i32_16x16x64_i8 v[144:147], a[12:15], v[72:75], v[144:147]\n"
    "v_mfma_i32_16x16x64_i8 v[144:147], a[36:39], v[76:79], v[144:147]\n"
    "ds_read_b128 v[104:107], v41 offset:10240\n"
    "v_mfma_i32_16x16x64_i8 v[140:143], a[0:3], v[68:71], v[140:143]\n"
    "v_mfma_i32_16x16x64_i8 v[140:143], a[8:11], v[72:75], v[140:143]\n"
    "ds_read_b128 v[108:111], v40 offset:18432\n"
    "v_mfma_i32_16x16x64_i8 v[140:143], a[32:35], v[76:79], v[140:143]\n"
    "s_waitcnt lgkmcnt(5)\n"
    "s_add_u32 s90, s64, 49152\n"
    "buffer_store_dwordx4 v[132:135], v22, s[44:47], s90 offen sc0 sc1\n"
    "v_mfma_i32_16x16x64_i8 v[140:143], a[40:43], v[80:83], v[140:143]\n"
    "ds_read_b128 v[112:115], v41 offset:18432\n"
    "v_mfma_i32_16x16x64_i8 v[140:143], a[64:67], v[84:87], v[140:143]\n"
    "v_mfma_i32_16x16x64_i8 v[140:143], a[72:75], v[88:91], v[140:143]\n"
    "ds_read_b128 v[52:55], v40 offset:4096\n"
    "v_mfma_i32_16x16x64_i8 v[144:147], a[44:47], v[80:83], v[144:147]\n"
    "buffer_store_dwordx4 v[136:139], v22, s[44:47], s90 offen offset:1024 sc0 sc1\n"
    "s_cmp_eq_u32 s84, 5\n"
    "s_cselect_b32 s91, s85, 0\n"
    "v_mfma_i32_16x16x64_i8 v[144:147], a[68:71], v[84:87], v[144:147]\n"
    "ds_read_b128 v[56:59], v41 offset:4096\n"
    "s_cmp_gt_u32 s91, 5\n"
    "s_cselect_b32 s82, 40960, s83\n"
    "s_addc_u32 s84, s84, 0\n"
    "v_mfma_i32_16x16x64_i8 v[144:147], a[76:79], v[88:91], v[144:147]\n"
    "s_waitcnt lgkmcnt(5)\n"
    "s_add_u32 m0, s11, 131072\n"
    "v_mfma_i32_16x16x64_i8 v[144:147], a[20:23], v[92:95], v[144:147]\n"
    "ds_read_b128 v[60:63], v40 offset:12288\n"
    "s_add_u32 s34, s8, s82\n"
    "v_mfma_i32_16x16x64_i8 v[144:147], a[28:31], v[96:99], v[144:147]\n"
    "buffer_load_dwordx4 v22, s[40:43], s34 offen sc0 sc1 lds\n"
    "v_mfma_i32_16x16x64_i8 v[144:147], a[52:55], v[100:103], v[144:147]\n"
    "ds_read_b128 v[64:67], v41 offset:12288\n"
    "v_mfma_i32_16x16x64_i8 v[140:143], a[16:19], v[92:95], v[140:143]\n"
    "s_cmp_lt_u32 s84, s96\n"
    "s_getpc_b64 s[88:89]\n"
    "s_cbranch_scc1 L_midb%=\n"
    "v_mfma_i32_16x16x64_i8 v[140:143], a[24:27], v[96:99], v[140:143]\n"
    "ds_read_b128 v[68:71], v40 offset:20480\n"
    "v_mfma_i32_16x16x64_i8 v[140:143], a[48:51], v[100:103], v[140:143]\n"
    "s_waitcnt lgkmcnt(5)\n"
    "buffer_load_dwordx4 v22, s[40:43], s34 offen offset:1024 sc0 sc1 lds\n"
    "v_mfma_i32_16x16x64_i8 v[140:143], a[56:59], v[104:107], v[140:143]\n"
    "ds_read_b128 v[72:75], v41 offset:20480\n"
    "v_mfma_i32_16x16x64_i8 v[140:143], a[80:83], v[108:111], v[140:143]\n"
    "s_add_u32 m0, s12, 131072\n"
    "s_add_u32 s34, s9, s82\n"
    "v_mfma_i32_16x16x64_i8 v[140:143], a[88:91], v[112:115], v[140:143]\n"
    "ds_read_b128 v[76:79], v40 offset:6144\n"
    "v_mfma_i32_16x16x64_i8 v[144:147], a[60:63], v[104:107], v[144:147]\n"
    "buffer_load_dwordx4 v22, s[40:43], s34 offen sc0 sc1 lds\n"
    "v_mfma_i32_16x16x64_i8 v[144:147], a[84:87], v[108:111], v[144:147]\n"
    "ds_read_b128 v[80:83], v41 offset:6144\n"
    "v_mfma_i32_16x16x64_i8 v[144:147], a[92:95], v[112:115], v[144:147]\n"
    "s_waitcnt lgkmcnt(5)\n"
    "buffer_load_dwordx4 v22, s[40:43], s34 offen offset:1024 sc0 sc1 lds\n"
    "v_mfma_i32_16x16x64_i8 v[144:147], a[100:103], v[52:55], v[144:147]\n"
    "ds_read_b128 v[84:87], v40 offset:14336\n"
    "v_mfma_i32_16x16x64_i8 v[144:147], a[108:111], v[56:59], v[144:147]\n"
    "s_add_u32 m0, s13, 131072\n"
    "s_add_u32 s34, s10, s82\n"
    "v_mfma_i32_16x16x64_i8 v[144:147], a[132:135], v[60:63], v[144:147]\n"
    "ds_read_b128 v[88:91], v41 offset:14336\n"
    "v_mfma_i32_16x16x64_i8 v[140:143], a[96:99], v[52:55], v[140:143]\n"
    "buffer_load_dwordx4 v22, s[40:43], s34 offen sc0 sc1 lds\n"
    "v_mfma_i32_16x16x64_i8 v[140:143], a[104:107], v[56:59], v[140:143]\n"
    "ds_read_b128 v[92:95], v40 offset:22528\n"
    "v_mfma_i32_16x16x64_i8 v[140:143], a[128:131], v[60:63], v[140:143]\n"
    "s_waitcnt lgkmcnt(5)\n"
    "v_mfma_i32_16x16x64_i8 v[140:143], a[136:139], v[64:67], v[140:143]\n"
    "ds_read_b128 v[96:99], v41 offset:22528\n"
    "v_mfma_i32_16x16x64_i8 v[140:143], a[160:163], v[68:71], v[140:143]\n"
    "buffer_load_dwordx4 v22, s[40:43], s34 offen offset:1024 sc0 sc1 lds\n"
    "s_mov_b32 s96, 6\n"
    "v_mfma_i32_16x16x64_i8 v[140:143], a[168:171], v[72:75], v[140:143]\n"
    "v_mfma_i32_16x16x64_i8 v[144:147], a[140:143], v[64:67], v[144:147]\n"
    "v_mfma_i32_16x16x64_i8 v[144:147], a[164:167], v[68:71], v[144:147]\n"
    "s_mov_b32 s95, 4\n"
    "s_and_b32 s91, s84, s25\n"
    "v_mfma_i32_16x16x64_i8 v[144:147], a[172:175], v[72:75], v[144:147]\n"
    "s_cmp_gt_u32 s91, 4\n"
    "s_getpc_b64 s[88:89]\n"
    "s_cbranch_scc0 L_slowb%=\n"
    "s_waitcnt vmcnt(9) lgkmcnt(0)\n"
    "s_barrier\n"
    "v_mfma_i32_16x16x64_i8 v[144:147], a[116:119], v[76:79], v[144:147]\n"
    "ds_read_b128 v[100:103], v40 offset:32768\n"
    "s_waitcnt vmcnt(8)\n"
    "v_mfma_i32_16x16x64_i8 v[144:147], a[124:127], v[80:83], v[144:147]\n"
    "ds_read_b128 v[104:107], v41 offset:32768\n"
    "v_cmp_le_u32 vcc, s17, v32\n"
    "v_mfma_i32_16x16x64_i8 v[144:147], a[148:151], v[84:87], v[144:147]\n"
    "ds_read_b128 v[108:111], v40 offset:40960\n"
    "v_mfma_i32_16x16x64_i8 v[140:143], a[112:115], v[76:79], v[140:143]\n"
    "ds_read_b128 v[112:115], v41 offset:40960\n"
    "s_lshr_b32 s87, vcc_lo, 16\n"
    "v_mfma_i32_16x16x64_i8 v[140:143], a[120:123], v[80:83], v[140:143]\n"
    "ds_read_b128 v[52:55], v40 offset:49152\n"
    "v_mfma_i32_16x16x64_i8 v[140:143], a[144:147], v[84:87], v[140:143]\n"
    "ds_read_b128 v[56:59], v41 offset:49152\n"
    "s_and_b32 s87, s87, vcc_lo\n"
    "v_mfma_i32_16x16x64_i8 v[140:143], a[152:155], v[88:91], v[140:143]\n"
    "ds_read_b128 v[60:63], v40 offset:34816\n"
    "v_mfma_i32_16x16x64_i8 v[140:143], a[176:179], v[92:95], v[140:143]\n"
    "ds_read_b128 v[64:67], v41 offset:34816\n"
    "s_and_b32 s87, s87, vcc_hi\n"
    "v_mfma_i32_16x16x64_i8 v[140:143], a[184:187], v[96:99], v[140:143]\n"
    "s_ff0_i32_b32 s85, s87\n"
    "v_mfma_i32_16x16x64_i8 v[144:147], a[156:159], v[88:91], v[144:147]\n"
    "s_max_u32 s85, s85, s86\n"
    "v_mfma_i32_16x16x64_i8 v[144:147], a[180:183], v[92:95], v[144:147]\n"
    "s_min_u32 s91, s85, s96\n"
    "v_mfma_i32_16x16x64_i8 v[144:147], a[188:191], v[96:99], v[144:147]\n"
    "s_cmp_lt_u32 s84, s91\n"
    "s_getpc_b64 s[88:89]\n"
    "s_cbranch_scc1 L_catchb%=\n"
    "s_waitcnt lgkmcnt(5)\n"
    "v_mfma_i32_16x16x64_i8 v[152:155], a[4:7], v[100:103], v[152:155]\n"
    "global_load_dword v32, v24, s[72:73] sc0 sc1\n"
    "ds_read_b128 v[68:71], v40 offset:43008\n"
    "v_mfma_i32_16x16x64_i8 v[152:155], a[12:15], v[104:107], v[152:155]\n"
    "v_mfma_i32_16x16x64_i8 v[152:155], a[36:39], v[108:111], v[152:155]\n"
    "ds_read_b128 v[72:75], v41 offset:43008\n"
    "v_mfma_i32_16x16x64_i8 v[148:151], a[0:3], v[100:103], v[148:151]\n"
    "v_mfma_i32_16x16x64_i8 v[148:151], a[8:11], v[104:107], v[148:151]\n"
    "ds_read_b128 v[76:79], v40 offset:51200\n"
    "v_mfma_i32_16x16x64_i8 v[148:151], a[32:35], v[108:111], v[148:151]\n"
    "s_waitcnt lgkmcnt(5)\n"
    "s_add_u32 s90, s64, 73728\n"
    "buffer_store_dwordx4 v[140:143], v22, s[44:47], s90 offen sc0 sc1\n"
    "v_mfma_i32_16x16x64_i8 v[148:151], a[40:43], v[112:115], v[148:151]\n"
    "ds_read_b128 v[80:83], v41 offset:51200\n"
    "v_mfma_i32_16x16x64_i8 v[148:151], a[64:67], v[52:55], v[148:151]\n"
    "v_mfma_i32_16x16x64_i8 v[148:151], a[72:75], v[56:59], v[148:151]\n"
    "ds_read_b128 v[84:87], v40 offset:36864\n"
    "v_mfma_i32_16x16x64_i8 v[152:155], a[44:47], v[112:115], v[152:155]\n"
    "buffer_store_dwordx4 v[144:147], v22, s[44:47], s90 offen offset:1024 sc0 sc1\n"
    "s_cmp_eq_u32 s84, 6\n"
    "s_cselect_b32 s91, s85, 0\n"
    "v_mfma_i32_16x16x64_i8 v[152:155], a[68:71], v[52:55], v[152:155]\n"
    "ds_read_b128 v[88:91], v41 offset:36864\n"
    "s_cmp_gt_u32 s91, 6\n"
    "s_cselect_b32 s82, 49152, s83\n"
    "s_addc_u32 s84, s84, 0\n"
    "v_mfma_i32_16x16x64_i8 v[152:155], a[76:79], v[56:59], v[152:155]\n"
    "s_waitcnt lgkmcnt(5)\n"
    "s_add_u32 m0, s11, 0\n"
    "v_mfma_i32_16x16x64_i8 v[152:155], a[20:23], v[60:63], v[152:155]\n"
    "ds_read_b128 v[92:95], v40 offset:45056\n"
    "s_add_u32 s34, s8, s82\n"
    "v_mfma_i32_16x16x64_i8 v[152:155], a[28:31], v[64:67], v[152:155]\n"
    "buffer_load_dwordx4 v22, s[40:43], s34 offen sc0 sc1 lds\n"
    "v_mfma_i32_16x16x64_i8 v[152:155], a[52:55], v[68:71], v[152:155]\n"
    "ds_read_b128 v[96:99], v41 offset:45056\n"
    "v_mfma_i32_16x16x64_i8 v[148:151], a[16:19], v[60:63], v[148:151]\n"
    "s_cmp_lt_u32 s84, s96\n"
    "s_getpc_b64 s[88:89]\n"
    "s_cbranch_scc1 L_midb%=\n"
    "v_mfma_i32_16x16x64_i8 v[148:151], a[24:27], v[64:67], v[148:151]\n"
    "ds_read_b128 v[100:103], v40 offset:53248\n"
    "v_mfma_i32_16x16x64_i8 v[148:151], a[48:51], v[68:71], v[148:151]\n"
    "s_waitcnt lgkmcnt(5)\n"
    "buffer_load_dwordx4 v22, s[40:43], s34 offen offset:1024 sc0 sc1 lds\n"
    "v_mfma_i32_16x16x64_i8 v[148:151], a[56:59], v[72:75], v[148:151]\n"
    "ds_read_b128 v[104:107], v41 offset:53248\n"
    "v_mfma_i32_16x16x64_i8 v[148:151], a[80:83], v[76:79], v[148:151]\n"
    "s_add_u32 m0, s12, 0\n"
    "s_add_u32 s34, s9, s82\n"
    "v_mfma_i32_16x16x64_i8 v[148:151], a[88:91], v[80:83], v[148:151]\n"
    "ds_read_b128 v[108:111], v40 offset:38912\n"
    "v_mfma_i32_16x16x64_i8 v[152:155], a[60:63], v[72:75], v[152:155]\n"
    "buffer_load_dwordx4 v22, s[40:43], s34 offen sc0 sc1 lds\n"
    "v_mfma_i32_16x16x64_i8 v[152:155], a[84:87], v[76:79], v[152:155]\n"
    "ds_read_b128 v[112:115], v41 offset:38912\n"
    "v_mfma_i32_16x16x64_i8 v[152:155], a[92:95], v[80:83], v[152:155]\n"
    "s_waitcnt lgkmcnt(5)\n"
    "buffer_load_dwordx4 v22, s[40:43], s34 offen offset:1024 sc0 sc1 lds\n"
    "v_mfma_i32_16x16x64_i8 v[152:155], a[100:103], v[84:87], v[152:155]\n"
    "ds_read_b128 v[52:55], v40 offset:47104\n"
    "v_mfma_i32_16x16x64_i8 v[152:155], a[108:111], v[88:91], v[152:155]\n"
    "s_add_u32 m0, s13, 0\n"
    "s_add_u32 s34, s10, s82\n"
    "v_mfma_i32_16x16x64_i8 v[152:155], a[132:135], v[92:95], v[152:155]\n"
    "ds_read_b128 v[56:59], v41 offset:47104\n"
    "v_mfma_i32_16x16x64_i8 v[148:151], a[96:99], v[84:87], v[148:151]\n"
    "buffer_load_dwordx4 v22, s[40:43], s34 offen sc0 sc1 lds\n"
    "v_mfma_i32_16x16x64_i8 v[148:151], a[104:107], v[88:91], v[148:151]\n"
    "ds_read_b128 v[60:63], v40 offset:55296\n"
    "v_mfma_i32_16x16x64_i8 v[148:151], a[128:131], v[92:95], v[148:151]\n"
    "s_waitcnt lgkmcnt(5)\n"
    "v_mfma_i32_16x16x64_i8 v[148:151], a[136:139], v[96:99], v[148:151]\n"
    "ds_read_b128 v[64:67], v41 offset:55296\n"
    "v_mfma_i32_16x16x64_i8 v[148:151], a[160:163], v[100:103], v[148:151]\n"
    "buffer_load_dwordx4 v22, s[40:43], s34 offen offset:1024 sc0 sc1 lds\n"
    "s_mov_b32 s96, 7\n"
    "v_mfma_i32_16x16x64_i8 v[148:151], a[168:171], v[104:107], v[148:151]\n"
    "v_mfma_i32_16x16x64_i8 v[152:155], a[140:143], v[96:99], v[152:155]\n"
    "v_mfma_i32_16x16x64_i8 v[152:155], a[164:167], v[100:103], v[152:155]\n"
    "s_mov_b32 s95, 5\n"
    "s_and_b32 s91, s84, s25\n"
    "v_mfma_i32_16x16x64_i8 v[152:155], a[172:175], v[104:107], v[152:155]\n"
    "s_cmp_gt_u32 s91, 5\n"
    "s_getpc_b64 s[88:89]\n"
    "s_cbranch_scc0 L_slowb%=\n"
    "s_waitcnt vmcnt(9) lgkmcnt(0)\n"
    "s_barrier\n"
    "v_mfma_i32_16x16x64_i8 v[152:155], a[116:119], v[108:111], v[152:155]\n"
    "ds_read_b128 v[68:71], v42\n"
    "s_waitcnt vmcnt(8)\n"
    "v_mfma_i32_16x16x64_i8 v[152:155], a[124:127], v[112:115], v[152:155]\n"
    "ds_read_b128 v[72:75], v43\n"
    "v_cmp_le_u32 vcc, s17, v32\n"
    "v_mfma_i32_16x16x64_i8 v[152:155], a[148:151], v[52:55], v[152:155]\n"
    "ds_read_b128 v[76:79], v42 offset:8192\n"
    "v_mfma_i32_16x16x64_i8 v[148:151], a[112:115], v[108:111], v[148:151]\n"
    "ds_read_b128 v[80:83], v43 offset:8192\n"
    "s_lshr_b32 s87, vcc_lo, 16\n"
    "v_mfma_i32_16x16x64_i8 v[148:151], a[120:123], v[112:115], v[148:151]\n"
    "ds_read_b128 v[84:87], v42 offset:16384\n"
    "v_mfma_i32_16x16x64_i8 v[148:151], a[144:147], v[52:55], v[148:151]\n"
    "ds_read_b128 v[88:91], v43 offset:16384\n"
    "s_and_b32 s87, s87, vcc_lo\n"
    "v_mfma_i32_16x16x64_i8 v[148:151], a[152:155], v[56:59], v[148:151]\n"
    "ds_read_b128 v[92:95], v42 offset:2048\n"
    "v_mfma_i32_16x16x64_i8 v[148:151], a[176:179], v[60:63], v[148:151]\n"
    "ds_read_b128 v[96:99], v43 offset:2048\n"
    "s_and_b32 s87, s87, vcc_hi\n"
    "v_mfma_i32_16x16x64_i8 v[148:151], a[184:187], v[64:67], v[148:151]\n"
    "s_ff0_i32_b32 s85, s87\n"
    "v_mfma_i32_16x16x64_i8 v[152:155], a[156:159], v[56:59], v[152:155]\n"
    "s_max_u32 s85, s85, s86\n"
    "v_mfma_i32_16x16x64_i8 v[152:155], a[180:183], v[60:63], v[152:155]\n"
    "s_min_u32 s91, s85, s96\n"
    "v_mfma_i32_16x16x64_i8 v[152:155], a[188:191], v[64:67], v[152:155]\n"
    "s_cmp_lt_u32 s84, s91\n"
    "s_getpc_b64 s[88:89]\n"
    "s_cbranch_scc1 L_catchb%=\n"
    "s_waitcnt lgkmcnt(5)\n"
    "v_mfma_i32_16x16x64_i8 v[160:163], a[4:7], v[68:71], v[160:163]\n"
    "global_load_dword v32, v24, s[72:73] sc0 sc1\n"
    "ds_read_b128 v[100:103], v42 offset:10240\n"
    "v_mfma_i32_16x16x64_i8 v[160:163], a[12:15], v[72:75], v[160:163]\n"
    "v_mfma_i32_16x16x64_i8 v[160:163], a[36:39], v[76:79], v[160:163]\n"
    "ds_read_b128 v[104:107], v43 offset:10240\n"
    "v_mfma_i32_16x16x64_i8 v[156:159], a[0:3], v[68:71], v[156:159]\n"
    "v_mfma_i32_16x16x64_i8 v[156:159], a[8:11], v[72:75], v[156:159]\n"
    "ds_read_b128 v[108:111], v42 offset:18432\n"
    "v_mfma_i32_16x16x64_i8 v[156:159], a[32:35], v[76:79], v[156:159]\n"
    "s_waitcnt lgkmcnt(5)\n"
    "global_load_dword v37, v26, s[72:73] sc0 sc1\n"
    "v_mfma_i32_16x16x64_i8 v[156:159], a[40:43], v[80:83], v[156:159]\n"
    "ds_read_b128 v[112:115], v43 offset:18432\n"
    "v_mfma_i32_16x16x64_i8 v[156:159], a[64:67], v[84:87], v[156:159]\n"
    "s_add_u32 s90, s64, 98304\n"
    "v_mfma_i32_16x16x64_i8 v[156:159], a[72:75], v[88:91], v[156:159]\n"
    "ds_read_b128 v[52:55], v42 offset:4096\n"
    "v_mfma_i32_16x16x64_i8 v[160:163], a[44:47], v[80:83], v[160:163]\n"
    "buffer_store_dwordx4 v[148:151], v22, s[44:47], s90 offen sc0 sc1\n"
    "v_mfma_i32_16x16x64_i8 v[160:163], a[68:71], v[84:87], v[160:163]\n"
    "ds_read_b128 v[56:59], v43 offset:4096\n"
    "v_mfma_i32_16x16x64_i8 v[160:163], a[76:79], v[88:91], v[160:163]\n"
    "s_waitcnt lgkmcnt(5)\n"
    "buffer_store_dwordx4 v[152:155], v22, s[44:47], s90 offen offset:1024 sc0 sc1\n"
    "v_mfma_i32_16x16x64_i8 v[160:163], a[20:23], v[92:95], v[160:163]\n"
    "ds_read_b128 v[60:63], v42 offset:12288\n"
    "s_cmp_eq_u32 s84, 7\n"
    "s_cselect_b32 s91, s85, 0\n"
    "v_mfma_i32_16x16x64_i8 v[160:163], a[28:31], v[96:99], v[160:163]\n"
    "s_cmp_gt_u32 s91, 7\n"
    "s_cselect_b32 s82, 57344, s83\n"
    "s_addc_u32 s84, s84, 0\n"
    "s_add_u32 m0, s11, 32768\n"
    "v_mfma_i32_16x16x64_i8 v[160:163], a[52:55], v[100:103], v[160:163]\n"
    "ds_read_b128 v[64:67], v43 offset:12288\n"
    "s_add_u32 s34, s8, s82\n"
    "v_mfma_i32_16x16x64_i8 v[156:159], a[16:19], v[92:95], v[156:159]\n"
    "buffer_load_dwordx4 v22, s[40:43], s34 offen sc0 sc1 lds\n"
    "s_cmp_lt_u32 s84, s96\n"
    "s_getpc_b64 s[88:89]\n"
    "s_cbranch_scc1 L_midb%=\n"
    "v_mfma_i32_16x16x64_i8 v[156:159], a[24:27], v[96:99], v[156:159]\n"
    "ds_read_b128 v[68:71], v42 offset:20480\n"
    "v_mfma_i32_16x16x64_i8 v[156:159], a[48:51], v[100:103], v[156:159]\n"
    "s_waitcnt lgkmcnt(5)\n"
    "v_mfma_i32_16x16x64_i8 v[156:159], a[56:59], v[104:107], v[156:159]\n"
    "ds_read_b128 v[72:75], v43 offset:20480\n"
    "v_mfma_i32_16x16x64_i8 v[156:159], a[80:83], v[108:111], v[156:159]\n"
    "buffer_load_dwordx4 v22, s[40:43], s34 offen offset:1024 sc0 sc1 lds\n"
    "s_add_u32 m0, s12, 32768\n"
    "v_mfma_i32_16x16x64_i8 v[156:159], a[88:91], v[112:115], v[156:159]\n"
    "ds_read_b128 v[76:79], v42 offset:6144\n"
    "s_add_u32 s34, s9, s82\n"
    "v_mfma_i32_16x16x64_i8 v[160:163], a[60:63], v[104:107], v[160:163]\n"
    "buffer_load_dwordx4 v22, s[40:43], s34 offen sc0 sc1 lds\n"
    "v_mfma_i32_16x16x64_i8 v[160:163], a[84:87], v[108:111], v[160:163]\n"
    "ds_read_b128 v[80:83], v43 offset:6144\n"
    "v_mfma_i32_16x16x64_i8 v[160:163], a[92:95], v[112:115], v[160:163]\n"
    "s_waitcnt lgkmcnt(5)\n"
    "v_mfma_i32_16x16x64_i8 v[160:163], a[100:103], v[52:55], v[160:163]\n"
    "ds_read_b128 v[84:87], v42 offset:14336\n"
    "v_mfma_i32_16x16x64_i8 v[160:163], a[108:111], v[56:59], v[160:163]\n"
    "buffer_load_dwordx4 v22, s[40:43], s34 offen offset:1024 sc0 sc1 lds\n"
    "v_mfma_i32_16x16x64_i8 v[160:163], a[132:135], v[60:63], v[160:163]\n"
    "ds_read_b128 v[88:91], v43 offset:14336\n"
    "v_mfma_i32_16x16x64_i8 v[156:159], a[96:99], v[52:55], v[156:159]\n"
    "s_add_u32 m0, s13, 32768\n"
    "s_add_u32 s34, s10, s82\n"
    "v_mfma_i32_16x16x64_i8 v[156:159], a[104:107], v[56:59], v[156:159]\n"
    "ds_read_b128 v[92:95], v42 offset:22528\n"
    "v_mfma_i32_16x16x64_i8 v[156:159], a[128:131], v[60:63], v[156:159]\n"
    "s_waitcnt lgkmcnt(5)\n"
    "buffer_load_dwordx4 v22, s[40:43], s34 offen sc0 sc1 lds\n"
    "v_mfma_i32_16x16x64_i8 v[156:159], a[136:139], v[64:67], v[156:159]\n"
    "ds_read_b128 v[96:99], v43 offset:22528\n"
    "v_mfma_i32_16x16x64_i8 v[156:159], a[160:163], v[68:71], v[156:159]\n"
    "buffer_load_dwordx4 v22, s[40:43], s34 offen offset:1024 sc0 sc1 lds\n"
    "s_mov_b32 s96, 8\n"
    "v_mfma_i32_16x16x64_i8 v[156:159], a[168:171], v[72:75], v[156:159]\n"
    "v_mfma_i32_16x16x64_i8 v[160:163], a[140:143], v[64:67], v[160:163]\n"
    "v_mfma_i32_16x16x64_i8 v[160:163], a[164:167], v[68:71], v[160:163]\n"
    "s_mov_b32 s95, 6\n"
    "s_and_b32 s91, s84, s25\n"
    "v_mfma_i32_16x16x64_i8 v[160:163], a[172:175], v[72:75], v[160:163]\n"
    "s_cmp_gt_u32 s91, 6\n"
    "s_getpc_b64 s[88:89]\n"
    "s_cbranch_scc0 L_slowb%=\n"
    "s_waitcnt vmcnt(10) lgkmcnt(0)\n"
    "s_barrier\n"
    "v_mfma_i32_16x16x64_i8 v[160:163], a[116:119], v[76:79], v[160:163]\n"
    "ds_read_b128 v[100:103], v38\n"
    "s_waitcnt vmcnt(9)\n"
    "v_mfma_i32_16x16x64_i8 v[160:163], a[124:127], v[80:83], v[160:163]\n"
    "ds_read_b128 v[104:107], v39\n"
    "v_cmp_le_u32 vcc, s17, v32\n"
    "v_mfma_i32_16x16x64_i8 v[160:163], a[148:151], v[84:87], v[160:163]\n"
    "ds_read_b128 v[108:111], v38 offset:8192\n"
    "s_lshr_b32 s87, vcc_lo, 16\n"
    "v_mfma_i32_16x16x64_i8 v[156:159], a[112:115], v[76:79], v[156:159]\n"
    "ds_read_b128 v[112:115], v39 offset:8192\n"
    "v_mfma_i32_16x16x64_i8 v[156:159], a[120:123], v[80:83], v[156:159]\n"
    "ds_read_b128 v[52:55], v38 offset:16384\n"
    "s_and_b32 s87, s87, vcc_lo\n"
    "v_mfma_i32_16x16x64_i8 v[156:159], a[144:147], v[84:87], v[156:159]\n"
    "ds_read_b128 v[56:59], v39 offset:16384\n"
    "s_and_b32 s87, s87, vcc_hi\n"
    "v_mfma_i32_16x16x64_i8 v[156:159], a[152:155], v[88:91], v[156:159]\n"
    "ds_read_b128 v[60:63], v38 offset:2048\n"
    "s_ff0_i32_b32 s85, s87\n"
    "v_mfma_i32_16x16x64_i8 v[156:159], a[176:179], v[92:95], v[156:159]\n"
    "ds_read_b128 v[64:67], v39 offset:2048\n"
    "v_mfma_i32_16x16x64_i8 v[156:159], a[184:187], v[96:99], v[156:159]\n"
    "s_max_u32 s85, s85, s86\n"
    "s_min_u32 s91, s85, s96\n"
    "v_mfma_i32_16x16x64_i8 v[160:163], a[156:159], v[88:91], v[160:163]\n"
    "s_cmp_lt_u32 s84, s91\n"
    "s_getpc_b64 s[88:89]\n"
    "s_cbranch_scc1 L_catchb%=\n"
    "v_mfma_i32_16x16x64_i8 v[160:163], a[180:183], v[92:95], v[160:163]\n"
    "s_waitcnt vmcnt(8)\n"
    "v_cmp_gt_u32 s[28:29], s17, v37\n"
    "v_mfma_i32_16x16x64_i8 v[160:163], a[188:191], v[96:99], v[160:163]\n"
    "s_cmp_lg_u64 s[28:29], 0\n"
    "s_cbranch_scc1 L_gateb%=\n"
    "L_rgb%=:\n"
    "s_waitcnt lgkmcnt(5)\n"
    "v_mfma_i32_16x16x64_i8 v[168:171], a[4:7], v[100:103], v[168:171]\n"
    "global_load_dword v32, v24, s[72:73] sc0 sc1\n"
    "ds_read_b128 v[68:71], v38 offset:10240\n"
    "v_mfma_i32_16x16x64_i8 v[168:171], a[12:15], v[104:107], v[168:171]\n"
    "s_add_u32 s90, s65, 0\n"
    "buffer_load_dwordx4 v[204:207], v22, s[44:47], s90 offen sc0 sc1\n"
    "v_mfma_i32_16x16x64_i8 v[168:171], a[36:39], v[108:111], v[168:171]\n"
    "ds_read_b128 v[72:75], v39 offset:10240\n"
    "v_mfma_i32_16x16x64_i8 v[164:167], a[0:3], v[100:103], v[164:167]\n"
    "v_mfma_i32_16x16x64_i8 v[164:167], a[8:11], v[104:107], v[164:167]\n"
    "ds_read_b128 v[76:79], v38 offset:18432\n"
    "v_mfma_i32_16x16x64_i8 v[164:167], a[32:35], v[108:111], v[164:167]\n"
    "s_waitcnt lgkmcnt(5)\n"
    "buffer_load_dwordx4 v[208:211], v22, s[44:47], s90 offen offset:1024 sc0 sc1\n"
    "v_mfma_i32_16x16x64_i8 v[164:167], a[40:43], v[112:115], v[164:167]\n"
    "ds_read_b128 v[80:83], v39 offset:18432\n"
    "s_add_u32 s90, s64, 114688\n"
    "v_mfma_i32_16x16x64_i8 v[164:167], a[64:67], v[52:55], v[164:167]\n"
    "buffer_store_dwordx4 v[156:159], v22, s[44:47], s90 offen sc0 sc1\n"
    "v_mfma_i32_16x16x64_i8 v[164:167], a[72:75], v[56:59], v[164:167]\n"
    "ds_read_b128 v[84:87], v38 offset:4096\n"
    "v_mfma_i32_16x16x64_i8 v[168:171], a[44:47], v[112:115], v[168:171]\n"
    "v_mfma_i32_16x16x64_i8 v[168:171], a[68:71], v[52:55], v[168:171]\n"
    "ds_read_b128 v[88:91], v39 offset:4096\n"
    "v_mfma_i32_16x16x64_i8 v[168:171], a[76:79], v[56:59], v[168:171]\n"
    "s_waitcnt lgkmcnt(5)\n"
    "buffer_store_dwordx4 v[160:163], v22, s[44:47], s90 offen offset:1024 sc0 sc1\n"
    "v_mfma_i32_16x16x64_i8 v[168:171], a[20:23], v[60:63], v[168:171]\n"
    "ds_read_b128 v[92:95], v38 offset:12288\n"
    "s_cmp_eq_u32 s84, 8\n"
    "s_cselect_b32 s91, s85, 0\n"
    "v_mfma_i32_16x16x64_i8 v[168:171], a[28:31], v[64:67], v[168:171]\n"
    "s_cmp_gt_u32 s91, 8\n"
    "s_cselect_b32 s82, 65536, s83\n"
    "s_addc_u32 s84, s84, 0\n"
    "s_add_u32 m0, s11, 65536\n"
    "v_mfma_i32_16x16x64_i8 v[168:171], a[52:55], v[68:71], v[168:171]\n"
    "ds_read_b128 v[96:99], v39 offset:12288\n"
    "s_add_u32 s34, s8, s82\n"
    "v_mfma_i32_16x16x64_i8 v[164:167], a[16:19], v[60:63], v[164:167]\n"
    "buffer_load_dwordx4 v22, s[40:43], s34 offen sc0 sc1 lds\n"
    "s_cmp_lt_u32 s84, s96\n"
    "s_getpc_b64 s[88:89]\n"
    "s_cbranch_scc1 L_midb%=\n"
    "v_mfma_i32_16x16x64_i8 v[164:167], a[24:27], v[64:67], v[164:167]\n"
    "ds_read_b128 v[100:103], v38 offset:20480\n"
    "v_mfma_i32_16x16x64_i8 v[164:167], a[48:51], v[68:71], v[164:167]\n"
    "s_waitcnt lgkmcnt(5)\n"
    "v_mfma_i32_16x16x64_i8 v[164:167], a[56:59], v[72:75], v[164:167]\n"
    "ds_read_b128 v[104:107], v39 offset:20480\n"
    "v_mfma_i32_16x16x64_i8 v[164:167], a[80:83], v[76:79], v[164:167]\n"
    "buffer_load_dwordx4 v22, s[40:43], s34 offen offset:1024 sc0 sc1 lds\n"
    "s_add_u32 m0, s12, 65536\n"
    "v_mfma_i32_16x16x64_i8 v[164:167], a[88:91], v[80:83], v[164:167]\n"
    "ds_read_b128 v[108:111], v38 offset:6144\n"
    "s_add_u32 s34, s9, s82\n"
    "v_mfma_i32_16x16x64_i8 v[168:171], a[60:63], v[72:75], v[168:171]\n"
    "buffer_load_dwordx4 v22, s[40:43], s34 offen sc0 sc1 lds\n"
    "v_mfma_i32_16x16x64_i8 v[168:171], a[84:87], v[76:79], v[168:171]\n"
    "ds_read_b128 v[112:115], v39 offset:6144\n"
    "v_mfma_i32_16x16x64_i8 v[168:171], a[92:95], v[80:83], v[168:171]\n"
    "s_waitcnt lgkmcnt(5)\n"
    "v_mfma_i32_16x16x64_i8 v[168:171], a[100:103], v[84:87], v[168:171]\n"
    "ds_read_b128 v[52:55], v38 offset:14336\n"
    "v_mfma_i32_16x16x64_i8 v[168:171], a[108:111], v[88:91], v[168:171]\n"
    "buffer_load_dwordx4 v22, s[40:43], s34 offen offset:1024 sc0 sc1 lds\n"
    "v_mfma_i32_16x16x64_i8 v[168:171], a[132:135], v[92:95], v[168:171]\n"
    "ds_read_b128 v[56:59], v39 offset:14336\n"
    "v_mfma_i32_16x16x64_i8 v[164:167], a[96:99], v[84:87], v[164:167]\n"
    "s_add_u32 m0, s13, 65536\n"
    "s_add_u32 s34, s10, s82\n"
    "v_mfma_i32_16x16x64_i8 v[164:167], a[104:107], v[88:91], v[164:167]\n"
    "ds_read_b128 v[60:63], v38 offset:22528\n"
    "v_mfma_i32_16x16x64_i8 v[164:167], a[128:131], v[92:95], v[164:167]\n"
    "s_waitcnt lgkmcnt(5)\n"
    "buffer_load_dwordx4 v22, s[40:43], s34 offen sc0 sc1 lds\n"
    "v_mfma_i32_16x16x64_i8 v[164:167], a[136:139], v[96:99], v[164:167]\n"
    "ds_read_b128 v[64:67], v39 offset:22528\n"
    "v_mfma_i32_16x16x64_i8 v[164:167], a[160:163], v[100:103], v[164:167]\n"
    "buffer_load_dwordx4 v22, s[40:43], s34 offen offset:1024 sc0 sc1 lds\n"
    "s_mov_b32 s96, 9\n"
    "v_mfma_i32_16x16x64_i8 v[164:167], a[168:171], v[104:107], v[164:167]\n"
    "v_mfma_i32_16x16x64_i8 v[168:171], a[140:143], v[96:99], v[168:171]\n"
    "v_mfma_i32_16x16x64_i8 v[168:171], a[164:167], v[100:103], v[168:171]\n"
    "s_mov_b32 s95, 7\n"
    "s_and_b32 s91, s84, s25\n"
    "v_mfma_i32_16x16x64_i8 v[168:171], a[172:175], v[104:107], v[168:171]\n"
    "s_cmp_gt_u32 s91, 7\n"
    "s_getpc_b64 s[88:89]\n"
    "s_cbranch_scc0 L_slowb%=\n"
    "s_waitcnt vmcnt(11) lgkmcnt(0)\n"
    "s_barrier\n"
    "v_mfma_i32_16x16x64_i8 v[168:171], a[116:119], v[108:111], v[168:171]\n"
    "ds_read_b128 v[68:71], v38 offset:32768\n"
    "s_waitcnt vmcnt(10)\n"
    "v_mfma_i32_16x16x64_i8 v[168:171], a[124:127], v[112:115], v[168:171]\n"
    "ds_read_b128 v[72:75], v39 offset:32768\n"
    "v_cmp_le_u32 vcc, s17, v32\n"
    "v_mfma_i32_16x16x64_i8 v[168:171], a[148:151], v[52:55], v[168:171]\n"
    "ds_read_b128 v[76:79], v38 offset:40960\n"
    "v_mfma_i32_16x16x64_i8 v[164:167], a[112:115], v[108:111], v[164:167]\n"
    "ds_read_b128 v[80:83], v39 offset:40960\n"
    "s_lshr_b32 s87, vcc_lo, 16\n"
    "v_mfma_i32_16x16x64_i8 v[164:167], a[120:123], v[112:115], v[164:167]\n"
    "ds_read_b128 v[84:87], v38 offset:49152\n"
    "v_mfma_i32_16x16x64_i8 v[164:167], a[144:147], v[52:55], v[164:167]\n"
    "ds_read_b128 v[88:91], v39 offset:49152\n"
    "s_and_b32 s87, s87, vcc_lo\n"
    "v_mfma_i32_16x16x64_i8 v[164:167], a[152:155], v[56:59], v[164:167]\n"
    "ds_read_b128 v[92:95], v38 offset:34816\n"
    "v_mfma_i32_16x16x64_i8 v[164:167], a[176:179], v[60:63], v[164:167]\n"
    "ds_read_b128 v[96:99], v39 offset:34816\n"
    "s_and_b32 s87, s87, vcc_hi\n"
    "v_mfma_i32_16x16x64_i8 v[164:167], a[184:187], v[64:67], v[164:167]\n"
    "s_ff0_i32_b32 s85, s87\n"
    "v_mfma_i32_16x16x64_i8 v[168:171], a[156:159], v[56:59], v[168:171]\n"
    "s_max_u32 s85, s85, s86\n"
    "v_mfma_i32_16x16x64_i8 v[168:171], a[180:183], v[60:63], v[168:171]\n"
    "s_min_u32 s91, s85, s96\n"
    "v_mfma_i32_16x16x64_i8 v[168:171], a[188:191], v[64:67], v[168:171]\n"
    "s_cmp_lt_u32 s84, s91\n"
    "s_getpc_b64 s[88:89]\n"
    "s_cbranch_scc1 L_catchb%=\n"
    "s_waitcnt lgkmcnt(5)\n"
    "v_mfma_i32_16x16x64_i8 v[176:179], a[4:7], v[68:71], v[176:179]\n"
    "global_load_dword v32, v24, s[72:73] sc0 sc1\n"
    "ds_read_b128 v[100:103], v38 offset:43008\n"
    "v_mfma_i32_16x16x64_i8 v[176:179], a[12:15], v[72:75], v[176:179]\n"
    "s_add_u32 s90, s65, 24576\n"
    "buffer_load_dwordx4 v[212:215], v22, s[44:47], s90 offen sc0 sc1\n"
    "v_mfma_i32_16x16x64_i8 v[176:179], a[36:39], v[76:79], v[176:179]\n"
    "ds_read_b128 v[104:107], v39 offset:43008\n"
    "v_mfma_i32_16x16x64_i8 v[172:175], a[0:3], v[68:71], v[172:175]\n"
    "v_mfma_i32_16x16x64_i8 v[172:175], a[8:11], v[72:75], v[172:175]\n"
    "ds_read_b128 v[108:111], v38 offset:51200\n"
    "v_mfma_i32_16x16x64_i8 v[172:175], a[32:35], v[76:79], v[172:175]\n"
    "s_waitcnt lgkmcnt(5)\n"
    "buffer_load_dwordx4 v[216:219], v22, s[44:47], s90 offen offset:1024 sc0 sc1\n"
    "s_waitcnt vmcnt(11)\n"
    "v_mfma_i32_16x16x64_i8 v[172:175], a[40:43], v[80:83], v[172:175]\n"
    "ds_read_b128 v[112:115], v39 offset:51200\n"
    "s_cmp_eq_u32 s84, 9\n"
    "s_cselect_b32 s91, s85, 0\n"
    "v_mfma_i32_16x16x64_i8 v[172:175], a[64:67], v[84:87], v[172:175]\n"
    "s_cmp_gt_u32 s91, 9\n"
    "s_cselect_b32 s82, 73728, s83\n"
    "s_addc_u32 s84, s84, 0\n"
    "s_add_u32 m0, s11, 98304\n"
    "v_mfma_i32_16x16x64_i8 v[172:175], a[72:75], v[88:91], v[172:175]\n"
    "ds_read_b128 v[52:55], v38 offset:36864\n"
    "s_add_u32 s34, s8, s82\n"
    "v_add_u32 v204, v164, v204\n"
    "v_mfma_i32_16x16x64_i8 v[176:179], a[44:47], v[80:83], v[176:179]\n"
    "buffer_load_dwordx4 v22, s[40:43], s34 offen sc0 sc1 lds\n"
    "v_add_u32 v205, v165, v205\n"
    "v_mfma_i32_16x16x64_i8 v[176:179], a[68:71], v[84:87], v[176:179]\n"
    "ds_read_b128 v[56:59], v39 offset:36864\n"
    "v_add_u32 v206, v166, v206\n"
    "v_mfma_i32_16x16x64_i8 v[176:179], a[76:79], v[88:91], v[176:179]\n"
    "s_waitcnt lgkmcnt(5)\n"
    "v_add_u32 v207, v167, v207\n"
    "v_mfma_i32_16x16x64_i8 v[176:179], a[20:23], v[92:95], v[176:179]\n"
    "ds_read_b128 v[60:63], v38 offset:45056\n"
    "v_add_u32 v208, v168, v208\n"
    "v_mfma_i32_16x16x64_i8 v[176:179], a[28:31], v[96:99], v[176:179]\n"
    "buffer_load_dwordx4 v22, s[40:43], s34 offen offset:1024 sc0 sc1 lds\n"
    "v_add_u32 v209, v169, v209\n"
    "v_add_u32 v210, v170, v210\n"
    "v_mfma_i32_16x16x64_i8 v[176:179], a[52:55], v[100:103], v[176:179]\n"
    "ds_read_b128 v[64:67], v39 offset:45056\n"
    "v_add_u32 v211, v171, v211\n"
    "v_mfma_i32_16x16x64_i8 v[172:175], a[16:19], v[92:95], v[172:175]\n"
    "s_cmp_lt_u32 s84, s96\n"
    "s_getpc_b64 s[88:89]\n"
    "s_cbranch_scc1 L_midb%=\n"
    "v_mfma_i32_16x16x64_i8 v[172:175], a[24:27], v[96:99], v[172:175]\n"
    "ds_read_b128 v[68:71], v38 offset:53248\n"
    "v_cvt_f32_i32 v204, v204\n"
    "v_mfma_i32_16x16x64_i8 v[172:175], a[48:51], v[100:103], v[172:175]\n"
    "s_waitcnt lgkmcnt(5)\n"
    "s_add_u32 m0, s12, 98304\n"
    "v_cvt_f32_i32 v205, v205\n"
    "v_mfma_i32_16x16x64_i8 v[172:175], a[56:59], v[104:107], v[172:175]\n"
    "ds_read_b128 v[72:75], v39 offset:53248\n"
    "s_add_u32 s34, s9, s82\n"
    "v_mfma_i32_16x16x64_i8 v[172:175], a[80:83], v[108:111], v[172:175]\n"
    "buffer_load_dwordx4 v22, s[40:43], s34 offen sc0 sc1 lds\n"
    "v_cvt_f32_i32 v206, v206\n"
    "v_mfma_i32_16x16x64_i8 v[172:175], a[88:91], v[112:115], v[172:175]\n"
    "ds_read_b128 v[76:79], v38 offset:38912\n"
    "v_cvt_f32_i32 v207, v207\n"
    "v_mfma_i32_16x16x64_i8 v[176:179], a[60:63], v[104:107], v[176:179]\n"
    "v_cvt_f32_i32 v208, v208\n"
    "v_cvt_f32_i32 v209, v209\n"
    "v_mfma_i32_16x16x64_i8 v[176:179], a[84:87], v[108:111], v[176:179]\n"
    "ds_read_b128 v[80:83], v39 offset:38912\n"
    "v_cvt_f32_i32 v210, v210\n"
    "v_mfma_i32_16x16x64_i8 v[176:179], a[92:95], v[112:115], v[176:179]\n"
    "s_waitcnt lgkmcnt(5)\n"
    "buffer_load_dwordx4 v22, s[40:43], s34 offen offset:1024 sc0 sc1 lds\n"
    "v_cvt_f32_i32 v211, v211\n"
    "v_mfma_i32_16x16x64_i8 v[176:179], a[100:103], v[52:55], v[176:179]\n"
    "ds_read_b128 v[84:87], v38 offset:47104\n"
    "v_max3_f32 v34, |v204|, |v205|, |v206|\n"
    "v_mfma_i32_16x16x64_i8 v[176:179], a[108:111], v[56:59], v[176:179]\n"
    "s_add_u32 m0, s13, 98304\n"
    "s_add_u32 s34, s10, s82\n"
    "v_max3_f32 v34, v34, |v207|, |v208|\n"
    "v_mfma_i32_16x16x64_i8 v[176:179], a[132:135], v[60:63], v[176:179]\n"
    "ds_read_b128 v[88:91], v39 offset:47104\n"
    "v_max3_f32 v34, v34, |v209|, |v210|\n"
    "v_mfma_i32_16x16x64_i8 v[172:175], a[96:99], v[52:55], v[172:175]\n"
    "buffer_load_dwordx4 v22, s[40:43], s34 offen sc0 sc1 lds\n"
    "v_max_f32 v34, v34, |v211|\n"
    "v_mfma_i32_16x16x64_i8 v[172:175], a[104:107], v[56:59], v[172:175]\n"
    "ds_read_b128 v[92:95], v38 offset:55296\n"
    "v_cmp_lt_f32 s[28:29], s97, v34\n"
    "v_mfma_i32_16x16x64_i8 v[172:175], a[128:131], v[60:63], v[172:175]\n"
    "s_waitcnt lgkmcnt(5)\n"
    "s_cmp_lg_u64 s[28:29], 0\n"
    "s_cselect_b32 s24, 64, 0\n"
    "s_or_b32 s81, s81, s24\n"
    "v_fma_f32 v204, v204, s18, v33\n"
    "v_mfma_i32_16x16x64_i8 v[172:175], a[136:139], v[64:67], v[172:175]\n"
    "ds_read_b128 v[96:99], v39 offset:55296\n"
    "v_fma_f32 v205, v205, s18, v33\n"
    "v_mfma_i32_16x16x64_i8 v[172:175], a[160:163], v[68:71], v[172:175]\n"
    "buffer_load_dwordx4 v22, s[40:43], s34 offen offset:1024 sc0 sc1 lds\n"
    "s_mov_b32 s96, 10\n"
    "v_mfma_i32_16x16x64_i8 v[172:175], a[168:171], v[72:75], v[172:175]\n"
    "v_fma_f32 v206, v206, s18, v33\n"
    "v_fma_f32 v207, v207, s18, v33\n"
    "v_mfma_i32_16x16x64_i8 v[176:179], a[140:143], v[64:67], v[176:179]\n"
    "v_fma_f32 v208, v208, s18, v33\n"
    "v_fma_f32 v209, v209, s18, v33\n"
    "v_mfma_i32_16x16x64_i8 v[176:179], a[164:167], v[68:71], v[176:179]\n"
    "s_mov_b32 s95, 8\n"
    "s_and_b32 s91, s84, s25\n"
    "v_mfma_i32_16x16x64_i8 v[176:179], a[172:175], v[72:75], v[176:179]\n"
    "s_cmp_gt_u32 s91, 8\n"
    "s_getpc_b64 s[88:89]\n"
    "s_cbranch_scc0 L_slowb%=\n"
    "s_waitcnt vmcnt(9) lgkmcnt(0)\n"
    "s_barrier\n"
    "v_mfma_i32_16x16x64_i8 v[176:179], a[116:119], v[76:79], v[176:179]\n"
    "ds_read_b128 v[100:103], v40\n"
    "s_waitcnt vmcnt(8)\n"
    "v_fma_f32 v210, v210, s18, v33\n"
    "v_mfma_i32_16x16x64_i8 v[176:179], a[124:127], v[80:83], v[176:179]\n"
    "ds_read_b128 v[104:107], v41\n"
    "v_cmp_le_u32 vcc, s17, v32\n"
    "v_fma_f32 v211, v211, s18, v33\n"
    "v_mfma_i32_16x16x64_i8 v[176:179], a[148:151], v[84:87], v[176:179]\n"
    "ds_read_b128 v[108:111], v40 offset:8192\n"
    "v_max_f32 v204, v204, v33\n"
    "v_mfma_i32_16x16x64_i8 v[172:175], a[112:115], v[76:79], v[172:175]\n"
    "ds_read_b128 v[112:115], v41 offset:8192\n"
    "s_lshr_b32 s87, vcc_lo, 16\n"
    "v_mfma_i32_16x16x64_i8 v[172:175], a[120:123], v[80:83], v[172:175]\n"
    "ds_read_b128 v[52:55], v40 offset:16384\n"
    "v_max_f32 v205, v205, v33\n"
    "v_max_f32 v206, v206, v33\n"
    "v_mfma_i32_16x16x64_i8 v[172:175], a[144:147], v[84:87], v[172:175]\n"
    "ds_read_b128 v[56:59], v41 offset:16384\n"
    "s_and_b32 s87, s87, vcc_lo\n"
    "v_mfma_i32_16x16x64_i8 v[172:175], a[152:155], v[88:91], v[172:175]\n"
    "ds_read_b128 v[60:63], v40 offset:2048\n"
    "v_max_f32 v207, v207, v33\n"
    "v_mfma_i32_16x16x64_i8 v[172:175], a[176:179], v[92:95], v[172:175]\n"
    "ds_read_b128 v[64:67], v41 offset:2048\n"
    "s_and_b32 s87, s87, vcc_hi\n"
    "v_max_f32 v208, v208, v33\n"
    "v_mfma_i32_16x16x64_i8 v[172:175], a[184:187], v[96:99], v[172:175]\n"
    "s_ff0_i32_b32 s85, s87\n"
    "v_max_f32 v209, v209, v33\n"
    "v_mfma_i32_16x16x64_i8 v[176:179], a[156:159], v[88:91], v[176:179]\n"
    "s_max_u32 s85, s85, s86\n"
    "v_max_f32 v210, v210, v33\n"
    "v_mfma_i32_16x16x64_i8 v[176:179], a[180:183], v[92:95], v[176:179]\n"
    "s_min_u32 s91, s85, s96\n"
    "v_max_f32 v211, v211, v33\n"
    "v_mfma_i32_16x16x64_i8 v[176:179], a[188:191], v[96:99], v[176:179]\n"
    "s_cmp_lt_u32 s84, s91\n"
    "s_getpc_b64 s[88:89]\n"
    "s_cbranch_scc1 L_catchb%=\n"
    "s_waitcnt lgkmcnt(5)\n"
    "v_mfma_i32_16x16x64_i8 v[184:187], a[4:7], v[100:103], v[184:187]\n"
    "global_load_dword v32, v24, s[72:73] sc0 sc1\n"
    "ds_read_b128 v[68:71], v40 offset:10240\n"
    "v_mfma_i32_16x16x64_i8 v[184:187], a[12:15], v[104:107], v[184:187]\n"
    "s_add_u32 s90, s65, 49152\n"
    "buffer_load_dwordx4 v[220:223], v22, s[44:47], s90 offen sc0 sc1\n"
    "v_mfma_i32_16x16x64_i8 v[184:187], a[36:39], v[108:111], v[184:187]\n"
    "ds_read_b128 v[72:75], v41 offset:10240\n"
    "v_mfma_i32_16x16x64_i8 v[180:183], a[0:3], v[100:103], v[180:183]\n"
    "v_mfma_i32_16x16x64_i8 v[180:183], a[8:11], v[104:107], v[180:183]\n"
    "ds_read_b128 v[76:79], v40 offset:18432\n"
    "v_mfma_i32_16x16x64_i8 v[180:183], a[32:35], v[108:111], v[180:183]\n"
    "s_waitcnt lgkmcnt(5)\n"
    "buffer_load_dwordx4 v[224:227], v22, s[44:47], s90 offen offset:1024 sc0 sc1\n"
    "s_add_u32 s90, s65, 0\n"
    "v_mfma_i32_16x16x64_i8 v[180:183], a[40:43], v[112:115], v[180:183]\n"
    "ds_read_b128 v[80:83], v41 offset:18432\n"
    "s_lshl_b32 s90, s90, 5\n"
    "s_waitcnt vmcnt(9)\n"
    "v_mfma_i32_16x16x64_i8 v[180:183], a[64:67], v[52:55], v[180:183]\n"
    "buffer_store_dwordx4 v[204:207], v23, s[48:51], s90 offen sc0 sc1\n"
    "v_add_u32 v212, v172, v212\n"
    "v_mfma_i32_16x16x64_i8 v[180:183], a[72:75], v[56:59], v[180:183]\n"
    "ds_read_b128 v[84:87], v40 offset:4096\n"
    "v_add_u32 v213, v173, v213\n"
    "v_add_u32 v214, v174, v214\n"
    "v_mfma_i32_16x16x64_i8 v[184:187], a[44:47], v[112:115], v[184:187]\n"
    "v_add_u32 v215, v175, v215\n"
    "v_mfma_i32_16x16x64_i8 v[184:187], a[68:71], v[52:55], v[184:187]\n"
    "ds_read_b128 v[88:91], v41 offset:4096\n"
    "v_add_u32 v216, v176, v216\n"
    "v_add_u32 v217, v177, v217\n"
    "v_mfma_i32_16x16x64_i8 v[184:187], a[76:79], v[56:59], v[184:187]\n"
    "s_waitcnt lgkmcnt(5)\n"
    "buffer_store_dwordx4 v[208:211], v23, s[48:51], s90 offen offset:64 sc0 sc1\n"
    "v_mfma_i32_16x16x64_i8 v[184:187], a[20:23], v[60:63], v[184:187]\n"
    "ds_read_b128 v[92:95], v40 offset:12288\n"
    "s_cmp_eq_u32 s84, 10\n"
    "s_cselect_b32 s91, s85, 0\n"
    "v_mfma_i32_16x16x64_i8 v[184:187], a[28:31], v[64:67], v[184:187]\n"
    "s_cmp_gt_u32 s91, 10\n"
    "s_cselect_b32 s82, 81920, s83\n"
    "s_addc_u32 s84, s84, 0\n"
    "s_add_u32 m0, s11, 131072\n"
    "v_mfma_i32_16x16x64_i8 v[184:187], a[52:55], v[68:71], v[184:187]\n"
    "ds_read_b128 v[96:99], v41 offset:12288\n"
    "s_add_u32 s34, s8, s82\n"
    "v_add_u32 v218, v178, v218\n"
    "v_mfma_i32_16x16x64_i8 v[180:183], a[16:19], v[60:63], v[180:183]\n"
    "buffer_load_dwordx4 v22, s[40:43], s34 offen sc0 sc1 lds\n"
    "s_cmp_lt_u32 s84, s96\n"
    "s_getpc_b64 s[88:89]\n"
    "s_cbranch_scc1 L_midb%=\n"
    "v_mfma_i32_16x16x64_i8 v[180:183], a[24:27], v[64:67], v[180:183]\n"
    "ds_read_b128 v[100:103], v40 offset:20480\n"
    "v_add_u32 v219, v179, v219\n"
    "v_mfma_i32_16x16x64_i8 v[180:183], a[48:51], v[68:71], v[180:183]\n"
    "s_waitcnt lgkmcnt(5)\n"
    "v_cvt_f32_i32 v212, v212\n"
    "v_mfma_i32_16x16x64_i8 v[180:183], a[56:59], v[72:75], v[180:183]\n"
    "ds_read_b128 v[104:107], v41 offset:20480\n"
    "v_cvt_f32_i32 v213, v213\n"
    "v_cvt_f32_i32 v214, v214\n"
    "v_mfma_i32_16x16x64_i8 v[180:183], a[80:83], v[76:79], v[180:183]\n"
    "buffer_load_dwordx4 v22, s[40:43], s34 offen offset:1024 sc0 sc1 lds\n"
    "s_add_u32 m0, s12, 131072\n"
    "v_mfma_i32_16x16x64_i8 v[180:183], a[88:91], v[80:83], v[180:183]\n"
    "ds_read_b128 v[108:111], v40 offset:6144\n"
    "s_add_u32 s34, s9, s82\n"
    "v_cvt_f32_i32 v215, v215\n"
    "v_mfma_i32_16x16x64_i8 v[184:187], a[60:63], v[72:75], v[184:187]\n"
    "buffer_load_dwordx4 v22, s[40:43], s34 offen sc0 sc1 lds\n"
    "v_cvt_f32_i32 v216, v216\n"
    "v_mfma_i32_16x16x64_i8 v[184:187], a[84:87], v[76:79], v[184:187]\n"
    "ds_read_b128 v[112:115], v41 offset:6144\n"
    "v_cvt_f32_i32 v217, v217\n"
    "v_mfma_i32_16x16x64_i8 v[184:187], a[92:95], v[80:83], v[184:187]\n"
    "s_waitcnt lgkmcnt(5)\n"
    "v_cvt_f32_i32 v218, v218\n"
    "v_cvt_f32_i32 v219, v219\n"
    "v_mfma_i32_16x16x64_i8 v[184:187], a[100:103], v[84:87], v[184:187]\n"
    "ds_read_b128 v[52:55], v40 offset:14336\n"
    "v_max3_f32 v34, |v212|, |v213|, |v214|\n"
    "v_mfma_i32_16x16x64_i8 v[184:187], a[108:111], v[88:91], v[184:187]\n"
    "buffer_load_dwordx4 v22, s[40:43], s34 offen offset:1024 sc0 sc1 lds\n"
    "v_max3_f32 v34, v34, |v215|, |v216|\n"
    "v_mfma_i32_16x16x64_i8 v[184:187], a[132:135], v[92:95], v[184:187]\n"
    "ds_read_b128 v[56:59], v41 offset:14336\n"
    "v_max3_f32 v34, v34, |v217|, |v218|\n"
    "v_mfma_i32_16x16x64_i8 v[180:183], a[96:99], v[84:87], v[180:183]\n"
    "s_add_u32 m0, s13, 131072\n"
    "s_add_u32 s34, s10, s82\n"
    "v_max_f32 v34, v34, |v219|\n"
    "v_mfma_i32_16x16x64_i8 v[180:183], a[104:107], v[88:91], v[180:183]\n"
    "ds_read_b128 v[60:63], v40 offset:22528\n"
    "v_cmp_lt_f32 s[28:29], s97, v34\n"
    "v_mfma_i32_16x16x64_i8 v[180:183], a[128:131], v[92:95], v[180:183]\n"
    "s_waitcnt lgkmcnt(5)\n"
    "buffer_load_dwordx4 v22, s[40:43], s34 offen sc0 sc1 lds\n"
    "s_cmp_lg_u64 s[28:29], 0\n"
    "s_cselect_b32 s24, 128, 0\n"
    "s_or_b32 s81, s81, s24\n"
    "v_mfma_i32_16x16x64_i8 v[180:183], a[136:139], v[96:99], v[180:183]\n"
    "ds_read_b128 v[64:67], v41 offset:22528\n"
    "v_fma_f32 v212, v212, s18, v33\n"
    "v_mfma_i32_16x16x64_i8 v[180:183], a[160:163], v[100:103], v[180:183]\n"
    "buffer_load_dwordx4 v22, s[40:43], s34 offen offset:1024 sc0 sc1 lds\n"
    "s_mov_b32 s96, 11\n"
    "v_fma_f32 v213, v213, s18, v33\n"
    "v_mfma_i32_16x16x64_i8 v[180:183], a[168:171], v[104:107], v[180:183]\n"
    "v_fma_f32 v214, v214, s18, v33\n"
    "v_mfma_i32_16x16x64_i8 v[184:187], a[140:143], v[96:99], v[184:187]\n"
    "v_fma_f32 v215, v215, s18, v33\n"
    "v_fma_f32 v216, v216, s18, v33\n"
    "v_mfma_i32_16x16x64_i8 v[184:187], a[164:167], v[100:103], v[184:187]\n"
    "s_mov_b32 s95, 9\n"
    "s_and_b32 s91, s84, s25\n"
    "v_fma_f32 v217, v217, s18, v33\n"
    "v_mfma_i32_16x16x64_i8 v[184:187], a[172:175], v[104:107], v[184:187]\n"
    "s_cmp_gt_u32 s91, 9\n"
    "s_getpc_b64 s[88:89]\n"
    "s_cbranch_scc0 L_slowb%=\n"
    "s_waitcnt vmcnt(11) lgkmcnt(0)\n"
    "s_barrier\n"
    "v_mfma_i32_16x16x64_i8 v[184:187], a[116:119], v[108:111], v[184:187]\n"
    "ds_read_b128 v[68:71], v40 offset:32768\n"
    "s_waitcnt vmcnt(10)\n"
    "v_fma_f32 v218, v218, s18, v33\n"
    "v_mfma_i32_16x16x64_i8 v[184:187], a[124:127], v[112:115], v[184:187]\n"
    "ds_read_b128 v[72:75], v41 offset:32768\n"
    "v_cmp_le_u32 vcc, s17, v32\n"
    "v_mfma_i32_16x16x64_i8 v[184:187], a[148:151], v[52:55], v[184:187]\n"
    "ds_read_b128 v[76:79], v40 offset:40960\n"
    "v_fma_f32 v219, v219, s18, v33\n"
    "v_mfma_i32_16x16x64_i8 v[180:183], a[112:115], v[108:111], v[180:183]\n"
    "ds_read_b128 v[80:83], v41 offset:40960\n"
    "s_lshr_b32 s87, vcc_lo, 16\n"
    "v_max_f32 v212, v212, v33\n"
    "v_mfma_i32_16x16x64_i8 v[180:183], a[120:123], v[112:115], v[180:183]\n"
    "ds_read_b128 v[84:87], v40 offset:49152\n"
    "v_max_f32 v213, v213, v33\n"
    "v_mfma_i32_16x16x64_i8 v[180:183], a[144:147], v[52:55], v[180:183]\n"
    "ds_read_b128 v[88:91], v41 offset:49152\n"
    "s_and_b32 s87, s87, vcc_lo\n"
    "v_max_f32 v214, v214, v33\n"
    "v_mfma_i32_16x16x64_i8 v[180:183], a[152:155], v[56:59], v[180:183]\n"
    "ds_read_b128 v[92:95], v40 offset:34816\n"
    "v_max_f32 v215, v215, v33\n"
    "v_mfma_i32_16x16x64_i8 v[180:183], a[176:179], v[60:63], v[180:183]\n"
    "ds_read_b128 v[96:99], v41 offset:34816\n"
    "s_and_b32 s87, s87, vcc_hi\n"
    "v_max_f32 v216, v216, v33\n"
    "v_mfma_i32_16x16x64_i8 v[180:183], a[184:187], v[64:67], v[180:183]\n"
    "s_ff0_i32_b32 s85, s87\n"
    "v_max_f32 v217, v217, v33\n"
    "v_mfma_i32_16x16x64_i8 v[184:187], a[156:159], v[56:59], v[184:187]\n"
    "s_max_u32 s85, s85, s86\n"
    "v_max_f32 v218, v218, v33\n"
    "v_mfma_i32_16x16x64_i8 v[184:187], a[180:183], v[60:63], v[184:187]\n"
    "s_min_u32 s91, s85, s96\n"
    "v_max_f32 v219, v219, v33\n"
    "v_mfma_i32_16x16x64_i8 v[184:187], a[188:191], v[64:67], v[184:187]\n"
    "s_cmp_lt_u32 s84, s91\n"
    "s_getpc_b64 s[88:89]\n"
    "s_cbranch_scc1 L_catchb%=\n"
    "s_waitcnt lgkmcnt(5)\n"
    "v_mfma_i32_16x16x64_i8 v[192:195], a[4:7], v[68:71], v[192:195]\n"
    "ds_read_b128 v[100:103], v40 offset:43008\n"
    "s_add_u32 s90, s65, 73728\n"
    "buffer_load_dwordx4 v[228:231], v22, s[44:47], s90 offen sc0 sc1\n"
    "v_mfma_i32_16x16x64_i8 v[192:195], a[12:15], v[72:75], v[192:195]\n"
    "v_mfma_i32_16x16x64_i8 v[192:195], a[36:39], v[76:79], v[192:195]\n"
    "ds_read_b128 v[104:107], v41 offset:43008\n"
    "v_mfma_i32_16x16x64_i8 v[188:191], a[0:3], v[68:71], v[188:191]\n"
    "buffer_load_dwordx4 v[232:235], v22, s[44:47], s90 offen offset:1024 sc0 sc1\n"
    "v_mfma_i32_16x16x64_i8 v[188:191], a[8:11], v[72:75], v[188:191]\n"
    "ds_read_b128 v[108:111], v40 offset:51200\n"
    "v_mfma_i32_16x16x64_i8 v[188:191], a[32:35], v[76:79], v[188:191]\n"
    "s_waitcnt lgkmcnt(5)\n"
    "s_add_u32 s90, s65, 24576\n"
    "v_mfma_i32_16x16x64_i8 v[188:191], a[40:43], v[80:83], v[188:191]\n"
    "ds_read_b128 v[112:115], v41 offset:51200\n"
    "s_lshl_b32 s90, s90, 5\n"
    "v_mfma_i32_16x16x64_i8 v[188:191], a[64:67], v[84:87], v[188:191]\n"
    "buffer_store_dwordx4 v[212:215], v23, s[48:51], s90 offen sc0 sc1\n"
    "s_waitcnt vmcnt(11)\n"
    "v_mfma_i32_16x16x64_i8 v[188:191], a[72:75], v[88:91], v[188:191]\n"
    "ds_read_b128 v[52:55], v40 offset:36864\n"
    "v_add_u32 v220, v180, v220\n"
    "v_mfma_i32_16x16x64_i8 v[192:195], a[44:47], v[80:83], v[192:195]\n"
    "v_add_u32 v221, v181, v221\n"
    "v_add_u32 v222, v182, v222\n"
    "v_mfma_i32_16x16x64_i8 v[192:195], a[68:71], v[84:87], v[192:195]\n"
    "ds_read_b128 v[56:59], v41 offset:36864\n"
    "v_mfma_i32_16x16x64_i8 v[192:195], a[76:79], v[88:91], v[192:195]\n"
    "s_waitcnt lgkmcnt(5)\n"
    "buffer_store_dwordx4 v[216:219], v23, s[48:51], s90 offen offset:64 sc0 sc1\n"
    "v_mfma_i32_16x16x64_i8 v[192:195], a[20:23], v[92:95], v[192:195]\n"
    "ds_read_b128 v[60:63], v40 offset:45056\n"
    "v_add_u32 v223, v183, v223\n"
    "v_mfma_i32_16x16x64_i8 v[192:195], a[28:31], v[96:99], v[192:195]\n"
    "v_add_u32 v224, v184, v224\n"
    "v_add_u32 v225, v185, v225\n"
    "v_mfma_i32_16x16x64_i8 v[192:195], a[52:55], v[100:103], v[192:195]\n"
    "ds_read_b128 v[64:67], v41 offset:45056\n"
    "v_add_u32 v226, v186, v226\n"
    "v_mfma_i32_16x16x64_i8 v[188:191], a[16:19], v[92:95], v[188:191]\n"
    "s_cmp_lt_u32 s84, s96\n"
    "s_getpc_b64 s[88:89]\n"
    "s_cbranch_scc1 L_midb%=\n"
    "v_mfma_i32_16x16x64_i8 v[188:191], a[24:27], v[96:99], v[188:191]\n"
    "ds_read_b128 v[68:71], v40 offset:53248\n"
    "v_mfma_i32_16x16x64_i8 v[188:191], a[48:51], v[100:103], v[188:191]\n"
    "s_waitcnt lgkmcnt(5)\n"
    "v_add_u32 v227, v187, v227\n"
    "v_mfma_i32_16x16x64_i8 v[188:191], a[56:59], v[104:107], v[188:191]\n"
    "ds_read_b128 v[72:75], v41 offset:53248\n"
    "v_cvt_f32_i32 v220, v220\n"
    "v_mfma_i32_16x16x64_i8 v[188:191], a[80:83], v[108:111], v[188:191]\n"
    "v_cvt_f32_i32 v221, v221\n"
    "v_mfma_i32_16x16x64_i8 v[188:191], a[88:91], v[112:115], v[188:191]\n"
    "ds_read_b128 v[76:79], v40 offset:38912\n"
    "v_cvt_f32_i32 v222, v222\n"
    "v_mfma_i32_16x16x64_i8 v[192:195], a[60:63], v[104:107], v[192:195]\n"
    "v_cvt_f32_i32 v223, v223\n"
    "v_cvt_f32_i32 v224, v224\n"
    "v_mfma_i32_16x16x64_i8 v[192:195], a[84:87], v[108:111], v[192:195]\n"
    "ds_read_b128 v[80:83], v41 offset:38912\n"
    "v_cvt_f32_i32 v225, v225\n"
    "v_mfma_i32_16x16x64_i8 v[192:195], a[92:95], v[112:115], v[192:195]\n"
    "s_waitcnt lgkmcnt(5)\n"
    "v_mfma_i32_16x16x64_i8 v[192:195], a[100:103], v[52:55], v[192:195]\n"
    "ds_read_b128 v[84:87], v40 offset:47104\n"
    "v_cvt_f32_i32 v226, v226\n"
    "v_mfma_i32_16x16x64_i8 v[192:195], a[108:111], v[56:59], v[192:195]\n"
    "v_cvt_f32_i32 v227, v227\n"
    "v_max3_f32 v34, |v220|, |v221|, |v222|\n"
    "v_mfma_i32_16x16x64_i8 v[192:195], a[132:135], v[60:63], v[192:195]\n"
    "ds_read_b128 v[88:91], v41 offset:47104\n"
    "v_mfma_i32_16x16x64_i8 v[188:191], a[96:99], v[52:55], v[188:191]\n"
    "v_max3_f32 v34, v34, |v223|, |v224|\n"
    "v_max3_f32 v34, v34, |v225|, |v226|\n"
    "v_mfma_i32_16x16x64_i8 v[188:191], a[104:107], v[56:59], v[188:191]\n"
    "ds_read_b128 v[92:95], v40 offset:55296\n"
    "v_max_f32 v34, v34, |v227|\n"
    "v_mfma_i32_16x16x64_i8 v[188:191], a[128:131], v[60:63], v[188:191]\n"
    "s_waitcnt lgkmcnt(5)\n"
    "v_cmp_lt_f32 s[28:29], s97, v34\n"
    "v_mfma_i32_16x16x64_i8 v[188:191], a[136:139], v[64:67], v[188:191]\n"
    "ds_read_b128 v[96:99], v41 offset:55296\n"
    "v_mfma_i32_16x16x64_i8 v[188:191], a[160:163], v[68:71], v[188:191]\n"
    "s_cmp_lg_u64 s[28:29], 0\n"
    "s_cselect_b32 s24, 256, 0\n"
    "s_or_b32 s81, s81, s24\n"
    "v_fma_f32 v220, v220, s18, v33\n"
    "v_mfma_i32_16x16x64_i8 v[188:191], a[168:171], v[72:75], v[188:191]\n"
    "v_fma_f32 v221, v221, s18, v33\n"
    "v_mfma_i32_16x16x64_i8 v[192:195], a[140:143], v[64:67], v[192:195]\n"
    "v_fma_f32 v222, v222, s18, v33\n"
    "v_fma_f32 v223, v223, s18, v33\n"
    "v_mfma_i32_16x16x64_i8 v[192:195], a[164:167], v[68:71], v[192:195]\n"
    "s_mov_b32 s95, 10\n"
    "s_and_b32 s91, s84, s25\n"
    "v_mfma_i32_16x16x64_i8 v[192:195], a[172:175], v[72:75], v[192:195]\n"
    "s_cmp_gt_u32 s91, 10\n"
    "s_getpc_b64 s[88:89]\n"
    "s_cbranch_scc0 L_slowb%=\n"
    "s_waitcnt vmcnt(4) lgkmcnt(0)\n"
    "s_barrier\n"
    "v_mfma_i32_16x16x64_i8 v[192:195], a[116:119], v[76:79], v[192:195]\n"
    "ds_read_b128 v[100:103], v42\n"
    "v_fma_f32 v224, v224, s18, v33\n"
    "v_mfma_i32_16x16x64_i8 v[192:195], a[124:127], v[80:83], v[192:195]\n"
    "ds_read_b128 v[104:107], v43\n"
    "v_fma_f32 v225, v225, s18, v33\n"
    "v_mfma_i32_16x16x64_i8 v[192:195], a[148:151], v[84:87], v[192:195]\n"
    "ds_read_b128 v[108:111], v42 offset:8192\n"
    "v_mfma_i32_16x16x64_i8 v[188:191], a[112:115], v[76:79], v[188:191]\n"
    "ds_read_b128 v[112:115], v43 offset:8192\n"
    "v_fma_f32 v226, v226, s18, v33\n"
    "v_mfma_i32_16x16x64_i8 v[188:191], a[120:123], v[80:83], v[188:191]\n"
    "ds_read_b128 v[52:55], v42 offset:16384\n"
    "v_fma_f32 v227, v227, s18, v33\n"
    "v_mfma_i32_16x16x64_i8 v[188:191], a[144:147], v[84:87], v[188:191]\n"
    "ds_read_b128 v[56:59], v43 offset:16384\n"
    "v_max_f32 v220, v220, v33\n"
    "v_mfma_i32_16x16x64_i8 v[188:191], a[152:155], v[88:91], v[188:191]\n"
    "ds_read_b128 v[60:63], v42 offset:2048\n"
    "v_mfma_i32_16x16x64_i8 v[188:191], a[176:179], v[92:95], v[188:191]\n"
    "ds_read_b128 v[64:67], v43 offset:2048\n"
    "v_max_f32 v221, v221, v33\n"
    "v_mfma_i32_16x16x64_i8 v[188:191], a[184:187], v[96:99], v[188:191]\n"
    "v_max_f32 v222, v222, v33\n"
    "v_max_f32 v223, v223, v33\n"
    "v_mfma_i32_16x16x64_i8 v[192:195], a[156:159], v[88:91], v[192:195]\n"
    "v_max_f32 v224, v224, v33\n"
    "v_mfma_i32_16x16x64_i8 v[192:195], a[180:183], v[92:95], v[192:195]\n"
    "v_max_f32 v225, v225, v33\n"
    "v_max_f32 v226, v226, v33\n"
    "v_mfma_i32_16x16x64_i8 v[192:195], a[188:191], v[96:99], v[192:195]\n"
    "v_max_f32 v227, v227, v33\n"
    "s_waitcnt lgkmcnt(5)\n"
    "v_mfma_i32_16x16x64_i8 v[200:203], a[4:7], v[100:103], v[200:203]\n"
    "global_load_dword v32, v24, s[72:73] sc0 sc1\n"
    "ds_read_b128 v[68:71], v42 offset:10240\n"
    "v_mfma_i32_16x16x64_i8 v[200:203], a[12:15], v[104:107], v[200:203]\n"
    "s_add_u32 s90, s65, 90112\n"
    "buffer_load_dwordx4 v[236:239], v22, s[44:47], s90 offen sc0 sc1\n"
    "v_mfma_i32_16x16x64_i8 v[200:203], a[36:39], v[108:111], v[200:203]\n"
    "ds_read_b128 v[72:75], v43 offset:10240\n"
    "v_mfma_i32_16x16x64_i8 v[196:199], a[0:3], v[100:103], v[196:199]\n"
    "v_mfma_i32_16x16x64_i8 v[196:199], a[8:11], v[104:107], v[196:199]\n"
    "ds_read_b128 v[76:79], v42 offset:18432\n"
    "v_mfma_i32_16x16x64_i8 v[196:199], a[32:35], v[108:111], v[196:199]\n"
    "s_waitcnt lgkmcnt(5)\n"
    "buffer_load_dwordx4 v[240:243], v22, s[44:47], s90 offen offset:1024 sc0 sc1\n"
    "s_waitcnt vmcnt(5)\n"
    "v_mfma_i32_16x16x64_i8 v[196:199], a[40:43], v[112:115], v[196:199]\n"
    "ds_read_b128 v[80:83], v43 offset:18432\n"
    "v_add_u32 v228, v188, v228\n"
    "v_mfma_i32_16x16x64_i8 v[196:199], a[64:67], v[52:55], v[196:199]\n"
    "s_add_u32 s90, s65, 49152\n"
    "s_lshl_b32 s90, s90, 5\n"
    "v_add_u32 v229, v189, v229\n"
    "v_mfma_i32_16x16x64_i8 v[196:199], a[72:75], v[56:59], v[196:199]\n"
    "ds_read_b128 v[84:87], v42 offset:4096\n"
    "v_add_u32 v230, v190, v230\n"
    "v_mfma_i32_16x16x64_i8 v[200:203], a[44:47], v[112:115], v[200:203]\n"
    "buffer_store_dwordx4 v[220:223], v23, s[48:51], s90 offen sc0 sc1\n"
    "v_add_u32 v231, v191, v231\n"
    "v_add_u32 v232, v192, v232\n"
    "v_mfma_i32_16x16x64_i8 v[200:203], a[68:71], v[52:55], v[200:203]\n"
    "ds_read_b128 v[88:91], v43 offset:4096\n"
    "v_add_u32 v233, v193, v233\n"
    "v_mfma_i32_16x16x64_i8 v[200:203], a[76:79], v[56:59], v[200:203]\n"
    "s_waitcnt lgkmcnt(5)\n"
    "buffer_store_dwordx4 v[224:227], v23, s[48:51], s90 offen offset:64 sc0 sc1\n"
    "v_add_u32 v234, v194, v234\n"
    "v_mfma_i32_16x16x64_i8 v[200:203], a[20:23], v[60:63], v[200:203]\n"
    "ds_read_b128 v[92:95], v42 offset:12288\n"
    "v_add_u32 v235, v195, v235\n"
    "v_mfma_i32_16x16x64_i8 v[200:203], a[28:31], v[64:67], v[200:203]\n"
    "v_cvt_f32_i32 v228, v228\n"
    "v_cvt_f32_i32 v229, v229\n"
    "v_mfma_i32_16x16x64_i8 v[200:203], a[52:55], v[68:71], v[200:203]\n"
    "ds_read_b128 v[96:99], v43 offset:12288\n"
    "v_cvt_f32_i32 v230, v230\n"
    "v_mfma_i32_16x16x64_i8 v[196:199], a[16:19], v[60:63], v[196:199]\n"
    "v_cvt_f32_i32 v231, v231\n"
    "v_cvt_f32_i32 v232, v232\n"
    "v_mfma_i32_16x16x64_i8 v[196:199], a[24:27], v[64:67], v[196:199]\n"
    "ds_read_b128 v[100:103], v42 offset:20480\n"
    "v_cvt_f32_i32 v233, v233\n"
    "v_mfma_i32_16x16x64_i8 v[196:199], a[48:51], v[68:71], v[196:199]\n"
    "s_waitcnt lgkmcnt(5)\n"
    "v_cvt_f32_i32 v234, v234\n"
    "v_cvt_f32_i32 v235, v235\n"
    "v_mfma_i32_16x16x64_i8 v[196:199], a[56:59], v[72:75], v[196:199]\n"
    "ds_read_b128 v[104:107], v43 offset:20480\n"
    "v_max3_f32 v34, |v228|, |v229|, |v230|\n"
    "v_mfma_i32_16x16x64_i8 v[196:199], a[80:83], v[76:79], v[196:199]\n"
    "v_max3_f32 v34, v34, |v231|, |v232|\n"
    "v_max3_f32 v34, v34, |v233|, |v234|\n"
    "v_mfma_i32_16x16x64_i8 v[196:199], a[88:91], v[80:83], v[196:199]\n"
    "ds_read_b128 v[108:111], v42 offset:6144\n"
    "v_max_f32 v34, v34, |v235|\n"
    "v_mfma_i32_16x16x64_i8 v[200:203], a[60:63], v[72:75], v[200:203]\n"
    "v_cmp_lt_f32 s[28:29], s97, v34\n"
    "s_cmp_lg_u64 s[28:29], 0\n"
    "s_cselect_b32 s24, 512, 0\n"
    "s_or_b32 s81, s81, s24\n"
    "v_mfma_i32_16x16x64_i8 v[200:203], a[84:87], v[76:79], v[200:203]\n"
    "ds_read_b128 v[112:115], v43 offset:6144\n"
    "v_fma_f32 v228, v228, s18, v33\n"
    "v_mfma_i32_16x16x64_i8 v[200:203], a[92:95], v[80:83], v[200:203]\n"
    "s_waitcnt lgkmcnt(5)\n"
    "v_fma_f32 v229, v229, s18, v33\n"
    "v_fma_f32 v230, v230, s18, v33\n"
    "v_mfma_i32_16x16x64_i8 v[200:203], a[100:103], v[84:87], v[200:203]\n"
    "ds_read_b128 v[52:55], v42 offset:14336\n"
    "v_fma_f32 v231, v231, s18, v33\n"
    "v_mfma_i32_16x16x64_i8 v[200:203], a[108:111], v[88:91], v[200:203]\n"
    "v_fma_f32 v232, v232, s18, v33\n"
    "v_fma_f32 v233, v233, s18, v33\n"
    "v_mfma_i32_16x16x64_i8 v[200:203], a[132:135], v[92:95], v[200:203]\n"
    "ds_read_b128 v[56:59], v43 offset:14336\n"
    "v_fma_f32 v234, v234, s18, v33\n"
    "v_mfma_i32_16x16x64_i8 v[196:199], a[96:99], v[84:87], v[196:199]\n"
    "v_fma_f32 v235, v235, s18, v33\n"
    "v_max_f32 v228, v228, v33\n"
    "v_mfma_i32_16x16x64_i8 v[196:199], a[104:107], v[88:91], v[196:199]\n"
    "ds_read_b128 v[60:63], v42 offset:22528\n"
    "v_max_f32 v229, v229, v33\n"
    "v_mfma_i32_16x16x64_i8 v[196:199], a[128:131], v[92:95], v[196:199]\n"
    "s_waitcnt lgkmcnt(5)\n"
    "v_max_f32 v230, v230, v33\n"
    "v_max_f32 v231, v231, v33\n"
    "v_mfma_i32_16x16x64_i8 v[196:199], a[136:139], v[96:99], v[196:199]\n"
    "ds_read_b128 v[64:67], v43 offset:22528\n"
    "v_max_f32 v232, v232, v33\n"
    "v_mfma_i32_16x16x64_i8 v[196:199], a[160:163], v[100:103], v[196:199]\n"
    "v_max_f32 v233, v233, v33\n"
    "v_max_f32 v234, v234, v33\n"
    "v_mfma_i32_16x16x64_i8 v[196:199], a[168:171], v[104:107], v[196:199]\n"
    "v_max_f32 v235, v235, v33\n"
    "s_add_u32 s90, s65, 73728\n"
    "v_mfma_i32_16x16x64_i8 v[200:203], a[140:143], v[96:99], v[200:203]\n"
    "s_lshl_b32 s90, s90, 5\n"
    "buffer_store_dwordx4 v[228:231], v23, s[48:51], s90 offen sc0 sc1\n"
    "v_mfma_i32_16x16x64_i8 v[200:203], a[164:167], v[100:103], v[200:203]\n"
    "buffer_store_dwordx4 v[232:235], v23, s[48:51], s90 offen offset:64 sc0 sc1\n"
    "v_mfma_i32_16x16x64_i8 v[200:203], a[172:175], v[104:107], v[200:203]\n"
    "s_waitcnt lgkmcnt(3)\n"
    "v_mfma_i32_16x16x64_i8 v[200:203], a[116:119], v[108:111], v[200:203]\n"
    "s_waitcnt vmcnt(6)\n"
    "v_mfma_i32_16x16x64_i8 v[200:203], a[124:127], v[112:115], v[200:203]\n"
    "v_cmp_le_u32 vcc, s17, v32\n"
    "v_mfma_i32_16x16x64_i8 v[200:203], a[148:151], v[52:55], v[200:203]\n"
    "s_lshr_b32 s87, vcc_lo, 16\n"
    "v_mfma_i32_16x16x64_i8 v[196:199], a[112:115], v[108:111], v[196:199]\n"
    "s_and_b32 s87, s87, vcc_lo\n"
    "v_mfma_i32_16x16x64_i8 v[196:199], a[120:123], v[112:115], v[196:199]\n"
    "v_mfma_i32_16x16x64_i8 v[196:199], a[144:147], v[52:55], v[196:199]\n"
    "s_waitcnt lgkmcnt(0)\n"
    "s_and_b32 s87, s87, vcc_hi\n"
    "v_mfma_i32_16x16x64_i8 v[196:199], a[152:155], v[56:59], v[196:199]\n"
    "s_ff0_i32_b32 s85, s87\n"
    "v_mfma_i32_16x16x64_i8 v[196:199], a[176:179], v[60:63], v[196:199]\n"
    "v_mfma_i32_16x16x64_i8 v[196:199], a[184:187], v[64:67], v[196:199]\n"
    "s_max_u32 s85, s85, s86\n"
    "v_mfma_i32_16x16x64_i8 v[200:203], a[156:159], v[56:59], v[200:203]\n"
    "s_bitcmp1_b32 vcc_hi, 24\n"
    "s_cselect_b32 s87, 0x7fc00000, 0\n"
    "v_mfma_i32_16x16x64_i8 v[200:203], a[180:183], v[60:63], v[200:203]\n"
    "v_mov_b32 v33, s87\n"
    "v_mfma_i32_16x16x64_i8 v[200:203], a[188:191], v[64:67], v[200:203]\n"
    "s_nop 7\n"
    "s_waitcnt vmcnt(4)\n"
    "v_add_u32 v236, v196, v236\n"
    "v_add_u32 v237, v197, v237\n"
    "v_add_u32 v238, v198, v238\n"
    "v_add_u32 v239, v199, v239\n"
    "v_add_u32 v240, v200, v240\n"
    "v_add_u32 v241, v201, v241\n"
    "v_add_u32 v242, v202, v242\n"
    "v_add_u32 v243, v203, v243\n"
    "v_cvt_f32_i32 v236, v236\n"
    "v_cvt_f32_i32 v237, v237\n"
    "v_cvt_f32_i32 v238, v238\n"
    "v_cvt_f32_i32 v239, v239\n"
    "v_cvt_f32_i32 v240, v240\n"
    "v_cvt_f32_i32 v241, v241\n"
    "v_cvt_f32_i32 v242, v242\n"
    "v_cvt_f32_i32 v243, v243\n"
    "v_max3_f32 v34, |v236|, |v237|, |v238|\n"
    "v_max3_f32 v34, v34, |v239|, |v240|\n"
    "v_max3_f32 v34, v34, |v241|, |v242|\n"
    "v_max_f32 v34, v34, |v243|\n"
    "v_cmp_lt_f32 s[28:29], s97, v34\n"
    "s_cmp_lg_u64 s[28:29], 0\n"
    "s_cselect_b32 s24, 1024, 0\n"
    "s_or_b32 s81, s81, s24\n"
    "v_fma_f32 v236, v236, s18, v33\n"
    "v_fma_f32 v237, v237, s18, v33\n"
    "v_fma_f32 v238, v238, s18, v33\n"
    "v_fma_f32 v239, v239, s18, v33\n"
    "v_fma_f32 v240, v240, s18, v33\n"
    "v_fma_f32 v241, v241, s18, v33\n"
    "v_fma_f32 v242, v242, s18, v33\n"
    "v_fma_f32 v243, v243, s18, v33\n"
    "v_max_f32 v236, v236, v33\n"
    "v_max_f32 v237, v237, v33\n"
    "v_max_f32 v238, v238, v33\n"
    "v_max_f32 v239, v239, v33\n"
    "v_max_f32 v240, v240, v33\n"
    "v_max_f32 v241, v241, v33\n"
    "v_max_f32 v242, v242, v33\n"
    "v_max_f32 v243, v243, v33\n"
    "s_add_u32 s90, s65, 90112\n"
    "s_lshl_b32 s90, s90, 5\n"
    "buffer_store_dwordx4 v[236:239], v23, s[48:51], s90 offen sc0 sc1\n"
    "buffer_store_dwordx4 v[240:243], v23, s[48:51], s90 offen offset:64 sc0 sc1\n"
    "s_cmp_lg_u32 s81, 0\n"
    "s_cbranch_scc1 L_redob%=\n"
    "L_finb%=:\n"
    "s_branch L_end%=\n"
    "L_redob%=:\n"
    "s_mov_b64 s[26:27], -1\n"
    "L_redo2b%=:\n"
    "s_bitcmp1_b32 s81, 6\n"
    "s_cbranch_scc0 L_rl6_b%=\n"
    "s_add_u32 s90, s65, 0\n"
    "buffer_load_dwordx4 v[204:207], v22, s[44:47], s90 offen sc0 sc1\n"
    "buffer_load_dwordx4 v[208:211], v22, s[44:47], s90 offen offset:1024 sc0 sc1\n"
    "L_rl6_b%=:\n"
    "s_bitcmp1_b32 s81, 7\n"
    "s_cbranch_scc0 L_rl7_b%=\n"
    "s_add_u32 s90, s65, 24576\n"
    "buffer_load_dwordx4 v[212:215], v22, s[44:47], s90 offen sc0 sc1\n"
    "buffer_load_dwordx4 v[216:219], v22, s[44:47], s90 offen offset:1024 sc0 sc1\n"
    "L_rl7_b%=:\n"
    "s_bitcmp1_b32 s81, 8\n"
    "s_cbranch_scc0 L_rl8_b%=\n"
    "s_add_u32 s90, s65, 49152\n"
    "buffer_load_dwordx4 v[220:223], v22, s[44:47], s90 offen sc0 sc1\n"
    "buffer_load_dwordx4 v[224:227], v22, s[44:47], s90 offen offset:1024 sc0 sc1\n"
    "L_rl8_b%=:\n"
    "s_bitcmp1_b32 s81, 9\n"
    "s_cbranch_scc0 L_rl9_b%=\n"
    "s_add_u32 s90, s65, 73728\n"
    "buffer_load_dwordx4 v[228:231], v22, s[44:47], s90 offen sc0 sc1\n"
    "buffer_load_dwordx4 v[232:235], v22, s[44:47], s90 offen offset:1024 sc0 sc1\n"
    "L_rl9_b%=:\n"
    "s_bitcmp1_b32 s81, 10\n"
    "s_cbranch_scc0 L_rl10_b%=\n"
    "s_add_u32 s90, s65, 90112\n"
    "buffer_load_dwordx4 v[236:239], v22, s[44:47], s90 offen sc0 sc1\n"
    "buffer_load_dwordx4 v[240:243], v22, s[44:47], s90 offen offset:1024 sc0 sc1\n"
    "L_rl10_b%=:\n"
    "s_waitcnt vmcnt(0)\n"
    "s_bitcmp1_b32 s81, 6\n"
    "s_cbranch_scc0 L_rn6_b%=\n"
    "v_add_u32 v204, v164, v204\n"
    "v_add_u32 v205, v165, v205\n"
    "v_add_u32 v206, v166, v206\n"
    "v_add_u32 v207, v167, v207\n"
    "v_add_u32 v208, v168, v208\n"
    "v_add_u32 v209, v169, v209\n"
    "v_add_u32 v210, v170, v210\n"
    "v_add_u32 v211, v171, v211\n"
    "v_cvt_f32_i32 v204, v204\n"
    "v_cvt_f32_i32 v205, v205\n"
    "v_cvt_f32_i32 v206, v206\n"
    "v_cvt_f32_i32 v207, v207\n"
    "v_cvt_f32_i32 v208, v208\n"
    "v_cvt_f32_i32 v209, v209\n"
    "v_cvt_f32_i32 v210, v210\n"
    "v_cvt_f32_i32 v211, v211\n"
    "v_max3_f32 v34, |v204|, |v205|, |v206|\n"
    "v_max3_f32 v34, v34, |v207|, |v208|\n"
    "v_max3_f32 v34, v34, |v209|, |v210|\n"
    "v_max_f32 v34, v34, |v211|\n"
    "v_cmp_lt_f32 s[28:29], s97, v34\n"
    "s_and_b64 s[28:29], s[28:29], s[26:27]\n"
    "s_cbranch_scc1 L_rn6_b%=\n"
    "v_fma_f32 v204, v204, s18, v33\n"
    "v_fma_f32 v205, v205, s18, v33\n"
    "v_fma_f32 v206, v206, s18, v33\n"
    "v_fma_f32 v207, v207, s18, v33\n"
    "v_fma_f32 v208, v208, s18, v33\n"
    "v_fma_f32 v209, v209, s18, v33\n"
    "v_fma_f32 v210, v210, s18, v33\n"
    "v_fma_f32 v211, v211, s18, v33\n"
    "v_max_f32 v204, v204, v33\n"
    "v_max_f32 v205, v205, v33\n"
    "v_max_f32 v206, v206, v33\n"
    "v_max_f32 v207, v207, v33\n"
    "v_max_f32 v208, v208, v33\n"
    "v_max_f32 v209, v209, v33\n"
    "v_max_f32 v210, v210, v33\n"
    "v_max_f32 v211, v211, v33\n"
    "s_add_u32 s90, s65, 0\n"
    "s_lshl_b32 s90, s90, 5\n"
    "buffer_store_dwordx4 v[204:207], v23, s[48:51], s90 offen sc0 sc1\n"
    "buffer_store_dwordx4 v[208:211], v23, s[48:51], s90 offen offset:64 sc0 sc1\n"
    "s_bitset0_b32 s81, 6\n"
    "L_rn6_b%=:\n"
    "s_bitcmp1_b32 s81, 7\n"
    "s_cbranch_scc0 L_rn7_b%=\n"
    "v_add_u32 v212, v172, v212\n"
    "v_add_u32 v213, v173, v213\n"
    "v_add_u32 v214, v174, v214\n"
    "v_add_u32 v215, v175, v215\n"
    "v_add_u32 v216, v176, v216\n"
    "v_add_u32 v217, v177, v217\n"
    "v_add_u32 v218, v178, v218\n"
    "v_add_u32 v219, v179, v219\n"
    "v_cvt_f32_i32 v212, v212\n"
    "v_cvt_f32_i32 v213, v213\n"
    "v_cvt_f32_i32 v214, v214\n"
    "v_cvt_f32_i32 v215, v215\n"
    "v_cvt_f32_i32 v216, v216\n"
    "v_cvt_f32_i32 v217, v217\n"
    "v_cvt_f32_i32 v218, v218\n"
    "v_cvt_f32_i32 v219, v219\n"
    "v_max3_f32 v34, |v212|, |v213|, |v214|\n"
    "v_max3_f32 v34, v34, |v215|, |v216|\n"
    "v_max3_f32 v34, v34, |v217|, |v218|\n"
    "v_max_f32 v34, v34, |v219|\n"
    "v_cmp_lt_f32 s[28:29], s97, v34\n"
    "s_and_b64 s[28:29], s[28:29], s[26:27]\n"
    "s_cbranch_scc1 L_rn7_b%=\n"
    "v_fma_f32 v212, v212, s18, v33\n"
    "v_fma_f32 v213, v213, s18, v33\n"
    "v_fma_f32 v214, v214, s18, v33\n"
    "v_fma_f32 v215, v215, s18, v33\n"
    "v_fma_f32 v216, v216, s18, v33\n"
    "v_fma_f32 v217, v217, s18, v33\n"
    "v_fma_f32 v218, v218, s18, v33\n"
    "v_fma_f32 v219, v219, s18, v33\n"
    "v_max_f32 v212, v212, v33\n"
    "v_max_f32 v213, v213, v33\n"
    "v_max_f32 v214, v214, v33\n"
    "v_max_f32 v215, v215, v33\n"
    "v_max_f32 v216, v216, v33\n"
    "v_max_f32 v217, v217, v33\n"
    "v_max_f32 v218, v218, v33\n"
    "v_max_f32 v219, v219, v33\n"
    "s_add_u32 s90, s65, 24576\n"
    "s_lshl_b32 s90, s90, 5\n"
    "buffer_store_dwordx4 v[212:215], v23, s[48:51], s90 offen sc0 sc1\n"
    "buffer_store_dwordx4 v[216:219], v23, s[48:51], s90 offen offset:64 sc0 sc1\n"
    "s_bitset0_b32 s81, 7\n"
    "L_rn7_b%=:\n"
    "s_bitcmp1_b32 s81, 8\n"
    "s_cbranch_scc0 L_rn8_b%=\n"
    "v_add_u32 v220, v180, v220\n"
    "v_add_u32 v221, v181, v221\n"
    "v_add_u32 v222, v182, v222\n"
    "v_add_u32 v223, v183, v223\n"
    "v_add_u32 v224, v184, v224\n"
    "v_add_u32 v225, v185, v225\n"
    "v_add_u32 v226, v186, v226\n"
    "v_add_u32 v227, v187, v227\n"
    "v_cvt_f32_i32 v220, v220\n"
    "v_cvt_f32_i32 v221, v221\n"
    "v_cvt_f32_i32 v222, v222\n"
    "v_cvt_f32_i32 v223, v223\n"
    "v_cvt_f32_i32 v224, v224\n"
    "v_cvt_f32_i32 v225, v225\n"
    "v_cvt_f32_i32 v226, v226\n"
    "v_cvt_f32_i32 v227, v227\n"
    "v_max3_f32 v34, |v220|, |v221|, |v222|\n"
    "v_max3_f32 v34, v34, |v223|, |v224|\n"
    "v_max3_f32 v34, v34, |v225|, |v226|\n"
    "v_max_f32 v34, v34, |v227|\n"
    "v_cmp_lt_f32 s[28:29], s97, v34\n"
    "s_and_b64 s[28:29], s[28:29], s[26:27]\n"
    "s_cbranch_scc1 L_rn8_b%=\n"
    "v_fma_f32 v220, v220, s18, v33\n"
    "v_fma_f32 v221, v221, s18, v33\n"
    "v_fma_f32 v222, v222, s18, v33\n"
    "v_fma_f32 v223, v223, s18, v33\n"
    "v_fma_f32 v224, v224, s18, v33\n"
    "v_fma_f32 v225, v225, s18, v33\n"
    "v_fma_f32 v226, v226, s18, v33\n"
    "v_fma_f32 v227, v227, s18, v33\n"
    "v_max_f32 v220, v220, v33\n"
    "v_max_f32 v221, v221, v33\n"
    "v_max_f32 v222, v222, v33\n"
    "v_max_f32 v223, v223, v33\n"
    "v_max_f32 v224, v224, v33\n"
    "v_max_f32 v225, v225, v33\n"
    "v_max_f32 v226, v226, v33\n"
    "v_max_f32 v227, v227, v33\n"
    "s_add_u32 s90, s65, 49152\n"
    "s_lshl_b32 s90, s90, 5\n"
    "buffer_store_dwordx4 v[220:223], v23, s[48:51], s90 offen sc0 sc1\n"
    "buffer_store_dwordx4 v[224:227], v23, s[48:51], s90 offen offset:64 sc0 sc1\n"
    "s_bitset0_b32 s81, 8\n"
    "L_rn8_b%=:\n"
    "s_bitcmp1_b32 s81, 9\n"
    "s_cbranch_scc0 L_rn9_b%=\n"
    "v_add_u32 v228, v188, v228\n"
    "v_add_u32 v229, v189, v229\n"
    "v_add_u32 v230, v190, v230\n"
    "v_add_u32 v231, v191, v231\n"
    "v_add_u32 v232, v192, v232\n"
    "v_add_u32 v233, v193, v233\n"
    "v_add_u32 v234, v194, v234\n"
    "v_add_u32 v235, v195, v235\n"
    "v_cvt_f32_i32 v228, v228\n"
    "v_cvt_f32_i32 v229, v229\n"
    "v_cvt_f32_i32 v230, v230\n"
    "v_cvt_f32_i32 v231, v231\n"
    "v_cvt_f32_i32 v232, v232\n"
    "v_cvt_f32_i32 v233, v233\n"
    "v_cvt_f32_i32 v234, v234\n"
    "v_cvt_f32_i32 v235, v235\n"
    "v_max3_f32 v34, |v228|, |v229|, |v230|\n"
    "v_max3_f32 v34, v34, |v231|, |v232|\n"
    "v_max3_f32 v34, v34, |v233|, |v234|\n"
    "v_max_f32 v34, v34, |v235|\n"
    "v_cmp_lt_f32 s[28:29], s97, v34\n"
    "s_and_b64 s[28:29], s[28:29], s[26:27]\n"
    "s_cbranch_scc1 L_rn9_b%=\n"
    "v_fma_f32 v228, v228, s18, v33\n"
    "v_fma_f32 v229, v229, s18, v33\n"
    "v_fma_f32 v230, v230, s18, v33\n"
    "v_fma_f32 v231, v231, s18, v33\n"
    "v_fma_f32 v232, v232, s18, v33\n"
    "v_fma_f32 v233, v233, s18, v33\n"
    "v_fma_f32 v234, v234, s18, v33\n"
    "v_fma_f32 v235, v235, s18, v33\n"
    "v_max_f32 v228, v228, v33\n"
    "v_max_f32 v229, v229, v33\n"
    "v_max_f32 v230, v230, v33\n"
    "v_max_f32 v231, v231, v33\n"
    "v_max_f32 v232, v232, v33\n"
    "v_max_f32 v233, v233, v33\n"
    "v_max_f32 v234, v234, v33\n"
    "v_max_f32 v235, v235, v33\n"
    "s_add_u32 s90, s65, 73728\n"
    "s_lshl_b32 s90, s90, 5\n"
    "buffer_store_dwordx4 v[228:231], v23, s[48:51], s90 offen sc0 sc1\n"
    "buffer_store_dwordx4 v[232:235], v23, s[48:51], s90 offen offset:64 sc0 sc1\n"
    "s_bitset0_b32 s81, 9\n"
    "L_rn9_b%=:\n"
    "s_bitcmp1_b32 s81, 10\n"
    "s_cbranch_scc0 L_rn10_b%=\n"
    "v_add_u32 v236, v196, v236\n"
    "v_add_u32 v237, v197, v237\n"
    "v_add_u32 v238, v198, v238\n"
    "v_add_u32 v239, v199, v239\n"
    "v_add_u32 v240, v200, v240\n"
    "v_add_u32 v241, v201, v241\n"
    "v_add_u32 v242, v202, v242\n"
    "v_add_u32 v243, v203, v243\n"
    "v_cvt_f32_i32 v236, v236\n"
    "v_cvt_f32_i32 v237, v237\n"
    "v_cvt_f32_i32 v238, v238\n"
    "v_cvt_f32_i32 v239, v239\n"
    "v_cvt_f32_i32 v240, v240\n"
    "v_cvt_f32_i32 v241, v241\n"
    "v_cvt_f32_i32 v242, v242\n"
    "v_cvt_f32_i32 v243, v243\n"
    "v_max3_f32 v34, |v236|, |v237|, |v238|\n"
    "v_max3_f32 v34, v34, |v239|, |v240|\n"
    "v_max3_f32 v34, v34, |v241|, |v242|\n"
    "v_max_f32 v34, v34, |v243|\n"
    "v_cmp_lt_f32 s[28:29], s97, v34\n"
    "s_and_b64 s[28:29], s[28:29], s[26:27]\n"
    "s_cbranch_scc1 L_rn10_b%=\n"
    "v_fma_f32 v236, v236, s18, v33\n"
    "v_fma_f32 v237, v237, s18, v33\n"
    "v_fma_f32 v238, v238, s18, v33\n"
    "v_fma_f32 v239, v239, s18, v33\n"
    "v_fma_f32 v240, v240, s18, v33\n"
    "v_fma_f32 v241, v241, s18, v33\n"
    "v_fma_f32 v242, v242, s18, v33\n"
    "v_fma_f32 v243, v243, s18, v33\n"
    "v_max_f32 v236, v236, v33\n"
    "v_max_f32 v237, v237, v33\n"
    "v_max_f32 v238, v238, v33\n"
    "v_max_f32 v239, v239, v33\n"
    "v_max_f32 v240, v240, v33\n"
    "v_max_f32 v241, v241, v33\n"
    "v_max_f32 v242, v242, v33\n"
    "v_max_f32 v243, v243, v33\n"
    "s_add_u32 s90, s65, 90112\n"
    "s_lshl_b32 s90, s90, 5\n"
    "buffer_store_dwordx4 v[236:239], v23, s[48:51], s90 offen sc0 sc1\n"
    "buffer_store_dwordx4 v[240:243], v23, s[48:51], s90 offen offset:64 sc0 sc1\n"
    "s_bitset0_b32 s81, 10\n"
    "L_rn10_b%=:\n"
    "s_cmp_eq_u32 s81, 0\n"
    "s_cbranch_scc1 L_finb%=\n"
    "s_ff1_i32_b32 s33, s81\n"
    "s_sub_u32 s33, s33, 6\n"
    "s_lshl_b32 s33, s33, 2\n"
    "s_lshr_b32 s33, 0xb9630, s33\n"
    "s_and_b32 s33, s33, 15\n"
    "s_lshl_b32 s33, s33, 13\n"
    "s_add_u32 s33, s33, s65\n"
    "s_call_b64 s[88:89], L_slowqb%=\n"
    "s_cselect_b64 s[26:27], 0, -1\n"
    "s_branch L_redo2b%=\n"
    "L_slowb%=:\n"
    "s_add_u32 s88, s88, 8\n"
    "s_addc_u32 s89, s89, 0\n"
    "s_mov_b32 s25, -1\n"
    "s_cmp_gt_u32 s84, s95\n"
    "s_cbranch_scc1 L_landb%=\n"
    "L_slow2b%=:\n"
    "global_load_dword v32, v24, s[72:73] sc0 sc1\n"
    "s_waitcnt vmcnt(0)\n"
    "v_cmp_le_u32 vcc, s17, v32\n"
    "s_lshr_b32 s94, vcc_lo, 16\n"
    "s_and_b32 s94, s94, vcc_lo\n"
    "s_and_b32 s94, s94, vcc_hi\n"
    "s_ff0_i32_b32 s85, s94\n"
    "s_max_u32 s85, s85, s86\n"
    "s_bitcmp1_b32 vcc_hi, 24\n"
    "s_cselect_b32 s94, 0x7fc00000, 0\n"
    "v_mov_b32 v33, s94\n"
    "s_cmp_gt_u32 s85, s95\n"
    "s_cbranch_scc1 L_reqb%=\n"
    "s_sleep 2\n"
    "s_memrealtime s[92:93]\n"
    "s_bitcmp1_b32 vcc_hi, 24\n"
    "s_cbranch_scc1 L_slowgb%=\n"
    "s_waitcnt lgkmcnt(0)\n"
    "s_sub_u32 s92, s92, s20\n"
    "s_cmp_lt_u32 s92, 50000000\n"
    "s_cbranch_scc1 L_slow2b%=\n"
    "L_slowgb%=:\n"
    "s_waitcnt lgkmcnt(0)\n"
    "global_store_dword v25, v36, s[72:73] sc1\n"
    "s_mov_b32 s86, 16\n"
    "s_waitcnt vmcnt(0)\n"
    "global_load_dword v32, v24, s[72:73] sc0 sc1\n"
    "s_waitcnt vmcnt(0)\n"
    "v_cmp_le_u32 vcc, s17, v32\n"
    "s_lshr_b32 s94, vcc_lo, 16\n"
    "s_and_b32 s94, s94, vcc_lo\n"
    "s_and_b32 s94, s94, vcc_hi\n"
    "s_ff0_i32_b32 s85, s94\n"
    "s_max_u32 s85, s85, s86\n"
    "s_bitcmp1_b32 vcc_hi, 24\n"
    "s_cselect_b32 s94, 0x7fc00000, 0\n"
    "v_mov_b32 v33, s94\n"
    "s_branch L_reqb%=\n"
    "L_midb%=:\n"
    "v_cmp_le_u32 vcc, s17, v32\n"
    "s_lshr_b32 s94, vcc_lo, 16\n"
    "s_and_b32 s94, s94, vcc_lo\n"
    "s_and_b32 s94, s94, vcc_hi\n"
    "s_ff0_i32_b32 s85, s94\n"
    "s_max_u32 s85, s85, s86\n"
    "global_load_dword v32, v24, s[72:73] sc0 sc1\n"
    "s_cmp_gt_u32 s85, s84\n"
    "s_cbranch_scc1 L_mid2b%=\n"
    "s_add_u32 s88, s88, 4\n"
    "s_addc_u32 s89, s89, 0\n"
    "s_setpc_b64 s[88:89]\n"
    "L_mid2b%=:\n"
    "s_mov_b32 s25, 0\n"
    "L_catchb%=:\n"
    "s_add_u32 s88, s88, 4\n"
    "s_addc_u32 s89, s89, 0\n"
    "L_catch2b%=:\n"
    "s_mov_b32 s95, -100\n"
    "L_reqb%=:\n"
    "s_mov_b32 s31, m0\n"
    "L_req2b%=:\n"
    "s_lshl_b32 s94, s84, 2\n"
    "s_lshr_b64 s[92:93], s[98:99], s94\n"
    "s_and_b32 s92, s92, 7\n"
    "s_lshl_b32 s92, s92, 15\n"
    "s_lshl_b32 s94, s84, 13\n"
    "s_add_u32 m0, s11, s92\n"
    "s_add_u32 s19, s8, s94\n"
    "buffer_load_dwordx4 v22, s[40:43], s19 offen sc0 sc1 lds\n"
    "buffer_load_dwordx4 v22, s[40:43], s19 offen offset:1024 sc0 sc1 lds\n"
    "s_add_u32 m0, s12, s92\n"
    "s_add_u32 s19, s9, s94\n"
    "buffer_load_dwordx4 v22, s[40:43], s19 offen sc0 sc1 lds\n"
    "buffer_load_dwordx4 v22, s[40:43], s19 offen offset:1024 sc0 sc1 lds\n"
    "s_add_u32 m0, s13, s92\n"
    "s_add_u32 s19, s10, s94\n"
    "buffer_load_dwordx4 v22, s[40:43], s19 offen sc0 sc1 lds\n"
    "buffer_load_dwordx4 v22, s[40:43], s19 offen offset:1024 sc0 sc1 lds\n"
    "s_add_u32 s84, s84, 1\n"
    "s_min_u32 s94, s85, s96\n"
    "s_cmp_lt_u32 s84, s94\n"
    "s_cbranch_scc1 L_req2b%=\n"
    "s_mov_b32 m0, s31\n"
    "s_sub_u32 s94, s84, s95\n"
    "s_cmp_gt_u32 s94, 50\n"
    "s_cbranch_scc0 L_w2b%=\n"
    "s_setpc_b64 s[88:89]\n"
    "L_w2b%=:\n"
    "s_waitcnt vmcnt(12) lgkmcnt(0)\n"
    "s_cmp_gt_u32 s94, 2\n"
    "s_cbranch_scc1 L_w0b%=\n"
    "s_waitcnt vmcnt(6)\n"
    "s_cmp_eq_u32 s94, 2\n"
    "s_cbranch_scc1 L_w0b%=\n"
    "s_sleep 6\n"
    "L_landb%=:\n"
    "s_waitcnt lgkmcnt(0)\n"
    "global_load_dword v32, v24, s[72:73] sc0 sc1\n"
    "s_waitcnt vmcnt(0)\n"
    "v_cmp_le_u32 vcc, s17, v32\n"
    "s_lshr_b32 s94, vcc_lo, 16\n"
    "s_and_b32 s94, s94, vcc_lo\n"
    "s_and_b32 s94, s94, vcc_hi\n"
    "s_ff0_i32_b32 s85, s94\n"
    "s_max_u32 s85, s85, s86\n"
    "s_bitcmp1_b32 vcc_hi, 24\n"
    "s_cselect_b32 s94, 0x7fc00000, 0\n"
    "v_mov_b32 v33, s94\n"
    "s_min_u32 s94, s85, s96\n"
    "s_cmp_lt_u32 s84, s94\n"
    "s_cbranch_scc1 L_catch2b%=\n"
    "s_setpc_b64 s[88:89]\n"
    "L_w0b%=:\n"
    "s_setpc_b64 s[88:89]\n"
    "L_gateb%=:\n"
    "s_sleep 4\n"
    "global_load_dword v37, v26, s[72:73] sc0 sc1\n"
    "global_load_dword v32, v24, s[72:73] sc0 sc1\n"
    "s_waitcnt vmcnt(0)\n"
    "v_cmp_le_u32 vcc, s17, v32\n"
    "s_lshr_b32 s94, vcc_lo, 16\n"
    "s_and_b32 s94, s94, vcc_lo\n"
    "s_and_b32 s94, s94, vcc_hi\n"
    "s_ff0_i32_b32 s85, s94\n"
    "s_max_u32 s85, s85, s86\n"
    "s_bitcmp1_b32 vcc_hi, 24\n"
    "s_cselect_b32 s94, 0x7fc00000, 0\n"
    "v_mov_b32 v33, s94\n"
    "v_cmp_gt_u32 s[28:29], s17, v37\n"
    "s_cmp_lg_u64 s[28:29], 0\n"
    "s_cbranch_scc0 L_rgb%=\n"
    "s_memrealtime s[92:93]\n"
    "s_bitcmp1_b32 vcc_hi, 24\n"
    "s_cbranch_scc1 L_ggb%=\n"
    "s_waitcnt lgkmcnt(0)\n"
    "s_sub_u32 s92, s92, s20\n"
    "s_cmp_lt_u32 s92, 50000000\n"
    "s_cbranch_scc1 L_gateb%=\n"
    "L_ggb%=:\n"
    "s_waitcnt lgkmcnt(0)\n"
    "global_store_dword v25, v36, s[72:73] sc1\n"
    "s_mov_b32 s86, 16\n"
    "s_waitcnt vmcnt(0)\n"
    "global_load_dword v32, v24, s[72:73] sc0 sc1\n"
    "s_waitcnt vmcnt(0)\n"
    "v_cmp_le_u32 vcc, s17, v32\n"
    "s_lshr_b32 s94, vcc_lo, 16\n"
    "s_and_b32 s94, s94, vcc_lo\n"
    "s_and_b32 s94, s94, vcc_hi\n"
    "s_ff0_i32_b32 s85, s94\n"
    "s_max_u32 s85, s85, s86\n"
    "s_bitcmp1_b32 vcc_hi, 24\n"
    "s_cselect_b32 s94, 0x7fc00000, 0\n"
    "v_mov_b32 v33, s94\n"
    "s_branch L_rgb%=\n"
    "L_slowqb%=:\n"
    "global_load_dword v32, v24, s[72:73] sc0 sc1\n"
    "buffer_load_dword v34, v35, s[44:47], s33 offen offset:2044 sc0 sc1\n"
    "s_waitcnt vmcnt(0)\n"
    "v_cmp_le_u32 vcc, s17, v32\n"
    "s_lshr_b32 s94, vcc_lo, 16\n"
    "s_and_b32 s94, s94, vcc_lo\n"
    "s_and_b32 s94, s94, vcc_hi\n"
    "s_ff0_i32_b32 s85, s94\n"
    "s_max_u32 s85, s85, s86\n"
    "s_bitcmp1_b32 vcc_hi, 24\n"
    "s_cselect_b32 s94, 0x7fc00000, 0\n"
    "v_mov_b32 v33, s94\n"
    "v_add_u32 v34, v34, v8\n"
    "v_cvt_f32_i32 v34, v34\n"
    "v_cmp_lt_f32 s[28:29], s97, |v34|\n"
    "s_memrealtime s[92:93]\n"
    "s_bitcmp1_b32 vcc_hi, 24\n"
    "s_cbranch_scc1 L_qgb%=\n"
    "s_waitcnt lgkmcnt(0)\n"
    "s_sub_u32 s92, s92, s20\n"
    "s_cmp_lt_u32 s92, 50000000\n"
    "s_cbranch_scc1 L_qbb%=\n"
    "L_qgb%=:\n"
    "s_waitcnt lgkmcnt(0)\n"
    "global_store_dword v25, v36, s[72:73] sc1\n"
    "s_mov_b32 s86, 16\n"
    "s_waitcnt vmcnt(0)\n"
    "global_load_dword v32, v24, s[72:73] sc0 sc1\n"
    "s_waitcnt vmcnt(0)\n"
    "v_cmp_le_u32 vcc, s17, v32\n"
    "s_lshr_b32 s94, vcc_lo, 16\n"
    "s_and_b32 s94, s94, vcc_lo\n"
    "s_and_b32 s94, s94, vcc_hi\n"
    "s_ff0_i32_b32 s85, s94\n"
    "s_max_u32 s85, s85, s86\n"
    "s_bitcmp1_b32 vcc_hi, 24\n"
    "s_cselect_b32 s94, 0x7fc00000, 0\n"
    "v_mov_b32 v33, s94\n"
    "s_cmp_eq_u32 0, 0\n"
    "s_setpc_b64 s[88:89]\n"
    "L_qbb%=:\n"
    "s_cmp_lg_u64 s[28:29], 0\n"
    "s_cbranch_scc0 L_qcb%=\n"
    "s_sleep 4\n"
    "s_branch L_slowqb%=\n"
    "L_qcb%=:\n"
    "s_setpc_b64 s[88:89]\n"
    "L_progc%=:\n"
    "s_mov_b32 s81, 0\n"
    "s_mov_b32 s25, -1\n"
    "v_and_b32 v4, 3, v1\n"
    "v_lshlrev_b32 v4, 9, v4\n"
    "v_add_u32 v26, 263060, v4\n"
    "s_movk_i32 s35, 2048\n"
    "s_cmp_lt_u32 s3, 3\n"
    "s_cselect_b32 s87, s35, 524288\n"
    "s_cselect_b32 s94, s35, 8192\n"
    "s_cselect_b32 s11, s16, 6144\n"
    "s_cselect_b32 s35, 6, 0\n"
    "s_lshl_b32 s8, s11, s35\n"
    "s_add_u32 s9, s8, s87\n"
    "s_add_u32 s12, s11, s94\n"
    "s_add_u32 s10, s9, s87\n"
    "s_add_u32 s13, s12, s94\n"
    "s_mov_b32 s8, s59\n"
    "s_mov_b32 s9, s60\n"
    "s_add_u32 s87, s9, 2048\n"
    "s_sub_u32 s94, 1591296, s8\n"
    "s_sub_u32 s94, s94, s9\n"
    "s_cmp_lt_u32 s3, 3\n"
    "s_cselect_b32 s10, s87, s94\n"
    "s_cmp_lt_u32 s3, 3\n"
    "s_cselect_b32 s94, 0, 512\n"
    "s_mov_b32 s42, 1179648\n"
    "v_lshrrev_b32 v4, 4, v1\n"
    "v_mul_u32_u24 v4, s94, v4\n"
    "v_and_b32 v5, 15, v1\n"
    "v_lshl_add_u32 v4, v5, 2, v4\n"
    "v_add_u32 v4, s67, v4\n"
    "v_mov_b32 v25, 8192\n"
    "v_cmp_gt_u32 vcc, 48, v1\n"
    "s_mov_b32 s98, 0x10432104\n"
    "s_mov_b32 s99, 0x32\n"
    "v_cndmask_b32 v24, v25, v4, vcc\n"
    "buffer_load_dword v12, v12, s[36:39], s95 offen offset:384\n"
    "global_load_dword v3, v2, s[72:73] sc1\n"
    "global_load_dword v14, v35, s[74:75]\n"
    "global_load_dword v15, v35, s[76:77]\n"
    "s_waitcnt vmcnt(12) lgkmcnt(0)\n"
    "s_barrier\n"
    "s_add_u32 m0, s16, 65536\n"
    "s_add_u32 s35, s55, 131072\n"
    "buffer_load_dwordx4 v16, s[36:39], s55 offen lds\n"
    "buffer_load_dwordx4 v17, s[36:39], s55 offen offset:1024 lds\n"
    "buffer_load_dwordx4 v18, s[36:39], s55 offen offset:2048 lds\n"
    "buffer_load_dwordx4 v19, s[36:39], s55 offen offset:3072 lds\n"
    "s_add_u32 m0, s16, 69632\n"
    "s_nop 0\n"
    "buffer_load_dwordx4 v16, s[36:39], s35 offen lds\n"
    "buffer_load_dwordx4 v17, s[36:39], s35 offen offset:1024 lds\n"
    "buffer_load_dwordx4 v18, s[36:39], s35 offen offset:2048 lds\n"
    "buffer_load_dwordx4 v19, s[36:39], s35 offen offset:3072 lds\n"
    "ds_read_b64_tr_b8 a[0:1], v44 offset:0\n"
    "ds_read_b64_tr_b8 a[2:3], v44 offset:1024\n"
    "ds_read_b64_tr_b8 a[4:5], v45 offset:0\n"
    "ds_read_b64_tr_b8 a[6:7], v45 offset:1024\n"
    "ds_read_b64_tr_b8 a[8:9], v44 offset:8192\n"
    "ds_read_b64_tr_b8 a[10:11], v44 offset:9216\n"
    "ds_read_b64_tr_b8 a[12:13], v45 offset:8192\n"
    "ds_read_b64_tr_b8 a[14:15], v45 offset:9216\n"
    "ds_read_b64_tr_b8 a[16:17], v44 offset:16384\n"
    "ds_read_b64_tr_b8 a[18:19], v44 offset:17408\n"
    "ds_read_b64_tr_b8 a[20:21], v45 offset:16384\n"
    "ds_read_b64_tr_b8 a[22:23], v45 offset:17408\n"
    "ds_read_b64_tr_b8 a[24:25], v44 offset:24576\n"
    "ds_read_b64_tr_b8 a[26:27], v44 offset:25600\n"
    "ds_read_b64_tr_b8 a[28:29], v45 offset:24576\n"
    "ds_read_b64_tr_b8 a[30:31], v45 offset:25600\n"
    "s_waitcnt vmcnt(12) lgkmcnt(0)\n"
    "s_barrier\n"
    "s_add_u32 m0, s16, 98304\n"
    "s_add_u32 s35, s56, 131072\n"
    "buffer_load_dwordx4 v16, s[36:39], s56 offen lds\n"
    "buffer_load_dwordx4 v17, s[36:39], s56 offen offset:1024 lds\n"
    "buffer_load_dwordx4 v18, s[36:39], s56 offen offset:2048 lds\n"
    "buffer_load_dwordx4 v19, s[36:39], s56 offen offset:3072 lds\n"
    "s_add_u32 m0, s16, 102400\n"
    "s_nop 0\n"
    "buffer_load_dwordx4 v16, s[36:39], s35 offen lds\n"
    "buffer_load_dwordx4 v17, s[36:39], s35 offen offset:1024 lds\n"
    "buffer_load_dwordx4 v18, s[36:39], s35 offen offset:2048 lds\n"
    "buffer_load_dwordx4 v19, s[36:39], s35 offen offset:3072 lds\n"
    "ds_read_b64_tr_b8 a[32:33], v44 offset:32768\n"
    "ds_read_b64_tr_b8 a[34:35], v44 offset:33792\n"
    "ds_read_b64_tr_b8 a[36:37], v45 offset:32768\n"
    "ds_read_b64_tr_b8 a[38:39], v45 offset:33792\n"
    "ds_read_b64_tr_b8 a[40:41], v44 offset:40960\n"
    "ds_read_b64_tr_b8 a[42:43], v44 offset:41984\n"
    "ds_read_b64_tr_b8 a[44:45], v45 offset:40960\n"
    "ds_read_b64_tr_b8 a[46:47], v45 offset:41984\n"
    "ds_read_b64_tr_b8 a[48:49], v44 offset:49152\n"
    "ds_read_b64_tr_b8 a[50:51], v44 offset:50176\n"
    "ds_read_b64_tr_b8 a[52:53], v45 offset:49152\n"
    "ds_read_b64_tr_b8 a[54:55], v45 offset:50176\n"
    "ds_read_b64_tr_b8 a[56:57], v44 offset:57344\n"
    "ds_read_b64_tr_b8 a[58:59], v44 offset:58368\n"
    "ds_read_b64_tr_b8 a[60:61], v45 offset:57344\n"
    "ds_read_b64_tr_b8 a[62:63], v45 offset:58368\n"
    "s_waitcnt vmcnt(8) lgkmcnt(0)\n"
    "s_barrier\n"
    "s_add_u32 m0, s16, 131072\n"
    "s_add_u32 s35, s52, 131072\n"
    "buffer_load_dwordx4 v16, s[36:39], s52 offen lds\n"
    "buffer_load_dwordx4 v17, s[36:39], s52 offen offset:1024 lds\n"
    "buffer_load_dwordx4 v18, s[36:39], s52 offen offset:2048 lds\n"
    "buffer_load_dwordx4 v19, s[36:39], s52 offen offset:3072 lds\n"
    "s_add_u32 m0, s16, 135168\n"
    "s_nop 0\n"
    "buffer_load_dwordx4 v16, s[36:39], s35 offen lds\n"
    "buffer_load_dwordx4 v17, s[36:39], s35 offen offset:1024 lds\n"
    "buffer_load_dwordx4 v18, s[36:39], s35 offen offset:2048 lds\n"
    "buffer_load_dwordx4 v19, s[36:39], s35 offen offset:3072 lds\n"
    "ds_read_b64_tr_b8 a[64:65], v46 offset:0\n"
    "ds_read_b64_tr_b8 a[66:67], v46 offset:1024\n"
    "ds_read_b64_tr_b8 a[68:69], v47 offset:0\n"
    "ds_read_b64_tr_b8 a[70:71], v47 offset:1024\n"
    "ds_read_b64_tr_b8 a[72:73], v46 offset:8192\n"
    "ds_read_b64_tr_b8 a[74:75], v46 offset:9216\n"
    "ds_read_b64_tr_b8 a[76:77], v47 offset:8192\n"
    "ds_read_b64_tr_b8 a[78:79], v47 offset:9216\n"
    "ds_read_b64_tr_b8 a[80:81], v46 offset:16384\n"
    "ds_read_b64_tr_b8 a[82:83], v46 offset:17408\n"
    "ds_read_b64_tr_b8 a[84:85], v47 offset:16384\n"
    "ds_read_b64_tr_b8 a[86:87], v47 offset:17408\n"
    "ds_read_b64_tr_b8 a[88:89], v46 offset:24576\n"
    "ds_read_b64_tr_b8 a[90:91], v46 offset:25600\n"
    "ds_read_b64_tr_b8 a[92:93], v47 offset:24576\n"
    "ds_read_b64_tr_b8 a[94:95], v47 offset:25600\n"
    "s_waitcnt vmcnt(24)\n"
    "v_add_u32 v36, 1, v3\n"
    "v_mul_f32 v14, v14, v15\n"
    "s_mov_b32 s97, 0x4d000000\n"
    "v_readfirstlane_b32 s17, v36\n"
    "v_readfirstlane_b32 s18, v14\n"
    "s_and_b32 s87, s17, 1\n"
    "s_lshl_b32 s87, s87, 22\n"
    "s_add_u32 s87, s87, s66\n"
    "s_add_u32 s40, s40, s87\n"
    "s_addc_u32 s41, s41, 0\n"
    "s_mul_hi_u32 s87, s17, 0x88888889\n"
    "s_lshr_b32 s87, s87, 3\n"
    "s_mul_i32 s87, s87, 15\n"
    "s_sub_u32 s87, s17, s87\n"
    "s_add_u32 s87, s87, 1\n"
    "s_lshl_b32 s87, s87, 28\n"
    "s_sub_u32 s94, 0, s87\n"
    "v_mov_b32 v4, s87\n"
    "v_mov_b32 v8, s94\n"
    "v_mov_b32 v5, s87\n"
    "v_mov_b32 v9, s94\n"
    "v_mov_b32 v6, s87\n"
    "v_mov_b32 v10, s94\n"
    "v_mov_b32 v7, s87\n"
    "v_mov_b32 v11, s94\n"
    "s_waitcnt vmcnt(8) lgkmcnt(0)\n"
    "s_barrier\n"
    "s_cmp_lg_u32 s3, 0\n"
    "s_cbranch_scc1 L_nocntc%=\n"
    "s_mov_b64 exec, 1\n"
    "global_store_dword v2, v36, s[72:73] sc1\n"
    "s_mov_b64 exec, -1\n"
    "L_nocntc%=:\n"
    "s_add_u32 m0, s14, 0\n"
    "s_nop 0\n"
    "buffer_load_dwordx4 v27, s[68:71], s61 offen lds\n"
    "buffer_load_dwordx4 v28, s[68:71], s61 offen offset:1024 lds\n"
    "buffer_load_dwordx4 v29, s[68:71], s61 offen offset:2048 lds\n"
    "buffer_load_dwordx4 v30, s[68:71], s61 offen offset:3072 lds\n"
    "s_add_u32 m0, s15, 0\n"
    "s_nop 0\n"
    "buffer_load_dwordx4 v27, s[68:71], s62 offen lds\n"
    "buffer_load_dwordx4 v28, s[68:71], s62 offen offset:1024 lds\n"
    "buffer_load_dwordx4 v29, s[68:71], s62 offen offset:2048 lds\n"
    "buffer_load_dwordx4 v30, s[68:71], s62 offen offset:3072 lds\n"
    "ds_read_b64_tr_b8 a[96:97], v46 offset:32768\n"
    "ds_read_b64_tr_b8 a[98:99], v46 offset:33792\n"
    "ds_read_b64_tr_b8 a[100:101], v47 offset:32768\n"
    "ds_read_b64_tr_b8 a[102:103], v47 offset:33792\n"
    "ds_read_b64_tr_b8 a[104:105], v46 offset:40960\n"
    "ds_read_b64_tr_b8 a[106:107], v46 offset:41984\n"
    "ds_read_b64_tr_b8 a[108:109], v47 offset:40960\n"
    "ds_read_b64_tr_b8 a[110:111], v47 offset:41984\n"
    "ds_read_b64_tr_b8 a[112:113], v46 offset:49152\n"
    "ds_read_b64_tr_b8 a[114:115], v46 offset:50176\n"
    "ds_read_b64_tr_b8 a[116:117], v47 offset:49152\n"
    "ds_read_b64_tr_b8 a[118:119], v47 offset:50176\n"
    "ds_read_b64_tr_b8 a[120:121], v46 offset:57344\n"
    "ds_read_b64_tr_b8 a[122:123], v46 offset:58368\n"
    "ds_read_b64_tr_b8 a[124:125], v47 offset:57344\n"
    "ds_read_b64_tr_b8 a[126:127], v47 offset:58368\n"
    "s_waitcnt vmcnt(8) lgkmcnt(0)\n"
    "s_barrier\n"
    "s_add_u32 m0, s16, 32768\n"
    "s_nop 0\n"
    "buffer_load_dwordx4 v27, s[68:71], s63 offen lds\n"
    "buffer_load_dwordx4 v28, s[68:71], s63 offen offset:1024 lds\n"
    "buffer_load_dwordx4 v29, s[68:71], s63 offen offset:2048 lds\n"
    "buffer_load_dwordx4 v30, s[68:71], s63 offen offset:3072 lds\n"
    "ds_read_b64_tr_b8 a[192:193], v48 offset:0\n"
    "ds_read_b64_tr_b8 a[194:195], v48 offset:1024\n"
    "ds_read_b64_tr_b8 a[196:197], v49 offset:0\n"
    "ds_read_b64_tr_b8 a[198:199], v49 offset:1024\n"
    "ds_read_b64_tr_b8 a[200:201], v48 offset:8192\n"
    "ds_read_b64_tr_b8 a[202:203], v48 offset:9216\n"
    "ds_read_b64_tr_b8 a[204:205], v49 offset:8192\n"
    "ds_read_b64_tr_b8 a[206:207], v49 offset:9216\n"
    "ds_read_b64_tr_b8 a[208:209], v48 offset:16384\n"
    "ds_read_b64_tr_b8 a[210:211], v48 offset:17408\n"
    "ds_read_b64_tr_b8 a[212:213], v49 offset:16384\n"
    "ds_read_b64_tr_b8 a[214:215], v49 offset:17408\n"
    "ds_read_b64_tr_b8 a[216:217], v48 offset:24576\n"
    "ds_read_b64_tr_b8 a[218:219], v48 offset:25600\n"
    "ds_read_b64_tr_b8 a[220:221], v49 offset:24576\n"
    "ds_read_b64_tr_b8 a[222:223], v49 offset:25600\n"
    "s_waitcnt vmcnt(4) lgkmcnt(0)\n"
    "s_barrier\n"
    "ds_read_b128 v[52:55], v38\n"
    "ds_read_b128 v[56:59], v39\n"
    "ds_read_b128 v[60:63], v38 offset:2048\n"
    "ds_read_b128 v[64:67], v39 offset:2048\n"
    "ds_read_b128 v[68:71], v38 offset:4096\n"
    "ds_read_b128 v[72:75], v39 offset:4096\n"
    "ds_read_b128 v[76:79], v38 offset:6144\n"
    "ds_read_b128 v[80:83], v39 offset:6144\n"
    "s_waitcnt lgkmcnt(4)\n"
    "v_mfma_i32_16x16x64_i8 v[116:119], a[192:195], v[52:55], v[4:7]\n"
    "ds_read_b128 v[84:87], v38 offset:8192\n"
    "s_add_u32 m0, s16, 65536\n"
    "s_add_u32 s35, s57, 131072\n"
    "buffer_load_dwordx4 v16, s[36:39], s57 offen lds\n"
    "v_mfma_i32_16x16x64_i8 v[116:119], a[200:203], v[56:59], v[116:119]\n"
    "v_mfma_i32_16x16x64_i8 v[116:119], a[208:211], v[60:63], v[116:119]\n"
    "ds_read_b128 v[88:91], v39 offset:8192\n"
    "v_mfma_i32_16x16x64_i8 v[116:119], a[216:219], v[64:67], v[116:119]\n"
    "buffer_load_dwordx4 v17, s[36:39], s57 offen offset:1024 lds\n"
    "v_mfma_i32_16x16x64_i8 v[120:123], a[196:199], v[52:55], v[4:7]\n"
    "ds_read_b128 v[92:95], v38 offset:10240\n"
    "v_mfma_i32_16x16x64_i8 v[120:123], a[204:207], v[56:59], v[120:123]\n"
    "v_mfma_i32_16x16x64_i8 v[120:123], a[212:215], v[60:63], v[120:123]\n"
    "ds_read_b128 v[96:99], v39 offset:10240\n"
    "v_mfma_i32_16x16x64_i8 v[120:123], a[220:223], v[64:67], v[120:123]\n"
    "s_waitcnt lgkmcnt(4)\n"
    "buffer_load_dwordx4 v18, s[36:39], s57 offen offset:2048 lds\n"
    "v_mfma_i32_16x16x64_i8 v[128:131], a[196:199], v[68:71], v[4:7]\n"
    "ds_read_b128 v[100:103], v38 offset:12288\n"
    "v_mfma_i32_16x16x64_i8 v[128:131], a[204:207], v[72:75], v[128:131]\n"
    "buffer_load_dwordx4 v19, s[36:39], s57 offen offset:3072 lds\n"
    "v_mfma_i32_16x16x64_i8 v[128:131], a[212:215], v[76:79], v[128:131]\n"
    "ds_read_b128 v[104:107], v39 offset:12288\n"
    "v_mfma_i32_16x16x64_i8 v[128:131], a[220:223], v[80:83], v[128:131]\n"
    "s_add_u32 m0, s16, 69632\n"
    "v_mfma_i32_16x16x64_i8 v[124:127], a[192:195], v[68:71], v[4:7]\n"
    "ds_read_b128 v[108:111], v38 offset:14336\n"
    "v_mfma_i32_16x16x64_i8 v[124:127], a[200:203], v[72:75], v[124:127]\n"
    "buffer_load_dwordx4 v16, s[36:39], s35 offen lds\n"
    "v_mfma_i32_16x16x64_i8 v[124:127], a[208:211], v[76:79], v[124:127]\n"
    "ds_read_b128 v[112:115], v39 offset:14336\n"
    "v_mfma_i32_16x16x64_i8 v[124:127], a[216:219], v[80:83], v[124:127]\n"
    "s_waitcnt lgkmcnt(4)\n"
    "buffer_load_dwordx4 v17, s[36:39], s35 offen offset:1024 lds\n"
    "v_mfma_i32_16x16x64_i8 v[132:135], a[192:195], v[84:87], v[4:7]\n"
    "ds_read_b128 v[52:55], v38 offset:16384\n"
    "v_mfma_i32_16x16x64_i8 v[132:135], a[200:203], v[88:91], v[132:135]\n"
    "v_mfma_i32_16x16x64_i8 v[132:135], a[208:211], v[92:95], v[132:135]\n"
    "ds_read_b128 v[56:59], v39 offset:16384\n"
    "v_mfma_i32_16x16x64_i8 v[132:135], a[216:219], v[96:99], v[132:135]\n"
    "buffer_load_dwordx4 v18, s[36:39], s35 offen offset:2048 lds\n"
    "v_mfma_i32_16x16x64_i8 v[136:139], a[196:199], v[84:87], v[4:7]\n"
    "ds_read_b128 v[60:63], v38 offset:18432\n"
    "v_mfma_i32_16x16x64_i8 v[136:139], a[204:207], v[88:91], v[136:139]\n"
    "buffer_load_dwordx4 v19, s[36:39], s35 offen offset:3072 lds\n"
    "v_mfma_i32_16x16x64_i8 v[136:139], a[212:215], v[92:95], v[136:139]\n"
    "ds_read_b128 v[64:67], v39 offset:18432\n"
    "v_mfma_i32_16x16x64_i8 v[136:139], a[220:223], v[96:99], v[136:139]\n"
    "s_waitcnt lgkmcnt(4)\n"
    "v_mfma_i32_16x16x64_i8 v[144:147], a[196:199], v[100:103], v[4:7]\n"
    "ds_read_b128 v[68:71], v38 offset:20480\n"
    "v_mfma_i32_16x16x64_i8 v[144:147], a[204:207], v[104:107], v[144:147]\n"
    "global_load_dword v32, v24, s[72:73] sc0 sc1\n"
    "s_add_u32 m0, s16, 98304\n"
    "v_mfma_i32_16x16x64_i8 v[144:147], a[212:215], v[108:111], v[144:147]\n"
    "ds_read_b128 v[72:75], v39 offset:20480\n"
    "s_add_u32 s35, s58, 131072\n"
    "v_mfma_i32_16x16x64_i8 v[144:147], a[220:223], v[112:115], v[144:147]\n"
    "buffer_load_dwordx4 v16, s[36:39], s58 offen lds\n"
    "v_mfma_i32_16x16x64_i8 v[140:143], a[192:195], v[100:103], v[4:7]\n"
    "ds_read_b128 v[76:79], v38 offset:22528\n"
    "v_mfma_i32_16x16x64_i8 v[140:143], a[200:203], v[104:107], v[140:143]\n"
    "v_mfma_i32_16x16x64_i8 v[140:143], a[208:211], v[108:111], v[140:143]\n"
    "ds_read_b128 v[80:83], v39 offset:22528\n"
    "v_mfma_i32_16x16x64_i8 v[140:143], a[216:219], v[112:115], v[140:143]\n"
    "s_waitcnt lgkmcnt(4)\n"
    "buffer_load_dwordx4 v17, s[36:39], s58 offen offset:1024 lds\n"
    "v_mfma_i32_16x16x64_i8 v[148:151], a[192:195], v[52:55], v[8:11]\n"
    "ds_read_b128 v[84:87], v38 offset:24576\n"
    "v_mfma_i32_16x16x64_i8 v[148:151], a[200:203], v[56:59], v[148:151]\n"
    "buffer_load_dwordx4 v18, s[36:39], s58 offen offset:2048 lds\n"
    "v_mfma_i32_16x16x64_i8 v[148:151], a[208:211], v[60:63], v[148:151]\n"
    "ds_read_b128 v[88:91], v39 offset:24576\n"
    "v_mfma_i32_16x16x64_i8 v[148:151], a[216:219], v[64:67], v[148:151]\n"
    "v_mfma_i32_16x16x64_i8 v[152:155], a[196:199], v[52:55], v[8:11]\n"
    "ds_read_b128 v[92:95], v38 offset:26624\n"
    "v_mfma_i32_16x16x64_i8 v[152:155], a[204:207], v[56:59], v[152:155]\n"
    "buffer_load_dwordx4 v19, s[36:39], s58 offen offset:3072 lds\n"
    "s_add_u32 m0, s16, 102400\n"
    "v_mfma_i32_16x16x64_i8 v[152:155], a[212:215], v[60:63], v[152:155]\n"
    "ds_read_b128 v[96:99], v39 offset:26624\n"
    "v_mfma_i32_16x16x64_i8 v[152:155], a[220:223], v[64:67], v[152:155]\n"
    "s_waitcnt lgkmcnt(4)\n"
    "buffer_load_dwordx4 v16, s[36:39], s35 offen lds\n"
    "v_mfma_i32_16x16x64_i8 v[160:163], a[196:199], v[68:71], v[8:11]\n"
    "ds_read_b128 v[100:103], v38 offset:28672\n"
    "v_mfma_i32_16x16x64_i8 v[160:163], a[204:207], v[72:75], v[160:163]\n"
    "v_mfma_i32_16x16x64_i8 v[160:163], a[212:215], v[76:79], v[160:163]\n"
    "ds_read_b128 v[104:107], v39 offset:28672\n"
    "v_mfma_i32_16x16x64_i8 v[160:163], a[220:223], v[80:83], v[160:163]\n"
    "buffer_load_dwordx4 v17, s[36:39], s35 offen offset:1024 lds\n"
    "v_mfma_i32_16x16x64_i8 v[156:159], a[192:195], v[68:71], v[8:11]\n"
    "ds_read_b128 v[108:111], v38 offset:30720\n"
    "v_mfma_i32_16x16x64_i8 v[156:159], a[200:203], v[72:75], v[156:159]\n"
    "buffer_load_dwordx4 v18, s[36:39], s35 offen offset:2048 lds\n"
    "v_mfma_i32_16x16x64_i8 v[156:159], a[208:211], v[76:79], v[156:159]\n"
    "ds_read_b128 v[112:115], v39 offset:30720\n"
    "v_mfma_i32_16x16x64_i8 v[156:159], a[216:219], v[80:83], v[156:159]\n"
    "s_waitcnt lgkmcnt(4)\n"
    "v_mfma_i32_16x16x64_i8 v[164:167], a[192:195], v[84:87], v[8:11]\n"
    "v_mfma_i32_16x16x64_i8 v[164:167], a[200:203], v[88:91], v[164:167]\n"
    "buffer_load_dwordx4 v19, s[36:39], s35 offen offset:3072 lds\n"
    "v_mfma_i32_16x16x64_i8 v[164:167], a[208:211], v[92:95], v[164:167]\n"
    "v_mfma_i32_16x16x64_i8 v[164:167], a[216:219], v[96:99], v[164:167]\n"
    "s_waitcnt vmcnt(17) lgkmcnt(0)\n"
    "s_barrier\n"
    "v_mfma_i32_16x16x64_i8 v[168:171], a[196:199], v[84:87], v[8:11]\n"
    "ds_read_b128 v[52:55], v38 offset:32768\n"
    "v_mfma_i32_16x16x64_i8 v[168:171], a[204:207], v[88:91], v[168:171]\n"
    "ds_read_b128 v[56:59], v39 offset:32768\n"
    "v_mfma_i32_16x16x64_i8 v[168:171], a[212:215], v[92:95], v[168:171]\n"
    "ds_read_b128 v[60:63], v38 offset:34816\n"
    "v_mfma_i32_16x16x64_i8 v[168:171], a[220:223], v[96:99], v[168:171]\n"
    "ds_read_b128 v[64:67], v39 offset:34816\n"
    "v_mfma_i32_16x16x64_i8 v[176:179], a[196:199], v[100:103], v[8:11]\n"
    "ds_read_b128 v[68:71], v38 offset:40960\n"
    "v_mfma_i32_16x16x64_i8 v[176:179], a[204:207], v[104:107], v[176:179]\n"
    "ds_read_b128 v[72:75], v39 offset:40960\n"
    "v_mfma_i32_16x16x64_i8 v[176:179], a[212:215], v[108:111], v[176:179]\n"
    "ds_read_b128 v[76:79], v38 offset:43008\n"
    "v_mfma_i32_16x16x64_i8 v[176:179], a[220:223], v[112:115], v[176:179]\n"
    "ds_read_b128 v[80:83], v39 offset:43008\n"
    "v_mfma_i32_16x16x64_i8 v[172:175], a[192:195], v[100:103], v[8:11]\n"
    "v_mfma_i32_16x16x64_i8 v[172:175], a[200:203], v[104:107], v[172:175]\n"
    "v_mfma_i32_16x16x64_i8 v[172:175], a[208:211], v[108:111], v[172:175]\n"
    "v_mfma_i32_16x16x64_i8 v[172:175], a[216:219], v[112:115], v[172:175]\n"
    "s_waitcnt lgkmcnt(4)\n"
    "v_mfma_i32_16x16x64_i8 v[180:183], a[192:195], v[52:55], v[8:11]\n"
    "s_waitcnt vmcnt(8)\n"
    "v_mfma_i32_16x16x64_i8 v[180:183], a[200:203], v[56:59], v[180:183]\n"
    "v_cmp_le_u32 vcc, s17, v32\n"
    "s_lshr_b32 s87, vcc_lo, 16\n"
    "v_mfma_i32_16x16x64_i8 v[180:183], a[208:211], v[60:63], v[180:183]\n"
    "s_and_b32 s87, s87, vcc_lo\n"
    "s_and_b32 s87, s87, vcc_hi\n"
    "v_mfma_i32_16x16x64_i8 v[180:183], a[216:219], v[64:67], v[180:183]\n"
    "s_ff0_i32_b32 s85, s87\n"
    "v_mfma_i32_16x16x64_i8 v[184:187], a[196:199], v[52:55], v[8:11]\n"
    "s_max_u32 s85, s85, s86\n"
    "s_cmp_eq_u32 s84, 0\n"
    "s_cselect_b32 s91, s85, 0\n"
    "v_mfma_i32_16x16x64_i8 v[184:187], a[204:207], v[56:59], v[184:187]\n"
    "s_cmp_gt_u32 s91, 0\n"
    "s_cselect_b32 s82, 0, s83\n"
    "s_addc_u32 s84, s84, 0\n"
    "s_add_u32 m0, s11, 131072\n"
    "v_mfma_i32_16x16x64_i8 v[184:187], a[212:215], v[60:63], v[184:187]\n"
    "s_add_u32 s34, s8, s82\n"
    "v_mfma_i32_16x16x64_i8 v[184:187], a[220:223], v[64:67], v[184:187]\n"
    "s_waitcnt lgkmcnt(0)\n"
    "buffer_load_dwordx4 v22, s[40:43], s34 offen sc0 sc1 lds\n"
    "v_mfma_i32_16x16x64_i8 v[192:195], a[196:199], v[68:71], v[8:11]\n"
    "buffer_load_dwordx4 v22, s[40:43], s34 offen offset:1024 sc0 sc1 lds\n"
    "s_add_u32 m0, s12, 131072\n"
    "s_add_u32 s34, s9, s82\n"
    "v_mfma_i32_16x16x64_i8 v[192:195], a[204:207], v[72:75], v[192:195]\n"
    "buffer_load_dwordx4 v22, s[40:43], s34 offen sc0 sc1 lds\n"
    "v_mfma_i32_16x16x64_i8 v[192:195], a[212:215], v[76:79], v[192:195]\n"
    "buffer_load_dwordx4 v22, s[40:43], s34 offen offset:1024 sc0 sc1 lds\n"
    "s_add_u32 m0, s13, 131072\n"
    "s_add_u32 s34, s10, s82\n"
    "v_mfma_i32_16x16x64_i8 v[192:195], a[220:223], v[80:83], v[192:195]\n"
    "buffer_load_dwordx4 v22, s[40:43], s34 offen sc0 sc1 lds\n"
    "v_mfma_i32_16x16x64_i8 v[188:191], a[192:195], v[68:71], v[8:11]\n"
    "v_mfma_i32_16x16x64_i8 v[188:191], a[200:203], v[72:75], v[188:191]\n"
    "buffer_load_dwordx4 v22, s[40:43], s34 offen offset:1024 sc0 sc1 lds\n"
    "s_mov_b32 s96, 1\n"
    "v_mfma_i32_16x16x64_i8 v[188:191], a[208:211], v[76:79], v[188:191]\n"
    "v_mfma_i32_16x16x64_i8 v[188:191], a[216:219], v[80:83], v[188:191]\n"
    "s_waitcnt vmcnt(15) lgkmcnt(0)\n"
    "s_barrier\n"
    "global_load_dword v32, v24, s[72:73] sc0 sc1\n"
    "s_cmp_eq_u32 s84, 1\n"
    "s_cselect_b32 s91, s85, 0\n"
    "s_cmp_gt_u32 s91, 1\n"
    "s_cselect_b32 s82, 8192, s83\n"
    "s_addc_u32 s84, s84, 0\n"
    "s_add_u32 m0, s11, 0\n"
    "s_add_u32 s34, s8, s82\n"
    "buffer_load_dwordx4 v22, s[40:43], s34 offen sc0 sc1 lds\n"
    "buffer_load_dwordx4 v22, s[40:43], s34 offen offset:1024 sc0 sc1 lds\n"
    "s_add_u32 m0, s12, 0\n"
    "s_add_u32 s34, s9, s82\n"
    "buffer_load_dwordx4 v22, s[40:43], s34 offen sc0 sc1 lds\n"
    "buffer_load_dwordx4 v22, s[40:43], s34 offen offset:1024 sc0 sc1 lds\n"
    "s_add_u32 m0, s13, 0\n"
    "s_add_u32 s34, s10, s82\n"
    "buffer_load_dwordx4 v22, s[40:43], s34 offen sc0 sc1 lds\n"
    "buffer_load_dwordx4 v22, s[40:43], s34 offen offset:1024 sc0 sc1 lds\n"
    "s_mov_b32 s96, 2\n"
    "ds_read_b64_tr_b8 a[128:129], v46 offset:0\n"
    "ds_read_b64_tr_b8 a[130:131], v46 offset:1024\n"
    "ds_read_b64_tr_b8 a[132:133], v47 offset:0\n"
    "ds_read_b64_tr_b8 a[134:135], v47 offset:1024\n"
    "ds_read_b64_tr_b8 a[136:137], v46 offset:8192\n"
    "ds_read_b64_tr_b8 a[138:139], v46 offset:9216\n"
    "ds_read_b64_tr_b8 a[140:141], v47 offset:8192\n"
    "ds_read_b64_tr_b8 a[142:143], v47 offset:9216\n"
    "ds_read_b64_tr_b8 a[144:145], v46 offset:16384\n"
    "ds_read_b64_tr_b8 a[146:147], v46 offset:17408\n"
    "ds_read_b64_tr_b8 a[148:149], v47 offset:16384\n"
    "ds_read_b64_tr_b8 a[150:151], v47 offset:17408\n"
    "ds_read_b64_tr_b8 a[152:153], v46 offset:24576\n"
    "ds_read_b64_tr_b8 a[154:155], v46 offset:25600\n"
    "ds_read_b64_tr_b8 a[156:157], v47 offset:24576\n"
    "ds_read_b64_tr_b8 a[158:159], v47 offset:25600\n"
    "s_waitcnt vmcnt(13) lgkmcnt(0)\n"
    "s_barrier\n"
    "s_cmp_lt_u32 s84, s96\n"
    "s_getpc_b64 s[88:89]\n"
    "s_cbranch_scc1 L_midc%=\n"
    "ds_read_b64_tr_b8 a[160:161], v46 offset:32768\n"
    "ds_read_b64_tr_b8 a[162:163], v46 offset:33792\n"
    "ds_read_b64_tr_b8 a[164:165], v47 offset:32768\n"
    "ds_read_b64_tr_b8 a[166:167], v47 offset:33792\n"
    "ds_read_b64_tr_b8 a[168:169], v46 offset:40960\n"
    "ds_read_b64_tr_b8 a[170:171], v46 offset:41984\n"
    "ds_read_b64_tr_b8 a[172:173], v47 offset:40960\n"
    "ds_read_b64_tr_b8 a[174:175], v47 offset:41984\n"
    "ds_read_b64_tr_b8 a[176:177], v46 offset:49152\n"
    "ds_read_b64_tr_b8 a[178:179], v46 offset:50176\n"
    "ds_read_b64_tr_b8 a[180:181], v47 offset:49152\n"
    "ds_read_b64_tr_b8 a[182:183], v47 offset:50176\n"
    "ds_read_b64_tr_b8 a[184:185], v46 offset:57344\n"
    "ds_read_b64_tr_b8 a[186:187], v46 offset:58368\n"
    "ds_read_b64_tr_b8 a[188:189], v47 offset:57344\n"
    "ds_read_b64_tr_b8 a[190:191], v47 offset:58368\n"
    "s_mov_b32 s95, 0\n"
    "s_and_b32 s91, s84, s25\n"
    "s_cmp_gt_u32 s91, 0\n"
    "s_getpc_b64 s[88:89]\n"
    "s_cbranch_scc0 L_slowc%=\n"
    "s_waitcnt vmcnt(7) lgkmcnt(0)\n"
    "s_barrier\n"
    "ds_read_b128 v[84:87], v42\n"
    "ds_read_b128 v[88:91], v43\n"
    "ds_read_b128 v[92:95], v42 offset:8192\n"
    "ds_read_b128 v[96:99], v43 offset:8192\n"
    "ds_read_b128 v[100:103], v42 offset:16384\n"
    "ds_read_b128 v[104:107], v43 offset:16384\n"
    "ds_read_b128 v[108:111], v42 offset:2048\n"
    "ds_read_b128 v[112:115], v43 offset:2048\n"
    "s_waitcnt lgkmcnt(5)\n"
    "v_mfma_i32_16x16x64_i8 v[116:119], a[0:3], v[84:87], v[116:119]\n"
    "s_waitcnt vmcnt(6)\n"
    "ds_read_b128 v[52:55], v42 offset:10240\n"
    "v_mfma_i32_16x16x64_i8 v[116:119], a[8:11], v[88:91], v[116:119]\n"
    "v_cmp_le_u32 vcc, s17, v32\n"
    "s_lshr_b32 s87, vcc_lo, 16\n"
    "s_and_b32 s87, s87, vcc_lo\n"
    "v_mfma_i32_16x16x64_i8 v[116:119], a[32:35], v[92:95], v[116:119]\n"
    "ds_read_b128 v[56:59], v43 offset:10240\n"
    "s_and_b32 s87, s87, vcc_hi\n"
    "v_mfma_i32_16x16x64_i8 v[120:123], a[4:7], v[84:87], v[120:123]\n"
    "s_ff0_i32_b32 s85, s87\n"
    "s_max_u32 s85, s85, s86\n"
    "s_min_u32 s91, s85, s96\n"
    "v_mfma_i32_16x16x64_i8 v[120:123], a[12:15], v[88:91], v[120:123]\n"
    "ds_read_b128 v[60:63], v42 offset:18432\n"
    "s_cmp_lt_u32 s84, s91\n"
    "s_getpc_b64 s[88:89]\n"
    "s_cbranch_scc1 L_catchc%=\n"
    "v_mfma_i32_16x16x64_i8 v[120:123], a[36:39], v[92:95], v[120:123]\n"
    "s_waitcnt lgkmcnt(5)\n"
    "global_load_dword v32, v24, s[72:73] sc0 sc1\n"
    "v_mfma_i32_16x16x64_i8 v[120:123], a[44:47], v[96:99], v[120:123]\n"
    "ds_read_b128 v[64:67], v43 offset:18432\n"
    "s_cmp_eq_u32 s84, 2\n"
    "s_cselect_b32 s91, s85, 0\n"
    "v_mfma_i32_16x16x64_i8 v[120:123], a[68:71], v[100:103], v[120:123]\n"
    "s_cmp_gt_u32 s91, 2\n"
    "s_cselect_b32 s82, 16384, s83\n"
    "s_addc_u32 s84, s84, 0\n"
    "s_add_u32 m0, s11, 32768\n"
    "v_mfma_i32_16x16x64_i8 v[120:123], a[76:79], v[104:107], v[120:123]\n"
    "ds_read_b128 v[68:71], v42 offset:4096\n"
    "s_add_u32 s34, s8, s82\n"
    "v_mfma_i32_16x16x64_i8 v[116:119], a[40:43], v[96:99], v[116:119]\n"
    "buffer_load_dwordx4 v22, s[40:43], s34 offen sc0 sc1 lds\n"
    "v_mfma_i32_16x16x64_i8 v[116:119], a[64:67], v[100:103], v[116:119]\n"
    "ds_read_b128 v[72:75], v43 offset:4096\n"
    "v_mfma_i32_16x16x64_i8 v[116:119], a[72:75], v[104:107], v[116:119]\n"
    "s_waitcnt lgkmcnt(5)\n"
    "buffer_load_dwordx4 v22, s[40:43], s34 offen offset:1024 sc0 sc1 lds\n"
    "v_mfma_i32_16x16x64_i8 v[116:119], a[16:19], v[108:111], v[116:119]\n"
    "ds_read_b128 v[76:79], v42 offset:12288\n"
    "v_mfma_i32_16x16x64_i8 v[116:119], a[24:27], v[112:115], v[116:119]\n"
    "s_add_u32 m0, s12, 32768\n"
    "s_add_u32 s34, s9, s82\n"
    "v_mfma_i32_16x16x64_i8 v[116:119], a[48:51], v[52:55], v[116:119]\n"
    "ds_read_b128 v[80:83], v43 offset:12288\n"
    "v_mfma_i32_16x16x64_i8 v[120:123], a[20:23], v[108:111], v[120:123]\n"
    "buffer_load_dwordx4 v22, s[40:43], s34 offen sc0 sc1 lds\n"
    "s_cmp_lt_u32 s84, s96\n"
    "s_getpc_b64 s[88:89]\n"
    "s_cbranch_scc1 L_midc%=\n"
    "v_mfma_i32_16x16x64_i8 v[120:123], a[28:31], v[112:115], v[120:123]\n"
    "ds_read_b128 v[84:87], v42 offset:20480\n"
    "v_mfma_i32_16x16x64_i8 v[120:123], a[52:55], v[52:55], v[120:123]\n"
    "s_waitcnt lgkmcnt(5)\n"
    "v_mfma_i32_16x16x64_i8 v[120:123], a[60:63], v[56:59], v[120:123]\n"
    "ds_read_b128 v[88:91], v43 offset:20480\n"
    "v_mfma_i32_16x16x64_i8 v[120:123], a[84:87], v[60:63], v[120:123]\n"
    "v_mfma_i32_16x16x64_i8 v[120:123], a[92:95], v[64:67], v[120:123]\n"
    "ds_read_b128 v[92:95], v42 offset:6144\n"
    "v_mfma_i32_16x16x64_i8 v[116:119], a[56:59], v[56:59], v[116:119]\n"
    "buffer_load_dwordx4 v22, s[40:43], s34 offen offset:1024 sc0 sc1 lds\n"
    "v_mfma_i32_16x16x64_i8 v[116:119], a[80:83], v[60:63], v[116:119]\n"
    "ds_read_b128 v[96:99], v43 offset:6144\n"
    "v_mfma_i32_16x16x64_i8 v[116:119], a[88:91], v[64:67], v[116:119]\n"
    "s_waitcnt lgkmcnt(5)\n"
    "s_add_u32 m0, s13, 32768\n"
    "v_mfma_i32_16x16x64_i8 v[116:119], a[96:99], v[68:71], v[116:119]\n"
    "ds_read_b128 v[100:103], v42 offset:14336\n"
    "s_add_u32 s34, s10, s82\n"
    "v_mfma_i32_16x16x64_i8 v[116:119], a[104:107], v[72:75], v[116:119]\n"
    "buffer_load_dwordx4 v22, s[40:43], s34 offen sc0 sc1 lds\n"
    "v_mfma_i32_16x16x64_i8 v[116:119], a[128:131], v[76:79], v[116:119]\n"
    "ds_read_b128 v[104:107], v43 offset:14336\n"
    "v_mfma_i32_16x16x64_i8 v[120:123], a[100:103], v[68:71], v[120:123]\n"
    "v_mfma_i32_16x16x64_i8 v[120:123], a[108:111], v[72:75], v[120:123]\n"
    "ds_read_b128 v[108:111], v42 offset:22528\n"
    "v_mfma_i32_16x16x64_i8 v[120:123], a[132:135], v[76:79], v[120:123]\n"
    "s_waitcnt lgkmcnt(5)\n"
    "v_mfma_i32_16x16x64_i8 v[120:123], a[140:143], v[80:83], v[120:123]\n"
    "ds_read_b128 v[112:115], v43 offset:22528\n"
    "v_mfma_i32_16x16x64_i8 v[120:123], a[164:167], v[84:87], v[120:123]\n"
    "buffer_load_dwordx4 v22, s[40:43], s34 offen offset:1024 sc0 sc1 lds\n"
    "s_mov_b32 s96, 3\n"
    "v_mfma_i32_16x16x64_i8 v[120:123], a[172:175], v[88:91], v[120:123]\n"
    "v_mfma_i32_16x16x64_i8 v[116:119], a[136:139], v[80:83], v[116:119]\n"
    "v_mfma_i32_16x16x64_i8 v[116:119], a[160:163], v[84:87], v[116:119]\n"
    "s_mov_b32 s95, 1\n"
    "s_and_b32 s91, s84, s25\n"
    "v_mfma_i32_16x16x64_i8 v[116:119], a[168:171], v[88:91], v[116:119]\n"
    "s_cmp_gt_u32 s91, 1\n"
    "s_getpc_b64 s[88:89]\n"
    "s_cbranch_scc0 L_slowc%=\n"
    "s_waitcnt vmcnt(7) lgkmcnt(0)\n"
    "s_barrier\n"
    "v_mfma_i32_16x16x64_i8 v[116:119], a[112:115], v[92:95], v[116:119]\n"
    "ds_read_b128 v[52:55], v38\n"
    "s_waitcnt vmcnt(6)\n"
    "v_mfma_i32_16x16x64_i8 v[116:119], a[120:123], v[96:99], v[116:119]\n"
    "ds_read_b128 v[56:59], v39\n"
    "v_cmp_le_u32 vcc, s17, v32\n"
    "v_mfma_i32_16x16x64_i8 v[116:119], a[144:147], v[100:103], v[116:119]\n"
    "ds_read_b128 v[60:63], v38 offset:8192\n"
    "v_mfma_i32_16x16x64_i8 v[120:123], a[116:119], v[92:95], v[120:123]\n"
    "ds_read_b128 v[64:67], v39 offset:8192\n"
    "s_lshr_b32 s87, vcc_lo, 16\n"
    "v_mfma_i32_16x16x64_i8 v[120:123], a[124:127], v[96:99], v[120:123]\n"
    "ds_read_b128 v[68:71], v38 offset:16384\n"
    "v_mfma_i32_16x16x64_i8 v[120:123], a[148:151], v[100:103], v[120:123]\n"
    "ds_read_b128 v[72:75], v39 offset:16384\n"
    "s_and_b32 s87, s87, vcc_lo\n"
    "v_mfma_i32_16x16x64_i8 v[120:123], a[156:159], v[104:107], v[120:123]\n"
    "ds_read_b128 v[76:79], v38 offset:2048\n"
    "v_mfma_i32_16x16x64_i8 v[120:123], a[180:183], v[108:111], v[120:123]\n"
    "ds_read_b128 v[80:83], v39 offset:2048\n"
    "s_and_b32 s87, s87, vcc_hi\n"
    "v_mfma_i32_16x16x64_i8 v[120:123], a[188:191], v[112:115], v[120:123]\n"
    "s_ff0_i32_b32 s85, s87\n"
    "v_mfma_i32_16x16x64_i8 v[116:119], a[152:155], v[104:107], v[116:119]\n"
    "s_max_u32 s85, s85, s86\n"
    "v_mfma_i32_16x16x64_i8 v[116:119], a[176:179], v[108:111], v[116:119]\n"
    "s_min_u32 s91, s85, s96\n"
    "v_mfma_i32_16x16x64_i8 v[116:119], a[184:187], v[112:115], v[116:119]\n"
    "s_cmp_lt_u32 s84, s91\n"
    "s_getpc_b64 s[88:89]\n"
    "s_cbranch_scc1 L_catchc%=\n"
    "s_waitcnt lgkmcnt(5)\n"
    "v_mfma_i32_16x16x64_i8 v[124:127], a[0:3], v[52:55], v[124:127]\n"
    "global_load_dword v32, v24, s[72:73] sc0 sc1\n"
    "ds_read_b128 v[84:87], v38 offset:10240\n"
    "v_mfma_i32_16x16x64_i8 v[124:127], a[8:11], v[56:59], v[124:127]\n"
    "v_mfma_i32_16x16x64_i8 v[124:127], a[32:35], v[60:63], v[124:127]\n"
    "ds_read_b128 v[88:91], v39 offset:10240\n"
    "v_mfma_i32_16x16x64_i8 v[128:131], a[4:7], v[52:55], v[128:131]\n"
    "v_mfma_i32_16x16x64_i8 v[128:131], a[12:15], v[56:59], v[128:131]\n"
    "ds_read_b128 v[92:95], v38 offset:18432\n"
    "v_mfma_i32_16x16x64_i8 v[128:131], a[36:39], v[60:63], v[128:131]\n"
    "s_waitcnt lgkmcnt(5)\n"
    "s_add_u32 s90, s64, 0\n"
    "buffer_store_dwordx4 v[116:119], v22, s[44:47], s90 offen sc0 sc1\n"
    "v_mfma_i32_16x16x64_i8 v[128:131], a[44:47], v[64:67], v[128:131]\n"
    "ds_read_b128 v[96:99], v39 offset:18432\n"
    "v_mfma_i32_16x16x64_i8 v[128:131], a[68:71], v[68:71], v[128:131]\n"
    "v_mfma_i32_16x16x64_i8 v[128:131], a[76:79], v[72:75], v[128:131]\n"
    "ds_read_b128 v[100:103], v38 offset:4096\n"
    "v_mfma_i32_16x16x64_i8 v[124:127], a[40:43], v[64:67], v[124:127]\n"
    "buffer_store_dwordx4 v[120:123], v22, s[44:47], s90 offen offset:1024 sc0 sc1\n"
    "s_cmp_eq_u32 s84, 3\n"
    "s_cselect_b32 s91, s85, 0\n"
    "v_mfma_i32_16x16x64_i8 v[124:127], a[64:67], v[68:71], v[124:127]\n"
    "ds_read_b128 v[104:107], v39 offset:4096\n"
    "s_cmp_gt_u32 s91, 3\n"
    "s_cselect_b32 s82, 24576, s83\n"
    "s_addc_u32 s84, s84, 0\n"
    "v_mfma_i32_16x16x64_i8 v[124:127], a[72:75], v[72:75], v[124:127]\n"
    "s_waitcnt lgkmcnt(5)\n"
    "s_add_u32 m0, s11, 65536\n"
    "v_mfma_i32_16x16x64_i8 v[124:127], a[16:19], v[76:79], v[124:127]\n"
    "ds_read_b128 v[108:111], v38 offset:12288\n"
    "s_add_u32 s34, s8, s82\n"
    "v_mfma_i32_16x16x64_i8 v[124:127], a[24:27], v[80:83], v[124:127]\n"
    "buffer_load_dwordx4 v22, s[40:43], s34 offen sc0 sc1 lds\n"
    "v_mfma_i32_16x16x64_i8 v[124:127], a[48:51], v[84:87], v[124:127]\n"
    "ds_read_b128 v[112:115], v39 offset:12288\n"
    "v_mfma_i32_16x16x64_i8 v[128:131], a[20:23], v[76:79], v[128:131]\n"
    "s_cmp_lt_u32 s84, s96\n"
    "s_getpc_b64 s[88:89]\n"
    "s_cbranch_scc1 L_midc%=\n"
    "v_mfma_i32_16x16x64_i8 v[128:131], a[28:31], v[80:83], v[128:131]\n"
    "ds_read_b128 v[52:55], v38 offset:20480\n"
    "v_mfma_i32_16x16x64_i8 v[128:131], a[52:55], v[84:87], v[128:131]\n"
    "s_waitcnt lgkmcnt(5)\n"
    "buffer_load_dwordx4 v22, s[40:43], s34 offen offset:1024 sc0 sc1 lds\n"
    "v_mfma_i32_16x16x64_i8 v[128:131], a[60:63], v[88:91], v[128:131]\n"
    "ds_read_b128 v[56:59], v39 offset:20480\n"
    "v_mfma_i32_16x16x64_i8 v[128:131], a[84:87], v[92:95], v[128:131]\n"
    "s_add_u32 m0, s12, 65536\n"
    "s_add_u32 s34, s9, s82\n"
    "v_mfma_i32_16x16x64_i8 v[128:131], a[92:95], v[96:99], v[128:131]\n"
    "ds_read_b128 v[60:63], v38 offset:6144\n"
    "v_mfma_i32_16x16x64_i8 v[124:127], a[56:59], v[88:91], v[124:127]\n"
    "buffer_load_dwordx4 v22, s[40:43], s34 offen sc0 sc1 lds\n"
    "v_mfma_i32_16x16x64_i8 v[124:127], a[80:83], v[92:95], v[124:127]\n"
    "ds_read_b128 v[64:67], v39 offset:6144\n"
    "v_mfma_i32_16x16x64_i8 v[124:127], a[88:91], v[96:99], v[124:127]\n"
    "s_waitcnt lgkmcnt(5)\n"
    "buffer_load_dwordx4 v22, s[40:43], s34 offen offset:1024 sc0 sc1 lds\n"
    "v_mfma_i32_16x16x64_i8 v[124:127], a[96:99], v[100:103], v[124:127]\n"
    "ds_read_b128 v[68:71], v38 offset:14336\n"
    "v_mfma_i32_16x16x64_i8 v[124:127], a[104:107], v[104:107], v[124:127]\n"
    "s_add_u32 m0, s13, 65536\n"
    "s_add_u32 s34, s10, s82\n"
    "v_mfma_i32_16x16x64_i8 v[124:127], a[128:131], v[108:111], v[124:127]\n"
    "ds_read_b128 v[72:75], v39 offset:14336\n"
    "v_mfma_i32_16x16x64_i8 v[128:131], a[100:103], v[100:103], v[128:131]\n"
    "buffer_load_dwordx4 v22, s[40:43], s34 offen sc0 sc1 lds\n"
    "v_mfma_i32_16x16x64_i8 v[128:131], a[108:111], v[104:107], v[128:131]\n"
    "ds_read_b128 v[76:79], v38 offset:22528\n"
    "v_mfma_i32_16x16x64_i8 v[128:131], a[132:135], v[108:111], v[128:131]\n"
    "s_waitcnt lgkmcnt(5)\n"
    "v_mfma_i32_16x16x64_i8 v[128:131], a[140:143], v[112:115], v[128:131]\n"
    "ds_read_b128 v[80:83], v39 offset:22528\n"
    "v_mfma_i32_16x16x64_i8 v[128:131], a[164:167], v[52:55], v[128:131]\n"
    "buffer_load_dwordx4 v22, s[40:43], s34 offen offset:1024 sc0 sc1 lds\n"
    "s_mov_b32 s96, 4\n"
    "v_mfma_i32_16x16x64_i8 v[128:131], a[172:175], v[56:59], v[128:131]\n"
    "v_mfma_i32_16x16x64_i8 v[124:127], a[136:139], v[112:115], v[124:127]\n"
    "v_mfma_i32_16x16x64_i8 v[124:127], a[160:163], v[52:55], v[124:127]\n"
    "s_mov_b32 s95, 2\n"
    "s_and_b32 s91, s84, s25\n"
    "v_mfma_i32_16x16x64_i8 v[124:127], a[168:171], v[56:59], v[124:127]\n"
    "s_cmp_gt_u32 s91, 2\n"
    "s_getpc_b64 s[88:89]\n"
    "s_cbranch_scc0 L_slowc%=\n"
    "s_waitcnt vmcnt(9) lgkmcnt(0)\n"
    "s_barrier\n"
    "v_mfma_i32_16x16x64_i8 v[124:127], a[112:115], v[60:63], v[124:127]\n"
    "ds_read_b128 v[84:87], v38 offset:32768\n"
    "s_waitcnt vmcnt(8)\n"
    "v_mfma_i32_16x16x64_i8 v[124:127], a[120:123], v[64:67], v[124:127]\n"
    "ds_read_b128 v[88:91], v39 offset:32768\n"
    "v_cmp_le_u32 vcc, s17, v32\n"
    "v_mfma_i32_16x16x64_i8 v[124:127], a[144:147], v[68:71], v[124:127]\n"
    "ds_read_b128 v[92:95], v38 offset:40960\n"
    "v_mfma_i32_16x16x64_i8 v[128:131], a[116:119], v[60:63], v[128:131]\n"
    "ds_read_b128 v[96:99], v39 offset:40960\n"
    "s_lshr_b32 s87, vcc_lo, 16\n"
    "v_mfma_i32_16x16x64_i8 v[128:131], a[124:127], v[64:67], v[128:131]\n"
    "ds_read_b128 v[100:103], v38 offset:49152\n"
    "v_mfma_i32_16x16x64_i8 v[128:131], a[148:151], v[68:71], v[128:131]\n"
    "ds_read_b128 v[104:107], v39 offset:49152\n"
    "s_and_b32 s87, s87, vcc_lo\n"
    "v_mfma_i32_16x16x64_i8 v[128:131], a[156:159], v[72:75], v[128:131]\n"
    "ds_read_b128 v[108:111], v38 offset:34816\n"
    "v_mfma_i32_16x16x64_i8 v[128:131], a[180:183], v[76:79], v[128:131]\n"
    "ds_read_b128 v[112:115], v39 offset:34816\n"
    "s_and_b32 s87, s87, vcc_hi\n"
    "v_mfma_i32_16x16x64_i8 v[128:131], a[188:191], v[80:83], v[128:131]\n"
    "s_ff0_i32_b32 s85, s87\n"
    "v_mfma_i32_16x16x64_i8 v[124:127], a[152:155], v[72:75], v[124:127]\n"
    "s_max_u32 s85, s85, s86\n"
    "v_mfma_i32_16x16x64_i8 v[124:127], a[176:179], v[76:79], v[124:127]\n"
    "s_min_u32 s91, s85, s96\n"
    "v_mfma_i32_16x16x64_i8 v[124:127], a[184:187], v[80:83], v[124:127]\n"
    "s_cmp_lt_u32 s84, s91\n"
    "s_getpc_b64 s[88:89]\n"
    "s_cbranch_scc1 L_catchc%=\n"
    "s_waitcnt lgkmcnt(5)\n"
    "v_mfma_i32_16x16x64_i8 v[132:135], a[0:3], v[84:87], v[132:135]\n"
    "global_load_dword v32, v24, s[72:73] sc0 sc1\n"
    "ds_read_b128 v[52:55], v38 offset:43008\n"
    "v_mfma_i32_16x16x64_i8 v[132:135], a[8:11], v[88:91], v[132:135]\n"
    "v_mfma_i32_16x16x64_i8 v[132:135], a[32:35], v[92:95], v[132:135]\n"
    "ds_read_b128 v[56:59], v39 offset:43008\n"
    "v_mfma_i32_16x16x64_i8 v[136:139], a[4:7], v[84:87], v[136:139]\n"
    "v_mfma_i32_16x16x64_i8 v[136:139], a[12:15], v[88:91], v[136:139]\n"
    "ds_read_b128 v[60:63], v38 offset:51200\n"
    "v_mfma_i32_16x16x64_i8 v[136:139], a[36:39], v[92:95], v[136:139]\n"
    "s_waitcnt lgkmcnt(5)\n"
    "s_add_u32 s90, s64, 24576\n"
    "buffer_store_dwordx4 v[124:127], v22, s[44:47], s90 offen sc0 sc1\n"
    "v_mfma_i32_16x16x64_i8 v[136:139], a[44:47], v[96:99], v[136:139]\n"
    "ds_read_b128 v[64:67], v39 offset:51200\n"
    "v_mfma_i32_16x16x64_i8 v[136:139], a[68:71], v[100:103], v[136:139]\n"
    "v_mfma_i32_16x16x64_i8 v[136:139], a[76:79], v[104:107], v[136:139]\n"
    "ds_read_b128 v[68:71], v38 offset:36864\n"
    "v_mfma_i32_16x16x64_i8 v[132:135], a[40:43], v[96:99], v[132:135]\n"
    "buffer_store_dwordx4 v[128:131], v22, s[44:47], s90 offen offset:1024 sc0 sc1\n"
    "s_cmp_eq_u32 s84, 4\n"
    "s_cselect_b32 s91, s85, 0\n"
    "v_mfma_i32_16x16x64_i8 v[132:135], a[64:67], v[100:103], v[132:135]\n"
    "ds_read_b128 v[72:75], v39 offset:36864\n"
    "s_cmp_gt_u32 s91, 4\n"
    "s_cselect_b32 s82, 32768, s83\n"
    "s_addc_u32 s84, s84, 0\n"
    "v_mfma_i32_16x16x64_i8 v[132:135], a[72:75], v[104:107], v[132:135]\n"
    "s_waitcnt lgkmcnt(5)\n"
    "s_add_u32 m0, s11, 98304\n"
    "v_mfma_i32_16x16x64_i8 v[132:135], a[16:19], v[108:111], v[132:135]\n"
    "ds_read_b128 v[76:79], v38 offset:45056\n"
    "s_add_u32 s34, s8, s82\n"
    "v_mfma_i32_16x16x64_i8 v[132:135], a[24:27], v[112:115], v[132:135]\n"
    "buffer_load_dwordx4 v22, s[40:43], s34 offen sc0 sc1 lds\n"
    "v_mfma_i32_16x16x64_i8 v[132:135], a[48:51], v[52:55], v[132:135]\n"
    "ds_read_b128 v[80:83], v39 offset:45056\n"
    "v_mfma_i32_16x16x64_i8 v[136:139], a[20:23], v[108:111], v[136:139]\n"
    "s_cmp_lt_u32 s84, s96\n"
    "s_getpc_b64 s[88:89]\n"
    "s_cbranch_scc1 L_midc%=\n"
    "v_mfma_i32_16x16x64_i8 v[136:139], a[28:31], v[112:115], v[136:139]\n"
    "ds_read_b128 v[84:87], v38 offset:53248\n"
    "v_mfma_i32_16x16x64_i8 v[136:139], a[52:55], v[52:55], v[136:139]\n"
    "s_waitcnt lgkmcnt(5)\n"
    "buffer_load_dwordx4 v22, s[40:43], s34 offen offset:1024 sc0 sc1 lds\n"
    "v_mfma_i32_16x16x64_i8 v[136:139], a[60:63], v[56:59], v[136:139]\n"
    "ds_read_b128 v[88:91], v39 offset:53248\n"
    "v_mfma_i32_16x16x64_i8 v[136:139], a[84:87], v[60:63], v[136:139]\n"
    "s_add_u32 m0, s12, 98304\n"
    "s_add_u32 s34, s9, s82\n"
    "v_mfma_i32_16x16x64_i8 v[136:139], a[92:95], v[64:67], v[136:139]\n"
    "ds_read_b128 v[92:95], v38 offset:38912\n"
    "v_mfma_i32_16x16x64_i8 v[132:135], a[56:59], v[56:59], v[132:135]\n"
    "buffer_load_dwordx4 v22, s[40:43], s34 offen sc0 sc1 lds\n"
    "v_mfma_i32_16x16x64_i8 v[132:135], a[80:83], v[60:63], v[132:135]\n"
    "ds_read_b128 v[96:99], v39 offset:38912\n"
    "v_mfma_i32_16x16x64_i8 v[132:135], a[88:91], v[64:67], v[132:135]\n"
    "s_waitcnt lgkmcnt(5)\n"
    "buffer_load_dwordx4 v22, s[40:43], s34 offen offset:1024 sc0 sc1 lds\n"
    "v_mfma_i32_16x16x64_i8 v[132:135], a[96:99], v[68:71], v[132:135]\n"
    "ds_read_b128 v[100:103], v38 offset:47104\n"
    "v_mfma_i32_16x16x64_i8 v[132:135], a[104:107], v[72:75], v[132:135]\n"
    "s_add_u32 m0, s13, 98304\n"
    "s_add_u32 s34, s10, s82\n"
    "v_mfma_i32_16x16x64_i8 v[132:135], a[128:131], v[76:79], v[132:135]\n"
    "ds_read_b128 v[104:107], v39 offset:47104\n"
    "v_mfma_i32_16x16x64_i8 v[136:139], a[100:103], v[68:71], v[136:139]\n"
    "buffer_load_dwordx4 v22, s[40:43], s34 offen sc0 sc1 lds\n"
    "v_mfma_i32_16x16x64_i8 v[136:139], a[108:111], v[72:75], v[136:139]\n"
    "ds_read_b128 v[108:111], v38 offset:55296\n"
    "v_mfma_i32_16x16x64_i8 v[136:139], a[132:135], v[76:79], v[136:139]\n"
    "s_waitcnt lgkmcnt(5)\n"
    "v_mfma_i32_16x16x64_i8 v[136:139], a[140:143], v[80:83], v[136:139]\n"
    "ds_read_b128 v[112:115], v39 offset:55296\n"
    "v_mfma_i32_16x16x64_i8 v[136:139], a[164:167], v[84:87], v[136:139]\n"
    "buffer_load_dwordx4 v22, s[40:43], s34 offen offset:1024 sc0 sc1 lds\n"
    "s_mov_b32 s96, 5\n"
    "v_mfma_i32_16x16x64_i8 v[136:139], a[172:175], v[88:91], v[136:139]\n"
    "v_mfma_i32_16x16x64_i8 v[132:135], a[136:139], v[80:83], v[132:135]\n"
    "v_mfma_i32_16x16x64_i8 v[132:135], a[160:163], v[84:87], v[132:135]\n"
    "s_mov_b32 s95, 3\n"
    "s_and_b32 s91, s84, s25\n"
    "v_mfma_i32_16x16x64_i8 v[132:135], a[168:171], v[88:91], v[132:135]\n"
    "s_cmp_gt_u32 s91, 3\n"
    "s_getpc_b64 s[88:89]\n"
    "s_cbranch_scc0 L_slowc%=\n"
    "s_waitcnt vmcnt(9) lgkmcnt(0)\n"
    "s_barrier\n"
    "v_mfma_i32_16x16x64_i8 v[132:135], a[112:115], v[92:95], v[132:135]\n"
    "ds_read_b128 v[52:55], v40\n"
    "s_waitcnt vmcnt(8)\n"
    "v_mfma_i32_16x16x64_i8 v[132:135], a[120:123], v[96:99], v[132:135]\n"
    "ds_read_b128 v[56:59], v41\n"
    "v_cmp_le_u32 vcc, s17, v32\n"
    "v_mfma_i32_16x16x64_i8 v[132:135], a[144:147], v[100:103], v[132:135]\n"
    "ds_read_b128 v[60:63], v40 offset:8192\n"
    "v_mfma_i32_16x16x64_i8 v[136:139], a[116:119], v[92:95], v[136:139]\n"
    "ds_read_b128 v[64:67], v41 offset:8192\n"
    "s_lshr_b32 s87, vcc_lo, 16\n"
    "v_mfma_i32_16x16x64_i8 v[136:139], a[124:127], v[96:99], v[136:139]\n"
    "ds_read_b128 v[68:71], v40 offset:16384\n"
    "v_mfma_i32_16x16x64_i8 v[136:139], a[148:151], v[100:103], v[136:139]\n"
    "ds_read_b128 v[72:75], v41 offset:16384\n"
    "s_and_b32 s87, s87, vcc_lo\n"
    "v_mfma_i32_16x16x64_i8 v[136:139], a[156:159], v[104:107], v[136:139]\n"
    "ds_read_b128 v[76:79], v40 offset:2048\n"
    "v_mfma_i32_16x16x64_i8 v[136:139], a[180:183], v[108:111], v[136:139]\n"
    "ds_read_b128 v[80:83], v41 offset:2048\n"
    "s_and_b32 s87, s87, vcc_hi\n"
    "v_mfma_i32_16x16x64_i8 v[136:139], a[188:191], v[112:115], v[136:139]\n"
    "s_ff0_i32_b32 s85, s87\n"
    "v_mfma_i32_16x16x64_i8 v[132:135], a[152:155], v[104:107], v[132:135]\n"
    "s_max_u32 s85, s85, s86\n"
    "v_mfma_i32_16x16x64_i8 v[132:135], a[176:179], v[108:111], v[132:135]\n"
    "s_min_u32 s91, s85, s96\n"
    "v_mfma_i32_16x16x64_i8 v[132:135], a[184:187], v[112:115], v[132:135]\n"
    "s_cmp_lt_u32 s84, s91\n"
    "s_getpc_b64 s[88:89]\n"
    "s_cbranch_scc1 L_catchc%=\n"
    "s_waitcnt lgkmcnt(5)\n"
    "v_mfma_i32_16x16x64_i8 v[140:143], a[0:3], v[52:55], v[140:143]\n"
    "global_load_dword v32, v24, s[72:73] sc0 sc1\n"
    "ds_read_b128 v[84:87], v40 offset:10240\n"
    "v_mfma_i32_16x16x64_i8 v[140:143], a[8:11], v[56:59], v[140:143]\n"
    "v_mfma_i32_16x16x64_i8 v[140:143], a[32:35], v[60:63], v[140:143]\n"
    "ds_read_b128 v[88:91], v41 offset:10240\n"
    "v_mfma_i32_16x16x64_i8 v[144:147], a[4:7], v[52:55], v[144:147]\n"
    "v_mfma_i32_16x16x64_i8 v[144:147], a[12:15], v[56:59], v[144:147]\n"
    "ds_read_b128 v[92:95], v40 offset:18432\n"
    "v_mfma_i32_16x16x64_i8 v[144:147], a[36:39], v[60:63], v[144:147]\n"
    "s_waitcnt lgkmcnt(5)\n"
    "global_load_dword v37, v26, s[72:73] sc0 sc1\n"
    "v_mfma_i32_16x16x64_i8 v[144:147], a[44:47], v[64:67], v[144:147]\n"
    "ds_read_b128 v[96:99], v41 offset:18432\n"
    "v_mfma_i32_16x16x64_i8 v[144:147], a[68:71], v[68:71], v[144:147]\n"
    "s_add_u32 s90, s64, 49152\n"
    "v_mfma_i32_16x16x64_i8 v[144:147], a[76:79], v[72:75], v[144:147]\n"
    "ds_read_b128 v[100:103], v40 offset:4096\n"
    "v_mfma_i32_16x16x64_i8 v[140:143], a[40:43], v[64:67], v[140:143]\n"
    "buffer_store_dwordx4 v[132:135], v22, s[44:47], s90 offen sc0 sc1\n"
    "v_mfma_i32_16x16x64_i8 v[140:143], a[64:67], v[68:71], v[140:143]\n"
    "ds_read_b128 v[104:107], v41 offset:4096\n"
    "v_mfma_i32_16x16x64_i8 v[140:143], a[72:75], v[72:75], v[140:143]\n"
    "s_waitcnt lgkmcnt(5)\n"
    "buffer_store_dwordx4 v[136:139], v22, s[44:47], s90 offen offset:1024 sc0 sc1\n"
    "v_mfma_i32_16x16x64_i8 v[140:143], a[16:19], v[76:79], v[140:143]\n"
    "ds_read_b128 v[108:111], v40 offset:12288\n"
    "s_cmp_eq_u32 s84, 5\n"
    "s_cselect_b32 s91, s85, 0\n"
    "v_mfma_i32_16x16x64_i8 v[140:143], a[24:27], v[80:83], v[140:143]\n"
    "s_cmp_gt_u32 s91, 5\n"
    "s_cselect_b32 s82, 40960, s83\n"
    "s_addc_u32 s84, s84, 0\n"
    "s_add_u32 m0, s11, 131072\n"
    "v_mfma_i32_16x16x64_i8 v[140:143], a[48:51], v[84:87], v[140:143]\n"
    "ds_read_b128 v[112:115], v41 offset:12288\n"
    "s_add_u32 s34, s8, s82\n"
    "v_mfma_i32_16x16x64_i8 v[144:147], a[20:23], v[76:79], v[144:147]\n"
    "buffer_load_dwordx4 v22, s[40:43], s34 offen sc0 sc1 lds\n"
    "s_cmp_lt_u32 s84, s96\n"
    "s_getpc_b64 s[88:89]\n"
    "s_cbranch_scc1 L_midc%=\n"
    "v_mfma_i32_16x16x64_i8 v[144:147], a[28:31], v[80:83], v[144:147]\n"
    "ds_read_b128 v[52:55], v40 offset:20480\n"
    "v_mfma_i32_16x16x64_i8 v[144:147], a[52:55], v[84:87], v[144:147]\n"
    "s_waitcnt lgkmcnt(5)\n"
    "v_mfma_i32_16x16x64_i8 v[144:147], a[60:63], v[88:91], v[144:147]\n"
    "ds_read_b128 v[56:59], v41 offset:20480\n"
    "v_mfma_i32_16x16x64_i8 v[144:147], a[84:87], v[92:95], v[144:147]\n"
    "buffer_load_dwordx4 v22, s[40:43], s34 offen offset:1024 sc0 sc1 lds\n"
    "s_add_u32 m0, s12, 131072\n"
    "v_mfma_i32_16x16x64_i8 v[144:147], a[92:95], v[96:99], v[144:147]\n"
    "ds_read_b128 v[60:63], v40 offset:6144\n"
    "s_add_u32 s34, s9, s82\n"
    "v_mfma_i32_16x16x64_i8 v[140:143], a[56:59], v[88:91], v[140:143]\n"
    "buffer_load_dwordx4 v22, s[40:43], s34 offen sc0 sc1 lds\n"
    "v_mfma_i32_16x16x64_i8 v[140:143], a[80:83], v[92:95], v[140:143]\n"
    "ds_read_b128 v[64:67], v41 offset:6144\n"
    "v_mfma_i32_16x16x64_i8 v[140:143], a[88:91], v[96:99], v[140:143]\n"
    "s_waitcnt lgkmcnt(5)\n"
    "v_mfma_i32_16x16x64_i8 v[140:143], a[96:99], v[100:103], v[140:143]\n"
    "ds_read_b128 v[68:71], v40 offset:14336\n"
    "v_mfma_i32_16x16x64_i8 v[140:143], a[104:107], v[104:107], v[140:143]\n"
    "buffer_load_dwordx4 v22, s[40:43], s34 offen offset:1024 sc0 sc1 lds\n"
    "v_mfma_i32_16x16x64_i8 v[140:143], a[128:131], v[108:111], v[140:143]\n"
    "ds_read_b128 v[72:75], v41 offset:14336\n"
    "v_mfma_i32_16x16x64_i8 v[144:147], a[100:103], v[100:103], v[144:147]\n"
    "s_add_u32 m0, s13, 131072\n"
    "s_add_u32 s34, s10, s82\n"
    "v_mfma_i32_16x16x64_i8 v[144:147], a[108:111], v[104:107], v[144:147]\n"
    "ds_read_b128 v[76:79], v40 offset:22528\n"
    "v_mfma_i32_16x16x64_i8 v[144:147], a[132:135], v[108:111], v[144:147]\n"
    "s_waitcnt lgkmcnt(5)\n"
    "buffer_load_dwordx4 v22, s[40:43], s34 offen sc0 sc1 lds\n"
    "v_mfma_i32_16x16x64_i8 v[144:147], a[140:143], v[112:115], v[144:147]\n"
    "ds_read_b128 v[80:83], v41 offset:22528\n"
    "v_mfma_i32_16x16x64_i8 v[144:147], a[164:167], v[52:55], v[144:147]\n"
    "buffer_load_dwordx4 v22, s[40:43], s34 offen offset:1024 sc0 sc1 lds\n"
    "s_mov_b32 s96, 6\n"
    "v_mfma_i32_16x16x64_i8 v[144:147], a[172:175], v[56:59], v[144:147]\n"
    "v_mfma_i32_16x16x64_i8 v[140:143], a[136:139], v[112:115], v[140:143]\n"
    "v_mfma_i32_16x16x64_i8 v[140:143], a[160:163], v[52:55], v[140:143]\n"
    "s_mov_b32 s95, 4\n"
    "s_and_b32 s91, s84, s25\n"
    "v_mfma_i32_16x16x64_i8 v[140:143], a[168:171], v[56:59], v[140:143]\n"
    "s_cmp_gt_u32 s91, 4\n"
    "s_getpc_b64 s[88:89]\n"
    "s_cbranch_scc0 L_slowc%=\n"
    "s_waitcnt vmcnt(10) lgkmcnt(0)\n"
    "s_barrier\n"
    "v_mfma_i32_16x16x64_i8 v[140:143], a[112:115], v[60:63], v[140:143]\n"
    "ds_read_b128 v[84:87], v40 offset:32768\n"
    "s_waitcnt vmcnt(9)\n"
    "v_mfma_i32_16x16x64_i8 v[140:143], a[120:123], v[64:67], v[140:143]\n"
    "ds_read_b128 v[88:91], v41 offset:32768\n"
    "v_cmp_le_u32 vcc, s17, v32\n"
    "v_mfma_i32_16x16x64_i8 v[140:143], a[144:147], v[68:71], v[140:143]\n"
    "ds_read_b128 v[92:95], v40 offset:40960\n"
    "s_lshr_b32 s87, vcc_lo, 16\n"
    "v_mfma_i32_16x16x64_i8 v[144:147], a[116:119], v[60:63], v[144:147]\n"
    "ds_read_b128 v[96:99], v41 offset:40960\n"
    "v_mfma_i32_16x16x64_i8 v[144:147], a[124:127], v[64:67], v[144:147]\n"
    "ds_read_b128 v[100:103], v40 offset:49152\n"
    "s_and_b32 s87, s87, vcc_lo\n"
    "v_mfma_i32_16x16x64_i8 v[144:147], a[148:151], v[68:71], v[144:147]\n"
    "ds_read_b128 v[104:107], v41 offset:49152\n"
    "s_and_b32 s87, s87, vcc_hi\n"
    "v_mfma_i32_16x16x64_i8 v[144:147], a[156:159], v[72:75], v[144:147]\n"
    "ds_read_b128 v[108:111], v40 offset:34816\n"
    "s_ff0_i32_b32 s85, s87\n"
    "v_mfma_i32_16x16x64_i8 v[144:147], a[180:183], v[76:79], v[144:147]\n"
    "ds_read_b128 v[112:115], v41 offset:34816\n"
    "v_mfma_i32_16x16x64_i8 v[144:147], a[188:191], v[80:83], v[144:147]\n"
    "s_max_u32 s85, s85, s86\n"
    "s_min_u32 s91, s85, s96\n"
    "v_mfma_i32_16x16x64_i8 v[140:143], a[152:155], v[72:75], v[140:143]\n"
    "s_cmp_lt_u32 s84, s91\n"
    "s_getpc_b64 s[88:89]\n"
    "s_cbranch_scc1 L_catchc%=\n"
    "v_mfma_i32_16x16x64_i8 v[140:143], a[176:179], v[76:79], v[140:143]\n"
    "s_waitcnt vmcnt(8)\n"
    "v_cmp_gt_u32 s[28:29], s17, v37\n"
    "v_mfma_i32_16x16x64_i8 v[140:143], a[184:187], v[80:83], v[140:143]\n"
    "s_cmp_lg_u64 s[28:29], 0\n"
    "s_cbranch_scc1 L_gatec%=\n"
    "L_rgc%=:\n"
    "s_waitcnt lgkmcnt(5)\n"
    "v_mfma_i32_16x16x64_i8 v[148:151], a[0:3], v[84:87], v[148:151]\n"
    "global_load_dword v32, v24, s[72:73] sc0 sc1\n"
    "ds_read_b128 v[52:55], v40 offset:43008\n"
    "v_mfma_i32_16x16x64_i8 v[148:151], a[8:11], v[88:91], v[148:151]\n"
    "s_add_u32 s90, s65, 0\n"
    "buffer_load_dwordx4 v[204:207], v22, s[44:47], s90 offen sc0 sc1\n"
    "v_mfma_i32_16x16x64_i8 v[148:151], a[32:35], v[92:95], v[148:151]\n"
    "ds_read_b128 v[56:59], v41 offset:43008\n"
    "v_mfma_i32_16x16x64_i8 v[152:155], a[4:7], v[84:87], v[152:155]\n"
    "v_mfma_i32_16x16x64_i8 v[152:155], a[12:15], v[88:91], v[152:155]\n"
    "ds_read_b128 v[60:63], v40 offset:51200\n"
    "v_mfma_i32_16x16x64_i8 v[152:155], a[36:39], v[92:95], v[152:155]\n"
    "s_waitcnt lgkmcnt(5)\n"
    "buffer_load_dwordx4 v[208:211], v22, s[44:47], s90 offen offset:1024 sc0 sc1\n"
    "v_mfma_i32_16x16x64_i8 v[152:155], a[44:47], v[96:99], v[152:155]\n"
    "ds_read_b128 v[64:67], v41 offset:51200\n"
    "s_add_u32 s90, s64, 73728\n"
    "v_mfma_i32_16x16x64_i8 v[152:155], a[68:71], v[100:103], v[152:155]\n"
    "buffer_store_dwordx4 v[140:143], v22, s[44:47], s90 offen sc0 sc1\n"
    "v_mfma_i32_16x16x64_i8 v[152:155], a[76:79], v[104:107], v[152:155]\n"
    "ds_read_b128 v[68:71], v40 offset:36864\n"
    "v_mfma_i32_16x16x64_i8 v[148:151], a[40:43], v[96:99], v[148:151]\n"
    "v_mfma_i32_16x16x64_i8 v[148:151], a[64:67], v[100:103], v[148:151]\n"
    "ds_read_b128 v[72:75], v41 offset:36864\n"
    "v_mfma_i32_16x16x64_i8 v[148:151], a[72:75], v[104:107], v[148:151]\n"
    "s_waitcnt lgkmcnt(5)\n"
    "buffer_store_dwordx4 v[144:147], v22, s[44:47], s90 offen offset:1024 sc0 sc1\n"
    "v_mfma_i32_16x16x64_i8 v[148:151], a[16:19], v[108:111], v[148:151]\n"
    "ds_read_b128 v[76:79], v40 offset:45056\n"
    "s_cmp_eq_u32 s84, 6\n"
    "s_cselect_b32 s91, s85, 0\n"
    "v_mfma_i32_16x16x64_i8 v[148:151], a[24:27], v[112:115], v[148:151]\n"
    "s_cmp_gt_u32 s91, 6\n"
    "s_cselect_b32 s82, 49152, s83\n"
    "s_addc_u32 s84, s84, 0\n"
    "s_add_u32 m0, s11, 0\n"
    "v_mfma_i32_16x16x64_i8 v[148:151], a[48:51], v[52:55], v[148:151]\n"
    "ds_read_b128 v[80:83], v41 offset:45056\n"
    "s_add_u32 s34, s8, s82\n"
    "v_mfma_i32_16x16x64_i8 v[152:155], a[20:23], v[108:111], v[152:155]\n"
    "buffer_load_dwordx4 v22, s[40:43], s34 offen sc0 sc1 lds\n"
    "s_cmp_lt_u32 s84, s96\n"
    "s_getpc_b64 s[88:89]\n"
    "s_cbranch_scc1 L_midc%=\n"
    "v_mfma_i32_16x16x64_i8 v[152:155], a[28:31], v[112:115], v[152:155]\n"
    "ds_read_b128 v[84:87], v40 offset:53248\n"
    "v_mfma_i32_16x16x64_i8 v[152:155], a[52:55], v[52:55], v[152:155]\n"
    "s_waitcnt lgkmcnt(5)\n"
    "v_mfma_i32_16x16x64_i8 v[152:155], a[60:63], v[56:59], v[152:155]\n"
    "ds_read_b128 v[88:91], v41 offset:53248\n"
    "v_mfma_i32_16x16x64_i8 v[152:155], a[84:87], v[60:63], v[152:155]\n"
    "buffer_load_dwordx4 v22, s[40:43], s34 offen offset:1024 sc0 sc1 lds\n"
    "s_add_u32 m0, s12, 0\n"
    "v_mfma_i32_16x16x64_i8 v[152:155], a[92:95], v[64:67], v[152:155]\n"
    "ds_read_b128 v[92:95], v40 offset:38912\n"
    "s_add_u32 s34, s9, s82\n"
    "v_mfma_i32_16x16x64_i8 v[148:151], a[56:59], v[56:59], v[148:151]\n"
    "buffer_load_dwordx4 v22, s[40:43], s34 offen sc0 sc1 lds\n"
    "v_mfma_i32_16x16x64_i8 v[148:151], a[80:83], v[60:63], v[148:151]\n"
    "ds_read_b128 v[96:99], v41 offset:38912\n"
    "v_mfma_i32_16x16x64_i8 v[148:151], a[88:91], v[64:67], v[148:151]\n"
    "s_waitcnt lgkmcnt(5)\n"
    "v_mfma_i32_16x16x64_i8 v[148:151], a[96:99], v[68:71], v[148:151]\n"
    "ds_read_b128 v[100:103], v40 offset:47104\n"
    "v_mfma_i32_16x16x64_i8 v[148:151], a[104:107], v[72:75], v[148:151]\n"
    "buffer_load_dwordx4 v22, s[40:43], s34 offen offset:1024 sc0 sc1 lds\n"
    "v_mfma_i32_16x16x64_i8 v[148:151], a[128:131], v[76:79], v[148:151]\n"
    "ds_read_b128 v[104:107], v41 offset:47104\n"
    "v_mfma_i32_16x16x64_i8 v[152:155], a[100:103], v[68:71], v[152:155]\n"
    "s_add_u32 m0, s13, 0\n"
    "s_add_u32 s34, s10, s82\n"
    "v_mfma_i32_16x16x64_i8 v[152:155], a[108:111], v[72:75], v[152:155]\n"
    "ds_read_b128 v[108:111], v40 offset:55296\n"
    "v_mfma_i32_16x16x64_i8 v[152:155], a[132:135], v[76:79], v[152:155]\n"
    "s_waitcnt lgkmcnt(5)\n"
    "buffer_load_dwordx4 v22, s[40:43], s34 offen sc0 sc1 lds\n"
    "v_mfma_i32_16x16x64_i8 v[152:155], a[140:143], v[80:83], v[152:155]\n"
    "ds_read_b128 v[112:115], v41 offset:55296\n"
    "v_mfma_i32_16x16x64_i8 v[152:155], a[164:167], v[84:87], v[152:155]\n"
    "buffer_load_dwordx4 v22, s[40:43], s34 offen offset:1024 sc0 sc1 lds\n"
    "s_mov_b32 s96, 7\n"
    "v_mfma_i32_16x16x64_i8 v[152:155], a[172:175], v[88:91], v[152:155]\n"
    "v_mfma_i32_16x16x64_i8 v[148:151], a[136:139], v[80:83], v[148:151]\n"
    "v_mfma_i32_16x16x64_i8 v[148:151], a[160:163], v[84:87], v[148:151]\n"
    "s_mov_b32 s95, 5\n"
    "s_and_b32 s91, s84, s25\n"
    "v_mfma_i32_16x16x64_i8 v[148:151], a[168:171], v[88:91], v[148:151]\n"
    "s_cmp_gt_u32 s91, 5\n"
    "s_getpc_b64 s[88:89]\n"
    "s_cbranch_scc0 L_slowc%=\n"
    "s_waitcnt vmcnt(11) lgkmcnt(0)\n"
    "s_barrier\n"
    "v_mfma_i32_16x16x64_i8 v[148:151], a[112:115], v[92:95], v[148:151]\n"
    "ds_read_b128 v[52:55], v42\n"
    "s_waitcnt vmcnt(10)\n"
    "v_mfma_i32_16x16x64_i8 v[148:151], a[120:123], v[96:99], v[148:151]\n"
    "ds_read_b128 v[56:59], v43\n"
    "v_cmp_le_u32 vcc, s17, v32\n"
    "v_mfma_i32_16x16x64_i8 v[148:151], a[144:147], v[100:103], v[148:151]\n"
    "ds_read_b128 v[60:63], v42 offset:8192\n"
    "v_mfma_i32_16x16x64_i8 v[152:155], a[116:119], v[92:95], v[152:155]\n"
    "ds_read_b128 v[64:67], v43 offset:8192\n"
    "s_lshr_b32 s87, vcc_lo, 16\n"
    "v_mfma_i32_16x16x64_i8 v[152:155], a[124:127], v[96:99], v[152:155]\n"
    "ds_read_b128 v[68:71], v42 offset:16384\n"
    "v_mfma_i32_16x16x64_i8 v[152:155], a[148:151], v[100:103], v[152:155]\n"
    "ds_read_b128 v[72:75], v43 offset:16384\n"
    "s_and_b32 s87, s87, vcc_lo\n"
    "v_mfma_i32_16x16x64_i8 v[152:155], a[156:159], v[104:107], v[152:155]\n"
    "ds_read_b128 v[76:79], v42 offset:2048\n"
    "v_mfma_i32_16x16x64_i8 v[152:155], a[180:183], v[108:111], v[152:155]\n"
    "ds_read_b128 v[80:83], v43 offset:2048\n"
    "s_and_b32 s87, s87, vcc_hi\n"
    "v_mfma_i32_16x16x64_i8 v[152:155], a[188:191], v[112:115], v[152:155]\n"
    "s_ff0_i32_b32 s85, s87\n"
    "v_mfma_i32_16x16x64_i8 v[148:151], a[152:155], v[104:107], v[148:151]\n"
    "s_max_u32 s85, s85, s86\n"
    "v_mfma_i32_16x16x64_i8 v[148:151], a[176:179], v[108:111], v[148:151]\n"
    "s_min_u32 s91, s85, s96\n"
    "v_mfma_i32_16x16x64_i8 v[148:151], a[184:187], v[112:115], v[148:151]\n"
    "s_cmp_lt_u32 s84, s91\n"
    "s_getpc_b64 s[88:89]\n"
    "s_cbranch_scc1 L_catchc%=\n"
    "s_waitcnt lgkmcnt(5)\n"
    "v_mfma_i32_16x16x64_i8 v[156:159], a[0:3], v[52:55], v[156:159]\n"
    "global_load_dword v32, v24, s[72:73] sc0 sc1\n"
    "ds_read_b128 v[84:87], v42 offset:10240\n"
    "v_mfma_i32_16x16x64_i8 v[156:159], a[8:11], v[56:59], v[156:159]\n"
    "s_add_u32 s90, s65, 8192\n"
    "buffer_load_dwordx4 v[212:215], v22, s[44:47], s90 offen sc0 sc1\n"
    "v_mfma_i32_16x16x64_i8 v[156:159], a[32:35], v[60:63], v[156:159]\n"
    "ds_read_b128 v[88:91], v43 offset:10240\n"
    "v_mfma_i32_16x16x64_i8 v[160:163], a[4:7], v[52:55], v[160:163]\n"
    "v_mfma_i32_16x16x64_i8 v[160:163], a[12:15], v[56:59], v[160:163]\n"
    "ds_read_b128 v[92:95], v42 offset:18432\n"
    "v_mfma_i32_16x16x64_i8 v[160:163], a[36:39], v[60:63], v[160:163]\n"
    "s_waitcnt lgkmcnt(5)\n"
    "buffer_load_dwordx4 v[216:219], v22, s[44:47], s90 offen offset:1024 sc0 sc1\n"
    "s_waitcnt vmcnt(11)\n"
    "v_mfma_i32_16x16x64_i8 v[160:163], a[44:47], v[64:67], v[160:163]\n"
    "ds_read_b128 v[96:99], v43 offset:18432\n"
    "s_cmp_eq_u32 s84, 7\n"
    "s_cselect_b32 s91, s85, 0\n"
    "v_mfma_i32_16x16x64_i8 v[160:163], a[68:71], v[68:71], v[160:163]\n"
    "s_cmp_gt_u32 s91, 7\n"
    "s_cselect_b32 s82, 57344, s83\n"
    "s_addc_u32 s84, s84, 0\n"
    "s_add_u32 m0, s11, 32768\n"
    "v_mfma_i32_16x16x64_i8 v[160:163], a[76:79], v[72:75], v[160:163]\n"
    "ds_read_b128 v[100:103], v42 offset:4096\n"
    "s_add_u32 s34, s8, s82\n"
    "v_add_u32 v204, v148, v204\n"
    "v_mfma_i32_16x16x64_i8 v[156:159], a[40:43], v[64:67], v[156:159]\n"
    "buffer_load_dwordx4 v22, s[40:43], s34 offen sc0 sc1 lds\n"
    "v_add_u32 v205, v149, v205\n"
    "v_mfma_i32_16x16x64_i8 v[156:159], a[64:67], v[68:71], v[156:159]\n"
    "ds_read_b128 v[104:107], v43 offset:4096\n"
    "v_add_u32 v206, v150, v206\n"
    "v_mfma_i32_16x16x64_i8 v[156:159], a[72:75], v[72:75], v[156:159]\n"
    "s_waitcnt lgkmcnt(5)\n"
    "v_add_u32 v207, v151, v207\n"
    "v_mfma_i32_16x16x64_i8 v[156:159], a[16:19], v[76:79], v[156:159]\n"
    "ds_read_b128 v[108:111], v42 offset:12288\n"
    "v_add_u32 v208, v152, v208\n"
    "v_mfma_i32_16x16x64_i8 v[156:159], a[24:27], v[80:83], v[156:159]\n"
    "buffer_load_dwordx4 v22, s[40:43], s34 offen offset:1024 sc0 sc1 lds\n"
    "v_add_u32 v209, v153, v209\n"
    "v_add_u32 v210, v154, v210\n"
    "v_mfma_i32_16x16x64_i8 v[156:159], a[48:51], v[84:87], v[156:159]\n"
    "ds_read_b128 v[112:115], v43 offset:12288\n"
    "v_add_u32 v211, v155, v211\n"
    "v_mfma_i32_16x16x64_i8 v[160:163], a[20:23], v[76:79], v[160:163]\n"
    "s_cmp_lt_u32 s84, s96\n"
    "s_getpc_b64 s[88:89]\n"
    "s_cbranch_scc1 L_midc%=\n"
    "v_mfma_i32_16x16x64_i8 v[160:163], a[28:31], v[80:83], v[160:163]\n"
    "ds_read_b128 v[52:55], v42 offset:20480\n"
    "v_cvt_f32_i32 v204, v204\n"
    "v_mfma_i32_16x16x64_i8 v[160:163], a[52:55], v[84:87], v[160:163]\n"
    "s_waitcnt lgkmcnt(5)\n"
    "s_add_u32 m0, s12, 32768\n"
    "v_cvt_f32_i32 v205, v205\n"
    "v_mfma_i32_16x16x64_i8 v[160:163], a[60:63], v[88:91], v[160:163]\n"
    "ds_read_b128 v[56:59], v43 offset:20480\n"
    "s_add_u32 s34, s9, s82\n"
    "v_mfma_i32_16x16x64_i8 v[160:163], a[84:87], v[92:95], v[160:163]\n"
    "buffer_load_dwordx4 v22, s[40:43], s34 offen sc0 sc1 lds\n"
    "v_cvt_f32_i32 v206, v206\n"
    "v_mfma_i32_16x16x64_i8 v[160:163], a[92:95], v[96:99], v[160:163]\n"
    "ds_read_b128 v[60:63], v42 offset:6144\n"
    "v_cvt_f32_i32 v207, v207\n"
    "v_mfma_i32_16x16x64_i8 v[156:159], a[56:59], v[88:91], v[156:159]\n"
    "v_cvt_f32_i32 v208, v208\n"
    "v_cvt_f32_i32 v209, v209\n"
    "v_mfma_i32_16x16x64_i8 v[156:159], a[80:83], v[92:95], v[156:159]\n"
    "ds_read_b128 v[64:67], v43 offset:6144\n"
    "v_cvt_f32_i32 v210, v210\n"
    "v_mfma_i32_16x16x64_i8 v[156:159], a[88:91], v[96:99], v[156:159]\n"
    "s_waitcnt lgkmcnt(5)\n"
    "buffer_load_dwordx4 v22, s[40:43], s34 offen offset:1024 sc0 sc1 lds\n"
    "v_cvt_f32_i32 v211, v211\n"
    "v_mfma_i32_16x16x64_i8 v[156:159], a[96:99], v[100:103], v[156:159]\n"
    "ds_read_b128 v[68:71], v42 offset:14336\n"
    "v_max3_f32 v34, |v204|, |v205|, |v206|\n"
    "v_mfma_i32_16x16x64_i8 v[156:159], a[104:107], v[104:107], v[156:159]\n"
    "s_add_u32 m0, s13, 32768\n"
    "s_add_u32 s34, s10, s82\n"
    "v_max3_f32 v34, v34, |v207|, |v208|\n"
    "v_mfma_i32_16x16x64_i8 v[156:159], a[128:131], v[108:111], v[156:159]\n"
    "ds_read_b128 v[72:75], v43 offset:14336\n"
    "v_max3_f32 v34, v34, |v209|, |v210|\n"
    "v_mfma_i32_16x16x64_i8 v[160:163], a[100:103], v[100:103], v[160:163]\n"
    "buffer_load_dwordx4 v22, s[40:43], s34 offen sc0 sc1 lds\n"
    "v_max_f32 v34, v34, |v211|\n"
    "v_mfma_i32_16x16x64_i8 v[160:163], a[108:111], v[104:107], v[160:163]\n"
    "ds_read_b128 v[76:79], v42 offset:22528\n"
    "v_cmp_lt_f32 s[28:29], s97, v34\n"
    "v_mfma_i32_16x16x64_i8 v[160:163], a[132:135], v[108:111], v[160:163]\n"
    "s_waitcnt lgkmcnt(5)\n"
    "s_cmp_lg_u64 s[28:29], 0\n"
    "s_cselect_b32 s24, 16, 0\n"
    "s_or_b32 s81, s81, s24\n"
    "v_fma_f32 v204, v204, s18, v33\n"
    "v_mfma_i32_16x16x64_i8 v[160:163], a[140:143], v[112:115], v[160:163]\n"
    "ds_read_b128 v[80:83], v43 offset:22528\n"
    "v_fma_f32 v205, v205, s18, v33\n"
    "v_mfma_i32_16x16x64_i8 v[160:163], a[164:167], v[52:55], v[160:163]\n"
    "buffer_load_dwordx4 v22, s[40:43], s34 offen offset:1024 sc0 sc1 lds\n"
    "s_mov_b32 s96, 8\n"
    "v_mfma_i32_16x16x64_i8 v[160:163], a[172:175], v[56:59], v[160:163]\n"
    "v_fma_f32 v206, v206, s18, v33\n"
    "v_fma_f32 v207, v207, s18, v33\n"
    "v_mfma_i32_16x16x64_i8 v[156:159], a[136:139], v[112:115], v[156:159]\n"
    "v_fma_f32 v208, v208, s18, v33\n"
    "v_fma_f32 v209, v209, s18, v33\n"
    "v_mfma_i32_16x16x64_i8 v[156:159], a[160:163], v[52:55], v[156:159]\n"
    "s_mov_b32 s95, 6\n"
    "s_and_b32 s91, s84, s25\n"
    "v_mfma_i32_16x16x64_i8 v[156:159], a[168:171], v[56:59], v[156:159]\n"
    "s_cmp_gt_u32 s91, 6\n"
    "s_getpc_b64 s[88:89]\n"
    "s_cbranch_scc0 L_slowc%=\n"
    "s_waitcnt vmcnt(9) lgkmcnt(0)\n"
    "s_barrier\n"
    "v_mfma_i32_16x16x64_i8 v[156:159], a[112:115], v[60:63], v[156:159]\n"
    "ds_read_b128 v[84:87], v38\n"
    "s_waitcnt vmcnt(8)\n"
    "v_fma_f32 v210, v210, s18, v33\n"
    "v_mfma_i32_16x16x64_i8 v[156:159], a[120:123], v[64:67], v[156:159]\n"
    "ds_read_b128 v[88:91], v39\n"
    "v_cmp_le_u32 vcc, s17, v32\n"
    "v_fma_f32 v211, v211, s18, v33\n"
    "v_mfma_i32_16x16x64_i8 v[156:159], a[144:147], v[68:71], v[156:159]\n"
    "ds_read_b128 v[92:95], v38 offset:8192\n"
    "v_max_f32 v204, v204, v33\n"
    "v_mfma_i32_16x16x64_i8 v[160:163], a[116:119], v[60:63], v[160:163]\n"
    "ds_read_b128 v[96:99], v39 offset:8192\n"
    "s_lshr_b32 s87, vcc_lo, 16\n"
    "v_mfma_i32_16x16x64_i8 v[160:163], a[124:127], v[64:67], v[160:163]\n"
    "ds_read_b128 v[100:103], v38 offset:16384\n"
    "v_max_f32 v205, v205, v33\n"
    "v_max_f32 v206, v206, v33\n"
    "v_mfma_i32_16x16x64_i8 v[160:163], a[148:151], v[68:71], v[160:163]\n"
    "ds_read_b128 v[104:107], v39 offset:16384\n"
    "s_and_b32 s87, s87, vcc_lo\n"
    "v_mfma_i32_16x16x64_i8 v[160:163], a[156:159], v[72:75], v[160:163]\n"
    "ds_read_b128 v[108:111], v38 offset:2048\n"
    "v_max_f32 v207, v207, v33\n"
    "v_mfma_i32_16x16x64_i8 v[160:163], a[180:183], v[76:79], v[160:163]\n"
    "ds_read_b128 v[112:115], v39 offset:2048\n"
    "s_and_b32 s87, s87, vcc_hi\n"
    "v_max_f32 v208, v208, v33\n"
    "v_mfma_i32_16x16x64_i8 v[160:163], a[188:191], v[80:83], v[160:163]\n"
    "s_ff0_i32_b32 s85, s87\n"
    "v_max_f32 v209, v209, v33\n"
    "v_mfma_i32_16x16x64_i8 v[156:159], a[152:155], v[72:75], v[156:159]\n"
    "s_max_u32 s85, s85, s86\n"
    "v_max_f32 v210, v210, v33\n"
    "v_mfma_i32_16x16x64_i8 v[156:159], a[176:179], v[76:79], v[156:159]\n"
    "s_min_u32 s91, s85, s96\n"
    "v_max_f32 v211, v211, v33\n"
    "v_mfma_i32_16x16x64_i8 v[156:159], a[184:187], v[80:83], v[156:159]\n"
    "s_cmp_lt_u32 s84, s91\n"
    "s_getpc_b64 s[88:89]\n"
    "s_cbranch_scc1 L_catchc%=\n"
    "s_waitcnt lgkmcnt(5)\n"
    "v_mfma_i32_16x16x64_i8 v[164:167], a[0:3], v[84:87], v[164:167]\n"
    "global_load_dword v32, v24, s[72:73] sc0 sc1\n"
    "ds_read_b128 v[52:55], v38 offset:10240\n"
    "v_mfma_i32_16x16x64_i8 v[164:167], a[8:11], v[88:91], v[164:167]\n"
    "s_add_u32 s90, s65, 16384\n"
    "buffer_load_dwordx4 v[220:223], v22, s[44:47], s90 offen sc0 sc1\n"
    "v_mfma_i32_16x16x64_i8 v[164:167], a[32:35], v[92:95], v[164:167]\n"
    "ds_read_b128 v[56:59], v39 offset:10240\n"
    "v_mfma_i32_16x16x64_i8 v[168:171], a[4:7], v[84:87], v[168:171]\n"
    "v_mfma_i32_16x16x64_i8 v[168:171], a[12:15], v[88:91], v[168:171]\n"
    "ds_read_b128 v[60:63], v38 offset:18432\n"
    "v_mfma_i32_16x16x64_i8 v[168:171], a[36:39], v[92:95], v[168:171]\n"
    "s_waitcnt lgkmcnt(5)\n"
    "buffer_load_dwordx4 v[224:227], v22, s[44:47], s90 offen offset:1024 sc0 sc1\n"
    "s_add_u32 s90, s65, 0\n"
    "v_mfma_i32_16x16x64_i8 v[168:171], a[44:47], v[96:99], v[168:171]\n"
    "ds_read_b128 v[64:67], v39 offset:18432\n"
    "s_lshl_b32 s90, s90, 5\n"
    "s_waitcnt vmcnt(9)\n"
    "v_mfma_i32_16x16x64_i8 v[168:171], a[68:71], v[100:103], v[168:171]\n"
    "buffer_store_dwordx4 v[204:207], v23, s[48:51], s90 offen sc0 sc1\n"
    "v_add_u32 v212, v156, v212\n"
    "v_mfma_i32_16x16x64_i8 v[168:171], a[76:79], v[104:107], v[168:171]\n"
    "ds_read_b128 v[68:71], v38 offset:4096\n"
    "v_add_u32 v213, v157, v213\n"
    "v_add_u32 v214, v158, v214\n"
    "v_mfma_i32_16x16x64_i8 v[164:167], a[40:43], v[96:99], v[164:167]\n"
    "v_add_u32 v215, v159, v215\n"
    "v_mfma_i32_16x16x64_i8 v[164:167], a[64:67], v[100:103], v[164:167]\n"
    "ds_read_b128 v[72:75], v39 offset:4096\n"
    "v_add_u32 v216, v160, v216\n"
    "v_add_u32 v217, v161, v217\n"
    "v_mfma_i32_16x16x64_i8 v[164:167], a[72:75], v[104:107], v[164:167]\n"
    "s_waitcnt lgkmcnt(5)\n"
    "buffer_store_dwordx4 v[208:211], v23, s[48:51], s90 offen offset:64 sc0 sc1\n"
    "v_mfma_i32_16x16x64_i8 v[164:167], a[16:19], v[108:111], v[164:167]\n"
    "ds_read_b128 v[76:79], v38 offset:12288\n"
    "s_cmp_eq_u32 s84, 8\n"
    "s_cselect_b32 s91, s85, 0\n"
    "v_mfma_i32_16x16x64_i8 v[164:167], a[24:27], v[112:115], v[164:167]\n"
    "s_cmp_gt_u32 s91, 8\n"
    "s_cselect_b32 s82, 65536, s83\n"
    "s_addc_u32 s84, s84, 0\n"
    "s_add_u32 m0, s11, 65536\n"
    "v_mfma_i32_16x16x64_i8 v[164:167], a[48:51], v[52:55], v[164:167]\n"
    "ds_read_b128 v[80:83], v39 offset:12288\n"
    "s_add_u32 s34, s8, s82\n"
    "v_add_u32 v218, v162, v218\n"
    "v_mfma_i32_16x16x64_i8 v[168:171], a[20:23], v[108:111], v[168:171]\n"
    "buffer_load_dwordx4 v22, s[40:43], s34 offen sc0 sc1 lds\n"
    "s_cmp_lt_u32 s84, s96\n"
    "s_getpc_b64 s[88:89]\n"
    "s_cbranch_scc1 L_midc%=\n"
    "v_mfma_i32_16x16x64_i8 v[168:171], a[28:31], v[112:115], v[168:171]\n"
    "ds_read_b128 v[84:87], v38 offset:20480\n"
    "v_add_u32 v219, v163, v219\n"
    "v_mfma_i32_16x16x64_i8 v[168:171], a[52:55], v[52:55], v[168:171]\n"
    "s_waitcnt lgkmcnt(5)\n"
    "v_cvt_f32_i32 v212, v212\n"
    "v_mfma_i32_16x16x64_i8 v[168:171], a[60:63], v[56:59], v[168:171]\n"
    "ds_read_b128 v[88:91], v39 offset:20480\n"
    "v_cvt_f32_i32 v213, v213\n"
    "v_cvt_f32_i32 v214, v214\n"
    "v_mfma_i32_16x16x64_i8 v[168:171], a[84:87], v[60:63], v[168:171]\n"
    "buffer_load_dwordx4 v22, s[40:43], s34 offen offset:1024 sc0 sc1 lds\n"
    "s_add_u32 m0, s12, 65536\n"
    "v_mfma_i32_16x16x64_i8 v[168:171], a[92:95], v[64:67], v[168:171]\n"
    "ds_read_b128 v[92:95], v38 offset:6144\n"
    "s_add_u32 s34, s9, s82\n"
    "v_cvt_f32_i32 v215, v215\n"
    "v_mfma_i32_16x16x64_i8 v[164:167], a[56:59], v[56:59], v[164:167]\n"
    "buffer_load_dwordx4 v22, s[40:43], s34 offen sc0 sc1 lds\n"
    "v_cvt_f32_i32 v216, v216\n"
    "v_mfma_i32_16x16x64_i8 v[164:167], a[80:83], v[60:63], v[164:167]\n"
    "ds_read_b128 v[96:99], v39 offset:6144\n"
    "v_cvt_f32_i32 v217, v217\n"
    "v_mfma_i32_16x16x64_i8 v[164:167], a[88:91], v[64:67], v[164:167]\n"
    "s_waitcnt lgkmcnt(5)\n"
    "v_cvt_f32_i32 v218, v218\n"
    "v_cvt_f32_i32 v219, v219\n"
    "v_mfma_i32_16x16x64_i8 v[164:167], a[96:99], v[68:71], v[164:167]\n"
    "ds_read_b128 v[100:103], v38 offset:14336\n"
    "v_max3_f32 v34, |v212|, |v213|, |v214|\n"
    "v_mfma_i32_16x16x64_i8 v[164:167], a[104:107], v[72:75], v[164:167]\n"
    "buffer_load_dwordx4 v22, s[40:43], s34 offen offset:1024 sc0 sc1 lds\n"
    "v_max3_f32 v34, v34, |v215|, |v216|\n"
    "v_mfma_i32_16x16x64_i8 v[164:167], a[128:131], v[76:79], v[164:167]\n"
    "ds_read_b128 v[104:107], v39 offset:14336\n"
    "v_max3_f32 v34, v34, |v217|, |v218|\n"
    "v_mfma_i32_16x16x64_i8 v[168:171], a[100:103], v[68:71], v[168:171]\n"
    "s_add_u32 m0, s13, 65536\n"
    "s_add_u32 s34, s10, s82\n"
    "v_max_f32 v34, v34, |v219|\n"
    "v_mfma_i32_16x16x64_i8 v[168:171], a[108:111], v[72:75], v[168:171]\n"
    "ds_read_b128 v[108:111], v38 offset:22528\n"
    "v_cmp_lt_f32 s[28:29], s97, v34\n"
    "v_mfma_i32_16x16x64_i8 v[168:171], a[132:135], v[76:79], v[168:171]\n"
    "s_waitcnt lgkmcnt(5)\n"
    "buffer_load_dwordx4 v22, s[40:43], s34 offen sc0 sc1 lds\n"
    "s_cmp_lg_u64 s[28:29], 0\n"
    "s_cselect_b32 s24, 32, 0\n"
    "s_or_b32 s81, s81, s24\n"
    "v_mfma_i32_16x16x64_i8 v[168:171], a[140:143], v[80:83], v[168:171]\n"
    "ds_read_b128 v[112:115], v39 offset:22528\n"
    "v_fma_f32 v212, v212, s18, v33\n"
    "v_mfma_i32_16x16x64_i8 v[168:171], a[164:167], v[84:87], v[168:171]\n"
    "buffer_load_dwordx4 v22, s[40:43], s34 offen offset:1024 sc0 sc1 lds\n"
    "s_mov_b32 s96, 9\n"
    "v_fma_f32 v213, v213, s18, v33\n"
    "v_mfma_i32_16x16x64_i8 v[168:171], a[172:175], v[88:91], v[168:171]\n"
    "v_fma_f32 v214, v214, s18, v33\n"
    "v_mfma_i32_16x16x64_i8 v[164:167], a[136:139], v[80:83], v[164:167]\n"
    "v_fma_f32 v215, v215, s18, v33\n"
    "v_fma_f32 v216, v216, s18, v33\n"
    "v_mfma_i32_16x16x64_i8 v[164:167], a[160:163], v[84:87], v[164:167]\n"
    "s_mov_b32 s95, 7\n"
    "s_and_b32 s91, s84, s25\n"
    "v_fma_f32 v217, v217, s18, v33\n"
    "v_mfma_i32_16x16x64_i8 v[164:167], a[168:171], v[88:91], v[164:167]\n"
    "s_cmp_gt_u32 s91, 7\n"
    "s_getpc_b64 s[88:89]\n"
    "s_cbranch_scc0 L_slowc%=\n"
    "s_waitcnt vmcnt(11) lgkmcnt(0)\n"
    "s_barrier\n"
    "v_mfma_i32_16x16x64_i8 v[164:167], a[112:115], v[92:95], v[164:167]\n"
    "ds_read_b128 v[52:55], v38 offset:32768\n"
    "s_waitcnt vmcnt(10)\n"
    "v_fma_f32 v218, v218, s18, v33\n"
    "v_mfma_i32_16x16x64_i8 v[164:167], a[120:123], v[96:99], v[164:167]\n"
    "ds_read_b128 v[56:59], v39 offset:32768\n"
    "v_cmp_le_u32 vcc, s17, v32\n"
    "v_mfma_i32_16x16x64_i8 v[164:167], a[144:147], v[100:103], v[164:167]\n"
    "ds_read_b128 v[60:63], v38 offset:40960\n"
    "v_fma_f32 v219, v219, s18, v33\n"
    "v_mfma_i32_16x16x64_i8 v[168:171], a[116:119], v[92:95], v[168:171]\n"
    "ds_read_b128 v[64:67], v39 offset:40960\n"
    "s_lshr_b32 s87, vcc_lo, 16\n"
    "v_max_f32 v212, v212, v33\n"
    "v_mfma_i32_16x16x64_i8 v[168:171], a[124:127], v[96:99], v[168:171]\n"
    "ds_read_b128 v[68:71], v38 offset:49152\n"
    "v_max_f32 v213, v213, v33\n"
    "v_mfma_i32_16x16x64_i8 v[168:171], a[148:151], v[100:103], v[168:171]\n"
    "ds_read_b128 v[72:75], v39 offset:49152\n"
    "s_and_b32 s87, s87, vcc_lo\n"
    "v_max_f32 v214, v214, v33\n"
    "v_mfma_i32_16x16x64_i8 v[168:171], a[156:159], v[104:107], v[168:171]\n"
    "ds_read_b128 v[76:79], v38 offset:34816\n"
    "v_max_f32 v215, v215, v33\n"
    "v_mfma_i32_16x16x64_i8 v[168:171], a[180:183], v[108:111], v[168:171]\n"
    "ds_read_b128 v[80:83], v39 offset:34816\n"
    "s_and_b32 s87, s87, vcc_hi\n"
    "v_max_f32 v216, v216, v33\n"
    "v_mfma_i32_16x16x64_i8 v[168:171], a[188:191], v[112:115], v[168:171]\n"
    "s_ff0_i32_b32 s85, s87\n"
    "v_max_f32 v217, v217, v33\n"
    "v_mfma_i32_16x16x64_i8 v[164:167], a[152:155], v[104:107], v[164:167]\n"
    "s_max_u32 s85, s85, s86\n"
    "v_max_f32 v218, v218, v33\n"
    "v_mfma_i32_16x16x64_i8 v[164:167], a[176:179], v[108:111], v[164:167]\n"
    "s_min_u32 s91, s85, s96\n"
    "v_max_f32 v219, v219, v33\n"
    "v_mfma_i32_16x16x64_i8 v[164:167], a[184:187], v[112:115], v[164:167]\n"
    "s_cmp_lt_u32 s84, s91\n"
    "s_getpc_b64 s[88:89]\n"
    "s_cbranch_scc1 L_catchc%=\n"
    "s_waitcnt lgkmcnt(5)\n"
    "v_mfma_i32_16x16x64_i8 v[172:175], a[0:3], v[52:55], v[172:175]\n"
    "global_load_dword v32, v24, s[72:73] sc0 sc1\n"
    "ds_read_b128 v[84:87], v38 offset:43008\n"
    "v_mfma_i32_16x16x64_i8 v[172:175], a[8:11], v[56:59], v[172:175]\n"
    "s_add_u32 s90, s65, 40960\n"
    "buffer_load_dwordx4 v[228:231], v22, s[44:47], s90 offen sc0 sc1\n"
    "v_mfma_i32_16x16x64_i8 v[172:175], a[32:35], v[60:63], v[172:175]\n"
    "ds_read_b128 v[88:91], v39 offset:43008\n"
    "v_mfma_i32_16x16x64_i8 v[176:179], a[4:7], v[52:55], v[176:179]\n"
    "v_mfma_i32_16x16x64_i8 v[176:179], a[12:15], v[56:59], v[176:179]\n"
    "ds_read_b128 v[92:95], v38 offset:51200\n"
    "v_mfma_i32_16x16x64_i8 v[176:179], a[36:39], v[60:63], v[176:179]\n"
    "s_waitcnt lgkmcnt(5)\n"
    "buffer_load_dwordx4 v[232:235], v22, s[44:47], s90 offen offset:1024 sc0 sc1\n"
    "s_add_u32 s90, s65, 8192\n"
    "v_mfma_i32_16x16x64_i8 v[176:179], a[44:47], v[64:67], v[176:179]\n"
    "ds_read_b128 v[96:99], v39 offset:51200\n"
    "s_lshl_b32 s90, s90, 5\n"
    "s_waitcnt vmcnt(11)\n"
    "v_mfma_i32_16x16x64_i8 v[176:179], a[68:71], v[68:71], v[176:179]\n"
    "buffer_store_dwordx4 v[212:215], v23, s[48:51], s90 offen sc0 sc1\n"
    "v_add_u32 v220, v164, v220\n"
    "v_mfma_i32_16x16x64_i8 v[176:179], a[76:79], v[72:75], v[176:179]\n"
    "ds_read_b128 v[100:103], v38 offset:36864\n"
    "v_add_u32 v221, v165, v221\n"
    "v_add_u32 v222, v166, v222\n"
    "v_mfma_i32_16x16x64_i8 v[172:175], a[40:43], v[64:67], v[172:175]\n"
    "v_add_u32 v223, v167, v223\n"
    "v_mfma_i32_16x16x64_i8 v[172:175], a[64:67], v[68:71], v[172:175]\n"
    "ds_read_b128 v[104:107], v39 offset:36864\n"
    "v_add_u32 v224, v168, v224\n"
    "v_add_u32 v225, v169, v225\n"
    "v_mfma_i32_16x16x64_i8 v[172:175], a[72:75], v[72:75], v[172:175]\n"
    "s_waitcnt lgkmcnt(5)\n"
    "buffer_store_dwordx4 v[216:219], v23, s[48:51], s90 offen offset:64 sc0 sc1\n"
    "v_mfma_i32_16x16x64_i8 v[172:175], a[16:19], v[76:79], v[172:175]\n"
    "ds_read_b128 v[108:111], v38 offset:45056\n"
    "s_cmp_eq_u32 s84, 9\n"
    "s_cselect_b32 s91, s85, 0\n"
    "v_mfma_i32_16x16x64_i8 v[172:175], a[24:27], v[80:83], v[172:175]\n"
    "s_cmp_gt_u32 s91, 9\n"
    "s_cselect_b32 s82, 73728, s83\n"
    "s_addc_u32 s84, s84, 0\n"
    "s_add_u32 m0, s11, 98304\n"
    "v_mfma_i32_16x16x64_i8 v[172:175], a[48:51], v[84:87], v[172:175]\n"
    "ds_read_b128 v[112:115], v39 offset:45056\n"
    "s_add_u32 s34, s8, s82\n"
    "v_add_u32 v226, v170, v226\n"
    "v_mfma_i32_16x16x64_i8 v[176:179], a[20:23], v[76:79], v[176:179]\n"
    "buffer_load_dwordx4 v22, s[40:43], s34 offen sc0 sc1 lds\n"
    "s_cmp_lt_u32 s84, s96\n"
    "s_getpc_b64 s[88:89]\n"
    "s_cbranch_scc1 L_midc%=\n"
    "v_mfma_i32_16x16x64_i8 v[176:179], a[28:31], v[80:83], v[176:179]\n"
    "ds_read_b128 v[52:55], v38 offset:53248\n"
    "v_add_u32 v227, v171, v227\n"
    "v_mfma_i32_16x16x64_i8 v[176:179], a[52:55], v[84:87], v[176:179]\n"
    "s_waitcnt lgkmcnt(5)\n"
    "v_cvt_f32_i32 v220, v220\n"
    "v_mfma_i32_16x16x64_i8 v[176:179], a[60:63], v[88:91], v[176:179]\n"
    "ds_read_b128 v[56:59], v39 offset:53248\n"
    "v_cvt_f32_i32 v221, v221\n"
    "v_cvt_f32_i32 v222, v222\n"
    "v_mfma_i32_16x16x64_i8 v[176:179], a[84:87], v[92:95], v[176:179]\n"
    "buffer_load_dwordx4 v22, s[40:43], s34 offen offset:1024 sc0 sc1 lds\n"
    "s_add_u32 m0, s12, 98304\n"
    "v_mfma_i32_16x16x64_i8 v[176:179], a[92:95], v[96:99], v[176:179]\n"
    "ds_read_b128 v[60:63], v38 offset:38912\n"
    "s_add_u32 s34, s9, s82\n"
    "v_cvt_f32_i32 v223, v223\n"
    "v_mfma_i32_16x16x64_i8 v[172:175], a[56:59], v[88:91], v[172:175]\n"
    "buffer_load_dwordx4 v22, s[40:43], s34 offen sc0 sc1 lds\n"
    "v_cvt_f32_i32 v224, v224\n"
    "v_mfma_i32_16x16x64_i8 v[172:175], a[80:83], v[92:95], v[172:175]\n"
    "ds_read_b128 v[64:67], v39 offset:38912\n"
    "v_cvt_f32_i32 v225, v225\n"
    "v_mfma_i32_16x16x64_i8 v[172:175], a[88:91], v[96:99], v[172:175]\n"
    "s_waitcnt lgkmcnt(5)\n"
    "v_cvt_f32_i32 v226, v226\n"
    "v_cvt_f32_i32 v227, v227\n"
    "v_mfma_i32_16x16x64_i8 v[172:175], a[96:99], v[100:103], v[172:175]\n"
    "ds_read_b128 v[68:71], v38 offset:47104\n"
    "v_max3_f32 v34, |v220|, |v221|, |v222|\n"
    "v_mfma_i32_16x16x64_i8 v[172:175], a[104:107], v[104:107], v[172:175]\n"
    "buffer_load_dwordx4 v22, s[40:43], s34 offen offset:1024 sc0 sc1 lds\n"
    "v_max3_f32 v34, v34, |v223|, |v224|\n"
    "v_mfma_i32_16x16x64_i8 v[172:175], a[128:131], v[108:111], v[172:175]\n"
    "ds_read_b128 v[72:75], v39 offset:47104\n"
    "v_max3_f32 v34, v34, |v225|, |v226|\n"
    "v_mfma_i32_16x16x64_i8 v[176:179], a[100:103], v[100:103], v[176:179]\n"
    "s_add_u32 m0, s13, 98304\n"
    "s_add_u32 s34, s10, s82\n"
    "v_max_f32 v34, v34, |v227|\n"
    "v_mfma_i32_16x16x64_i8 v[176:179], a[108:111], v[104:107], v[176:179]\n"
    "ds_read_b128 v[76:79], v38 offset:55296\n"
    "v_cmp_lt_f32 s[28:29], s97, v34\n"
    "v_mfma_i32_16x16x64_i8 v[176:179], a[132:135], v[108:111], v[176:179]\n"
    "s_waitcnt lgkmcnt(5)\n"
    "buffer_load_dwordx4 v22, s[40:43], s34 offen sc0 sc1 lds\n"
    "s_cmp_lg_u64 s[28:29], 0\n"
    "s_cselect_b32 s24, 64, 0\n"
    "s_or_b32 s81, s81, s24\n"
    "v_mfma_i32_16x16x64_i8 v[176:179], a[140:143], v[112:115], v[176:179]\n"
    "ds_read_b128 v[80:83], v39 offset:55296\n"
    "v_fma_f32 v220, v220, s18, v33\n"
    "v_mfma_i32_16x16x64_i8 v[176:179], a[164:167], v[52:55], v[176:179]\n"
    "buffer_load_dwordx4 v22, s[40:43], s34 offen offset:1024 sc0 sc1 lds\n"
    "s_mov_b32 s96, 10\n"
    "v_fma_f32 v221, v221, s18, v33\n"
    "v_mfma_i32_16x16x64_i8 v[176:179], a[172:175], v[56:59], v[176:179]\n"
    "v_fma_f32 v222, v222, s18, v33\n"
    "v_mfma_i32_16x16x64_i8 v[172:175], a[136:139], v[112:115], v[172:175]\n"
    "v_fma_f32 v223, v223, s18, v33\n"
    "v_fma_f32 v224, v224, s18, v33\n"
    "v_mfma_i32_16x16x64_i8 v[172:175], a[160:163], v[52:55], v[172:175]\n"
    "s_mov_b32 s95, 8\n"
    "s_and_b32 s91, s84, s25\n"
    "v_fma_f32 v225, v225, s18, v33\n"
    "v_mfma_i32_16x16x64_i8 v[172:175], a[168:171], v[56:59], v[172:175]\n"
    "s_cmp_gt_u32 s91, 8\n"
    "s_getpc_b64 s[88:89]\n"
    "s_cbranch_scc0 L_slowc%=\n"
    "s_waitcnt vmcnt(11) lgkmcnt(0)\n"
    "s_barrier\n"
    "v_mfma_i32_16x16x64_i8 v[172:175], a[112:115], v[60:63], v[172:175]\n"
    "ds_read_b128 v[84:87], v40\n"
    "s_waitcnt vmcnt(10)\n"
    "v_fma_f32 v226, v226, s18, v33\n"
    "v_mfma_i32_16x16x64_i8 v[172:175], a[120:123], v[64:67], v[172:175]\n"
    "ds_read_b128 v[88:91], v41\n"
    "v_cmp_le_u32 vcc, s17, v32\n"
    "v_mfma_i32_16x16x64_i8 v[172:175], a[144:147], v[68:71], v[172:175]\n"
    "ds_read_b128 v[92:95], v40 offset:8192\n"
    "v_fma_f32 v227, v227, s18, v33\n"
    "v_mfma_i32_16x16x64_i8 v[176:179], a[116:119], v[60:63], v[176:179]\n"
    "ds_read_b128 v[96:99], v41 offset:8192\n"
    "s_lshr_b32 s87, vcc_lo, 16\n"
    "v_max_f32 v220, v220, v33\n"
    "v_mfma_i32_16x16x64_i8 v[176:179], a[124:127], v[64:67], v[176:179]\n"
    "ds_read_b128 v[100:103], v40 offset:16384\n"
    "v_max_f32 v221, v221, v33\n"
    "v_mfma_i32_16x16x64_i8 v[176:179], a[148:151], v[68:71], v[176:179]\n"
    "ds_read_b128 v[104:107], v41 offset:16384\n"
    "s_and_b32 s87, s87, vcc_lo\n"
    "v_max_f32 v222, v222, v33\n"
    "v_mfma_i32_16x16x64_i8 v[176:179], a[156:159], v[72:75], v[176:179]\n"
    "ds_read_b128 v[108:111], v40 offset:2048\n"
    "v_max_f32 v223, v223, v33\n"
    "v_mfma_i32_16x16x64_i8 v[176:179], a[180:183], v[76:79], v[176:179]\n"
    "ds_read_b128 v[112:115], v41 offset:2048\n"
    "s_and_b32 s87, s87, vcc_hi\n"
    "v_max_f32 v224, v224, v33\n"
    "v_mfma_i32_16x16x64_i8 v[176:179], a[188:191], v[80:83], v[176:179]\n"
    "s_ff0_i32_b32 s85, s87\n"
    "v_max_f32 v225, v225, v33\n"
    "v_mfma_i32_16x16x64_i8 v[172:175], a[152:155], v[72:75], v[172:175]\n"
    "s_max_u32 s85, s85, s86\n"
    "v_max_f32 v226, v226, v33\n"
    "v_mfma_i32_16x16x64_i8 v[172:175], a[176:179], v[76:79], v[172:175]\n"
    "s_min_u32 s91, s85, s96\n"
    "v_max_f32 v227, v227, v33\n"
    "v_mfma_i32_16x16x64_i8 v[172:175], a[184:187], v[80:83], v[172:175]\n"
    "s_cmp_lt_u32 s84, s91\n"
    "s_getpc_b64 s[88:89]\n"
    "s_cbranch_scc1 L_catchc%=\n"
    "s_waitcnt lgkmcnt(5)\n"
    "v_mfma_i32_16x16x64_i8 v[180:183], a[0:3], v[84:87], v[180:183]\n"
    "ds_read_b128 v[52:55], v40 offset:10240\n"
    "s_add_u32 s90, s65, 65536\n"
    "buffer_load_dwordx4 v[236:239], v22, s[44:47], s90 offen sc0 sc1\n"
    "v_mfma_i32_16x16x64_i8 v[180:183], a[8:11], v[88:91], v[180:183]\n"
    "v_mfma_i32_16x16x64_i8 v[180:183], a[32:35], v[92:95], v[180:183]\n"
    "ds_read_b128 v[56:59], v41 offset:10240\n"
    "v_mfma_i32_16x16x64_i8 v[184:187], a[4:7], v[84:87], v[184:187]\n"
    "buffer_load_dwordx4 v[240:243], v22, s[44:47], s90 offen offset:1024 sc0 sc1\n"
    "v_mfma_i32_16x16x64_i8 v[184:187], a[12:15], v[88:91], v[184:187]\n"
    "ds_read_b128 v[60:63], v40 offset:18432\n"
    "v_mfma_i32_16x16x64_i8 v[184:187], a[36:39], v[92:95], v[184:187]\n"
    "s_waitcnt lgkmcnt(5)\n"
    "s_add_u32 s90, s65, 16384\n"
    "v_mfma_i32_16x16x64_i8 v[184:187], a[44:47], v[96:99], v[184:187]\n"
    "ds_read_b128 v[64:67], v41 offset:18432\n"
    "s_lshl_b32 s90, s90, 5\n"
    "v_mfma_i32_16x16x64_i8 v[184:187], a[68:71], v[100:103], v[184:187]\n"
    "buffer_store_dwordx4 v[220:223], v23, s[48:51], s90 offen sc0 sc1\n"
    "s_waitcnt vmcnt(11)\n"
    "v_mfma_i32_16x16x64_i8 v[184:187], a[76:79], v[104:107], v[184:187]\n"
    "ds_read_b128 v[68:71], v40 offset:4096\n"
    "v_add_u32 v228, v172, v228\n"
    "v_mfma_i32_16x16x64_i8 v[180:183], a[40:43], v[96:99], v[180:183]\n"
    "v_add_u32 v229, v173, v229\n"
    "v_add_u32 v230, v174, v230\n"
    "v_mfma_i32_16x16x64_i8 v[180:183], a[64:67], v[100:103], v[180:183]\n"
    "ds_read_b128 v[72:75], v41 offset:4096\n"
    "v_mfma_i32_16x16x64_i8 v[180:183], a[72:75], v[104:107], v[180:183]\n"
    "s_waitcnt lgkmcnt(5)\n"
    "buffer_store_dwordx4 v[224:227], v23, s[48:51], s90 offen offset:64 sc0 sc1\n"
    "v_mfma_i32_16x16x64_i8 v[180:183], a[16:19], v[108:111], v[180:183]\n"
    "ds_read_b128 v[76:79], v40 offset:12288\n"
    "v_add_u32 v231, v175, v231\n"
    "v_mfma_i32_16x16x64_i8 v[180:183], a[24:27], v[112:115], v[180:183]\n"
    "v_add_u32 v232, v176, v232\n"
    "v_add_u32 v233, v177, v233\n"
    "v_mfma_i32_16x16x64_i8 v[180:183], a[48:51], v[52:55], v[180:183]\n"
    "ds_read_b128 v[80:83], v41 offset:12288\n"
    "v_add_u32 v234, v178, v234\n"
    "v_mfma_i32_16x16x64_i8 v[184:187], a[20:23], v[108:111], v[184:187]\n"
    "s_cmp_lt_u32 s84, s96\n"
    "s_getpc_b64 s[88:89]\n"
    "s_cbranch_scc1 L_midc%=\n"
    "v_mfma_i32_16x16x64_i8 v[184:187], a[28:31], v[112:115], v[184:187]\n"
    "ds_read_b128 v[84:87], v40 offset:20480\n"
    "v_mfma_i32_16x16x64_i8 v[184:187], a[52:55], v[52:55], v[184:187]\n"
    "s_waitcnt lgkmcnt(5)\n"
    "v_add_u32 v235, v179, v235\n"
    "v_mfma_i32_16x16x64_i8 v[184:187], a[60:63], v[56:59], v[184:187]\n"
    "ds_read_b128 v[88:91], v41 offset:20480\n"
    "v_cvt_f32_i32 v228, v228\n"
    "v_mfma_i32_16x16x64_i8 v[184:187], a[84:87], v[60:63], v[184:187]\n"
    "v_cvt_f32_i32 v229, v229\n"
    "v_mfma_i32_16x16x64_i8 v[184:187], a[92:95], v[64:67], v[184:187]\n"
    "ds_read_b128 v[92:95], v40 offset:6144\n"
    "v_cvt_f32_i32 v230, v230\n"
    "v_mfma_i32_16x16x64_i8 v[180:183], a[56:59], v[56:59], v[180:183]\n"
    "v_cvt_f32_i32 v231, v231\n"
    "v_cvt_f32_i32 v232, v232\n"
    "v_mfma_i32_16x16x64_i8 v[180:183], a[80:83], v[60:63], v[180:183]\n"
    "ds_read_b128 v[96:99], v41 offset:6144\n"
    "v_cvt_f32_i32 v233, v233\n"
    "v_mfma_i32_16x16x64_i8 v[180:183], a[88:91], v[64:67], v[180:183]\n"
    "s_waitcnt lgkmcnt(5)\n"
    "v_mfma_i32_16x16x64_i8 v[180:183], a[96:99], v[68:71], v[180:183]\n"
    "ds_read_b128 v[100:103], v40 offset:14336\n"
    "v_cvt_f32_i32 v234, v234\n"
    "v_mfma_i32_16x16x64_i8 v[180:183], a[104:107], v[72:75], v[180:183]\n"
    "v_cvt_f32_i32 v235, v235\n"
    "v_max3_f32 v34, |v228|, |v229|, |v230|\n"
    "v_mfma_i32_16x16x64_i8 v[180:183], a[128:131], v[76:79], v[180:183]\n"
    "ds_read_b128 v[104:107], v41 offset:14336\n"
    "v_mfma_i32_16x16x64_i8 v[184:187], a[100:103], v[68:71], v[184:187]\n"
    "v_max3_f32 v34, v34, |v231|, |v232|\n"
    "v_max3_f32 v34, v34, |v233|, |v234|\n"
    "v_mfma_i32_16x16x64_i8 v[184:187], a[108:111], v[72:75], v[184:187]\n"
    "ds_read_b128 v[108:111], v40 offset:22528\n"
    "v_max_f32 v34, v34, |v235|\n"
    "v_mfma_i32_16x16x64_i8 v[184:187], a[132:135], v[76:79], v[184:187]\n"
    "s_waitcnt lgkmcnt(5)\n"
    "v_cmp_lt_f32 s[28:29], s97, v34\n"
    "v_mfma_i32_16x16x64_i8 v[184:187], a[140:143], v[80:83], v[184:187]\n"
    "ds_read_b128 v[112:115], v41 offset:22528\n"
    "v_mfma_i32_16x16x64_i8 v[184:187], a[164:167], v[84:87], v[184:187]\n"
    "s_cmp_lg_u64 s[28:29], 0\n"
    "s_cselect_b32 s24, 128, 0\n"
    "s_or_b32 s81, s81, s24\n"
    "v_fma_f32 v228, v228, s18, v33\n"
    "v_mfma_i32_16x16x64_i8 v[184:187], a[172:175], v[88:91], v[184:187]\n"
    "v_fma_f32 v229, v229, s18, v33\n"
    "v_mfma_i32_16x16x64_i8 v[180:183], a[136:139], v[80:83], v[180:183]\n"
    "v_fma_f32 v230, v230, s18, v33\n"
    "v_fma_f32 v231, v231, s18, v33\n"
    "v_mfma_i32_16x16x64_i8 v[180:183], a[160:163], v[84:87], v[180:183]\n"
    "s_mov_b32 s95, 9\n"
    "s_and_b32 s91, s84, s25\n"
    "v_mfma_i32_16x16x64_i8 v[180:183], a[168:171], v[88:91], v[180:183]\n"
    "s_cmp_gt_u32 s91, 9\n"
    "s_getpc_b64 s[88:89]\n"
    "s_cbranch_scc0 L_slowc%=\n"
    "s_waitcnt vmcnt(4) lgkmcnt(0)\n"
    "s_barrier\n"
    "v_mfma_i32_16x16x64_i8 v[180:183], a[112:115], v[92:95], v[180:183]\n"
    "ds_read_b128 v[52:55], v40 offset:32768\n"
    "v_fma_f32 v232, v232, s18, v33\n"
    "v_mfma_i32_16x16x64_i8 v[180:183], a[120:123], v[96:99], v[180:183]\n"
    "ds_read_b128 v[56:59], v41 offset:32768\n"
    "v_fma_f32 v233, v233, s18, v33\n"
    "v_mfma_i32_16x16x64_i8 v[180:183], a[144:147], v[100:103], v[180:183]\n"
    "ds_read_b128 v[60:63], v40 offset:40960\n"
    "v_mfma_i32_16x16x64_i8 v[184:187], a[116:119], v[92:95], v[184:187]\n"
    "ds_read_b128 v[64:67], v41 offset:40960\n"
    "v_fma_f32 v234, v234, s18, v33\n"
    "v_mfma_i32_16x16x64_i8 v[184:187], a[124:127], v[96:99], v[184:187]\n"
    "ds_read_b128 v[68:71], v40 offset:49152\n"
    "v_fma_f32 v235, v235, s18, v33\n"
    "v_mfma_i32_16x16x64_i8 v[184:187], a[148:151], v[100:103], v[184:187]\n"
    "ds_read_b128 v[72:75], v41 offset:49152\n"
    "v_max_f32 v228, v228, v33\n"
    "v_mfma_i32_16x16x64_i8 v[184:187], a[156:159], v[104:107], v[184:187]\n"
    "ds_read_b128 v[76:79], v40 offset:34816\n"
    "v_mfma_i32_16x16x64_i8 v[184:187], a[180:183], v[108:111], v[184:187]\n"
    "ds_read_b128 v[80:83], v41 offset:34816\n"
    "v_max_f32 v229, v229, v33\n"
    "v_mfma_i32_16x16x64_i8 v[184:187], a[188:191], v[112:115], v[184:187]\n"
    "v_max_f32 v230, v230, v33\n"
    "v_max_f32 v231, v231, v33\n"
    "v_mfma_i32_16x16x64_i8 v[180:183], a[152:155], v[104:107], v[180:183]\n"
    "v_max_f32 v232, v232, v33\n"
    "v_mfma_i32_16x16x64_i8 v[180:183], a[176:179], v[108:111], v[180:183]\n"
    "v_max_f32 v233, v233, v33\n"
    "v_max_f32 v234, v234, v33\n"
    "v_mfma_i32_16x16x64_i8 v[180:183], a[184:187], v[112:115], v[180:183]\n"
    "v_max_f32 v235, v235, v33\n"
    "s_waitcnt lgkmcnt(5)\n"
    "v_mfma_i32_16x16x64_i8 v[188:191], a[0:3], v[52:55], v[188:191]\n"
    "global_load_dword v32, v24, s[72:73] sc0 sc1\n"
    "ds_read_b128 v[84:87], v40 offset:43008\n"
    "v_mfma_i32_16x16x64_i8 v[188:191], a[8:11], v[56:59], v[188:191]\n"
    "s_add_u32 s90, s65, 90112\n"
    "buffer_load_dwordx4 v[244:247], v22, s[44:47], s90 offen sc0 sc1\n"
    "v_mfma_i32_16x16x64_i8 v[188:191], a[32:35], v[60:63], v[188:191]\n"
    "ds_read_b128 v[88:91], v41 offset:43008\n"
    "v_mfma_i32_16x16x64_i8 v[192:195], a[4:7], v[52:55], v[192:195]\n"
    "v_mfma_i32_16x16x64_i8 v[192:195], a[12:15], v[56:59], v[192:195]\n"
    "ds_read_b128 v[92:95], v40 offset:51200\n"
    "v_mfma_i32_16x16x64_i8 v[192:195], a[36:39], v[60:63], v[192:195]\n"
    "s_waitcnt lgkmcnt(5)\n"
    "buffer_load_dwordx4 v[248:251], v22, s[44:47], s90 offen offset:1024 sc0 sc1\n"
    "s_waitcnt vmcnt(5)\n"
    "v_mfma_i32_16x16x64_i8 v[192:195], a[44:47], v[64:67], v[192:195]\n"
    "ds_read_b128 v[96:99], v41 offset:51200\n"
    "v_add_u32 v236, v180, v236\n"
    "v_mfma_i32_16x16x64_i8 v[192:195], a[68:71], v[68:71], v[192:195]\n"
    "s_add_u32 s90, s65, 40960\n"
    "s_lshl_b32 s90, s90, 5\n"
    "v_add_u32 v237, v181, v237\n"
    "v_mfma_i32_16x16x64_i8 v[192:195], a[76:79], v[72:75], v[192:195]\n"
    "ds_read_b128 v[100:103], v40 offset:36864\n"
    "v_add_u32 v238, v182, v238\n"
    "v_mfma_i32_16x16x64_i8 v[188:191], a[40:43], v[64:67], v[188:191]\n"
    "buffer_store_dwordx4 v[228:231], v23, s[48:51], s90 offen sc0 sc1\n"
    "v_add_u32 v239, v183, v239\n"
    "v_add_u32 v240, v184, v240\n"
    "v_mfma_i32_16x16x64_i8 v[188:191], a[64:67], v[68:71], v[188:191]\n"
    "ds_read_b128 v[104:107], v41 offset:36864\n"
    "v_add_u32 v241, v185, v241\n"
    "v_mfma_i32_16x16x64_i8 v[188:191], a[72:75], v[72:75], v[188:191]\n"
    "s_waitcnt lgkmcnt(5)\n"
    "buffer_store_dwordx4 v[232:235], v23, s[48:51], s90 offen offset:64 sc0 sc1\n"
    "v_add_u32 v242, v186, v242\n"
    "v_mfma_i32_16x16x64_i8 v[188:191], a[16:19], v[76:79], v[188:191]\n"
    "ds_read_b128 v[108:111], v40 offset:45056\n"
    "v_add_u32 v243, v187, v243\n"
    "v_mfma_i32_16x16x64_i8 v[188:191], a[24:27], v[80:83], v[188:191]\n"
    "v_cvt_f32_i32 v236, v236\n"
    "v_cvt_f32_i32 v237, v237\n"
    "v_mfma_i32_16x16x64_i8 v[188:191], a[48:51], v[84:87], v[188:191]\n"
    "ds_read_b128 v[112:115], v41 offset:45056\n"
    "v_cvt_f32_i32 v238, v238\n"
    "v_mfma_i32_16x16x64_i8 v[192:195], a[20:23], v[76:79], v[192:195]\n"
    "v_cvt_f32_i32 v239, v239\n"
    "v_cvt_f32_i32 v240, v240\n"
    "v_mfma_i32_16x16x64_i8 v[192:195], a[28:31], v[80:83], v[192:195]\n"
    "ds_read_b128 v[52:55], v40 offset:53248\n"
    "v_cvt_f32_i32 v241, v241\n"
    "v_mfma_i32_16x16x64_i8 v[192:195], a[52:55], v[84:87], v[192:195]\n"
    "s_waitcnt lgkmcnt(5)\n"
    "v_cvt_f32_i32 v242, v242\n"
    "v_cvt_f32_i32 v243, v243\n"
    "v_mfma_i32_16x16x64_i8 v[192:195], a[60:63], v[88:91], v[192:195]\n"
    "ds_read_b128 v[56:59], v41 offset:53248\n"
    "v_max3_f32 v34, |v236|, |v237|, |v238|\n"
    "v_mfma_i32_16x16x64_i8 v[192:195], a[84:87], v[92:95], v[192:195]\n"
    "v_max3_f32 v34, v34, |v239|, |v240|\n"
    "v_max3_f32 v34, v34, |v241|, |v242|\n"
    "v_mfma_i32_16x16x64_i8 v[192:195], a[92:95], v[96:99], v[192:195]\n"
    "ds_read_b128 v[60:63], v40 offset:38912\n"
    "v_max_f32 v34, v34, |v243|\n"
    "v_mfma_i32_16x16x64_i8 v[188:191], a[56:59], v[88:91], v[188:191]\n"
    "v_cmp_lt_f32 s[28:29], s97, v34\n"
    "s_cmp_lg_u64 s[28:29], 0\n"
    "s_cselect_b32 s24, 256, 0\n"
    "s_or_b32 s81, s81, s24\n"
    "v_mfma_i32_16x16x64_i8 v[188:191], a[80:83], v[92:95], v[188:191]\n"
    "ds_read_b128 v[64:67], v41 offset:38912\n"
    "v_fma_f32 v236, v236, s18, v33\n"
    "v_mfma_i32_16x16x64_i8 v[188:191], a[88:91], v[96:99], v[188:191]\n"
    "s_waitcnt lgkmcnt(5)\n"
    "v_fma_f32 v237, v237, s18, v33\n"
    "v_fma_f32 v238, v238, s18, v33\n"
    "v_mfma_i32_16x16x64_i8 v[188:191], a[96:99], v[100:103], v[188:191]\n"
    "ds_read_b128 v[68:71], v40 offset:47104\n"
    "v_fma_f32 v239, v239, s18, v33\n"
    "v_mfma_i32_16x16x64_i8 v[188:191], a[104:107], v[104:107], v[188:191]\n"
    "v_fma_f32 v240, v240, s18, v33\n"
    "v_fma_f32 v241, v241, s18, v33\n"
    "v_mfma_i32_16x16x64_i8 v[188:191], a[128:131], v[108:111], v[188:191]\n"
    "ds_read_b128 v[72:75], v41 offset:47104\n"
    "v_fma_f32 v242, v242, s18, v33\n"
    "v_mfma_i32_16x16x64_i8 v[192:195], a[100:103], v[100:103], v[192:195]\n"
    "v_fma_f32 v243, v243, s18, v33\n"
    "v_max_f32 v236, v236, v33\n"
    "v_mfma_i32_16x16x64_i8 v[192:195], a[108:111], v[104:107], v[192:195]\n"
    "ds_read_b128 v[76:79], v40 offset:55296\n"
    "v_max_f32 v237, v237, v33\n"
    "v_mfma_i32_16x16x64_i8 v[192:195], a[132:135], v[108:111], v[192:195]\n"
    "s_waitcnt lgkmcnt(5)\n"
    "v_max_f32 v238, v238, v33\n"
    "v_max_f32 v239, v239, v33\n"
    "v_mfma_i32_16x16x64_i8 v[192:195], a[140:143], v[112:115], v[192:195]\n"
    "ds_read_b128 v[80:83], v41 offset:55296\n"
    "v_max_f32 v240, v240, v33\n"
    "v_mfma_i32_16x16x64_i8 v[192:195], a[164:167], v[52:55], v[192:195]\n"
    "v_max_f32 v241, v241, v33\n"
    "v_max_f32 v242, v242, v33\n"
    "v_mfma_i32_16x16x64_i8 v[192:195], a[172:175], v[56:59], v[192:195]\n"
    "v_max_f32 v243, v243, v33\n"
    "s_add_u32 s90, s65, 65536\n"
    "v_mfma_i32_16x16x64_i8 v[188:191], a[136:139], v[112:115], v[188:191]\n"
    "s_lshl_b32 s90, s90, 5\n"
    "buffer_store_dwordx4 v[236:239], v23, s[48:51], s90 offen sc0 sc1\n"
    "v_mfma_i32_16x16x64_i8 v[188:191], a[160:163], v[52:55], v[188:191]\n"
    "buffer_store_dwordx4 v[240:243], v23, s[48:51], s90 offen offset:64 sc0 sc1\n"
    "v_mfma_i32_16x16x64_i8 v[188:191], a[168:171], v[56:59], v[188:191]\n"
    "s_waitcnt lgkmcnt(3)\n"
    "v_mfma_i32_16x16x64_i8 v[188:191], a[112:115], v[60:63], v[188:191]\n"
    "s_waitcnt vmcnt(6)\n"
    "v_mfma_i32_16x16x64_i8 v[188:191], a[120:123], v[64:67], v[188:191]\n"
    "v_cmp_le_u32 vcc, s17, v32\n"
    "v_mfma_i32_16x16x64_i8 v[188:191], a[144:147], v[68:71], v[188:191]\n"
    "s_lshr_b32 s87, vcc_lo, 16\n"
    "v_mfma_i32_16x16x64_i8 v[192:195], a[116:119], v[60:63], v[192:195]\n"
    "s_and_b32 s87, s87, vcc_lo\n"
    "v_mfma_i32_16x16x64_i8 v[192:195], a[124:127], v[64:67], v[192:195]\n"
    "v_mfma_i32_16x16x64_i8 v[192:195], a[148:151], v[68:71], v[192:195]\n"
    "s_waitcnt lgkmcnt(0)\n"
    "s_and_b32 s87, s87, vcc_hi\n"
    "v_mfma_i32_16x16x64_i8 v[192:195], a[156:159], v[72:75], v[192:195]\n"
    "s_ff0_i32_b32 s85, s87\n"
    "v_mfma_i32_16x16x64_i8 v[192:195], a[180:183], v[76:79], v[192:195]\n"
    "v_mfma_i32_16x16x64_i8 v[192:195], a[188:191], v[80:83], v[192:195]\n"
    "s_max_u32 s85, s85, s86\n"
    "v_mfma_i32_16x16x64_i8 v[188:191], a[152:155], v[72:75], v[188:191]\n"
    "s_bitcmp1_b32 vcc_hi, 24\n"
    "s_cselect_b32 s87, 0x7fc00000, 0\n"
    "v_mfma_i32_16x16x64_i8 v[188:191], a[176:179], v[76:79], v[188:191]\n"
    "v_mov_b32 v33, s87\n"
    "v_mfma_i32_16x16x64_i8 v[188:191], a[184:187], v[80:83], v[188:191]\n"
    "s_nop 7\n"
    "s_waitcnt vmcnt(4)\n"
    "v_add_u32 v244, v188, v244\n"
    "v_add_u32 v245, v189, v245\n"
    "v_add_u32 v246, v190, v246\n"
    "v_add_u32 v247, v191, v247\n"
    "v_add_u32 v248, v192, v248\n"
    "v_add_u32 v249, v193, v249\n"
    "v_add_u32 v250, v194, v250\n"
    "v_add_u32 v251, v195, v251\n"
    "v_cvt_f32_i32 v244, v244\n"
    "v_cvt_f32_i32 v245, v245\n"
    "v_cvt_f32_i32 v246, v246\n"
    "v_cvt_f32_i32 v247, v247\n"
    "v_cvt_f32_i32 v248, v248\n"
    "v_cvt_f32_i32 v249, v249\n"
    "v_cvt_f32_i32 v250, v250\n"
    "v_cvt_f32_i32 v251, v251\n"
    "v_max3_f32 v34, |v244|, |v245|, |v246|\n"
    "v_max3_f32 v34, v34, |v247|, |v248|\n"
    "v_max3_f32 v34, v34, |v249|, |v250|\n"
    "v_max_f32 v34, v34, |v251|\n"
    "v_cmp_lt_f32 s[28:29], s97, v34\n"
    "s_cmp_lg_u64 s[28:29], 0\n"
    "s_cselect_b32 s24, 512, 0\n"
    "s_or_b32 s81, s81, s24\n"
    "v_fma_f32 v244, v244, s18, v33\n"
    "v_fma_f32 v245, v245, s18, v33\n"
    "v_fma_f32 v246, v246, s18, v33\n"
    "v_fma_f32 v247, v247, s18, v33\n"
    "v_fma_f32 v248, v248, s18, v33\n"
    "v_fma_f32 v249, v249, s18, v33\n"
    "v_fma_f32 v250, v250, s18, v33\n"
    "v_fma_f32 v251, v251, s18, v33\n"
    "v_max_f32 v244, v244, v33\n"
    "v_max_f32 v245, v245, v33\n"
    "v_max_f32 v246, v246, v33\n"
    "v_max_f32 v247, v247, v33\n"
    "v_max_f32 v248, v248, v33\n"
    "v_max_f32 v249, v249, v33\n"
    "v_max_f32 v250, v250, v33\n"
    "v_max_f32 v251, v251, v33\n"
    "s_add_u32 s90, s65, 90112\n"
    "s_lshl_b32 s90, s90, 5\n"
    "buffer_store_dwordx4 v[244:247], v23, s[48:51], s90 offen sc0 sc1\n"
    "buffer_store_dwordx4 v[248:251], v23, s[48:51], s90 offen offset:64 sc0 sc1\n"
    "s_cmp_lg_u32 s81, 0\n"
    "s_cbranch_scc1 L_redoc%=\n"
    "L_finc%=:\n"
    "s_branch L_end%=\n"
    "L_redoc%=:\n"
    "s_mov_b64 s[26:27], -1\n"
    "L_redo2c%=:\n"
    "s_bitcmp1_b32 s81, 4\n"
    "s_cbranch_scc0 L_rl4_c%=\n"
    "s_add_u32 s90, s65, 0\n"
    "buffer_load_dwordx4 v[204:207], v22, s[44:47], s90 offen sc0 sc1\n"
    "buffer_load_dwordx4 v[208:211], v22, s[44:47], s90 offen offset:1024 sc0 sc1\n"
    "L_rl4_c%=:\n"
    "s_bitcmp1_b32 s81, 5\n"
    "s_cbranch_scc0 L_rl5_c%=\n"
    "s_add_u32 s90, s65, 8192\n"
    "buffer_load_dwordx4 v[212:215], v22, s[44:47], s90 offen sc0 sc1\n"
    "buffer_load_dwordx4 v[216:219], v22, s[44:47], s90 offen offset:1024 sc0 sc1\n"
    "L_rl5_c%=:\n"
    "s_bitcmp1_b32 s81, 6\n"
    "s_cbranch_scc0 L_rl6_c%=\n"
    "s_add_u32 s90, s65, 16384\n"
    "buffer_load_dwordx4 v[220:223], v22, s[44:47], s90 offen sc0 sc1\n"
    "buffer_load_dwordx4 v[224:227], v22, s[44:47], s90 offen offset:1024 sc0 sc1\n"
    "L_rl6_c%=:\n"
    "s_bitcmp1_b32 s81, 7\n"
    "s_cbranch_scc0 L_rl7_c%=\n"
    "s_add_u32 s90, s65, 40960\n"
    "buffer_load_dwordx4 v[228:231], v22, s[44:47], s90 offen sc0 sc1\n"
    "buffer_load_dwordx4 v[232:235], v22, s[44:47], s90 offen offset:1024 sc0 sc1\n"
    "L_rl7_c%=:\n"
    "s_bitcmp1_b32 s81, 8\n"
    "s_cbranch_scc0 L_rl8_c%=\n"
    "s_add_u32 s90, s65, 65536\n"
    "buffer_load_dwordx4 v[236:239], v22, s[44:47], s90 offen sc0 sc1\n"
    "buffer_load_dwordx4 v[240:243], v22, s[44:47], s90 offen offset:1024 sc0 sc1\n"
    "L_rl8_c%=:\n"
    "s_bitcmp1_b32 s81, 9\n"
    "s_cbranch_scc0 L_rl9_c%=\n"
    "s_add_u32 s90, s65, 90112\n"
    "buffer_load_dwordx4 v[244:247], v22, s[44:47], s90 offen sc0 sc1\n"
    "buffer_load_dwordx4 v[248:251], v22, s[44:47], s90 offen offset:1024 sc0 sc1\n"
    "L_rl9_c%=:\n"
    "s_waitcnt vmcnt(0)\n"
    "s_bitcmp1_b32 s81, 4\n"
    "s_cbranch_scc0 L_rn4_c%=\n"
    "v_add_u32 v204, v148, v204\n"
    "v_add_u32 v205, v149, v205\n"
    "v_add_u32 v206, v150, v206\n"
    "v_add_u32 v207, v151, v207\n"
    "v_add_u32 v208, v152, v208\n"
    "v_add_u32 v209, v153, v209\n"
    "v_add_u32 v210, v154, v210\n"
    "v_add_u32 v211, v155, v211\n"
    "v_cvt_f32_i32 v204, v204\n"
    "v_cvt_f32_i32 v205, v205\n"
    "v_cvt_f32_i32 v206, v206\n"
    "v_cvt_f32_i32 v207, v207\n"
    "v_cvt_f32_i32 v208, v208\n"
    "v_cvt_f32_i32 v209, v209\n"
    "v_cvt_f32_i32 v210, v210\n"
    "v_cvt_f32_i32 v211, v211\n"
    "v_max3_f32 v34, |v204|, |v205|, |v206|\n"
    "v_max3_f32 v34, v34, |v207|, |v208|\n"
    "v_max3_f32 v34, v34, |v209|, |v210|\n"
    "v_max_f32 v34, v34, |v211|\n"
    "v_cmp_lt_f32 s[28:29], s97, v34\n"
    "s_and_b64 s[28:29], s[28:29], s[26:27]\n"
    "s_cbranch_scc1 L_rn4_c%=\n"
    "v_fma_f32 v204, v204, s18, v33\n"
    "v_fma_f32 v205, v205, s18, v33\n"
    "v_fma_f32 v206, v206, s18, v33\n"
    "v_fma_f32 v207, v207, s18, v33\n"
    "v_fma_f32 v208, v208, s18, v33\n"
    "v_fma_f32 v209, v209, s18, v33\n"
    "v_fma_f32 v210, v210, s18, v33\n"
    "v_fma_f32 v211, v211, s18, v33\n"
    "v_max_f32 v204, v204, v33\n"
    "v_max_f32 v205, v205, v33\n"
    "v_max_f32 v206, v206, v33\n"
    "v_max_f32 v207, v207, v33\n"
    "v_max_f32 v208, v208, v33\n"
    "v_max_f32 v209, v209, v33\n"
    "v_max_f32 v210, v210, v33\n"
    "v_max_f32 v211, v211, v33\n"
    "s_add_u32 s90, s65, 0\n"
    "s_lshl_b32 s90, s90, 5\n"
    "buffer_store_dwordx4 v[204:207], v23, s[48:51], s90 offen sc0 sc1\n"
    "buffer_store_dwordx4 v[208:211], v23, s[48:51], s90 offen offset:64 sc0 sc1\n"
    "s_bitset0_b32 s81, 4\n"
    "L_rn4_c%=:\n"
    "s_bitcmp1_b32 s81, 5\n"
    "s_cbranch_scc0 L_rn5_c%=\n"
    "v_add_u32 v212, v156, v212\n"
    "v_add_u32 v213, v157, v213\n"
    "v_add_u32 v214, v158, v214\n"
    "v_add_u32 v215, v159, v215\n"
    "v_add_u32 v216, v160, v216\n"
    "v_add_u32 v217, v161, v217\n"
    "v_add_u32 v218, v162, v218\n"
    "v_add_u32 v219, v163, v219\n"
    "v_cvt_f32_i32 v212, v212\n"
    "v_cvt_f32_i32 v213, v213\n"
    "v_cvt_f32_i32 v214, v214\n"
    "v_cvt_f32_i32 v215, v215\n"
    "v_cvt_f32_i32 v216, v216\n"
    "v_cvt_f32_i32 v217, v217\n"
    "v_cvt_f32_i32 v218, v218\n"
    "v_cvt_f32_i32 v219, v219\n"
    "v_max3_f32 v34, |v212|, |v213|, |v214|\n"
    "v_max3_f32 v34, v34, |v215|, |v216|\n"
    "v_max3_f32 v34, v34, |v217|, |v218|\n"
    "v_max_f32 v34, v34, |v219|\n"
    "v_cmp_lt_f32 s[28:29], s97, v34\n"
    "s_and_b64 s[28:29], s[28:29], s[26:27]\n"
    "s_cbranch_scc1 L_rn5_c%=\n"
    "v_fma_f32 v212, v212, s18, v33\n"
    "v_fma_f32 v213, v213, s18, v33\n"
    "v_fma_f32 v214, v214, s18, v33\n"
    "v_fma_f32 v215, v215, s18, v33\n"
    "v_fma_f32 v216, v216, s18, v33\n"
    "v_fma_f32 v217, v217, s18, v33\n"
    "v_fma_f32 v218, v218, s18, v33\n"
    "v_fma_f32 v219, v219, s18, v33\n"
    "v_max_f32 v212, v212, v33\n"
    "v_max_f32 v213, v213, v33\n"
    "v_max_f32 v214, v214, v33\n"
    "v_max_f32 v215, v215, v33\n"
    "v_max_f32 v216, v216, v33\n"
    "v_max_f32 v217, v217, v33\n"
    "v_max_f32 v218, v218, v33\n"
    "v_max_f32 v219, v219, v33\n"
    "s_add_u32 s90, s65, 8192\n"
    "s_lshl_b32 s90, s90, 5\n"
    "buffer_store_dwordx4 v[212:215], v23, s[48:51], s90 offen sc0 sc1\n"
    "buffer_store_dwordx4 v[216:219], v23, s[48:51], s90 offen offset:64 sc0 sc1\n"
    "s_bitset0_b32 s81, 5\n"
    "L_rn5_c%=:\n"
    "s_bitcmp1_b32 s81, 6\n"
    "s_cbranch_scc0 L_rn6_c%=\n"
    "v_add_u32 v220, v164, v220\n"
    "v_add_u32 v221, v165, v221\n"
    "v_add_u32 v222, v166, v222\n"
    "v_add_u32 v223, v167, v223\n"
    "v_add_u32 v224, v168, v224\n"
    "v_add_u32 v225, v169, v225\n"
    "v_add_u32 v226, v170, v226\n"
    "v_add_u32 v227, v171, v227\n"
    "v_cvt_f32_i32 v220, v220\n"
    "v_cvt_f32_i32 v221, v221\n"
    "v_cvt_f32_i32 v222, v222\n"
    "v_cvt_f32_i32 v223, v223\n"
    "v_cvt_f32_i32 v224, v224\n"
    "v_cvt_f32_i32 v225, v225\n"
    "v_cvt_f32_i32 v226, v226\n"
    "v_cvt_f32_i32 v227, v227\n"
    "v_max3_f32 v34, |v220|, |v221|, |v222|\n"
    "v_max3_f32 v34, v34, |v223|, |v224|\n"
    "v_max3_f32 v34, v34, |v225|, |v226|\n"
    "v_max_f32 v34, v34, |v227|\n"
    "v_cmp_lt_f32 s[28:29], s97, v34\n"
    "s_and_b64 s[28:29], s[28:29], s[26:27]\n"
    "s_cbranch_scc1 L_rn6_c%=\n"
    "v_fma_f32 v220, v220, s18, v33\n"
    "v_fma_f32 v221, v221, s18, v33\n"
    "v_fma_f32 v222, v222, s18, v33\n"
    "v_fma_f32 v223, v223, s18, v33\n"
    "v_fma_f32 v224, v224, s18, v33\n"
    "v_fma_f32 v225, v225, s18, v33\n"
    "v_fma_f32 v226, v226, s18, v33\n"
    "v_fma_f32 v227, v227, s18, v33\n"
    "v_max_f32 v220, v220, v33\n"
    "v_max_f32 v221, v221, v33\n"
    "v_max_f32 v222, v222, v33\n"
    "v_max_f32 v223, v223, v33\n"
    "v_max_f32 v224, v224, v33\n"
    "v_max_f32 v225, v225, v33\n"
    "v_max_f32 v226, v226, v33\n"
    "v_max_f32 v227, v227, v33\n"
    "s_add_u32 s90, s65, 16384\n"
    "s_lshl_b32 s90, s90, 5\n"
    "buffer_store_dwordx4 v[220:223], v23, s[48:51], s90 offen sc0 sc1\n"
    "buffer_store_dwordx4 v[224:227], v23, s[48:51], s90 offen offset:64 sc0 sc1\n"
    "s_bitset0_b32 s81, 6\n"
    "L_rn6_c%=:\n"
    "s_bitcmp1_b32 s81, 7\n"
    "s_cbranch_scc0 L_rn7_c%=\n"
    "v_add_u32 v228, v172, v228\n"
    "v_add_u32 v229, v173, v229\n"
    "v_add_u32 v230, v174, v230\n"
    "v_add_u32 v231, v175, v231\n"
    "v_add_u32 v232, v176, v232\n"
    "v_add_u32 v233, v177, v233\n"
    "v_add_u32 v234, v178, v234\n"
    "v_add_u32 v235, v179, v235\n"
    "v_cvt_f32_i32 v228, v228\n"
    "v_cvt_f32_i32 v229, v229\n"
    "v_cvt_f32_i32 v230, v230\n"
    "v_cvt_f32_i32 v231, v231\n"
    "v_cvt_f32_i32 v232, v232\n"
    "v_cvt_f32_i32 v233, v233\n"
    "v_cvt_f32_i32 v234, v234\n"
    "v_cvt_f32_i32 v235, v235\n"
    "v_max3_f32 v34, |v228|, |v229|, |v230|\n"
    "v_max3_f32 v34, v34, |v231|, |v232|\n"
    "v_max3_f32 v34, v34, |v233|, |v234|\n"
    "v_max_f32 v34, v34, |v235|\n"
    "v_cmp_lt_f32 s[28:29], s97, v34\n"
    "s_and_b64 s[28:29], s[28:29], s[26:27]\n"
    "s_cbranch_scc1 L_rn7_c%=\n"
    "v_fma_f32 v228, v228, s18, v33\n"
    "v_fma_f32 v229, v229, s18, v33\n"
    "v_fma_f32 v230, v230, s18, v33\n"
    "v_fma_f32 v231, v231, s18, v33\n"
    "v_fma_f32 v232, v232, s18, v33\n"
    "v_fma_f32 v233, v233, s18, v33\n"
    "v_fma_f32 v234, v234, s18, v33\n"
    "v_fma_f32 v235, v235, s18, v33\n"
    "v_max_f32 v228, v228, v33\n"
    "v_max_f32 v229, v229, v33\n"
    "v_max_f32 v230, v230, v33\n"
    "v_max_f32 v231, v231, v33\n"
    "v_max_f32 v232, v232, v33\n"
    "v_max_f32 v233, v233, v33\n"
    "v_max_f32 v234, v234, v33\n"
    "v_max_f32 v235, v235, v33\n"
    "s_add_u32 s90, s65, 40960\n"
    "s_lshl_b32 s90, s90, 5\n"
    "buffer_store_dwordx4 v[228:231], v23, s[48:51], s90 offen sc0 sc1\n"
    "buffer_store_dwordx4 v[232:235], v23, s[48:51], s90 offen offset:64 sc0 sc1\n"
    "s_bitset0_b32 s81, 7\n"
    "L_rn7_c%=:\n"
    "s_bitcmp1_b32 s81, 8\n"
    "s_cbranch_scc0 L_rn8_c%=\n"
    "v_add_u32 v236, v180, v236\n"
    "v_add_u32 v237, v181, v237\n"
    "v_add_u32 v238, v182, v238\n"
    "v_add_u32 v239, v183, v239\n"
    "v_add_u32 v240, v184, v240\n"
    "v_add_u32 v241, v185, v241\n"
    "v_add_u32 v242, v186, v242\n"
    "v_add_u32 v243, v187, v243\n"
    "v_cvt_f32_i32 v236, v236\n"
    "v_cvt_f32_i32 v237, v237\n"
    "v_cvt_f32_i32 v238, v238\n"
    "v_cvt_f32_i32 v239, v239\n"
    "v_cvt_f32_i32 v240, v240\n"
    "v_cvt_f32_i32 v241, v241\n"
    "v_cvt_f32_i32 v242, v242\n"
    "v_cvt_f32_i32 v243, v243\n"
    "v_max3_f32 v34, |v236|, |v237|, |v238|\n"
    "v_max3_f32 v34, v34, |v239|, |v240|\n"
    "v_max3_f32 v34, v34, |v241|, |v242|\n"
    "v_max_f32 v34, v34, |v243|\n"
    "v_cmp_lt_f32 s[28:29], s97, v34\n"
    "s_and_b64 s[28:29], s[28:29], s[26:27]\n"
    "s_cbranch_scc1 L_rn8_c%=\n"
    "v_fma_f32 v236, v236, s18, v33\n"
    "v_fma_f32 v237, v237, s18, v33\n"
    "v_fma_f32 v238, v238, s18, v33\n"
    "v_fma_f32 v239, v239, s18, v33\n"
    "v_fma_f32 v240, v240, s18, v33\n"
    "v_fma_f32 v241, v241, s18, v33\n"
    "v_fma_f32 v242, v242, s18, v33\n"
    "v_fma_f32 v243, v243, s18, v33\n"
    "v_max_f32 v236, v236, v33\n"
    "v_max_f32 v237, v237, v33\n"
    "v_max_f32 v238, v238, v33\n"
    "v_max_f32 v239, v239, v33\n"
    "v_max_f32 v240, v240, v33\n"
    "v_max_f32 v241, v241, v33\n"
    "v_max_f32 v242, v242, v33\n"
    "v_max_f32 v243, v243, v33\n"
    "s_add_u32 s90, s65, 65536\n"
    "s_lshl_b32 s90, s90, 5\n"
    "buffer_store_dwordx4 v[236:239], v23, s[48:51], s90 offen sc0 sc1\n"
    "buffer_store_dwordx4 v[240:243], v23, s[48:51], s90 offen offset:64 sc0 sc1\n"
    "s_bitset0_b32 s81, 8\n"
    "L_rn8_c%=:\n"
    "s_bitcmp1_b32 s81, 9\n"
    "s_cbranch_scc0 L_rn9_c%=\n"
    "v_add_u32 v244, v188, v244\n"
    "v_add_u32 v245, v189, v245\n"
    "v_add_u32 v246, v190, v246\n"
    "v_add_u32 v247, v191, v247\n"
    "v_add_u32 v248, v192, v248\n"
    "v_add_u32 v249, v193, v249\n"
    "v_add_u32 v250, v194, v250\n"
    "v_add_u32 v251, v195, v251\n"
    "v_cvt_f32_i32 v244, v244\n"
    "v_cvt_f32_i32 v245, v245\n"
    "v_cvt_f32_i32 v246, v246\n"
    "v_cvt_f32_i32 v247, v247\n"
    "v_cvt_f32_i32 v248, v248\n"
    "v_cvt_f32_i32 v249, v249\n"
    "v_cvt_f32_i32 v250, v250\n"
    "v_cvt_f32_i32 v251, v251\n"
    "v_max3_f32 v34, |v244|, |v245|, |v246|\n"
    "v_max3_f32 v34, v34, |v247|, |v248|\n"
    "v_max3_f32 v34, v34, |v249|, |v250|\n"
    "v_max_f32 v34, v34, |v251|\n"
    "v_cmp_lt_f32 s[28:29], s97, v34\n"
    "s_and_b64 s[28:29], s[28:29], s[26:27]\n"
    "s_cbranch_scc1 L_rn9_c%=\n"
    "v_fma_f32 v244, v244, s18, v33\n"
    "v_fma_f32 v245, v245, s18, v33\n"
    "v_fma_f32 v246, v246, s18, v33\n"
    "v_fma_f32 v247, v247, s18, v33\n"
    "v_fma_f32 v248, v248, s18, v33\n"
    "v_fma_f32 v249, v249, s18, v33\n"
    "v_fma_f32 v250, v250, s18, v33\n"
    "v_fma_f32 v251, v251, s18, v33\n"
    "v_max_f32 v244, v244, v33\n"
    "v_max_f32 v245, v245, v33\n"
    "v_max_f32 v246, v246, v33\n"
    "v_max_f32 v247, v247, v33\n"
    "v_max_f32 v248, v248, v33\n"
    "v_max_f32 v249, v249, v33\n"
    "v_max_f32 v250, v250, v33\n"
    "v_max_f32 v251, v251, v33\n"
    "s_add_u32 s90, s65, 90112\n"
    "s_lshl_b32 s90, s90, 5\n"
    "buffer_store_dwordx4 v[244:247], v23, s[48:51], s90 offen sc0 sc1\n"
    "buffer_store_dwordx4 v[248:251], v23, s[48:51], s90 offen offset:64 sc0 sc1\n"
    "s_bitset0_b32 s81, 9\n"
    "L_rn9_c%=:\n"
    "s_cmp_eq_u32 s81, 0\n"
    "s_cbranch_scc1 L_finc%=\n"
    "s_ff1_i32_b32 s33, s81\n"
    "s_sub_u32 s33, s33, 4\n"
    "s_lshl_b32 s33, s33, 2\n"
    "s_lshr_b32 s33, 0xb85210, s33\n"
    "s_and_b32 s33, s33, 15\n"
    "s_lshl_b32 s33, s33, 13\n"
    "s_add_u32 s33, s33, s65\n"
    "s_call_b64 s[88:89], L_slowqc%=\n"
    "s_cselect_b64 s[26:27], 0, -1\n"
    "s_branch L_redo2c%=\n"
    "L_slowc%=:\n"
    "s_add_u32 s88, s88, 8\n"
    "s_addc_u32 s89, s89, 0\n"
    "s_mov_b32 s25, -1\n"
    "s_cmp_gt_u32 s84, s95\n"
    "s_cbranch_scc1 L_landc%=\n"
    "L_slow2c%=:\n"
    "global_load_dword v32, v24, s[72:73] sc0 sc1\n"
    "s_waitcnt vmcnt(0)\n"
    "v_cmp_le_u32 vcc, s17, v32\n"
    "s_lshr_b32 s94, vcc_lo, 16\n"
    "s_and_b32 s94, s94, vcc_lo\n"
    "s_and_b32 s94, s94, vcc_hi\n"
    "s_ff0_i32_b32 s85, s94\n"
    "s_max_u32 s85, s85, s86\n"
    "s_bitcmp1_b32 vcc_hi, 24\n"
    "s_cselect_b32 s94, 0x7fc00000, 0\n"
    "v_mov_b32 v33, s94\n"
    "s_cmp_gt_u32 s85, s95\n"
    "s_cbranch_scc1 L_reqc%=\n"
    "s_sleep 2\n"
    "s_memrealtime s[92:93]\n"
    "s_bitcmp1_b32 vcc_hi, 24\n"
    "s_cbranch_scc1 L_slowgc%=\n"
    "s_waitcnt lgkmcnt(0)\n"
    "s_sub_u32 s92, s92, s20\n"
    "s_cmp_lt_u32 s92, 50000000\n"
    "s_cbranch_scc1 L_slow2c%=\n"
    "L_slowgc%=:\n"
    "s_waitcnt lgkmcnt(0)\n"
    "global_store_dword v25, v36, s[72:73] sc1\n"
    "s_mov_b32 s86, 16\n"
    "s_waitcnt vmcnt(0)\n"
    "global_load_dword v32, v24, s[72:73] sc0 sc1\n"
    "s_waitcnt vmcnt(0)\n"
    "v_cmp_le_u32 vcc, s17, v32\n"
    "s_lshr_b32 s94, vcc_lo, 16\n"
    "s_and_b32 s94, s94, vcc_lo\n"
    "s_and_b32 s94, s94, vcc_hi\n"
    "s_ff0_i32_b32 s85, s94\n"
    "s_max_u32 s85, s85, s86\n"
    "s_bitcmp1_b32 vcc_hi, 24\n"
    "s_cselect_b32 s94, 0x7fc00000, 0\n"
    "v_mov_b32 v33, s94\n"
    "s_branch L_reqc%=\n"
    "L_midc%=:\n"
    "v_cmp_le_u32 vcc, s17, v32\n"
    "s_lshr_b32 s94, vcc_lo, 16\n"
    "s_and_b32 s94, s94, vcc_lo\n"
    "s_and_b32 s94, s94, vcc_hi\n"
    "s_ff0_i32_b32 s85, s94\n"
    "s_max_u32 s85, s85, s86\n"
    "global_load_dword v32, v24, s[72:73] sc0 sc1\n"
    "s_cmp_gt_u32 s85, s84\n"
    "s_cbranch_scc1 L_mid2c%=\n"
    "s_add_u32 s88, s88, 4\n"
    "s_addc_u32 s89, s89, 0\n"
    "s_setpc_b64 s[88:89]\n"
    "L_mid2c%=:\n"
    "s_mov_b32 s25, 0\n"
    "L_catchc%=:\n"
    "s_add_u32 s88, s88, 4\n"
    "s_addc_u32 s89, s89, 0\n"
    "L_catch2c%=:\n"
    "s_mov_b32 s95, -100\n"
    "L_reqc%=:\n"
    "s_mov_b32 s31, m0\n"
    "L_req2c%=:\n"
    "s_lshl_b32 s94, s84, 2\n"
    "s_lshr_b64 s[92:93], s[98:99], s94\n"
    "s_and_b32 s92, s92, 7\n"
    "s_lshl_b32 s92, s92, 15\n"
    "s_lshl_b32 s94, s84, 13\n"
    "s_add_u32 m0, s11, s92\n"
    "s_add_u32 s19, s8, s94\n"
    "buffer_load_dwordx4 v22, s[40:43], s19 offen sc0 sc1 lds\n"
    "buffer_load_dwordx4 v22, s[40:43], s19 offen offset:1024 sc0 sc1 lds\n"
    "s_add_u32 m0, s12, s92\n"
    "s_add_u32 s19, s9, s94\n"
    "buffer_load_dwordx4 v22, s[40:43], s19 offen sc0 sc1 lds\n"
    "buffer_load_dwordx4 v22, s[40:43], s19 offen offset:1024 sc0 sc1 lds\n"
    "s_add_u32 m0, s13, s92\n"
    "s_add_u32 s19, s10, s94\n"
    "buffer_load_dwordx4 v22, s[40:43], s19 offen sc0 sc1 lds\n"
    "buffer_load_dwordx4 v22, s[40:43], s19 offen offset:1024 sc0 sc1 lds\n"
    "s_add_u32 s84, s84, 1\n"
    "s_min_u32 s94, s85, s96\n"
    "s_cmp_lt_u32 s84, s94\n"
    "s_cbranch_scc1 L_req2c%=\n"
    "s_mov_b32 m0, s31\n"
    "s_sub_u32 s94, s84, s95\n"
    "s_cmp_gt_u32 s94, 50\n"
    "s_cbranch_scc0 L_w2c%=\n"
    "s_setpc_b64 s[88:89]\n"
    "L_w2c%=:\n"
    "s_waitcnt vmcnt(12) lgkmcnt(0)\n"
    "s_cmp_gt_u32 s94, 2\n"
    "s_cbranch_scc1 L_w0c%=\n"
    "s_waitcnt vmcnt(6)\n"
    "s_cmp_eq_u32 s94, 2\n"
    "s_cbranch_scc1 L_w0c%=\n"
    "s_sleep 6\n"
    "L_landc%=:\n"
    "s_waitcnt lgkmcnt(0)\n"
    "global_load_dword v32, v24, s[72:73] sc0 sc1\n"
    "s_waitcnt vmcnt(0)\n"
    "v_cmp_le_u32 vcc, s17, v32\n"
    "s_lshr_b32 s94, vcc_lo, 16\n"
    "s_and_b32 s94, s94, vcc_lo\n"
    "s_and_b32 s94, s94, vcc_hi\n"
    "s_ff0_i32_b32 s85, s94\n"
    "s_max_u32 s85, s85, s86\n"
    "s_bitcmp1_b32 vcc_hi, 24\n"
    "s_cselect_b32 s94, 0x7fc00000, 0\n"
    "v_mov_b32 v33, s94\n"
    "s_min_u32 s94, s85, s96\n"
    "s_cmp_lt_u32 s84, s94\n"
    "s_cbranch_scc1 L_catch2c%=\n"
    "s_setpc_b64 s[88:89]\n"
    "L_w0c%=:\n"
    "s_setpc_b64 s[88:89]\n"
    "L_gatec%=:\n"
    "s_sleep 4\n"
    "global_load_dword v37, v26, s[72:73] sc0 sc1\n"
    "global_load_dword v32, v24, s[72:73] sc0 sc1\n"
    "s_waitcnt vmcnt(0)\n"
    "v_cmp_le_u32 vcc, s17, v32\n"
    "s_lshr_b32 s94, vcc_lo, 16\n"
    "s_and_b32 s94, s94, vcc_lo\n"
    "s_and_b32 s94, s94, vcc_hi\n"
    "s_ff0_i32_b32 s85, s94\n"
    "s_max_u32 s85, s85, s86\n"
    "s_bitcmp1_b32 vcc_hi, 24\n"
    "s_cselect_b32 s94, 0x7fc00000, 0\n"
    "v_mov_b32 v33, s94\n"
    "v_cmp_gt_u32 s[28:29], s17, v37\n"
    "s_cmp_lg_u64 s[28:29], 0\n"
    "s_cbranch_scc0 L_rgc%=\n"
    "s_memrealtime s[92:93]\n"
    "s_bitcmp1_b32 vcc_hi, 24\n"
    "s_cbranch_scc1 L_ggc%=\n"
    "s_waitcnt lgkmcnt(0)\n"
    "s_sub_u32 s92, s92, s20\n"
    "s_cmp_lt_u32 s92, 50000000\n"
    "s_cbranch_scc1 L_gatec%=\n"
    "L_ggc%=:\n"
    "s_waitcnt lgkmcnt(0)\n"
    "global_store_dword v25, v36, s[72:73] sc1\n"
    "s_mov_b32 s86, 16\n"
    "s_waitcnt vmcnt(0)\n"
    "global_load_dword v32, v24, s[72:73] sc0 sc1\n"
    "s_waitcnt vmcnt(0)\n"
    "v_cmp_le_u32 vcc, s17, v32\n"
    "s_lshr_b32 s94, vcc_lo, 16\n"
    "s_and_b32 s94, s94, vcc_lo\n"
    "s_and_b32 s94, s94, vcc_hi\n"
    "s_ff0_i32_b32 s85, s94\n"
    "s_max_u32 s85, s85, s86\n"
    "s_bitcmp1_b32 vcc_hi, 24\n"
    "s_cselect_b32 s94, 0x7fc00000, 0\n"
    "v_mov_b32 v33, s94\n"
    "s_branch L_rgc%=\n"
    "L_slowqc%=:\n"
    "global_load_dword v32, v24, s[72:73] sc0 sc1\n"
    "buffer_load_dword v34, v35, s[44:47], s33 offen offset:2044 sc0 sc1\n"
    "s_waitcnt vmcnt(0)\n"
    "v_cmp_le_u32 vcc, s17, v32\n"
    "s_lshr_b32 s94, vcc_lo, 16\n"
    "s_and_b32 s94, s94, vcc_lo\n"
    "s_and_b32 s94, s94, vcc_hi\n"
    "s_ff0_i32_b32 s85, s94\n"
    "s_max_u32 s85, s85, s86\n"
    "s_bitcmp1_b32 vcc_hi, 24\n"
    "s_cselect_b32 s94, 0x7fc00000, 0\n"
    "v_mov_b32 v33, s94\n"
    "v_add_u32 v34, v34, v8\n"
    "v_cvt_f32_i32 v34, v34\n"
    "v_cmp_lt_f32 s[28:29], s97, |v34|\n"
    "s_memrealtime s[92:93]\n"
    "s_bitcmp1_b32 vcc_hi, 24\n"
    "s_cbranch_scc1 L_qgc%=\n"
    "s_waitcnt lgkmcnt(0)\n"
    "s_sub_u32 s92, s92, s20\n"
    "s_cmp_lt_u32 s92, 50000000\n"
    "s_cbranch_scc1 L_qbc%=\n"
    "L_qgc%=:\n"
    "s_waitcnt lgkmcnt(0)\n"
    "global_store_dword v25, v36, s[72:73] sc1\n"
    "s_mov_b32 s86, 16\n"
    "s_waitcnt vmcnt(0)\n"
    "global_load_dword v32, v24, s[72:73] sc0 sc1\n"
    "s_waitcnt vmcnt(0)\n"
    "v_cmp_le_u32 vcc, s17, v32\n"
    "s_lshr_b32 s94, vcc_lo, 16\n"
    "s_and_b32 s94, s94, vcc_lo\n"
    "s_and_b32 s94, s94, vcc_hi\n"
    "s_ff0_i32_b32 s85, s94\n"
    "s_max_u32 s85, s85, s86\n"
    "s_bitcmp1_b32 vcc_hi, 24\n"
    "s_cselect_b32 s94, 0x7fc00000, 0\n"
    "v_mov_b32 v33, s94\n"
    "s_cmp_eq_u32 0, 0\n"
    "s_setpc_b64 s[88:89]\n"
    "L_qbc%=:\n"
    "s_cmp_lg_u64 s[28:29], 0\n"
    "s_cbranch_scc0 L_qcc%=\n"
    "s_sleep 4\n"
    "s_branch L_slowqc%=\n"
    "L_qcc%=:\n"
    "s_setpc_b64 s[88:89]\n"
    "L_end%=:\n"
    :
    : "{s[0:1]}"(__builtin_amdgcn_kernarg_segment_ptr()), "{s2}"(blockIdx.x), "{v0}"(threadIdx.x)
    :
      "memory", "vcc", "scc", "s3", "s4", "s5", "s6", "s7", "s8", "s9", "s10", "s11", "s12", "s13", "s14", "s15",
      "s16", "s17", "s18", "s19", "s20", "s21", "s22", "s23", "s24", "s25", "s26", "s27", "s28", "s29", "s30", "s31",
      "s33", "s34", "s35", "s36", "s37", "s38", "s39", "s40", "s41", "s42", "s43", "s44", "s45", "s46", "s47", "s48",
      "s49", "s50", "s51", "s52", "s53", "s54", "s55", "s56", "s57", "s58", "s59", "s60", "s61", "s62", "s63", "s64",
      "s65", "s66", "s67", "s68", "s69", "s70", "s71", "s72", "s73", "s74", "s75", "s76", "s77", "s78", "s79", "s80",
      "s81", "s82", "s83", "s84", "s85", "s86", "s87", "s88", "s89", "s90", "s91", "s92", "s93", "s94", "s95", "s96",
      "s97", "s98", "s99", "v1", "v2", "v3", "v4", "v5", "v6", "v7", "v8", "v9", "v10", "v11", "v12", "v13",
      "v14", "v15", "v16", "v17", "v18", "v19", "v20", "v21", "v22", "v23", "v24", "v25", "v26", "v27", "v28", "v29",
      "v30", "v31", "v32", "v33", "v34", "v35", "v36", "v37", "v38", "v39", "v40", "v41", "v42", "v43", "v44", "v45",
      "v46", "v47", "v48", "v49", "v50", "v51", "v52", "v53", "v54", "v55", "v56", "v57", "v58", "v59", "v60", "v61",
      "v62", "v63", "v64", "v65", "v66", "v67", "v68", "v69", "v70", "v71", "v72", "v73", "v74", "v75", "v76", "v77",
      "v78", "v79", "v80", "v81", "v82", "v83", "v84", "v85", "v86", "v87", "v88", "v89", "v90", "v91", "v92", "v93",
      "v94", "v95", "v96", "v97", "v98", "v99", "v100", "v101", "v102", "v103", "v104", "v105", "v106", "v107", "v108", "v109",
      "v110", "v111", "v112", "v113", "v114", "v115", "v116", "v117", "v118", "v119", "v120", "v121", "v122", "v123", "v124", "v125",
      "v126", "v127", "v128", "v129", "v130", "v131", "v132", "v133", "v134", "v135", "v136", "v137", "v138", "v139", "v140", "v141",
      "v142", "v143", "v144", "v145", "v146", "v147", "v148", "v149", "v150", "v151", "v152", "v153", "v154", "v155", "v156", "v157",
      "v158", "v159", "v160", "v161", "v162", "v163", "v164", "v165", "v166", "v167", "v168", "v169", "v170", "v171", "v172", "v173",
      "v174", "v175", "v176", "v177", "v178", "v179", "v180", "v181", "v182", "v183", "v184", "v185", "v186", "v187", "v188", "v189",
      "v190", "v191", "v192", "v193", "v194", "v195", "v196", "v197", "v198", "v199", "v200", "v201", "v202", "v203", "v204", "v205",
      "v206", "v207", "v208", "v209", "v210", "v211", "v212", "v213", "v214", "v215", "v216", "v217", "v218", "v219", "v220", "v221",
      "v222", "v223", "v224", "v225", "v226", "v227", "v228", "v229", "v230", "v231", "v232", "v233", "v234", "v235", "v236", "v237",
      "v238", "v239", "v240", "v241", "v242", "v243", "v244", "v245", "v246", "v247", "v248", "v249", "v250", "v251", "a0", "a1",
      "a2", "a3", "a4", "a5", "a6", "a7", "a8", "a9", "a10", "a11", "a12", "a13", "a14", "a15", "a16", "a17",
      "a18", "a19", "a20", "a21", "a22", "a23", "a24", "a25", "a26", "a27", "a28", "a29", "a30", "a31", "a32", "a33",
      "a34", "a35", "a36", "a37", "a38", "a39", "a40", "a41", "a42", "a43", "a44", "a45", "a46", "a47", "a48", "a49",
      "a50", "a51", "a52", "a53", "a54", "a55", "a56", "a57", "a58", "a59", "a60", "a61", "a62", "a63", "a64", "a65",
      "a66", "a67", "a68", "a69", "a70", "a71", "a72", "a73", "a74", "a75", "a76", "a77", "a78", "a79", "a80", "a81",
      "a82", "a83", "a84", "a85", "a86", "a87", "a88", "a89", "a90", "a91", "a92", "a93", "a94", "a95", "a96", "a97",
      "a98", "a99", "a100", "a101", "a102", "a103", "a104", "a105", "a106", "a107", "a108", "a109", "a110", "a111", "a112", "a113",
      "a114", "a115", "a116", "a117", "a118", "a119", "a120", "a121", "a122", "a123", "a124", "a125", "a126", "a127", "a128", "a129",
      "a130", "a131", "a132", "a133", "a134", "a135", "a136", "a137", "a138", "a139", "a140", "a141", "a142", "a143", "a144", "a145",
      "a146", "a147", "a148", "a149", "a150", "a151", "a152", "a153", "a154", "a155", "a156", "a157", "a158", "a159", "a160", "a161",
      "a162", "a163", "a164", "a165", "a166", "a167", "a168", "a169", "a170", "a171", "a172", "a173", "a174", "a175", "a176", "a177",
      "a178", "a179", "a180", "a181", "a182", "a183", "a184", "a185", "a186", "a187", "a188", "a189", "a190", "a191", "a192", "a193",
      "a194", "a195", "a196", "a197", "a198", "a199", "a200", "a201", "a202", "a203", "a204", "a205", "a206", "a207", "a208", "a209",
      "a210", "a211", "a212", "a213", "a214", "a215", "a216", "a217", "a218", "a219", "a220", "a221", "a222", "a223", "a224", "a225",
      "a226", "a227", "a228", "a229", "a230", "a231", "a232", "a233", "a234", "a235", "a236", "a237", "a238", "a239", "a240", "a241",
      "a242", "a243", "a244", "a245", "a246", "a247", "a248", "a249", "a250", "a251", "a252", "a253", "a254", "a255");
}

static int blk_a(int g, int o) { return 4 * o + g; }
static int blk_b(int g, int o) {
  static const int b11[11] = {16, 19, 22, 25, 28, 30, 3, 6, 9, 12, 14}, b10[10] = {18, 21, 24, 27, 0, 1, 2, 5, 8, 11};
  return g == 2 ? b10[o] : b11[o] + g;
}
static void put_srd(unsigned* d, const void* p, unsigned bytes) {
  const unsigned long long v = (unsigned long long)p;
  d[0] = (unsigned)v; d[1] = (unsigned)(v >> 32) & 0xffffu; d[2] = bytes; d[3] = 0x00020000u;
}
static void put_ptr(unsigned* d, const void* p) { const unsigned long long v = (unsigned long long)p; d[0] = (unsigned)v; d[1] = (unsigned)(v >> 32); }

extern "C" void kernel_launch(int rank, int nranks, ncclComm_t comm, hipStream_t stream,
                              void* const* d_in, const int* in_sizes, int n_in,
                              void* d_out, int out_size, const int* in_dtypes, int out_dtype,
                              void* const* d_out_all, void* const* d_ws_all, size_t ws_bytes) {
  if (nranks != NR || (unsigned)rank >= (unsigned)NR || ws_bytes < WS_NEED || n_in != 4 ||
      in_sizes[0] != KTOT * KS || in_sizes[1] != KTOT * NTOT || out_size != MLOC * NTOT) {
    fprintf(stderr, "unexpected shapes\n");
    return;
  }
  unsigned KA[4] = {KA0, KA1, KA2, KA3}, KB[3] = {KB0, KB1, KB2};
  int SB[3][3] = {{0, 1, 2}, {1, 0, 2}, {0, 2, 1}};
  const char* x = (const char*)d_in[0];
  char* ws = (char*)d_ws_all[rank];
  Args a;
  memset(&a, 0, sizeof a);
  put_srd(a.common + 0, d_in[1], KTOT * NTOT);
  put_srd(a.common + 4, ws + A_OFF, 0);
  put_srd(a.common + 8, ws + P_OFF, 32 * 8192);
  put_srd(a.common + 12, d_out, MLOC * NTOT * 4);
  put_srd(a.c2, x + (size_t)rank * MLOC * KS - 4096, MLOC * KS + 4096);
  put_ptr(a.c2 + 4, ws); put_ptr(a.c2 + 6, d_in[2]); put_ptr(a.c2 + 8, d_in[3]);
  a.c2[10] = rank * 8;
  for (int l = 0; l < 7; ++l) {
    const int d = (rank + 1 + l) & 7;
    put_ptr(a.link[l], x + (size_t)d * MLOC * KS - 512); a.link[l][1] &= 0xffffu;
    put_ptr(a.link[l] + 2, d_ws_all[d]);
  }
  for (int lb = 0; lb < 2; ++lb)
    for (int u = 0; u < 32; ++u) {
      const int st = lb ? u / 3 : u / 4, gp = lb ? u % 3 : u % 4;
      const unsigned kp = lb ? KB[gp] : KA[gp];
      for (int i = 0; i < 4; ++i) {
        const int o = lb ? TAB_B[st][i][0] : TAB_A[st][i][0], qs = lb ? TAB_B[st][i][1] : TAB_A[st][i][1];
        const int fr = lb ? blk_b(gp, o) : blk_a(gp, o), k0 = ((kp >> (8 * qs)) & 7) * 64, k1 = ((kp >> (8 * qs + 4)) & 7) * 64;
        a.snd[lb * 16 + (u & 15)][(u >> 4) * 4 + i] = (unsigned)(fr * 8192 + k0) | ((unsigned)(k1 - k0) << 20);
      }
    }
  for (int kind = 0; kind < 7; ++kind)
    for (int w = 0; w < 4; ++w) {
      const bool tb = kind >= 4; const int g = kind & 3;
      unsigned* t = a.tab[kind * 4 + w];
      const unsigned kp = tb ? KB[g] : KA[g];
      t[0] = (rank * 512 + (tb ? 256 : 0) + w * 64) * 4096;
      for (int c = 1; c < (tb ? 7 : 9); ++c) {
        const int n = 4 * (c - 1) + w;
        int ks = tb ? n / 3 : c - 1, src = tb ? 5 + n % 3 : 1 + w;
        if (1 && tb) { ks = (c - 1) / 3 * 4 + w; src = 5 + SB[g][(c - 1) % 3]; }
        t[c] = (((rank + src) & 7) * 512 + ((kp >> (4 * ks)) & 7) * 64) * 4096;
      }
      if (1 && tb) { t[7] = w < 3 ? SB[g][w] * 524288 : SB[g][0] * 524288 + 6144; t[8] = w < 3 ? t[7] + 2048 : SB[g][1] * 524288 + 6144; }
      const int ko = tb ? 256 : 0, m8 = g == 2 ? 9 : 10;
      t[9] = (tb ? blk_b(g, 2 * w) : blk_a(g, 2 * w)) * 8192 + ko;
      t[10] = (tb ? blk_b(g, 2 * w + 1) : blk_a(g, 2 * w + 1)) * 8192 + ko;
      t[11] = tb ? blk_b(g, 8 + w < m8 ? 8 + w : m8) * 8192 + ko : 0;
      t[12] = (tb ? blk_b(g, 0) : blk_a(g, 0)) * 8192; t[13] = (tb ? blk_b(g, g == 2 ? 4 : 6) : blk_a(g, 4)) * 8192;
      t[14] = ((tb ? 5 : 1 + w) * 4 + g) * 131072;
      t[15] = FLAG_OFF + ((tb ? (w < 3 ? 5 + (1 ? SB[g][w] : w) : 5) : 1 + w) * 4 + g) * 128;
    }
  (void)hipFuncSetAttribute((const void*)a2a_gemm, hipFuncAttributeMaxDynamicSharedMemorySize, L_TOTAL);
  hipLaunchKernelGGL(a2a_gemm, dim3(256), dim3(256), L_TOTAL, stream, a);
}
```
